```python
import numpy as np

kernel_name = "gpu_gemm_ar_m8192_k8192_n8192_bf16_relu"


ABSMAX_RTOL = 0.008

COMPUTE_DTYPE = "bfloat16"
EPILOGUE = "relu"
IN_QUANT = None
OUT_QUANT = None
IN_DTYPES = {"x": "bfloat16", "w_mat": "bfloat16"}
WORLD_SIZE = 8

def _to_bf16(x):
    u = np.ascontiguousarray(x, dtype=np.float32).view(np.uint32)
    return ((u + 0x7FFF + ((u >> 16) & 1)) & 0xFFFF0000).view(np.float32)

def setup_inputs():

    x = _to_bf16(np.random.randn(8192, 8192).astype(np.float32))
    w_mat = _to_bf16((np.random.randn(8192, 8192) / np.sqrt(8192)).astype(np.float32))
    return {"x": x, "w_mat": w_mat}

def _epilogue(y):
    return np.maximum(y, 0.0).astype(np.float32)

def reference(x, w_mat):
    y = (x @ w_mat).astype(np.float32)
    return _epilogue(y)
```

```cpp
#include <hip/hip_runtime.h>
#include <rccl/rccl.h>
#include <cstdio>
#include <cstdint>

#define HIP_CHECK(expr) do { hipError_t _e = (expr); if (_e != hipSuccess) { \
    fprintf(stderr, "HIP error %s at %s:%d\n", hipGetErrorString(_e), __FILE__, __LINE__); \
    return; } } while (0)

#define LAS __attribute__((address_space(3)))
#define GAS __attribute__((address_space(1)))
typedef unsigned short u16;
typedef unsigned long long u64;
typedef short bf16x8 __attribute__((ext_vector_type(8)));
typedef short s16x4 __attribute__((ext_vector_type(4)));
typedef float f32x4 __attribute__((ext_vector_type(4)));
typedef unsigned u32x4 __attribute__((ext_vector_type(4)));
typedef unsigned u32x2 __attribute__((ext_vector_type(2)));
typedef u16 u16x2 __attribute__((ext_vector_type(2)));
typedef GAS unsigned gu32;
typedef GAS u64 gu64;

#define FORCE_RAW_IN 0
#define FORCE_RAW_OUT 0
#define AW 3

namespace {

constexpr int NR = 8;
constexpr int ND = 8192;
constexpr int KS = 1024;
constexpr int RROWS = 4096, RCOLS = 2048;
constexpr int NSTEP = 128;
constexpr int NWG = 256;
constexpr int NW = 16 * AW;
constexpr int NENC = 1152 + 112 * (8 - AW);
constexpr unsigned long long DEADLINE = 40000000ull;

constexpr unsigned OFF_CALL = 0x0;
constexpr unsigned OFF_TMO = 0x80;
constexpr unsigned OFF_ECNT = 0x1000;
constexpr unsigned OFF_FIN = 0x8000;
constexpr unsigned OFF_XF = 0x10000;
constexpr unsigned OFF_WF = 0x18000;
constexpr unsigned OFF_FOUT = 0x20000;
constexpr unsigned USLOT = 8192 + 128;
constexpr unsigned OFF_EX = 1u << 20;
constexpr unsigned OFF_EW = OFF_EX + (17u << 20);
constexpr unsigned OFF_XL = OFF_EW + (17u << 20);
constexpr unsigned OFF_WL = OFF_XL + (64u << 20);
constexpr unsigned SSLOT = 12416;
constexpr unsigned OFF_EO = OFF_WL + (32u << 20);
constexpr unsigned WS_USED = OFF_EO + (13u << 20);
static_assert(2048u * USLOT <= (17u << 20), "unit area");
static_assert(1024u * SSLOT <= (13u << 20), "strip area");
constexpr unsigned OUT_BYTES = (unsigned)ND * ND * 2u;

constexpr int SMEM_BYTES = 131072 + 256;
constexpr int SM_SEQ = 131072, SM_AVAIL = 131072 + 16, SM_ZERO = 131072 + 64;

struct Params {
    const u16* x; const u16* w; u16* out;
    void* const* out_all; void* const* ws_all;
    int rank;
};

typedef __amdgpu_buffer_rsrc_t rsrc_t;
__device__ __forceinline__ rsrc_t make_rsrc(const void* base, unsigned bytes) {
    const u64 b = (u64)base;
    const unsigned lo = __builtin_amdgcn_readfirstlane((unsigned)b), hi = __builtin_amdgcn_readfirstlane((unsigned)(b >> 32));
    return __builtin_amdgcn_make_buffer_rsrc((void*)(((u64)hi << 32) | lo), 0, bytes, 0x00020000);
}
constexpr int SC1 = 16, SYS = 17;
template <int AUX> __device__ __forceinline__ u32x4 bld16(rsrc_t r, unsigned off) { return __builtin_amdgcn_raw_buffer_load_b128(r, off, 0, AUX); }
template <int AUX> __device__ __forceinline__ unsigned bld4(rsrc_t r, unsigned off) { return __builtin_amdgcn_raw_buffer_load_b32(r, off, 0, AUX); }
template <int AUX> __device__ __forceinline__ unsigned bld2(rsrc_t r, unsigned off) { return __builtin_amdgcn_raw_buffer_load_b16(r, off, 0, AUX); }
template <int AUX> __device__ __forceinline__ void bst16(rsrc_t r, unsigned off, u32x4 v) { __builtin_amdgcn_raw_buffer_store_b128(v, r, off, 0, AUX); }
template <int AUX> __device__ __forceinline__ void bst4(rsrc_t r, unsigned off, unsigned v) { __builtin_amdgcn_raw_buffer_store_b32(v, r, off, 0, AUX); }
template <int AUX> __device__ __forceinline__ void bst2(rsrc_t r, unsigned off, unsigned v) { __builtin_amdgcn_raw_buffer_store_b16((u16)v, r, off, 0, AUX); }

#define WAIT_V(n) asm volatile("s_waitcnt vmcnt(" #n ")" ::: "memory")
#define WAIT_L(n) asm volatile("s_waitcnt lgkmcnt(" #n ")" ::: "memory")
#define BAR __builtin_amdgcn_s_barrier()
#define SCHED __builtin_amdgcn_sched_barrier(0)
#define CFENCE asm volatile("" ::: "memory")

__device__ __forceinline__ unsigned rfl(unsigned v) { return __builtin_amdgcn_readfirstlane(v); }
__device__ __forceinline__ unsigned pkmax(unsigned a, unsigned b) {
    u16x2 x = __builtin_bit_cast(u16x2, a), y = __builtin_bit_cast(u16x2, b);
    return __builtin_bit_cast(unsigned, __builtin_elementwise_max(x, y));
}
__device__ __forceinline__ unsigned wave_max(unsigned v) {
#pragma unroll
    for (int o = 32; o >= 1; o >>= 1) { const unsigned t = (unsigned)__shfl_xor((int)v, o); v = v > t ? v : t; }
    return rfl(v);
}
__device__ __forceinline__ unsigned cvt_pk_bf16(float lo, float hi) { unsigned r; asm volatile("v_cvt_pk_bf16_f32 %0, %1, %2" : "=v"(r) : "v"(lo), "v"(hi)); return r; }
__device__ __forceinline__ int lds_byte(int r, int c) { const int st = (r >> 4) * 2 + (c >> 5), rr = r & 15, cc = c & 31, ob = rr * 64 + cc * 2; return st * 1024 + (ob ^ (((ob >> 9) & 1) << 5)); }

__device__ __forceinline__ bool give_up(gu32* tmo, unsigned seq, u64 t0) {
    bool dead = rfl(__hip_atomic_load(tmo, __ATOMIC_RELAXED, __HIP_MEMORY_SCOPE_AGENT)) == seq;
    if (!dead && __builtin_amdgcn_s_memrealtime() - t0 > DEADLINE) {
        __hip_atomic_store(tmo, seq, __ATOMIC_RELAXED, __HIP_MEMORY_SCOPE_AGENT);
        dead = true;
    }
    return dead;
}
__device__ __forceinline__ bool wait_sys32(gu32* flag, unsigned seq, gu32* tmo) {
    bool dead = false;
    if (rfl(__hip_atomic_load(flag, __ATOMIC_RELAXED, __HIP_MEMORY_SCOPE_SYSTEM)) < seq) {
        const u64 t0 = __builtin_amdgcn_s_memrealtime(); unsigned n = 0;
        while (rfl(__hip_atomic_load(flag, __ATOMIC_RELAXED, __HIP_MEMORY_SCOPE_SYSTEM)) < seq) {
            if ((++n & 63u) == 0u && give_up(tmo, seq, t0)) { dead = true; break; }
            __builtin_amdgcn_s_sleep(2);
        }
    }
    __builtin_amdgcn_fence(__ATOMIC_ACQUIRE, "wavefront");
    return dead;
}

__device__ __forceinline__ void pack_unit(const Params& p, char* ws, rsrc_t wsr, LAS unsigned char* lw, int t, int lane, unsigned seq) {
    const int j = t >> 8, r = t & 255; const bool isx = r >= 128; const int ub = r & 127;
    const int ra = p.rank >> 2, rb = p.rank & 3;
    u32x4 v[8];
    if (isx) {
        const u16* src = p.x + (size_t)(ub * 64 + (lane & 15)) * KS + j * 64 + (lane >> 4) * 8;
#pragma unroll
        for (int c = 0; c < 8; ++c) v[c] = *(const u32x4*)(src + (size_t)(c >> 1) * 16 * KS + (c & 1) * 32);
    } else {
        constexpr int P = 144;
        const u16* src = p.w + (size_t)(j * 64 + (lane >> 3)) * ND + ub * 64 + (lane & 7) * 8;
        u32x4 tmp[8];
#pragma unroll
        for (int i = 0; i < 8; ++i) tmp[i] = *(const u32x4*)(src + (size_t)i * 8 * ND);
#pragma unroll
        for (int i = 0; i < 8; ++i) *(LAS u32x4*)(lw + (i * 8 + (lane >> 3)) * P + (lane & 7) * 16) = tmp[i];
        const int q = (lane >> 2) & 3, pp = lane & 3;
#pragma unroll
        for (int c = 0; c < 8; ++c) {
            const int k0 = (c & 1) * 32 + (lane >> 4) * 8, rbk = c >> 1;
            const int col = (rbk >> 1) * 32 + 8 * pp + 4 * (rbk & 1);
            const s16x4 a = __builtin_amdgcn_ds_read_tr16_b64_v4i16((LAS s16x4*)(lw + (k0 + q) * P + col * 2));
            const s16x4 b = __builtin_amdgcn_ds_read_tr16_b64_v4i16((LAS s16x4*)(lw + (k0 + 4 + q) * P + col * 2));
            const u32x2 a2 = __builtin_bit_cast(u32x2, a), b2 = __builtin_bit_cast(u32x2, b);
            v[c] = (u32x4){a2.x, a2.y, b2.x, b2.y};
        }
    }
    const unsigned loff = (lane & 15) * 64 + (((lane >> 4) * 16) ^ (((lane >> 3) & 1) << 5));
    const int cls = isx ? (ub >> 6) : (ub >> 5);
    const bool own = isx ? (cls == ra) : (cls == rb);
    const int li = isx ? (ub & 63) : (ub & 31);
    if (own) {
        const unsigned dst = (isx ? OFF_XL + (unsigned)(j * 64 + li) * 8192u : OFF_WL + (unsigned)(j * 32 + li) * 8192u) + loff;
#pragma unroll
        for (int c = 0; c < 8; ++c) bst16<SC1>(wsr, dst + c * 1024, v[c]);
    }
    unsigned mx = 0;
#pragma unroll
    for (int c = 0; c < 8; ++c)
#pragma unroll
        for (int k = 0; k < 4; ++k) mx = pkmax(mx, v[c][k] & 0x7fff7fffu);
    mx = (mx & 0xffffu) > (mx >> 16) ? (mx & 0xffffu) : (mx >> 16);
    unsigned b7 = wave_max(mx) >> 8; b7 = b7 < 7u ? 7u : b7;
    const unsigned bq = b7 * 0x01010101u;
    unsigned lo[16], nb[8], bad = 0;
#pragma unroll
    for (int c = 0; c < 8; ++c) {
        lo[2 * c] = __builtin_amdgcn_perm(v[c].y, v[c].x, 0x06040200u); lo[2 * c + 1] = __builtin_amdgcn_perm(v[c].w, v[c].z, 0x06040200u);
        const unsigned ha = __builtin_amdgcn_perm(v[c].y, v[c].x, 0x07050301u), hb = __builtin_amdgcn_perm(v[c].w, v[c].z, 0x07050301u);
        const unsigned da = bq - (ha & 0x7f7f7f7fu), db = bq - (hb & 0x7f7f7f7fu);
        bad |= (da | db) & 0x78787878u;
        nb[c] = ((da & 0x07070707u) | ((ha >> 4) & 0x08080808u)) | (((db & 0x07070707u) << 4) | (hb & 0x80808080u));
    }
    unsigned cnt = 0;
    const unsigned slot0 = (isx ? OFF_EX : OFF_EW) + (unsigned)(j * 128 + ub) * USLOT;
    if (__builtin_amdgcn_ballot_w64(bad != 0u)) {
#pragma unroll
        for (int c = 0; c < 8; ++c)
#pragma unroll
            for (int e = 0; e < 8; ++e) {
                const unsigned val = (v[c][e >> 1] >> ((e & 1) * 16)) & 0xffffu;
                const bool ex = b7 - ((val >> 8) & 0x7fu) > 7u;
                const u64 m = __builtin_amdgcn_ballot_w64(ex);
                const unsigned sl = cnt + 1 + __builtin_amdgcn_mbcnt_hi((unsigned)(m >> 32), __builtin_amdgcn_mbcnt_lo((unsigned)m, 0u));
                if (ex && sl < 16u) bst4<SYS>(wsr, slot0 + 64 + sl * 4, ((((unsigned)c * 1024u + loff) / 2u + e) << 16) | val);
                cnt += (unsigned)__builtin_popcountll(m);
            }
    }
    const bool raw = FORCE_RAW_IN || cnt > 15u;
    if (lane == 0) bst4<SYS>(wsr, slot0 + 64, b7 | ((raw ? 0u : cnt) << 8) | ((raw ? 1u : 0u) << 16));
    const unsigned pay = slot0 + 128 + lane * 16;
    if (raw) {
#pragma unroll
        for (int c = 0; c < 8; ++c) bst16<SYS>(wsr, pay + c * 1024, v[c]);
    } else {
#pragma unroll
        for (int i = 0; i < 4; ++i) bst16<SYS>(wsr, pay + i * 1024, (u32x4){lo[4 * i], lo[4 * i + 1], lo[4 * i + 2], lo[4 * i + 3]});
#pragma unroll
        for (int k = 0; k < 2; ++k) bst16<SYS>(wsr, pay + 4096 + k * 1024, (u32x4){nb[4 * k], nb[4 * k + 1], nb[4 * k + 2], nb[4 * k + 3]});
    }
    WAIT_V(0);
    if (own && lane == 0) {
        const unsigned f = isx ? OFF_XF + (unsigned)(((li >> 2) * NSTEP + j) * 4 + (li & 3)) * 4u : OFF_WF + (unsigned)(((li >> 2) * NSTEP + j) * 4 + (li & 3)) * 4u;
        __hip_atomic_store((gu32*)(ws + f), seq, __ATOMIC_RELAXED, __HIP_MEMORY_SCOPE_AGENT);
    }
    unsigned ticket = 0;
    if (lane == 0) ticket = __hip_atomic_fetch_add((gu32*)(ws + OFF_ECNT + (unsigned)(j * 6 + (isx ? cls : 2 + cls)) * 128u), 1u, __ATOMIC_RELAXED, __HIP_MEMORY_SCOPE_AGENT);
    ticket = rfl(ticket);
    const unsigned per = isx ? 64u : 32u;
    if ((ticket & (per - 1u)) == per - 1u) {
        const bool want = lane < NR && lane != p.rank && (isx ? ((lane >> 2) == cls) : ((lane & 3) == cls));
        if (want) {
            char* pw = (char*)p.ws_all[lane];
            __hip_atomic_store((gu32*)(pw + OFF_FIN + (unsigned)p.rank * 128u + (unsigned)j * 8u + (isx ? 4u : 0u)), seq, __ATOMIC_RELAXED, __HIP_MEMORY_SCOPE_SYSTEM);
        }
    }
}

__device__ __forceinline__ void pull_unit(char* ws, rsrc_t wsr, rsrc_t pr, int rank, int step, int j, bool isx, int li, int lane, unsigned seq) {
    const int ub = isx ? (rank >> 2) * 64 + li : (rank & 3) * 32 + li;
    const unsigned slot0 = (isx ? OFF_EX : OFF_EW) + (unsigned)(j * 128 + ub) * USLOT;
    const unsigned pay = slot0 + 128 + lane * 16;
    const unsigned hdr = bld4<SYS>(pr, slot0 + 64 + (lane & 15) * 4);
    u32x4 a[6];
#pragma unroll
    for (int i = 0; i < 6; ++i) a[i] = bld16<SYS>(pr, pay + i * 1024);
    const unsigned loff = (lane & 15) * 64 + (((lane >> 4) * 16) ^ (((lane >> 3) & 1) << 5));
    const unsigned dst0 = isx ? OFF_XL + (unsigned)(step * 64 + li) * 8192u : OFF_WL + (unsigned)(step * 32 + li) * 8192u;
    const unsigned dst = dst0 + loff;
    const unsigned h0 = rfl(hdr);
    if (h0 >> 16) {
        const u32x4 a6 = bld16<SYS>(pr, pay + 6 * 1024), a7 = bld16<SYS>(pr, pay + 7 * 1024);
#pragma unroll
        for (int c = 0; c < 6; ++c) bst16<SC1>(wsr, dst + c * 1024, a[c]);
        bst16<SC1>(wsr, dst + 6 * 1024, a6); bst16<SC1>(wsr, dst + 7 * 1024, a7);
    } else {
        const unsigned bq = (h0 & 0xffu) * 0x01010101u;
#pragma unroll
        for (int c = 0; c < 8; ++c) {
            const unsigned la = a[c >> 1][(c & 1) * 2], lb = a[c >> 1][(c & 1) * 2 + 1], n = a[4 + (c >> 2)][c & 3];
            const unsigned ha = (bq - (n & 0x07070707u)) | ((n & 0x08080808u) << 4);
            const unsigned hb = (bq - ((n >> 4) & 0x07070707u)) | (n & 0x80808080u);
            u32x4 o;
            o.x = __builtin_amdgcn_perm(ha, la, 0x05010400u); o.y = __builtin_amdgcn_perm(ha, la, 0x07030602u);
            o.z = __builtin_amdgcn_perm(hb, lb, 0x05010400u); o.w = __builtin_amdgcn_perm(hb, lb, 0x07030602u);
            bst16<SC1>(wsr, dst + c * 1024, o);
        }
        const unsigned cnt = (h0 >> 8) & 0xffu;
        if (cnt) {
            WAIT_V(0);
            if (lane >= 1 && (unsigned)lane <= cnt && lane < 16) bst2<SC1>(wsr, dst0 + ((hdr >> 16) & 4095u) * 2u, hdr & 0xffffu);
        }
    }
    WAIT_V(0);
    if (lane == 0) {
        const unsigned f = (isx ? OFF_XF : OFF_WF) + (unsigned)(((li >> 2) * NSTEP + step) * 4 + (li & 3)) * 4u;
        __hip_atomic_store((gu32*)(ws + f), seq, __ATOMIC_RELAXED, __HIP_MEMORY_SCOPE_AGENT);
    }
}

__device__ __forceinline__ unsigned strip_out_off(int r, int s, int lane) {
    const int tile = s >> 3, tm = tile >> 3, tn = tile & 7;
    const int row = (r >> 2) * RROWS + tm * 256 + (s & 7) * 32 + (lane >> 5);
    const int col = (r & 3) * RCOLS + tn * 256 + (lane & 31) * 8;
    return ((unsigned)row * ND + col) * 2u;
}

__device__ __forceinline__ void pull_strip(rsrc_t outr, rsrc_t pr, rsrc_t por, LAS unsigned char* smem, unsigned lbase, int peer, int s, u64 fl, bool dead, int lane) {
    const unsigned oo = strip_out_off(peer, s, lane);
    const unsigned info = (unsigned)fl;
    const unsigned T = info & 0x3fffu, b7 = (info >> 14) & 0x7fu, cnt = (info >> 21) & 15u, raw = (info >> 25) & 1u;
    if (dead) {
#pragma unroll
        for (int q = 0; q < 16; ++q) bst16<0>(outr, oo + q * 2 * ND * 2, (u32x4){0x7fc07fc0u, 0x7fc07fc0u, 0x7fc07fc0u, 0x7fc07fc0u});
        return;
    }
    if (raw) {
#pragma unroll
        for (int h = 0; h < 2; ++h) {
            u32x4 a[8];
#pragma unroll
            for (int q = 0; q < 8; ++q) a[q] = bld16<SYS>(por, oo + (h * 8 + q) * 2 * ND * 2);
#pragma unroll
            for (int q = 0; q < 8; ++q) bst16<0>(outr, oo + (h * 8 + q) * 2 * ND * 2, a[q]);
        }
        return;
    }
    const unsigned slot = OFF_EO + (unsigned)s * SSLOT;
    const u32x4 mk = bld16<SYS>(pr, slot + lane * 16);
    unsigned exc = 0;
    if (cnt) exc = bld4<SYS>(pr, slot + 12288 + (lane & 15) * 4);
    u32x4 lo[8]; unsigned p2[8], p1[8];
#pragma unroll
    for (int r = 0; r < 8; ++r) {
        if ((unsigned)r * 1024u < T) {
            if ((unsigned)(r * 1024 + lane * 16) < T) {
                lo[r] = bld16<SYS>(pr, slot + 4096 + r * 1024 + lane * 16);
                p2[r] = bld4<SYS>(pr, slot + 2048 + r * 256 + lane * 4);
                p1[r] = bld2<SYS>(pr, slot + 1024 + r * 128 + lane * 2);
            }
        }
    }
    const unsigned bq = b7 * 0x01010101u;
#pragma unroll
    for (int r = 0; r < 8; ++r) {
        if ((unsigned)r * 1024u < T) {
            if ((unsigned)(r * 1024 + lane * 16) < T) {
                const unsigned z = __builtin_amdgcn_perm(0u, p1[r], 0x01010000u);
                const unsigned x1 = (z & 0x000f000fu) | ((z >> 4) & 0x0f000f00u);
                u32x4 o[2];
#pragma unroll
                for (int k = 0; k < 4; ++k) {
                    const unsigned d = ((p2[r] >> (2 * k)) & 0x03030303u) | (((x1 >> k) & 0x01010101u) << 2);
                    const unsigned h = bq - d;
                    o[k >> 1][(k & 1) * 2] = __builtin_amdgcn_perm(h, lo[r][k], 0x05010400u);
                    o[k >> 1][(k & 1) * 2 + 1] = __builtin_amdgcn_perm(h, lo[r][k], 0x07030602u);
                }
                *(LAS u32x4*)(smem + lbase + r * 2048 + lane * 32) = o[0];
                *(LAS u32x4*)(smem + lbase + r * 2048 + lane * 32 + 16) = o[1];
            }
        }
    }
    if (cnt) { if ((unsigned)lane < cnt) *(LAS u16*)(smem + lbase + ((exc >> 16) & 8191u) * 2u) = (u16)exc; }
    unsigned pos = lbase >> 1;
#pragma unroll
    for (int q = 0; q < 16; ++q) {
        unsigned val[8];
#pragma unroll
        for (int e = 0; e < 8; ++e) {
            const int d = (q * 8 + e) * 2;
            const unsigned ml = __builtin_amdgcn_readlane(mk[d >> 6], d & 63), mh = __builtin_amdgcn_readlane(mk[d >> 6], (d & 63) + 1);
            const unsigned idx = __builtin_amdgcn_mbcnt_hi(mh, __builtin_amdgcn_mbcnt_lo(ml, pos));
            const bool on = __builtin_amdgcn_inverse_ballot_w64(((u64)mh << 32) | ml);
            const unsigned a = on ? idx * 2u : (unsigned)SM_ZERO;
            val[e] = *(const LAS u16*)(smem + a);
            pos += (unsigned)__builtin_popcount(ml) + (unsigned)__builtin_popcount(mh);
        }
        bst16<0>(outr, oo + q * 2 * ND * 2, (u32x4){val[0] | (val[1] << 16), val[2] | (val[3] << 16), val[4] | (val[5] << 16), val[6] | (val[7] << 16)});
    }
}

__device__ __forceinline__ void pack_strip(const Params& p, rsrc_t wsr, rsrc_t outr, LAS unsigned char* smem, int tile, int wid, int lane, unsigned seq) {
    const int s = tile * 8 + wid;
    const unsigned oo = strip_out_off(p.rank, s, lane);
    const unsigned lbase = (unsigned)wid * 16384u;
    unsigned pos = lbase >> 1, mx = 0, sg = 0;
    u32x4 mk = {0u, 0u, 0u, 0u};
#pragma unroll
    for (int q = 0; q < 16; ++q) {
        const int row = wid * 32 + 2 * q + (lane >> 5);
        const u32x4 v = *(const LAS u32x4*)(smem + row * 512 + (((lane & 31) ^ (row & 7)) * 16));
        bst16<SYS>(outr, oo + q * 2 * ND * 2, v);
#pragma unroll
        for (int k = 0; k < 4; ++k) { mx = pkmax(mx, v[k]); sg |= v[k]; }
#pragma unroll
        for (int e = 0; e < 8; ++e) {
            const unsigned val = (v[e >> 1] >> ((e & 1) * 16)) & 0xffffu;
            const bool nz = val != 0u;
            const u64 m = __builtin_amdgcn_ballot_w64(nz);
            const unsigned ml = (unsigned)m, mh = (unsigned)(m >> 32);
            const unsigned idx = __builtin_amdgcn_mbcnt_hi(mh, __builtin_amdgcn_mbcnt_lo(ml, pos));
            if (nz) *(LAS u16*)(smem + idx * 2u) = (u16)val;
            const int d = (q * 8 + e) * 2;
            { unsigned mv = mk[d >> 6];
              asm volatile("s_nop 1\n\tv_writelane_b32 %0, %1, %3\n\tv_writelane_b32 %0, %2, %4" : "+v"(mv) : "s"(ml), "s"(mh), "i"(d & 63), "i"((d & 63) + 1));
              mk[d >> 6] = mv; }
            pos += (unsigned)__builtin_popcount(ml) + (unsigned)__builtin_popcount(mh);
        }
    }
    const unsigned T = pos - (lbase >> 1);
    mx = (mx & 0xffffu) > (mx >> 16) ? (mx & 0xffffu) : (mx >> 16);
    unsigned b7 = (wave_max(mx) >> 8) & 0x7fu; b7 = b7 < 7u ? 7u : b7;
    bool raw = FORCE_RAW_OUT || __builtin_amdgcn_ballot_w64((sg & 0x80008000u) != 0u) != 0ull;
    const unsigned slot = OFF_EO + (unsigned)s * SSLOT;
    unsigned cnt = 0;
    if (!raw) {
        if (lane < 16 && T + lane < 8192u) *(LAS u16*)(smem + lbase + (T + lane) * 2u) = (u16)(b7 << 8);
        const unsigned bq = b7 * 0x01010101u;
        bst16<SYS>(wsr, slot + lane * 16, mk);
        for (unsigned r = 0; r * 1024u < T; ++r) {
            const bool act = r * 1024u + lane * 16u < T;
            u32x4 a, b;
            a = *(const LAS u32x4*)(smem + lbase + r * 2048u + lane * 32u); b = *(const LAS u32x4*)(smem + lbase + r * 2048u + lane * 32u + 16u);
            const unsigned w8[8] = {a.x, a.y, a.z, a.w, b.x, b.y, b.z, b.w};
            u32x4 lo; unsigned p2 = 0, x1 = 0, bad = 0;
#pragma unroll
            for (int k = 0; k < 4; ++k) {
                lo[k] = __builtin_amdgcn_perm(w8[2 * k + 1], w8[2 * k], 0x06040200u);
                const unsigned d = bq - __builtin_amdgcn_perm(w8[2 * k + 1], w8[2 * k], 0x07050301u);
                bad |= d & 0xf8f8f8f8u;
                p2 |= (d & 0x03030303u) << (2 * k); x1 |= ((d >> 2) & 0x01010101u) << k;
            }
            const unsigned y = x1 | (x1 >> 4);
            const unsigned p1 = __builtin_amdgcn_perm(0u, y, 0x0c0c0200u);
            if (act) {
                bst16<SYS>(wsr, slot + 4096 + r * 1024u + lane * 16u, lo);
                bst4<SYS>(wsr, slot + 2048 + r * 256u + lane * 4u, p2);
                bst2<SYS>(wsr, slot + 1024 + r * 128u + lane * 2u, p1);
            }
            if (__builtin_amdgcn_ballot_w64(act && bad != 0u)) {
#pragma unroll
                for (int e = 0; e < 16; ++e) {
                    const unsigned val = (w8[e >> 1] >> ((e & 1) * 16)) & 0xffffu;
                    const bool ex = act && (b7 - (val >> 8) > 7u);
                    const u64 m = __builtin_amdgcn_ballot_w64(ex);
                    const unsigned sl = cnt + __builtin_amdgcn_mbcnt_hi((unsigned)(m >> 32), __builtin_amdgcn_mbcnt_lo((unsigned)m, 0u));
                    if (ex && sl < 15u) bst4<SYS>(wsr, slot + 12288 + sl * 4, ((r * 1024u + lane * 16u + e) << 16) | val);
                    cnt += (unsigned)__builtin_popcountll(m);
                }
            }
        }
        if (cnt > 15u) raw = true;
    }
    WAIT_V(0);
    if (lane < NR && lane != p.rank) {
        const unsigned info = raw ? (1u << 25) : (T | (b7 << 14) | (cnt << 21));
        char* pw = (char*)p.ws_all[lane];
        __hip_atomic_store((gu64*)(pw + OFF_FOUT + ((unsigned)p.rank * 1024u + (unsigned)s) * 8u), ((u64)seq << 32) | info, __ATOMIC_RELAXED, __HIP_MEMORY_SCOPE_SYSTEM);
    }
}

}

__global__ void __launch_bounds__(512, 2) gemm_ar_fused(Params p) {
    __shared__ __attribute__((aligned(16))) unsigned char smem_[SMEM_BYTES];
    LAS unsigned char* smem = (LAS unsigned char*)smem_;
    const int tid = threadIdx.x, wid = rfl(tid >> 6), lane = tid & 63;
    char* ws = (char*)p.ws_all[p.rank];
    const rsrc_t wsr = make_rsrc(ws, WS_USED);
    const rsrc_t outr = make_rsrc(p.out, OUT_BYTES);
    gu32* tmo = (gu32*)(ws + OFF_TMO);
    if (tid == 0) {
        const unsigned ticket = __hip_atomic_fetch_add((gu32*)(ws + OFF_CALL), 1u, __ATOMIC_RELAXED, __HIP_MEMORY_SCOPE_AGENT);
        *(LAS unsigned*)(smem + SM_SEQ) = ticket / NWG + 1u;
        *(LAS unsigned*)(smem + SM_ZERO) = 0u;
    }
    __syncthreads();
    const unsigned seq = rfl(*(LAS unsigned*)(smem + SM_SEQ));
    const int xcd = blockIdx.x & 7, wi = blockIdx.x >> 3;
    const bool is_gemm = wi < 16;
    const int lq = (wi - 16) & 7, lsub = xcd * 2 + ((wi - 16) >> 3);
    const bool is_link = !is_gemm && lq < 7 && wid < AW;

    if (!is_link) {
        int e;
        if (is_gemm) e = (wi * 8 + xcd) * 8 + wid;
        else if (lq == 7) e = 1024 + lsub * 8 + wid;
        else e = 1152 + (lsub * 7 + lq) * (8 - AW) + (wid - AW);
        for (int t = e; t < 4096; t += NENC) pack_unit(p, ws, wsr, smem + wid * 16384, t, lane, seq);
        if (!is_gemm) return;
    } else {
        const int peer = (p.rank + 1 + lq) & 7;
        const rsrc_t pr = make_rsrc(p.ws_all[peer], WS_USED);
        const rsrc_t por = make_rsrc(p.out_all[peer], OUT_BYTES);
        const int lw = lsub * AW + wid;
        bool dead = false;
        int okw = -1, okx = -1;
        for (int i = lw; i < 1536; i += NW) {
            const int j = i / 96, r = i - j * 96; const bool isx = r >= 32; const int li = isx ? r - 32 : r;
            if (isx ? (okx < j) : (okw < j)) {
                if (!dead) dead = wait_sys32((gu32*)(ws + OFF_FIN + (unsigned)peer * 128u + (unsigned)j * 8u + (isx ? 4u : 0u)), seq, tmo);
                if (isx) okx = j; else okw = j;
            }
            pull_unit(ws, wsr, pr, p.rank, 16 + j * 7 + lq, j, isx, li, lane, seq);
        }
        for (int s = lw; s < 1024; s += NW) {
            gu64* fp = (gu64*)(ws + OFF_FOUT + ((unsigned)peer * 1024u + (unsigned)s) * 8u);
            u64 fl = 0;
            if (!dead) {
                const u64 t0 = __builtin_amdgcn_s_memrealtime(); unsigned n = 0;
                for (;;) {
                    fl = __hip_atomic_load(fp, __ATOMIC_RELAXED, __HIP_MEMORY_SCOPE_SYSTEM);
                    fl = ((u64)rfl((unsigned)(fl >> 32)) << 32) | rfl((unsigned)fl);
                    if ((unsigned)(fl >> 32) >= seq) break;
                    if ((++n & 63u) == 0u && give_up(tmo, seq, t0)) { dead = true; break; }
                    __builtin_amdgcn_s_sleep(2);
                }
                __builtin_amdgcn_fence(__ATOMIC_ACQUIRE, "wavefront");
            }
            pull_strip(outr, pr, por, smem, (unsigned)wid * 16384u, peer, s, fl, dead, lane);
        }
        return;
    }

    __syncthreads();
    const int tm = (xcd >> 1) * 4 + (wi >> 2), tn = (xcd & 1) * 4 + (wi & 3);
    const int wr = wid >> 2, wc = wid & 3, fr = lane & 15, fq = lane >> 4;
    constexpr int HTB = 16384;
    const char* cA = ws + OFF_XL + (size_t)tm * 32768 + tid * 16;
    const char* cB = ws + OFF_WL + (size_t)tn * 32768 + tid * 16;
    constexpr size_t ASTEP = 64 * 8192, BSTEP = 32 * 8192;
    const unsigned ldsw = (unsigned)wid * 1024u;
    const int aoff = lds_byte(wr * 64 + fr, fq * 8), boff = lds_byte(wc * 32 + fr, fq * 8);
    const unsigned pollo = (lane < 32 ? OFF_XF + (unsigned)(tm * NSTEP + lane) * 16u : OFF_WF + (unsigned)(tn * NSTEP + lane - 32) * 16u);
    unsigned avail = 0;
#define SA(b, h) (((b) * 2 + (h)) * HTB)
#define SB(b, h) ((4 + (b) * 2 + (h)) * HTB)
#define STAGE(bufoff, gptr) do { _Pragma("unroll") for (int _i = 0; _i < 2; ++_i) \
        __builtin_amdgcn_global_load_lds((const unsigned*)((gptr) + _i * 8192), (LAS unsigned*)(smem + (bufoff) + ldsw + _i * 8192), 16, 0, SC1); } while (0)
#define LDA(dst, b, h) do { _Pragma("unroll") for (int m = 0; m < 4; ++m) _Pragma("unroll") for (int k = 0; k < 2; ++k) dst[m][k] = *(const LAS bf16x8*)(smem + SA(b, h) + aoff + m * 2048 + k * 1024); } while (0)
#define LDB(dst, b, h) do { _Pragma("unroll") for (int n = 0; n < 2; ++n) _Pragma("unroll") for (int k = 0; k < 2; ++k) dst[n][k] = *(const LAS bf16x8*)(smem + SB(b, h) + boff + n * 2048 + k * 1024); } while (0)
#define MMA(ai, bj, At, Bt) do { __builtin_amdgcn_s_setprio(1); _Pragma("unroll") for (int m = 0; m < 4; ++m) _Pragma("unroll") for (int n = 0; n < 2; ++n) _Pragma("unroll") for (int k = 0; k < 2; ++k) \
        acc[ai][bj][m][n] = __builtin_amdgcn_mfma_f32_16x16x32_bf16(Bt[n][k], At[m][k], acc[ai][bj][m][n], 0, 0, 0); __builtin_amdgcn_s_setprio(0); } while (0)
#define NEED(s) do { if ((avail & 0xffffu) <= (unsigned)(s)) { \
        if (wid == 0) { const unsigned s0 = avail & 0xffffu; unsigned av; const u64 t0 = __builtin_amdgcn_s_memrealtime(); unsigned n = 0; \
            for (;;) { const u32x4 f = bld16<SC1>(wsr, pollo + s0 * 16u); \
                const bool ok = (s0 + (lane & 31) >= (unsigned)NSTEP) || (f.x >= seq && f.y >= seq && f.z >= seq && f.w >= seq); \
                const u64 mm = __builtin_amdgcn_ballot_w64(ok); const unsigned both = (unsigned)mm & (unsigned)(mm >> 32); \
                av = s0 + (both == 0xffffffffu ? 32u : (unsigned)__builtin_ctz(~both)); \
                if (av > (unsigned)(s)) break; \
                if ((++n & 63u) == 0u && give_up(tmo, seq, t0)) { av = 0x10000u | NSTEP; break; } \
                __builtin_amdgcn_s_sleep(1); } \
            asm volatile("ds_write_b32 %0, %1\n\ts_waitcnt lgkmcnt(0)" :: "v"(smbase + SM_AVAIL), "v"(av) : "memory"); } \
        CFENCE; BAR; CFENCE; \
        unsigned av2; asm volatile("ds_read_b32 %0, %1\n\ts_waitcnt lgkmcnt(0)" : "=v"(av2) : "v"(smbase + SM_AVAIL) : "memory"); \
        avail = rfl(av2); } } while (0)
    const unsigned smbase = (unsigned)(uintptr_t)smem;

    f32x4 acc[2][2][4][2];
#pragma unroll
    for (int a = 0; a < 2; ++a)
#pragma unroll
        for (int b = 0; b < 2; ++b)
#pragma unroll
            for (int m = 0; m < 4; ++m)
#pragma unroll
                for (int n = 0; n < 2; ++n) acc[a][b][m][n] = (f32x4){0.f, 0.f, 0.f, 0.f};
    bf16x8 At[4][2], B0[2][2], B1[2][2];
    NEED(1);
    STAGE(SB(0, 0), cB); STAGE(SA(0, 0), cA); STAGE(SB(0, 1), cB + HTB); STAGE(SA(0, 1), cA + HTB);
    if (wr == 1) BAR;
    WAIT_V(4); BAR;
    STAGE(SB(1, 0), cB + BSTEP); STAGE(SA(1, 0), cA + ASTEP); STAGE(SB(1, 1), cB + BSTEP + HTB);
    WAIT_V(6); BAR;
    for (int t = 0; t < NSTEP; t += 2) {
        const bool last = (t == NSTEP - 2);
        const char* a1 = cA + (size_t)(t + 1) * ASTEP;
        const char* a2 = last ? cA : cA + (size_t)(t + 2) * ASTEP; const char* b2 = last ? cB : cB + (size_t)(t + 2) * BSTEP;
        const char* a3 = a2 + ASTEP; const char* b3 = b2 + BSTEP;
        if (!last) NEED(t + 2);
        LDB(B0, 0, 0); SCHED; LDA(At, 0, 0); STAGE(SA(1, 1), a1 + HTB);
        WAIT_L(8); BAR; WAIT_L(0); MMA(0, 0, At, B0); BAR; SCHED;
        LDB(B1, 0, 1); STAGE(SB(0, 0), b2);
        BAR; WAIT_L(0); MMA(0, 1, At, B1); BAR;
        LDA(At, 0, 1); STAGE(SA(0, 0), a2);
        BAR; WAIT_L(0); MMA(1, 0, At, B0); BAR; SCHED;
        STAGE(SB(0, 1), b2 + HTB);
        WAIT_V(6); BAR; MMA(1, 1, At, B1); BAR;
        if (!last) NEED(t + 3);
        LDB(B0, 1, 0); SCHED; LDA(At, 1, 0); STAGE(SA(0, 1), a2 + HTB);
        WAIT_L(8); BAR; WAIT_L(0); MMA(0, 0, At, B0); BAR; SCHED;
        LDB(B1, 1, 1); STAGE(SB(1, 0), b3);
        BAR; WAIT_L(0); MMA(0, 1, At, B1); BAR;
        LDA(At, 1, 1); STAGE(SA(1, 0), a3);
        BAR; WAIT_L(0); MMA(1, 0, At, B0); BAR; SCHED;
        STAGE(SB(1, 1), b3 + HTB);
        WAIT_V(6); BAR; MMA(1, 1, At, B1); BAR;
    }
    WAIT_V(0);
    if (wr == 0) BAR;
    BAR;
    {
        int tz = threadIdx.x; asm volatile("" : "+v"(tz));
        const int lane2 = tz & 63, fr2 = lane2 & 15, fq2 = lane2 >> 4;
        const bool dead = (avail >> 16) != 0u || rfl(__hip_atomic_load(tmo, __ATOMIC_RELAXED, __HIP_MEMORY_SCOPE_AGENT)) == seq;
#pragma unroll
        for (int ai = 0; ai < 2; ++ai)
#pragma unroll
            for (int m = 0; m < 4; ++m) {
                const int row = ai * 128 + wr * 64 + m * 16 + fr2;
#pragma unroll
                for (int bj = 0; bj < 2; ++bj) {
                    f32x4 v0 = acc[ai][bj][m][0], v1 = acc[ai][bj][m][1];
#pragma unroll
                    for (int k = 0; k < 4; ++k) { v0[k] = v0[k] <= 0.f ? 0.f : v0[k]; v1[k] = v1[k] <= 0.f ? 0.f : v1[k]; }
                    u32x4 w; w.x = cvt_pk_bf16(v0[0], v0[1]); w.y = cvt_pk_bf16(v0[2], v0[3]); w.z = cvt_pk_bf16(v1[0], v1[1]); w.w = cvt_pk_bf16(v1[2], v1[3]);
                    if (dead) w = (u32x4){0x7fc07fc0u, 0x7fc07fc0u, 0x7fc07fc0u, 0x7fc07fc0u};
                    const int chunk = bj * 16 + wc * 4 + fq2;
                    *(LAS u32x4*)(smem + row * 512 + ((chunk ^ (row & 7)) * 16)) = w;
                }
            }
        __syncthreads();
        pack_strip(p, wsr, outr, smem, tm * 8 + tn, wid, lane2, seq);
    }
}

extern "C" void kernel_launch(int rank, int nranks, ncclComm_t comm,
                              hipStream_t stream,
                              void* const* d_in, const int* in_sizes, int n_in,
                              void* d_out, int out_size,
                              const int* in_dtypes, int out_dtype,
                              void* const* d_out_all,
                              void* const* d_ws_all, size_t ws_bytes) {
    (void)comm; (void)in_dtypes; (void)out_dtype;
    if (nranks != NR || n_in != 2 || (unsigned)rank >= (unsigned)NR || ws_bytes < WS_USED || in_sizes[0] != ND * KS || in_sizes[1] != KS * ND || out_size != ND * ND) {
        fprintf(stderr, "kernel_launch: unexpected problem shape\n");
        return;
    }
    Params p;
    p.x = (const u16*)d_in[0]; p.w = (const u16*)d_in[1]; p.out = (u16*)d_out;
    p.out_all = d_out_all; p.ws_all = d_ws_all; p.rank = rank;
    hipLaunchKernelGGL(gemm_ar_fused, dim3(NWG), dim3(512), 0, stream, p);
    HIP_CHECK(hipGetLastError());
}
```

```cpp
#include <hip/hip_runtime.h>
#include <rccl/rccl.h>
#include <cstdio>
#include <cstdint>

#define HIP_CHECK(expr) do { hipError_t _e = (expr); if (_e != hipSuccess) { \
    fprintf(stderr, "HIP error %s at %s:%d\n", hipGetErrorString(_e), __FILE__, __LINE__); \
    return; } } while (0)

#define LAS __attribute__((address_space(3)))
#define GAS __attribute__((address_space(1)))
typedef unsigned short u16;
typedef unsigned long long u64;
typedef short bf16x8 __attribute__((ext_vector_type(8)));
typedef short s16x4 __attribute__((ext_vector_type(4)));
typedef float f32x4 __attribute__((ext_vector_type(4)));
typedef unsigned u32x4 __attribute__((ext_vector_type(4)));
typedef unsigned u32x2 __attribute__((ext_vector_type(2)));
typedef u16 u16x2 __attribute__((ext_vector_type(2)));
typedef GAS unsigned gu32;
typedef GAS u64 gu64;

#define FORCE_RAW_IN 0
#define FORCE_RAW_OUT 0
#define LW_IN 18
#define LW_OUT 18
#define PW 4
#define STG 2

namespace {

constexpr int NR = 8;
constexpr int ND = 8192;
constexpr int KS = 1024;
constexpr int RROWS = 4096, RCOLS = 2048;
constexpr int NSTEP = 128;
constexpr int NWG = 256;
constexpr int NENC = 128 * PW;
static_assert(LW_IN <= 16 * (8 - PW) && LW_OUT <= 128, "wave roles");
constexpr unsigned long long DEADLINE = 40000000ull;

constexpr unsigned OFF_TMO = 0x80;
constexpr unsigned OFF_Q = 0x400;
constexpr unsigned OFF_ECNT = 0x1000;
constexpr unsigned OFF_FIN = 0x8000;
constexpr unsigned OFF_SEQ = 0xa000;
constexpr unsigned OFF_XF = 0x10000;
constexpr unsigned OFF_WF = 0x18000;
constexpr unsigned OFF_FOUT = 0x20000;
constexpr unsigned USLOT = 8192 + 128;
constexpr unsigned OFF_EX = 1u << 20;
constexpr unsigned OFF_EW = OFF_EX + (17u << 20);
constexpr unsigned OFF_XL = OFF_EW + (17u << 20);
constexpr unsigned OFF_WL = OFF_XL + (64u << 20);
constexpr unsigned SSLOT = 12416;
constexpr unsigned OFF_EO = OFF_WL + (32u << 20);
constexpr unsigned WS_USED = OFF_EO + (13u << 20);
static_assert(2048u * USLOT <= (17u << 20), "unit area");
static_assert(1024u * SSLOT <= (13u << 20), "strip area");
constexpr unsigned OUT_BYTES = (unsigned)ND * ND * 2u;

constexpr int SMEM_BYTES = 131072 + 256 + 4096;
constexpr int SM_AVAIL = 131072 + 16, SM_DUMMY = 131072 + 64, SM_LUT = 131072 + 256;

struct Params {
    const u16* x; const u16* w; u16* out;
    void* const* out_all; void* const* ws_all;
    int rank;
};

typedef __amdgpu_buffer_rsrc_t rsrc_t;
__device__ __forceinline__ rsrc_t make_rsrc(const void* base, unsigned bytes) {
    const u64 b = (u64)base;
    const unsigned lo = __builtin_amdgcn_readfirstlane((unsigned)b), hi = __builtin_amdgcn_readfirstlane((unsigned)(b >> 32));
    return __builtin_amdgcn_make_buffer_rsrc((void*)(((u64)hi << 32) | lo), 0, bytes, 0x00020000);
}
constexpr int SC1 = 16, SYS = 17;
template <int AUX> __device__ __forceinline__ u32x4 bld16(rsrc_t r, unsigned off) { return __builtin_amdgcn_raw_buffer_load_b128(r, off, 0, AUX); }
template <int AUX> __device__ __forceinline__ unsigned bld4(rsrc_t r, unsigned off) { return __builtin_amdgcn_raw_buffer_load_b32(r, off, 0, AUX); }
template <int AUX> __device__ __forceinline__ unsigned bld2(rsrc_t r, unsigned off) { return __builtin_amdgcn_raw_buffer_load_b16(r, off, 0, AUX); }
template <int AUX> __device__ __forceinline__ void bst16(rsrc_t r, unsigned off, u32x4 v) { __builtin_amdgcn_raw_buffer_store_b128(v, r, off, 0, AUX); }
template <int AUX> __device__ __forceinline__ void bst4(rsrc_t r, unsigned off, unsigned v) { __builtin_amdgcn_raw_buffer_store_b32(v, r, off, 0, AUX); }
template <int AUX> __device__ __forceinline__ void bst2(rsrc_t r, unsigned off, unsigned v) { __builtin_amdgcn_raw_buffer_store_b16((u16)v, r, off, 0, AUX); }

__device__ __forceinline__ u32x4 make_desc(const void* base, unsigned bytes) {
    const u64 b = (u64)base;
    return (u32x4){(unsigned)__builtin_amdgcn_readfirstlane((unsigned)b), (unsigned)__builtin_amdgcn_readfirstlane((unsigned)(b >> 32)), bytes, 0x00020000u};
}
#define ALD16(dst, desc, off, imm) asm volatile("buffer_load_dwordx4 %0, %1, %2, 0 offen offset:%3 sc0 sc1" : "+v"(dst) : "v"(off), "s"(desc), "i"(imm) : "memory")
#define ALD4(dst, desc, off, imm) asm volatile("buffer_load_dword %0, %1, %2, 0 offen offset:%3 sc0 sc1" : "+v"(dst) : "v"(off), "s"(desc), "i"(imm) : "memory")
#define ALD2(dst, desc, off, imm) asm volatile("buffer_load_ushort %0, %1, %2, 0 offen offset:%3 sc0 sc1" : "+v"(dst) : "v"(off), "s"(desc), "i"(imm) : "memory")
constexpr unsigned OOB = 0xfffff000u;

#define WAIT_V(n) asm volatile("s_waitcnt vmcnt(" #n ")" ::: "memory")
#define WAIT_L(n) asm volatile("s_waitcnt lgkmcnt(" #n ")" ::: "memory")
#define BAR __builtin_amdgcn_s_barrier()
#define SCHED __builtin_amdgcn_sched_barrier(0)
#define CFENCE asm volatile("" ::: "memory")
#define STAMP(k) do { } while (0)

__device__ __forceinline__ unsigned rfl(unsigned v) { return __builtin_amdgcn_readfirstlane(v); }
__device__ __forceinline__ unsigned pkmax(unsigned a, unsigned b) {
    u16x2 x = __builtin_bit_cast(u16x2, a), y = __builtin_bit_cast(u16x2, b);
    return __builtin_bit_cast(unsigned, __builtin_elementwise_max(x, y));
}
__device__ __forceinline__ unsigned pkmin(unsigned a, unsigned b) {
    u16x2 x = __builtin_bit_cast(u16x2, a), y = __builtin_bit_cast(u16x2, b);
    return __builtin_bit_cast(unsigned, __builtin_elementwise_min(x, y));
}
__device__ __forceinline__ unsigned wave_max(unsigned v) {
#pragma unroll
    for (int o = 32; o >= 1; o >>= 1) { const unsigned t = (unsigned)__shfl_xor((int)v, o); v = v > t ? v : t; }
    return rfl(v);
}
__device__ __forceinline__ unsigned cvt_pk_bf16(float lo, float hi) { unsigned r; asm volatile("v_cvt_pk_bf16_f32 %0, %1, %2" : "=v"(r) : "v"(lo), "v"(hi)); return r; }
__device__ __forceinline__ int lds_byte(int r, int c) { const int st = (r >> 4) * 2 + (c >> 5), rr = r & 15, cc = c & 31, ob = rr * 64 + cc * 2; return st * 1024 + (ob ^ (((ob >> 9) & 1) << 5)); }

__device__ __forceinline__ unsigned wave_scan(unsigned x) {
    x += (unsigned)__builtin_amdgcn_update_dpp(0, (int)x, 0x111, 0xf, 0xf, false);
    x += (unsigned)__builtin_amdgcn_update_dpp(0, (int)x, 0x112, 0xf, 0xf, false);
    x += (unsigned)__builtin_amdgcn_update_dpp(0, (int)x, 0x114, 0xf, 0xf, false);
    x += (unsigned)__builtin_amdgcn_update_dpp(0, (int)x, 0x118, 0xf, 0xf, false);
    x += (unsigned)__builtin_amdgcn_update_dpp(0, (int)x, 0x142, 0xa, 0xf, false);
    x += (unsigned)__builtin_amdgcn_update_dpp(0, (int)x, 0x143, 0xc, 0xf, false);
    return x;
}
struct Order {
    int ph, start;
    template <int NWU, int NXU> __device__ __forceinline__ void locate(int i, int& j, int& g, int& u) {
        for (;;) {
            int c = (ph < 16) ? NWU : 0;
#pragma unroll
            for (int k = 0; k < 4; ++k) c += ((unsigned)(ph - k * STG) < 16u) ? NXU : 0;
            if (i < start + c) break;
            start += c; ++ph;
        }
        u = i - start; g = 0; j = ph;
        if (ph < 16) { if (u < NWU) return; u -= NWU; }
#pragma unroll
        for (int k = 0; k < 4; ++k) {
            if ((unsigned)(ph - k * STG) < 16u) { if (u < NXU) { g = 1 + k; j = ph - k * STG; return; } u -= NXU; }
        }
    }
};

__device__ __forceinline__ bool give_up(gu32* tmo, unsigned seq, u64 t0) {
    bool dead = rfl(__hip_atomic_load(tmo, __ATOMIC_RELAXED, __HIP_MEMORY_SCOPE_AGENT)) == seq;
    if (!dead && __builtin_amdgcn_s_memrealtime() - t0 > DEADLINE) {
        __hip_atomic_store(tmo, seq, __ATOMIC_RELAXED, __HIP_MEMORY_SCOPE_AGENT);
        dead = true;
    }
    return dead;
}
__device__ __forceinline__ bool wait_sys32(gu32* flag, unsigned seq, gu32* tmo) {
    bool dead = false;
    if (rfl(__hip_atomic_load(flag, __ATOMIC_RELAXED, __HIP_MEMORY_SCOPE_SYSTEM)) < seq) {
        const u64 t0 = __builtin_amdgcn_s_memrealtime(); unsigned n = 0;
        while (rfl(__hip_atomic_load(flag, __ATOMIC_RELAXED, __HIP_MEMORY_SCOPE_SYSTEM)) < seq) {
            if ((++n & 63u) == 0u && give_up(tmo, seq, t0)) { dead = true; break; }
            __builtin_amdgcn_s_sleep(2);
        }
    }
    __builtin_amdgcn_fence(__ATOMIC_ACQUIRE, "wavefront");
    return dead;
}

__device__ __forceinline__ void pack_unit(const Params& p, char* ws, rsrc_t wsr, LAS unsigned char* lw, int j, int g, int u, int lane, unsigned seqv) {
    const bool isx = g != 0; const int ub = isx ? (u >> 4) * 64 + (g - 1) * 16 + (u & 15) : u;
    const int ra = p.rank >> 2, rb = p.rank & 3;
    u32x4 v[8];
    if (isx) {
        const u16* src = p.x + (size_t)(ub * 64 + (lane & 15)) * KS + j * 64 + (lane >> 4) * 8;
#pragma unroll
        for (int c = 0; c < 8; ++c) v[c] = *(const u32x4*)(src + (size_t)(c >> 1) * 16 * KS + (c & 1) * 32);
    } else {
        constexpr int P = 144;
        const u16* src = p.w + (size_t)(j * 64 + (lane >> 3)) * ND + ub * 64 + (lane & 7) * 8;
        u32x4 tmp[8];
#pragma unroll
        for (int i = 0; i < 8; ++i) tmp[i] = *(const u32x4*)(src + (size_t)i * 8 * ND);
#pragma unroll
        for (int i = 0; i < 8; ++i) *(LAS u32x4*)(lw + (i * 8 + (lane >> 3)) * P + (lane & 7) * 16) = tmp[i];
        const int q = (lane >> 2) & 3, pp = lane & 3;
#pragma unroll
        for (int c = 0; c < 8; ++c) {
            const int k0 = (c & 1) * 32 + (lane >> 4) * 8, rbk = c >> 1;
            const int col = (rbk >> 1) * 32 + 8 * pp + 4 * (rbk & 1);
            const s16x4 a = __builtin_amdgcn_ds_read_tr16_b64_v4i16((LAS s16x4*)(lw + (k0 + q) * P + col * 2));
            const s16x4 b = __builtin_amdgcn_ds_read_tr16_b64_v4i16((LAS s16x4*)(lw + (k0 + 4 + q) * P + col * 2));
            const u32x2 a2 = __builtin_bit_cast(u32x2, a), b2 = __builtin_bit_cast(u32x2, b);
            v[c] = (u32x4){a2.x, a2.y, b2.x, b2.y};
        }
    }
    const unsigned loff = (lane & 15) * 64 + (((lane >> 4) * 16) ^ (((lane >> 3) & 1) << 5));
    const int cls = isx ? (ub >> 6) : (ub >> 5);
    const bool own = isx ? (cls == ra) : (cls == rb);
    const int li = isx ? (ub & 63) : (ub & 31);
    if (own) {
        const unsigned dst = (isx ? OFF_XL + (unsigned)(j * 8 * 64 + li) * 8192u : OFF_WL + (unsigned)(j * 8 * 32 + li) * 8192u) + loff;
#pragma unroll
        for (int c = 0; c < 8; ++c) bst16<SC1>(wsr, dst + c * 1024, v[c]);
    }
    unsigned mx = 0;
#pragma unroll
    for (int c = 0; c < 8; ++c)
#pragma unroll
        for (int k = 0; k < 4; ++k) mx = pkmax(mx, v[c][k] & 0x7fff7fffu);
    mx = (mx & 0xffffu) > (mx >> 16) ? (mx & 0xffffu) : (mx >> 16);
    unsigned b7 = wave_max(mx) >> 8; b7 = b7 < 7u ? 7u : b7;
    const unsigned bq = b7 * 0x01010101u;
    unsigned lo[16], nb[8], bad = 0;
#pragma unroll
    for (int c = 0; c < 8; ++c) {
        lo[2 * c] = __builtin_amdgcn_perm(v[c].y, v[c].x, 0x06040200u); lo[2 * c + 1] = __builtin_amdgcn_perm(v[c].w, v[c].z, 0x06040200u);
        const unsigned ha = __builtin_amdgcn_perm(v[c].y, v[c].x, 0x07050301u), hb = __builtin_amdgcn_perm(v[c].w, v[c].z, 0x07050301u);
        const unsigned da = bq - (ha & 0x7f7f7f7fu), db = bq - (hb & 0x7f7f7f7fu);
        bad |= (da | db) & 0x78787878u;
        nb[c] = ((da & 0x07070707u) | ((ha >> 4) & 0x08080808u)) | (((db & 0x07070707u) << 4) | (hb & 0x80808080u));
    }
    unsigned cnt = 0;
    const unsigned slot0 = (isx ? OFF_EX : OFF_EW) + (unsigned)(j * 128 + ub) * USLOT;
    if (__builtin_amdgcn_ballot_w64(bad != 0u)) {
#pragma unroll
        for (int c = 0; c < 8; ++c)
#pragma unroll
            for (int e = 0; e < 8; ++e) {
                const unsigned val = (v[c][e >> 1] >> ((e & 1) * 16)) & 0xffffu;
                const bool ex = b7 - ((val >> 8) & 0x7fu) > 7u;
                const u64 m = __builtin_amdgcn_ballot_w64(ex);
                const unsigned sl = cnt + 1 + __builtin_amdgcn_mbcnt_hi((unsigned)(m >> 32), __builtin_amdgcn_mbcnt_lo((unsigned)m, 0u));
                if (ex && sl < 16u) bst4<SYS>(wsr, slot0 + 64 + sl * 4, ((((unsigned)c * 1024u + loff) / 2u + e) << 16) | val);
                cnt += (unsigned)__builtin_popcountll(m);
            }
    }
    const bool raw = FORCE_RAW_IN || cnt > 15u;
    if (lane == 0) bst4<SYS>(wsr, slot0 + 64, b7 | ((raw ? 0u : cnt) << 8) | ((raw ? 1u : 0u) << 16));
    const unsigned pay = slot0 + 128 + lane * 16;
    if (raw) {
#pragma unroll
        for (int c = 0; c < 8; ++c) bst16<SYS>(wsr, pay + c * 1024, v[c]);
    } else {
#pragma unroll
        for (int i = 0; i < 4; ++i) bst16<SYS>(wsr, pay + i * 1024, (u32x4){lo[4 * i], lo[4 * i + 1], lo[4 * i + 2], lo[4 * i + 3]});
#pragma unroll
        for (int k = 0; k < 2; ++k) bst16<SYS>(wsr, pay + 4096 + k * 1024, (u32x4){nb[4 * k], nb[4 * k + 1], nb[4 * k + 2], nb[4 * k + 3]});
    }
    WAIT_V(0);
    const unsigned seq = rfl(seqv) + 1u;
    if (own && lane == 0) {
        const unsigned f = (isx ? OFF_XF : OFF_WF) + (unsigned)(((li >> 2) * NSTEP + j * 8) * 4 + (li & 3)) * 4u;
        __hip_atomic_store((gu32*)(ws + f), seq, __ATOMIC_RELAXED, __HIP_MEMORY_SCOPE_AGENT);
    }
    unsigned ticket = 0;
    if (lane == 0) ticket = __hip_atomic_fetch_add((gu32*)(ws + OFF_ECNT + (unsigned)(j * 12 + (isx ? cls * 4 + g - 1 : 8 + cls)) * 128u), 1u, __ATOMIC_RELAXED, __HIP_MEMORY_SCOPE_AGENT);
    ticket = rfl(ticket);
    const unsigned per = isx ? 16u : 32u;
    if ((ticket & (per - 1u)) == per - 1u) {
        const bool want = lane < NR && lane != p.rank && (isx ? ((lane >> 2) == cls) : ((lane & 3) == cls));
        if (want) {
            char* pw = (char*)p.ws_all[lane];
            __hip_atomic_store((gu32*)(pw + OFF_FIN + (unsigned)p.rank * 512u + (unsigned)j * 32u + (unsigned)g * 4u), seq, __ATOMIC_RELAXED, __HIP_MEMORY_SCOPE_SYSTEM);
        }
    }
}

struct UnitRegs { u32x4 a[6]; unsigned hdr; };
__device__ __forceinline__ unsigned unit_slot(int rank, int j, int g, int u) {
    const int ub = g ? (rank >> 2) * 64 + (g - 1) * 16 + u : (rank & 3) * 32 + u;
    return (g ? OFF_EX : OFF_EW) + (unsigned)(j * 128 + ub) * USLOT;
}
__device__ __forceinline__ void unit_issue(UnitRegs& r, u32x4 pd, unsigned slot0, int lane) {
    const unsigned oh = slot0 + 64 + (lane & 15) * 4, o0 = slot0 + 128 + lane * 16, o1 = o0 + 4096;
    ALD4(r.hdr, pd, oh, 0);
    ALD16(r.a[0], pd, o0, 0); ALD16(r.a[1], pd, o0, 1024); ALD16(r.a[2], pd, o0, 2048); ALD16(r.a[3], pd, o0, 3072);
    ALD16(r.a[4], pd, o1, 0); ALD16(r.a[5], pd, o1, 1024);
}
#define UNIT_LANDED(r, n) asm volatile("s_waitcnt vmcnt(0)" : "+v"(r.a[0]), "+v"(r.a[1]), "+v"(r.a[2]), "+v"(r.a[3]), "+v"(r.a[4]), "+v"(r.a[5]), "+v"(r.hdr), "+v"(n) :: "memory")
__device__ __forceinline__ unsigned unit_unpack(const UnitRegs& r, rsrc_t wsr, rsrc_t pr, unsigned slot0, unsigned dst0, int lane) {
    const unsigned dst = dst0 + (lane & 15) * 64 + (((lane >> 4) * 16) ^ (((lane >> 3) & 1) << 5));
    const unsigned h0 = rfl(r.hdr);
    if (h0 >> 16) {
        const u32x4 a6 = bld16<SYS>(pr, slot0 + 128 + lane * 16 + 6 * 1024), a7 = bld16<SYS>(pr, slot0 + 128 + lane * 16 + 7 * 1024);
#pragma unroll
        for (int c = 0; c < 6; ++c) bst16<SC1>(wsr, dst + c * 1024, r.a[c]);
        bst16<SC1>(wsr, dst + 6 * 1024, a6); bst16<SC1>(wsr, dst + 7 * 1024, a7);
        return 0u;
    }
    const unsigned bq = (h0 & 0xffu) * 0x01010101u;
#pragma unroll
    for (int c = 0; c < 8; ++c) {
        const unsigned la = r.a[c >> 1][(c & 1) * 2], lb = r.a[c >> 1][(c & 1) * 2 + 1], n = r.a[4 + (c >> 2)][c & 3];
        const unsigned ha = (bq - (n & 0x07070707u)) | ((n & 0x08080808u) << 4);
        const unsigned hb = (bq - ((n >> 4) & 0x07070707u)) | (n & 0x80808080u);
        u32x4 o;
        o.x = __builtin_amdgcn_perm(ha, la, 0x05010400u); o.y = __builtin_amdgcn_perm(ha, la, 0x07030602u);
        o.z = __builtin_amdgcn_perm(hb, lb, 0x05010400u); o.w = __builtin_amdgcn_perm(hb, lb, 0x07030602u);
        bst16<SC1>(wsr, dst + c * 1024, o);
    }
    return (h0 >> 8) & 0xffu;
}

__device__ __forceinline__ unsigned strip_out_off(int r, int s, int lane) {
    const int tile = s >> 3, tm = tile >> 3, tn = tile & 7;
    const int row = (r >> 2) * RROWS + tm * 256 + (s & 7) * 32 + (lane >> 5);
    const int col = (r & 3) * RCOLS + tn * 256 + (lane & 31) * 8;
    return ((unsigned)row * ND + col) * 2u;
}

struct StripRegs { u32x4 mk; u32x4 lo[8]; unsigned p2[8], p1[8], exc; };
__device__ __forceinline__ void strip_issue(StripRegs& r, u32x4 pd, unsigned s, unsigned info, int lane) {
    if (info >> 25) return;
    const unsigned T = info & 0x3fffu, slot = OFF_EO + s * SSLOT;
    const unsigned om = slot + lane * 16;
    ALD16(r.mk, pd, om, 0);
    if ((info >> 21) & 15u) { const unsigned oe = slot + 12288 + (lane & 15) * 4; ALD4(r.exc, pd, oe, 0); }
#define STRIP_ROUND(k) if ((unsigned)(k) * 1024u < T) { const bool act = (unsigned)((k) * 1024 + lane * 16) < T; \
        const unsigned ol = act ? om + 4096 + ((k) >> 2) * 4096 : OOB, o2 = act ? slot + lane * 4 : OOB, o1 = act ? slot + lane * 2 : OOB; \
        ALD16(r.lo[k], pd, ol, ((k) & 3) * 1024); ALD4(r.p2[k], pd, o2, 2048 + (k) * 256); ALD2(r.p1[k], pd, o1, 1024 + (k) * 128); }
    STRIP_ROUND(0) STRIP_ROUND(1) STRIP_ROUND(2) STRIP_ROUND(3) STRIP_ROUND(4) STRIP_ROUND(5) STRIP_ROUND(6) STRIP_ROUND(7)
}
#define STRIP_LANDED(r, fn, n, cnt) do { \
    asm volatile("s_waitcnt vmcnt(" #cnt ")" : "+v"(r.lo[0]), "+v"(r.lo[1]), "+v"(r.lo[2]), "+v"(r.lo[3]), "+v"(r.lo[4]), "+v"(r.lo[5]), "+v"(r.lo[6]), "+v"(r.lo[7]), \
        "+v"(r.mk), "+v"(r.exc), "+v"(fn), "+v"(n) :: "memory"); \
    asm volatile("" : "+v"(r.p2[0]), "+v"(r.p2[1]), "+v"(r.p2[2]), "+v"(r.p2[3]), "+v"(r.p2[4]), "+v"(r.p2[5]), "+v"(r.p2[6]), "+v"(r.p2[7]), \
        "+v"(r.p1[0]), "+v"(r.p1[1]), "+v"(r.p1[2]), "+v"(r.p1[3]), "+v"(r.p1[4]), "+v"(r.p1[5]), "+v"(r.p1[6]), "+v"(r.p1[7]) :: "memory"); } while (0)
__device__ __forceinline__ void strip_stage1(const StripRegs& r, LAS unsigned char* smem, unsigned lbase, unsigned info, int lane) {
    if (info >> 25) return;
    const unsigned T = info & 0x3fffu, cnt = (info >> 21) & 15u;
    const unsigned bq = ((info >> 14) & 0x7fu) * 0x01010101u;
#pragma unroll
    for (int k = 0; k < 8; ++k) {
        if ((unsigned)k * 1024u < T) {
            if ((unsigned)(k * 1024 + lane * 16) < T) {
                const unsigned z = __builtin_amdgcn_perm(0u, r.p1[k], 0x01010000u);
                const unsigned x1 = (z & 0x000f000fu) | ((z >> 4) & 0x0f000f00u);
                u32x4 h;
#pragma unroll
                for (int i = 0; i < 4; ++i) h[i] = bq - (((r.p2[k] >> (2 * i)) & 0x03030303u) | (((x1 >> i) & 0x01010101u) << 2));
                *(LAS u32x4*)(smem + lbase + k * 2048 + lane * 32) = (u32x4){r.lo[k].x, h.x, r.lo[k].y, h.y};
                *(LAS u32x4*)(smem + lbase + k * 2048 + lane * 32 + 16) = (u32x4){r.lo[k].z, h.z, r.lo[k].w, h.w};
            }
        }
    }
    if (cnt) { const unsigned ix = (r.exc >> 16) & 8191u;
        if ((unsigned)lane < cnt) *(LAS unsigned char*)(smem + lbase + (ix & ~3u) * 2u + 4u + (ix & 3u)) = (unsigned char)(r.exc >> 8); }
}
__device__ __forceinline__ void strip_stage2(u32x4 mk, rsrc_t outr, rsrc_t por, LAS unsigned char* smem, unsigned lbase, int peer, unsigned s, unsigned info, int lane) {
    const unsigned oo = strip_out_off(peer, (int)s, lane);
    if (info >> 26) {
#pragma unroll
        for (int q = 0; q < 16; ++q) bst16<0>(outr, oo + q * 2 * ND * 2, (u32x4){0x7fc07fc0u, 0x7fc07fc0u, 0x7fc07fc0u, 0x7fc07fc0u});
        return;
    }
    if (info >> 25) {
#pragma unroll
        for (int h = 0; h < 4; ++h) {
            u32x4 a[4];
#pragma unroll
            for (int q = 0; q < 4; ++q) a[q] = bld16<SYS>(por, oo + (h * 4 + q) * 2 * ND * 2);
#pragma unroll
            for (int q = 0; q < 4; ++q) bst16<0>(outr, oo + (h * 4 + q) * 2 * ND * 2, a[q]);
        }
        return;
    }
    unsigned c2[8], x[8];
#pragma unroll
    for (int i = 0; i < 8; ++i) {
        const unsigned m16 = mk[i >> 1] >> ((i & 1) * 16);
        c2[i] = (unsigned)__builtin_popcount(m16 & 0xffu) | ((unsigned)__builtin_popcount((m16 >> 8) & 0xffu) << 16);
        x[i] = c2[i];
    }
#define SCAN_STEP(ctrl, rmask) _Pragma("unroll") for (int i = 0; i < 8; ++i) x[i] += (unsigned)__builtin_amdgcn_update_dpp(0, (int)x[i], ctrl, rmask, 0xf, false);
    SCAN_STEP(0x111, 0xf) SCAN_STEP(0x112, 0xf) SCAN_STEP(0x114, 0xf) SCAN_STEP(0x118, 0xf) SCAN_STEP(0x142, 0xa) SCAN_STEP(0x143, 0xc)
    unsigned pos = 0;
#pragma unroll
    for (int bt = 0; bt < 4; ++bt) {
        u32x4 sel[4], d[8]; unsigned off[8];
#pragma unroll
        for (int k = 0; k < 4; ++k) {
            const int i = bt * 2 + (k >> 1), hq = k & 1;
            const unsigned m8 = (mk[bt] >> (k * 8)) & 0xffu;
            sel[k] = *(const LAS u32x4*)(smem + SM_LUT + m8 * 16u);
            const unsigned tot = __builtin_amdgcn_readlane(x[i], 63);
            off[2 * k] = pos + (((x[i] - c2[i]) >> (hq * 16)) & 0xffffu);
            off[2 * k + 1] = off[2 * k] + (unsigned)__builtin_popcount(m8 & 15u);
            pos += (tot >> (hq * 16)) & 0xffffu;
        }
#pragma unroll
        for (int g = 0; g < 8; ++g) {
            const LAS u32x2* pl = (const LAS u32x2*)(smem + lbase + (off[g] & ~3u) * 2u);
            const u32x2 d0 = pl[0], d1 = pl[1];
            d[g] = (u32x4){d0.x, d0.y, d1.x, d1.y};
        }
#pragma unroll
        for (int k = 0; k < 4; ++k) {
            u32x4 o;
#pragma unroll
            for (int g = 0; g < 2; ++g) {
                const u32x4 dd = d[2 * k + g];
                const unsigned l4 = __builtin_amdgcn_alignbyte(dd.z, dd.x, off[2 * k + g]), h4 = __builtin_amdgcn_alignbyte(dd.w, dd.y, off[2 * k + g]);
                o[2 * g] = __builtin_amdgcn_perm(h4, l4, sel[k][2 * g]); o[2 * g + 1] = __builtin_amdgcn_perm(h4, l4, sel[k][2 * g + 1]);
            }
            bst16<0>(outr, oo + (bt * 4 + k) * 2 * ND * 2, o);
        }
    }
}

__device__ __forceinline__ void pack_strip(const Params& p, rsrc_t wsr, rsrc_t outr, LAS unsigned char* smem, int tile, int wid, int lane, unsigned seq) {
    const int s = tile * 8 + wid;
    const unsigned oo = strip_out_off(p.rank, s, lane);
    const unsigned lbase = (unsigned)wid * 16384u;
    unsigned pos = lbase, mx = 0, sg = 0;
    const unsigned dummy = SM_DUMMY + lane * 2;
    u32x4 mk = {0u, 0u, 0u, 0u};
#pragma unroll
    for (int q2 = 0; q2 < 8; ++q2) {
        u32x4 v[2]; unsigned m8[2];
#pragma unroll
        for (int hq = 0; hq < 2; ++hq) {
            const int q = q2 * 2 + hq, row = wid * 32 + 2 * q + (lane >> 5);
            v[hq] = *(const LAS u32x4*)(smem + row * 512 + (((lane & 31) ^ (row & 7)) * 16));
            bst16<SYS>(outr, oo + q * 2 * ND * 2, v[hq]);
            unsigned m = 0;
#pragma unroll
            for (int k = 0; k < 4; ++k) {
                mx = pkmax(mx, v[hq][k]); sg |= v[hq][k];
                const unsigned t = pkmin(v[hq][k], 0x00010001u);
                m |= ((t | (t >> 15)) & 3u) << (2 * k);
            }
            m8[hq] = m;
        }
        mk[q2 >> 1] |= (m8[0] | (m8[1] << 8)) << ((q2 & 1) * 16);
        const unsigned c2 = (unsigned)__builtin_popcount(m8[0]) | ((unsigned)__builtin_popcount(m8[1]) << 16);
        const unsigned inc = wave_scan(c2), exl = inc - c2, tot = __builtin_amdgcn_readlane(inc, 63);
#pragma unroll
        for (int hq = 0; hq < 2; ++hq) {
            unsigned a = pos + ((exl >> (hq * 16)) & 0xffffu) * 2u;
#pragma unroll
            for (int e = 0; e < 8; ++e) {
                const unsigned bit = (m8[hq] >> e) & 1u;
                *(LAS u16*)(smem + (bit ? a : dummy)) = (u16)(v[hq][e >> 1] >> ((e & 1) * 16));
                a += bit * 2u;
            }
            pos += ((tot >> (hq * 16)) & 0xffffu) * 2u;
        }
    }
    const unsigned T = (pos - lbase) >> 1;
    mx = (mx & 0xffffu) > (mx >> 16) ? (mx & 0xffffu) : (mx >> 16);
    unsigned b7 = (wave_max(mx) >> 8) & 0x7fu; b7 = b7 < 7u ? 7u : b7;
    bool raw = FORCE_RAW_OUT || __builtin_amdgcn_ballot_w64((sg & 0x80008000u) != 0u) != 0ull;
    const unsigned slot = OFF_EO + (unsigned)s * SSLOT;
    unsigned cnt = 0;
    if (!raw) {
        if (lane < 16 && T + lane < 8192u) *(LAS u16*)(smem + lbase + (T + lane) * 2u) = (u16)(b7 << 8);
        const unsigned bq = b7 * 0x01010101u;
        bst16<SYS>(wsr, slot + lane * 16, mk);
        for (unsigned r = 0; r * 1024u < T; ++r) {
            const bool act = r * 1024u + lane * 16u < T;
            u32x4 a, b;
            a = *(const LAS u32x4*)(smem + lbase + r * 2048u + lane * 32u); b = *(const LAS u32x4*)(smem + lbase + r * 2048u + lane * 32u + 16u);
            const unsigned w8[8] = {a.x, a.y, a.z, a.w, b.x, b.y, b.z, b.w};
            u32x4 lo; unsigned p2 = 0, x1 = 0, bad = 0;
#pragma unroll
            for (int k = 0; k < 4; ++k) {
                lo[k] = __builtin_amdgcn_perm(w8[2 * k + 1], w8[2 * k], 0x06040200u);
                const unsigned d = bq - __builtin_amdgcn_perm(w8[2 * k + 1], w8[2 * k], 0x07050301u);
                bad |= d & 0xf8f8f8f8u;
                p2 |= (d & 0x03030303u) << (2 * k); x1 |= ((d >> 2) & 0x01010101u) << k;
            }
            const unsigned y = x1 | (x1 >> 4);
            const unsigned p1 = __builtin_amdgcn_perm(0u, y, 0x0c0c0200u);
            if (act) {
                bst16<SYS>(wsr, slot + 4096 + r * 1024u + lane * 16u, lo);
                bst4<SYS>(wsr, slot + 2048 + r * 256u + lane * 4u, p2);
                bst2<SYS>(wsr, slot + 1024 + r * 128u + lane * 2u, p1);
            }
            if (__builtin_amdgcn_ballot_w64(act && bad != 0u)) {
#pragma unroll
                for (int e = 0; e < 16; ++e) {
                    const unsigned val = (w8[e >> 1] >> ((e & 1) * 16)) & 0xffffu;
                    const bool ex = act && (b7 - (val >> 8) > 7u);
                    const u64 m = __builtin_amdgcn_ballot_w64(ex);
                    const unsigned sl = cnt + __builtin_amdgcn_mbcnt_hi((unsigned)(m >> 32), __builtin_amdgcn_mbcnt_lo((unsigned)m, 0u));
                    if (ex && sl < 15u) bst4<SYS>(wsr, slot + 12288 + sl * 4, ((r * 1024u + lane * 16u + e) << 16) | val);
                    cnt += (unsigned)__builtin_popcountll(m);
                }
            }
        }
        if (cnt > 15u) raw = true;
    }
    WAIT_V(0);
    if (lane < NR && lane != p.rank) {
        const unsigned info = raw ? (1u << 25) : (T | (b7 << 14) | (cnt << 21));
        char* pw = (char*)p.ws_all[lane];
        __hip_atomic_store((gu64*)(pw + OFF_FOUT + ((unsigned)p.rank * 1024u + (unsigned)s) * 8u), ((u64)seq << 32) | info, __ATOMIC_RELAXED, __HIP_MEMORY_SCOPE_SYSTEM);
    }
}

}

__global__ void __launch_bounds__(512, 2) gemm_ar_fused(Params p) {
    __shared__ __attribute__((aligned(16))) unsigned char smem_[SMEM_BYTES];
    LAS unsigned char* smem = (LAS unsigned char*)smem_;
    const int tid = threadIdx.x, wid = rfl(tid >> 6), lane = tid & 63;
    char* ws = (char*)p.ws_all[p.rank];
    const rsrc_t wsr = make_rsrc(ws, WS_USED);
    const rsrc_t outr = make_rsrc(p.out, OUT_BYTES);
    gu32* tmo = (gu32*)(ws + OFF_TMO);
    gu32* seqp = (gu32*)(ws + OFF_SEQ + (unsigned)(blockIdx.x * 8 + wid) * 4u);
    const unsigned seqv = __hip_atomic_load(seqp, __ATOMIC_RELAXED, __HIP_MEMORY_SCOPE_AGENT);
    STAMP(0);
    const int xcd = blockIdx.x & 7, wi = blockIdx.x >> 3;
    const bool is_gemm = wi < 16;
    if (!is_gemm && wid >= 8 - PW) {
        {
            Order od{0, 0};
            for (int t = (wid - (8 - PW)) * 128 + (wi - 16) * 8 + xcd; t < 4096; t += NENC) {
                int j, g, u; od.locate<128, 32>(t, j, g, u);
                pack_unit(p, ws, wsr, smem + wid * 16384, j, g, u, lane, seqv);
            }
            STAMP(7);
        }
    }
    const unsigned seq = rfl(seqv) + 1u;
    if (lane == 0) __hip_atomic_store(seqp, seq, __ATOMIC_RELAXED, __HIP_MEMORY_SCOPE_AGENT);
    if (!is_gemm) {
        const int lq = (wi - 16) & 7;
        if (lq == 7 || wid * 16 >= (LW_IN > LW_OUT ? LW_IN : LW_OUT)) return;
        const int peer = (p.rank + 1 + lq) & 7;
        const rsrc_t pr = make_rsrc(p.ws_all[peer], WS_USED);
        const rsrc_t por = make_rsrc(p.out_all[peer], OUT_BYTES);
#pragma unroll
        for (int i = 0; i < 4; ++i) {
            const unsigned m8 = (unsigned)(i * 64 + lane); u32x4 sel;
#pragma unroll
            for (int g = 0; g < 2; ++g) {
                unsigned rk = 0;
#pragma unroll
                for (int h = 0; h < 2; ++h) {
                    unsigned w = 0;
#pragma unroll
                    for (int e = 0; e < 2; ++e) { const unsigned bit = (m8 >> (g * 4 + h * 2 + e)) & 1u; w |= (bit ? (rk | ((rk + 4u) << 8)) : 0x0c0cu) << (16 * e); rk += bit; }
                    sel[g * 2 + h] = w;
                }
            }
            *(LAS u32x4*)(smem + SM_LUT + m8 * 16u) = sel;
        }
        bool dead = false;
        const int leng = wid * 16 + ((((wi - 16) >> 3) * 8 + xcd + 2 * lq) & 15);
        if (leng >= LW_OUT && leng >= LW_IN) return;
        const u32x4 pd = make_desc(p.ws_all[peer], WS_USED);
        const unsigned one = 1u;
#define QNEXT(dst) do { if (lane == 0) asm volatile("global_atomic_add %0, %1, %2, off sc0" : "+v"(dst) : "v"(qp), "v"(one) : "memory"); } while (0)
        if (leng < LW_IN) {
            gu32* qp = (gu32*)(ws + OFF_Q + (unsigned)lq * 128u);
            const unsigned qbase = (seq - 1u) * (1536u + LW_IN);
            Order od{0, 0};
            unsigned okm = 0;
            unsigned nq = 0;
            UnitRegs ur = {}; unsigned slot0 = 0, dst0 = 0, fo = 0;
            QNEXT(nq);
            UNIT_LANDED(ur, nq);
#define UNIT_ISSUE(i) do { int j, g, u; od.locate<32, 16>((int)(i), j, g, u); \
                if (((okm >> (5 * g)) & 31u) <= (unsigned)j) { \
                    if (!dead) dead = wait_sys32((gu32*)(ws + OFF_FIN + (unsigned)peer * 512u + (unsigned)j * 32u + (unsigned)g * 4u), seq, tmo); \
                    okm = (okm & ~(31u << (5 * g))) | ((unsigned)(j + 1) << (5 * g)); } \
                const int step = j * 8 + 1 + lq, li = g ? (g - 1) * 16 + u : u; \
                slot0 = unit_slot(p.rank, j, g, u); \
                dst0 = g ? OFF_XL + (unsigned)(step * 64 + li) * 8192u : OFF_WL + (unsigned)(step * 32 + li) * 8192u; \
                fo = (g ? OFF_XF : OFF_WF) + (unsigned)(((li >> 2) * NSTEP + step) * 4 + (li & 3)) * 4u; \
                unit_issue(ur, pd, slot0, lane); QNEXT(nq); } while (0)
            const unsigned i0 = rfl(nq) - qbase;
            if (i0 < 1536u) {
                UNIT_ISSUE(i0);
                STAMP(1);
                for (;;) {
                    UNIT_LANDED(ur, nq);
                    const unsigned pdst = dst0, pfo = fo; unsigned phdr = ur.hdr;
                    asm volatile("" : "+v"(phdr));
                    const unsigned cnt = unit_unpack(ur, wsr, pr, slot0, dst0, lane);
                    const unsigned i = rfl(nq) - qbase;
                    const bool have = i < 1536u;
                    if (have) { UNIT_ISSUE(i); WAIT_V(8); } else WAIT_V(0);
                    if (cnt) {
                        if (lane >= 1 && (unsigned)lane <= cnt && lane < 16) bst2<SC1>(wsr, pdst + ((phdr >> 16) & 4095u) * 2u, phdr & 0xffffu);
                        WAIT_V(0);
                    }
                    if (lane == 0) __hip_atomic_store((gu32*)(ws + pfo), seq, __ATOMIC_RELAXED, __HIP_MEMORY_SCOPE_AGENT);
                    if (!have) break;
                }
            }
            STAMP(3);
        }
        if (leng < LW_OUT) {
            gu32* qp = (gu32*)(ws + OFF_Q + (unsigned)(7 + lq) * 128u);
            const unsigned qbase = (seq - 1u) * (1024u + 2u * LW_OUT);
            gu64* fbase = (gu64*)(ws + OFF_FOUT + (unsigned)peer * 8192u);
            const unsigned lbase = (unsigned)wid * 16384u;
            unsigned n1 = 0, nq = 0; u64 fn = 0;
            StripRegs sr = {}; unsigned s = 0, info = 0;
            QNEXT(n1); QNEXT(nq);
            asm volatile("s_waitcnt vmcnt(0)" : "+v"(n1), "+v"(nq) :: "memory");
            unsigned sn = rfl(n1) - qbase;
#define STRIP_ISSUE() do { s = sn; \
                unsigned fh = rfl((unsigned)(fn >> 32)); info = rfl((unsigned)fn); \
                if (fh < seq && !dead) { \
                    const u64 t0 = __builtin_amdgcn_s_memrealtime(); unsigned n = 0; \
                    for (;;) { \
                        const u64 f2 = __hip_atomic_load(fbase + s, __ATOMIC_RELAXED, __HIP_MEMORY_SCOPE_SYSTEM); \
                        fh = rfl((unsigned)(f2 >> 32)); info = rfl((unsigned)f2); \
                        if (fh >= seq) break; \
                        if ((++n & 63u) == 0u && give_up(tmo, seq, t0)) { dead = true; break; } \
                        __builtin_amdgcn_s_sleep(4); } } \
                __builtin_amdgcn_fence(__ATOMIC_ACQUIRE, "wavefront"); \
                if (dead) info = 3u << 25; \
                sn = rfl(nq) - qbase; \
                strip_issue(sr, pd, s, info, lane); \
                { gu64* fa = fbase + (sn & 1023u); asm volatile("global_load_dwordx2 %0, %1, off sc0 sc1" : "+v"(fn) : "v"(fa) : "memory"); } \
                QNEXT(nq); } while (0)
            if (sn < 1024u) {
                STRIP_ISSUE();
                STAMP(4);
                WAIT_V(0);
                for (;;) {
                    STRIP_LANDED(sr, fn, nq, 16);
                    const unsigned ps = s, pinfo = info;
                    strip_stage1(sr, smem, lbase, info, lane);
                    u32x4 pmk = sr.mk;
                    asm volatile("" : "+v"(pmk));
                    const bool have = sn < 1024u;
                    if (have) STRIP_ISSUE();
                    strip_stage2(pmk, outr, por, smem, lbase, peer, ps, pinfo, lane);
                    if (!have) break;
                }
            }
            STAMP(6);
        }
        return;
    }

    const int tm = (xcd >> 1) * 4 + (wi >> 2), tn = (xcd & 1) * 4 + (wi & 3);
    const int wr = wid >> 2, wc = wid & 3, fr = lane & 15, fq = lane >> 4;
    constexpr int HTB = 16384;
    const char* cA = ws + OFF_XL + (size_t)tm * 32768 + tid * 16;
    const char* cB = ws + OFF_WL + (size_t)tn * 32768 + tid * 16;
    constexpr size_t ASTEP = 64 * 8192, BSTEP = 32 * 8192;
    const unsigned ldsw = (unsigned)wid * 1024u;
    const int aoff = lds_byte(wr * 64 + fr, fq * 8), boff = lds_byte(wc * 32 + fr, fq * 8);
    const unsigned pollo = (lane < 32 ? OFF_XF + (unsigned)(tm * NSTEP + lane) * 16u : OFF_WF + (unsigned)(tn * NSTEP + lane - 32) * 16u);
    unsigned avail = 0;
#define SA(b, h) (((b) * 2 + (h)) * HTB)
#define SB(b, h) ((4 + (b) * 2 + (h)) * HTB)
#define STAGE(bufoff, gptr) do { _Pragma("unroll") for (int _i = 0; _i < 2; ++_i) \
        __builtin_amdgcn_global_load_lds((const unsigned*)((gptr) + _i * 8192), (LAS unsigned*)(smem + (bufoff) + ldsw + _i * 8192), 16, 0, SC1); } while (0)
#define LDA(dst, b, h) do { _Pragma("unroll") for (int m = 0; m < 4; ++m) _Pragma("unroll") for (int k = 0; k < 2; ++k) dst[m][k] = *(const LAS bf16x8*)(smem + SA(b, h) + aoff + m * 2048 + k * 1024); } while (0)
#define LDB(dst, b, h) do { _Pragma("unroll") for (int n = 0; n < 2; ++n) _Pragma("unroll") for (int k = 0; k < 2; ++k) dst[n][k] = *(const LAS bf16x8*)(smem + SB(b, h) + boff + n * 2048 + k * 1024); } while (0)
#define MMA(ai, bj, At, Bt) do { __builtin_amdgcn_s_setprio(1); _Pragma("unroll") for (int m = 0; m < 4; ++m) _Pragma("unroll") for (int n = 0; n < 2; ++n) _Pragma("unroll") for (int k = 0; k < 2; ++k) \
        acc[ai][bj][m][n] = __builtin_amdgcn_mfma_f32_16x16x32_bf16(Bt[n][k], At[m][k], acc[ai][bj][m][n], 0, 0, 0); __builtin_amdgcn_s_setprio(0); } while (0)
#define NEED(s) do { if ((avail & 0xffffu) <= (unsigned)(s)) { \
        if (wid == 0) { const unsigned s0 = avail & 0xffffu; unsigned av; const u64 t0 = __builtin_amdgcn_s_memrealtime(); unsigned n = 0; \
            for (;;) { const u32x4 f = bld16<SC1>(wsr, pollo + s0 * 16u); \
                const bool ok = (s0 + (lane & 31) >= (unsigned)NSTEP) || (f.x >= seq && f.y >= seq && f.z >= seq && f.w >= seq); \
                const u64 mm = __builtin_amdgcn_ballot_w64(ok); const unsigned both = (unsigned)mm & (unsigned)(mm >> 32); \
                av = s0 + (both == 0xffffffffu ? 32u : (unsigned)__builtin_ctz(~both)); \
                if (av > (unsigned)(s)) break; \
                if ((++n & 63u) == 0u && give_up(tmo, seq, t0)) { av = 0x10000u | NSTEP; break; } \
                __builtin_amdgcn_s_sleep(1); } \
            asm volatile("ds_write_b32 %0, %1\n\ts_waitcnt lgkmcnt(0)" :: "v"(smbase + SM_AVAIL), "v"(av) : "memory"); } \
        CFENCE; BAR; CFENCE; \
        unsigned av2; asm volatile("ds_read_b32 %0, %1\n\ts_waitcnt lgkmcnt(0)" : "=v"(av2) : "v"(smbase + SM_AVAIL) : "memory"); \
        avail = rfl(av2); } } while (0)
    const unsigned smbase = (unsigned)(uintptr_t)smem;

    f32x4 acc[2][2][4][2];
#pragma unroll
    for (int a = 0; a < 2; ++a)
#pragma unroll
        for (int b = 0; b < 2; ++b)
#pragma unroll
            for (int m = 0; m < 4; ++m)
#pragma unroll
                for (int n = 0; n < 2; ++n) acc[a][b][m][n] = (f32x4){0.f, 0.f, 0.f, 0.f};
    bf16x8 At[4][2], B0[2][2], B1[2][2];
    NEED(1);
    STAMP(2);
    STAGE(SB(0, 0), cB); STAGE(SA(0, 0), cA); STAGE(SB(0, 1), cB + HTB); STAGE(SA(0, 1), cA + HTB);
    if (wr == 1) BAR;
    WAIT_V(4); BAR;
    STAGE(SB(1, 0), cB + BSTEP); STAGE(SA(1, 0), cA + ASTEP); STAGE(SB(1, 1), cB + BSTEP + HTB);
    WAIT_V(6); BAR;
    for (int t = 0; t < NSTEP; t += 2) {
        const bool last = (t == NSTEP - 2);
        const char* a1 = cA + (size_t)(t + 1) * ASTEP;
        const char* a2 = last ? cA : cA + (size_t)(t + 2) * ASTEP; const char* b2 = last ? cB : cB + (size_t)(t + 2) * BSTEP;
        const char* a3 = a2 + ASTEP; const char* b3 = b2 + BSTEP;
        if (!last) NEED(t + 2);
        LDB(B0, 0, 0); SCHED; LDA(At, 0, 0); STAGE(SA(1, 1), a1 + HTB);
        WAIT_L(8); BAR; WAIT_L(0); MMA(0, 0, At, B0); BAR; SCHED;
        LDB(B1, 0, 1); STAGE(SB(0, 0), b2);
        BAR; WAIT_L(0); MMA(0, 1, At, B1); BAR;
        LDA(At, 0, 1); STAGE(SA(0, 0), a2);
        BAR; WAIT_L(0); MMA(1, 0, At, B0); BAR; SCHED;
        STAGE(SB(0, 1), b2 + HTB);
        WAIT_V(6); BAR; MMA(1, 1, At, B1); BAR;
        if (!last) NEED(t + 3);
        LDB(B0, 1, 0); SCHED; LDA(At, 1, 0); STAGE(SA(0, 1), a2 + HTB);
        WAIT_L(8); BAR; WAIT_L(0); MMA(0, 0, At, B0); BAR; SCHED;
        LDB(B1, 1, 1); STAGE(SB(1, 0), b3);
        BAR; WAIT_L(0); MMA(0, 1, At, B1); BAR;
        LDA(At, 1, 1); STAGE(SA(1, 0), a3);
        BAR; WAIT_L(0); MMA(1, 0, At, B0); BAR; SCHED;
        STAGE(SB(1, 1), b3 + HTB);
        WAIT_V(6); BAR; MMA(1, 1, At, B1); BAR;
    }
    WAIT_V(0);
    if (wr == 0) BAR;
    BAR;
    STAMP(3);
    {
        int tz = threadIdx.x; asm volatile("" : "+v"(tz));
        const int lane2 = tz & 63, fr2 = lane2 & 15, fq2 = lane2 >> 4;
        const bool dead = (avail >> 16) != 0u || rfl(__hip_atomic_load(tmo, __ATOMIC_RELAXED, __HIP_MEMORY_SCOPE_AGENT)) == seq;
#pragma unroll
        for (int ai = 0; ai < 2; ++ai)
#pragma unroll
            for (int m = 0; m < 4; ++m) {
                const int row = ai * 128 + wr * 64 + m * 16 + fr2;
#pragma unroll
                for (int bj = 0; bj < 2; ++bj) {
                    f32x4 v0 = acc[ai][bj][m][0], v1 = acc[ai][bj][m][1];
#pragma unroll
                    for (int k = 0; k < 4; ++k) { v0[k] = v0[k] <= 0.f ? 0.f : v0[k]; v1[k] = v1[k] <= 0.f ? 0.f : v1[k]; }
                    u32x4 w; w.x = cvt_pk_bf16(v0[0], v0[1]); w.y = cvt_pk_bf16(v0[2], v0[3]); w.z = cvt_pk_bf16(v1[0], v1[1]); w.w = cvt_pk_bf16(v1[2], v1[3]);
                    if (dead) w = (u32x4){0x7fc07fc0u, 0x7fc07fc0u, 0x7fc07fc0u, 0x7fc07fc0u};
                    const int chunk = bj * 16 + wc * 4 + fq2;
                    *(LAS u32x4*)(smem + row * 512 + ((chunk ^ (row & 7)) * 16)) = w;
                }
            }
        __syncthreads();
        STAMP(4);
        pack_strip(p, wsr, outr, smem, tm * 8 + tn, wid, lane2, seq);
        STAMP(5);
    }
}

extern "C" void kernel_launch(int rank, int nranks, ncclComm_t comm,
                              hipStream_t stream,
                              void* const* d_in, const int* in_sizes, int n_in,
                              void* d_out, int out_size,
                              const int* in_dtypes, int out_dtype,
                              void* const* d_out_all,
                              void* const* d_ws_all, size_t ws_bytes) {
    (void)comm; (void)in_dtypes; (void)out_dtype;
    if (nranks != NR || n_in != 2 || (unsigned)rank >= (unsigned)NR || ws_bytes < WS_USED || in_sizes[0] != ND * KS || in_sizes[1] != KS * ND || out_size != ND * ND) {
        fprintf(stderr, "kernel_launch: unexpected problem shape\n");
        return;
    }
    Params p;
    p.x = (const u16*)d_in[0]; p.w = (const u16*)d_in[1]; p.out = (u16*)d_out;
    p.out_all = d_out_all; p.ws_all = d_ws_all; p.rank = rank;
    hipLaunchKernelGGL(gemm_ar_fused, dim3(NWG), dim3(512), 0, stream, p);
    HIP_CHECK(hipGetLastError());
}
```

```cpp
#include <hip/hip_runtime.h>
#include <rccl/rccl.h>
#include <cstdio>
#include <cstdint>

#define HIP_CHECK(expr) do { hipError_t _e = (expr); if (_e != hipSuccess) { \
    fprintf(stderr, "HIP error %s at %s:%d\n", hipGetErrorString(_e), __FILE__, __LINE__); \
    return; } } while (0)

#define LAS __attribute__((address_space(3)))
#define GAS __attribute__((address_space(1)))
typedef unsigned short u16;
typedef unsigned long long u64;
typedef short bf16x8 __attribute__((ext_vector_type(8)));
typedef short s16x4 __attribute__((ext_vector_type(4)));
typedef float f32x4 __attribute__((ext_vector_type(4)));
typedef unsigned u32x4 __attribute__((ext_vector_type(4)));
typedef unsigned u32x2 __attribute__((ext_vector_type(2)));
typedef u16 u16x2 __attribute__((ext_vector_type(2)));
typedef GAS unsigned gu32;
typedef GAS u64 gu64;

#define FORCE_RAW_IN 0
#define FORCE_RAW_OUT 0
#define NT 12
#define NWV 4
#define PW 4
#define LAG1 0
#define LAG2 4
#define LAG3 7

namespace {

constexpr int NR = 8;
constexpr int ND = 8192;
constexpr int KS = 1024;
constexpr int RROWS = 4096, RCOLS = 2048;
constexpr int NSTEP = 128;
constexpr int NWG = 256;
constexpr int NENC = 96 * PW;
static_assert(NT <= 13 && NWV + PW <= 8, "wave roles");
__device__ __forceinline__ constexpr int lag(int k) { return k == 0 ? 0 : k == 1 ? LAG1 : k == 2 ? LAG2 : LAG3; }
constexpr unsigned long long DEADLINE = 40000000ull;

constexpr unsigned OFF_TMO = 0x80;
constexpr unsigned OFF_Q = 0x400;
constexpr unsigned OFF_ECNT = 0x1000;
constexpr unsigned OFF_FIN = 0x8000;
constexpr unsigned OFF_SEQ = 0xa000;
constexpr unsigned OFF_XF = 0x10000;
constexpr unsigned OFF_WF = 0x18000;
constexpr unsigned OFF_FOUT = 0x20000;
constexpr unsigned USLOT = 8192 + 128;
constexpr unsigned OFF_EX = 1u << 20;
constexpr unsigned OFF_EW = OFF_EX + (17u << 20);
constexpr unsigned OFF_XL = OFF_EW + (17u << 20);
constexpr unsigned OFF_WL = OFF_XL + (64u << 20);
constexpr unsigned SSLOT = 11520;
constexpr unsigned OFF_EO = OFF_WL + (32u << 20);
constexpr unsigned WS_USED = OFF_EO + (13u << 20);
static_assert(2048u * USLOT <= (17u << 20), "unit area");
static_assert(1024u * SSLOT <= (13u << 20), "strip area");
constexpr unsigned OUT_BYTES = (unsigned)ND * ND * 2u;

constexpr int SM_AVAIL = 131072 + 16, SM_TOKEN = 131072 + 32, SM_DUMMY = 131072 + 64, SM_LUT = 131072 + 256, SM_LUTE = SM_LUT + 4096, SM_ESC = SM_LUTE + 128;
constexpr int SMEM_BYTES = SM_ESC + 8 * 320;

struct Params {
    const u16* x; const u16* w; u16* out;
    void* const* out_all; void* const* ws_all;
    int rank;
};

typedef __amdgpu_buffer_rsrc_t rsrc_t;
__device__ __forceinline__ rsrc_t make_rsrc(const void* base, unsigned bytes) {
    const u64 b = (u64)base;
    const unsigned lo = __builtin_amdgcn_readfirstlane((unsigned)b), hi = __builtin_amdgcn_readfirstlane((unsigned)(b >> 32));
    return __builtin_amdgcn_make_buffer_rsrc((void*)(((u64)hi << 32) | lo), 0, bytes, 0x00020000);
}
constexpr int SC1 = 16, SYS = 17;
template <int AUX> __device__ __forceinline__ u32x4 bld16(rsrc_t r, unsigned off) { return __builtin_amdgcn_raw_buffer_load_b128(r, off, 0, AUX); }
template <int AUX> __device__ __forceinline__ unsigned bld4(rsrc_t r, unsigned off) { return __builtin_amdgcn_raw_buffer_load_b32(r, off, 0, AUX); }
template <int AUX> __device__ __forceinline__ unsigned bld2(rsrc_t r, unsigned off) { return __builtin_amdgcn_raw_buffer_load_b16(r, off, 0, AUX); }
template <int AUX> __device__ __forceinline__ void bst16(rsrc_t r, unsigned off, u32x4 v) { __builtin_amdgcn_raw_buffer_store_b128(v, r, off, 0, AUX); }
template <int AUX> __device__ __forceinline__ void bst8(rsrc_t r, unsigned off, u32x2 v) { __builtin_amdgcn_raw_buffer_store_b64(v, r, off, 0, AUX); }
template <int AUX> __device__ __forceinline__ void bst4(rsrc_t r, unsigned off, unsigned v) { __builtin_amdgcn_raw_buffer_store_b32(v, r, off, 0, AUX); }
template <int AUX> __device__ __forceinline__ void bst2(rsrc_t r, unsigned off, unsigned v) { __builtin_amdgcn_raw_buffer_store_b16((u16)v, r, off, 0, AUX); }

__device__ __forceinline__ u32x4 make_desc(const void* base, unsigned bytes) {
    const u64 b = (u64)base;
    return (u32x4){(unsigned)__builtin_amdgcn_readfirstlane((unsigned)b), (unsigned)__builtin_amdgcn_readfirstlane((unsigned)(b >> 32)), bytes, 0x00020000u};
}
#define ALD16(dst, desc, off, imm) asm volatile("buffer_load_dwordx4 %0, %1, %2, 0 offen offset:%3 sc0 sc1" : "+v"(dst) : "v"(off), "s"(desc), "i"(imm) : "memory")
#define ALD8(dst, desc, off, imm) asm volatile("buffer_load_dwordx2 %0, %1, %2, 0 offen offset:%3 sc0 sc1" : "+v"(dst) : "v"(off), "s"(desc), "i"(imm) : "memory")
#define ALD4(dst, desc, off, imm) asm volatile("buffer_load_dword %0, %1, %2, 0 offen offset:%3 sc0 sc1" : "+v"(dst) : "v"(off), "s"(desc), "i"(imm) : "memory")
#define ALD2(dst, desc, off, imm) asm volatile("buffer_load_ushort %0, %1, %2, 0 offen offset:%3 sc0 sc1" : "+v"(dst) : "v"(off), "s"(desc), "i"(imm) : "memory")
constexpr unsigned OOB = 0xfffff000u;

#define WAIT_V(n) asm volatile("s_waitcnt vmcnt(" #n ")" ::: "memory")
#define WAIT_L(n) asm volatile("s_waitcnt lgkmcnt(" #n ")" ::: "memory")
#define BAR __builtin_amdgcn_s_barrier()
#define SCHED __builtin_amdgcn_sched_barrier(0)
#define CFENCE asm volatile("" ::: "memory")
#define STAMP(k) do { } while (0)

__device__ __forceinline__ unsigned rfl(unsigned v) { return __builtin_amdgcn_readfirstlane(v); }
__device__ __forceinline__ unsigned pkmax(unsigned a, unsigned b) {
    u16x2 x = __builtin_bit_cast(u16x2, a), y = __builtin_bit_cast(u16x2, b);
    return __builtin_bit_cast(unsigned, __builtin_elementwise_max(x, y));
}
__device__ __forceinline__ unsigned pkmin(unsigned a, unsigned b) {
    u16x2 x = __builtin_bit_cast(u16x2, a), y = __builtin_bit_cast(u16x2, b);
    return __builtin_bit_cast(unsigned, __builtin_elementwise_min(x, y));
}
__device__ __forceinline__ unsigned wave_max(unsigned v) {
#pragma unroll
    for (int o = 32; o >= 1; o >>= 1) { const unsigned t = (unsigned)__shfl_xor((int)v, o); v = v > t ? v : t; }
    return rfl(v);
}
__device__ __forceinline__ unsigned cvt_pk_bf16(float lo, float hi) { unsigned r; asm volatile("v_cvt_pk_bf16_f32 %0, %1, %2" : "=v"(r) : "v"(lo), "v"(hi)); return r; }
__device__ __forceinline__ int lds_byte(int r, int c) { const int st = (r >> 4) * 2 + (c >> 5), rr = r & 15, cc = c & 31, ob = rr * 64 + cc * 2; return st * 1024 + (ob ^ (((ob >> 9) & 1) << 5)); }

__device__ __forceinline__ unsigned wave_scan(unsigned x) {
    x += (unsigned)__builtin_amdgcn_update_dpp(0, (int)x, 0x111, 0xf, 0xf, false);
    x += (unsigned)__builtin_amdgcn_update_dpp(0, (int)x, 0x112, 0xf, 0xf, false);
    x += (unsigned)__builtin_amdgcn_update_dpp(0, (int)x, 0x114, 0xf, 0xf, false);
    x += (unsigned)__builtin_amdgcn_update_dpp(0, (int)x, 0x118, 0xf, 0xf, false);
    x += (unsigned)__builtin_amdgcn_update_dpp(0, (int)x, 0x142, 0xa, 0xf, false);
    x += (unsigned)__builtin_amdgcn_update_dpp(0, (int)x, 0x143, 0xc, 0xf, false);
    return x;
}
__device__ __forceinline__ unsigned wave_sum(unsigned v) { return __builtin_amdgcn_readlane(wave_scan(v), 63); }
__device__ __forceinline__ unsigned pksub(unsigned a, unsigned b) {
    u16x2 x = __builtin_bit_cast(u16x2, a), y = __builtin_bit_cast(u16x2, b);
    return __builtin_bit_cast(unsigned, (u16x2)(x - y));
}
__device__ __forceinline__ unsigned bfi(unsigned mask, unsigned a, unsigned b) { return (a & mask) | (b & ~mask); }
__device__ __forceinline__ unsigned window_top(unsigned esum, unsigned n, unsigned emax) {
    unsigned t = n ? (esum * 32u + n * 86u) / (n * 32u) : 7u;
    t = t > emax ? emax : t;
    return t < 7u ? 7u : t;
}
template <int NQ> __device__ __forceinline__ unsigned apply_escapes(unsigned (&e)[NQ], const unsigned (&m)[NQ], LAS unsigned char* smem, unsigned off) {
    unsigned sel[NQ], o[NQ]; u32x2 w[NQ];
#pragma unroll
    for (int q = 0; q < NQ; ++q) {
        sel[q] = *(const LAS unsigned*)(smem + SM_LUTE + __builtin_amdgcn_udot4(m[q], 0x08040201u, 0u, false));
        o[q] = off; off += (unsigned)__builtin_popcount(m[q]);
        const LAS unsigned* pw = (const LAS unsigned*)(smem + (o[q] & ~3u));
        w[q] = (u32x2){pw[0], pw[1]};
    }
#pragma unroll
    for (int q = 0; q < NQ; ++q) e[q] = __builtin_amdgcn_perm(__builtin_amdgcn_alignbyte(w[q].y, w[q].x, o[q]), e[q], sel[q]);
    return off;
}
struct Order {
    int ph, start;
    template <int NWU, int NXU> __device__ __forceinline__ void locate(int i, int& j, int& g, int& u) {
        for (;;) {
            int c = (ph < 16) ? NWU : 0;
#pragma unroll
            for (int k = 0; k < 4; ++k) c += ((unsigned)(ph - lag(k)) < 16u) ? NXU : 0;
            if (i < start + c) break;
            start += c; ++ph;
        }
        u = i - start; g = 0; j = ph;
        if (ph < 16) { if (u < NWU) return; u -= NWU; }
#pragma unroll
        for (int k = 0; k < 4; ++k) {
            if ((unsigned)(ph - lag(k)) < 16u) { if (u < NXU) { g = 1 + k; j = ph - lag(k); return; } u -= NXU; }
        }
    }
};

__device__ __forceinline__ bool give_up(gu32* tmo, unsigned seq, u64 t0) {
    bool dead = rfl(__hip_atomic_load(tmo, __ATOMIC_RELAXED, __HIP_MEMORY_SCOPE_AGENT)) == seq;
    if (!dead && __builtin_amdgcn_s_memrealtime() - t0 > DEADLINE) {
        __hip_atomic_store(tmo, seq, __ATOMIC_RELAXED, __HIP_MEMORY_SCOPE_AGENT);
        dead = true;
    }
    return dead;
}
__device__ __forceinline__ bool wait_sys32(gu32* flag, unsigned seq, gu32* tmo) {
    bool dead = false;
    if (rfl(__hip_atomic_load(flag, __ATOMIC_RELAXED, __HIP_MEMORY_SCOPE_SYSTEM)) < seq) {
        const u64 t0 = __builtin_amdgcn_s_memrealtime(); unsigned n = 0;
        while (rfl(__hip_atomic_load(flag, __ATOMIC_RELAXED, __HIP_MEMORY_SCOPE_SYSTEM)) < seq) {
            if ((++n & 63u) == 0u && give_up(tmo, seq, t0)) { dead = true; break; }
            __builtin_amdgcn_s_sleep(2);
        }
    }
    __builtin_amdgcn_fence(__ATOMIC_ACQUIRE, "wavefront");
    return dead;
}

__device__ __forceinline__ void pack_unit(const Params& p, char* ws, rsrc_t wsr, LAS unsigned char* lw, int j, int g, int u, int lane, unsigned seqv) {
    const bool isx = g != 0; const int ub = isx ? (u >> 4) * 64 + (g - 1) * 16 + (u & 15) : u;
    const int ra = p.rank >> 2, rb = p.rank & 3;
    u32x4 v[8];
    if (isx) {
        const u16* src = p.x + (size_t)(ub * 64 + (lane & 15)) * KS + j * 64 + (lane >> 4) * 8;
#pragma unroll
        for (int c = 0; c < 8; ++c) v[c] = *(const u32x4*)(src + (size_t)(c >> 1) * 16 * KS + (c & 1) * 32);
    } else {
        constexpr int P = 144;
        const u16* src = p.w + (size_t)(j * 64 + (lane >> 3)) * ND + ub * 64 + (lane & 7) * 8;
        u32x4 tmp[8];
#pragma unroll
        for (int i = 0; i < 8; ++i) tmp[i] = *(const u32x4*)(src + (size_t)i * 8 * ND);
#pragma unroll
        for (int i = 0; i < 8; ++i) *(LAS u32x4*)(lw + (i * 8 + (lane >> 3)) * P + (lane & 7) * 16) = tmp[i];
        const int q = (lane >> 2) & 3, pp = lane & 3;
#pragma unroll
        for (int c = 0; c < 8; ++c) {
            const int k0 = (c & 1) * 32 + (lane >> 4) * 8, rbk = c >> 1;
            const int col = (rbk >> 1) * 32 + 8 * pp + 4 * (rbk & 1);
            const s16x4 a = __builtin_amdgcn_ds_read_tr16_b64_v4i16((LAS s16x4*)(lw + (k0 + q) * P + col * 2));
            const s16x4 b = __builtin_amdgcn_ds_read_tr16_b64_v4i16((LAS s16x4*)(lw + (k0 + 4 + q) * P + col * 2));
            const u32x2 a2 = __builtin_bit_cast(u32x2, a), b2 = __builtin_bit_cast(u32x2, b);
            v[c] = (u32x4){a2.x, a2.y, b2.x, b2.y};
        }
    }
    const unsigned loff = (lane & 15) * 64 + (((lane >> 4) * 16) ^ (((lane >> 3) & 1) << 5));
    const int cls = isx ? (ub >> 6) : (ub >> 5);
    const bool own = isx ? (cls == ra) : (cls == rb);
    const int li = isx ? (ub & 63) : (ub & 31);
    if (own) {
        const unsigned dst = (isx ? OFF_XL + (unsigned)(j * 8 * 64 + li) * 8192u : OFF_WL + (unsigned)(j * 8 * 32 + li) * 8192u) + loff;
#pragma unroll
        for (int c = 0; c < 8; ++c) bst16<SC1>(wsr, dst + c * 1024, v[c]);
    }
    unsigned e4[16], sm[16], mx = 0, esum = 0;
#pragma unroll
    for (int q = 0; q < 16; ++q) {
        const unsigned a = v[q >> 1][(q & 1) * 2], b = v[q >> 1][(q & 1) * 2 + 1];
        const unsigned lo = __builtin_amdgcn_perm(b, a, 0x06040200u), hi = __builtin_amdgcn_perm(b, a, 0x07050301u);
        e4[q] = bfi(0x01010101u, lo >> 7, hi << 1);
        sm[q] = bfi(0x7f7f7f7fu, lo, hi);
        esum = __builtin_amdgcn_udot4(e4[q], 0x01010101u, esum, false);
        mx = pkmax(mx, a & 0x7fff7fffu); mx = pkmax(mx, b & 0x7fff7fffu);
    }
    mx = (mx & 0xffffu) > (mx >> 16) ? (mx & 0xffffu) : (mx >> 16);
    const unsigned top = window_top(wave_sum(esum), 4096u, wave_max(mx) >> 7);
    const unsigned top2 = top * 0x00010001u;
    unsigned p2[4] = {0u, 0u, 0u, 0u}, p1[2] = {0u, 0u}, em[2] = {0u, 0u};
#pragma unroll
    for (int q = 0; q < 16; ++q) {
        const unsigned da = pkmin(pksub(top2, e4[q] & 0x00ff00ffu), 0x00070007u), db = pkmin(pksub(top2, (e4[q] >> 8) & 0x00ff00ffu), 0x00070007u);
        const unsigned d = da | (db << 8);
        p2[q >> 2] |= (d & 0x03030303u) << (2 * (q & 3));
        p1[q >> 3] |= ((d >> 2) & 0x01010101u) << (q & 7);
        em[q >> 3] |= (__builtin_amdgcn_udot4((d + 0x01010101u) & 0x08080808u, 0x08040201u, 0u, false) >> 3) << (4 * (q & 7));
    }
    const unsigned ec = (unsigned)__builtin_popcount(em[0]) + (unsigned)__builtin_popcount(em[1]);
    const unsigned einc = wave_scan(ec), cnt = __builtin_amdgcn_readlane(einc, 63);
    const bool raw = FORCE_RAW_IN || cnt > 252u;
    const unsigned slot0 = (isx ? OFF_EX : OFF_EW) + (unsigned)(j * 128 + ub) * USLOT;
    const unsigned pay = slot0 + 128 + lane * 16;
    if (raw) {
#pragma unroll
        for (int c = 0; c < 8; ++c) bst16<SYS>(wsr, pay + c * 1024, v[c]);
        if (lane == 0) bst4<SYS>(wsr, slot0, 1u << 8);
    } else {
#pragma unroll
        for (int i = 0; i < 4; ++i) bst16<SYS>(wsr, pay + i * 1024, (u32x4){sm[4 * i], sm[4 * i + 1], sm[4 * i + 2], sm[4 * i + 3]});
        bst16<SYS>(wsr, pay + 4096, (u32x4){p2[0], p2[1], p2[2], p2[3]});
        bst8<SYS>(wsr, slot0 + 128 + 5120 + lane * 8, (u32x2){p1[0], p1[1]});
        LAS unsigned char* le = lw + 10240;
        LAS unsigned char* ll = lw + 14336;
#pragma unroll
        for (int i = 0; i < 4; ++i) *(LAS u32x4*)(le + lane * 64 + i * 16) = (u32x4){e4[4 * i], e4[4 * i + 1], e4[4 * i + 2], e4[4 * i + 3]};
        if (lane == 0) *(LAS unsigned*)ll = top | (cnt << 16);
        unsigned eo = 4u + einc - ec;
#pragma unroll
        for (int h = 0; h < 2; ++h) {
            unsigned mk = em[h];
            while (mk) { const unsigned bt = (unsigned)__builtin_ctz(mk); mk &= mk - 1u; ll[eo++] = le[lane * 64 + h * 32 + bt]; }
        }
        const unsigned lv = *(const LAS unsigned*)(ll + lane * 4);
        if (lane < 32) bst4<SYS>(wsr, slot0 + lane * 4, lv);
        else if (cnt > 124u) bst4<SYS>(wsr, slot0 + 128 + 5632 + (lane - 32) * 4, lv);
    }
    WAIT_V(0);
    const unsigned seq = rfl(seqv) + 1u;
    if (own && lane == 0) {
        const unsigned f = (isx ? OFF_XF : OFF_WF) + (unsigned)(((li >> 2) * NSTEP + j * 8) * 4 + (li & 3)) * 4u;
        __hip_atomic_store((gu32*)(ws + f), seq, __ATOMIC_RELAXED, __HIP_MEMORY_SCOPE_AGENT);
    }
    unsigned ticket = 0;
    if (lane == 0) ticket = __hip_atomic_fetch_add((gu32*)(ws + OFF_ECNT + (unsigned)(j * 12 + (isx ? cls * 4 + g - 1 : 8 + cls)) * 128u), 1u, __ATOMIC_RELAXED, __HIP_MEMORY_SCOPE_AGENT);
    ticket = rfl(ticket);
    const unsigned per = isx ? 16u : 32u;
    if ((ticket & (per - 1u)) == per - 1u) {
        const bool want = lane < NR && lane != p.rank && (isx ? ((lane >> 2) == cls) : ((lane & 3) == cls));
        if (want) {
            char* pw = (char*)p.ws_all[lane];
            __hip_atomic_store((gu32*)(pw + OFF_FIN + (unsigned)p.rank * 512u + (unsigned)j * 32u + (unsigned)g * 4u), seq, __ATOMIC_RELAXED, __HIP_MEMORY_SCOPE_SYSTEM);
        }
    }
}

struct UnitRegs { u32x4 a[5]; u32x2 p1; unsigned hdr; };
__device__ __forceinline__ unsigned unit_slot(int rank, int j, int g, int u) {
    const int ub = g ? (rank >> 2) * 64 + (g - 1) * 16 + u : (rank & 3) * 32 + u;
    return (g ? OFF_EX : OFF_EW) + (unsigned)(j * 128 + ub) * USLOT;
}
__device__ __forceinline__ void unit_issue(UnitRegs& r, u32x4 pd, unsigned slot0, int lane) {
    const unsigned oh = slot0 + (lane & 31) * 4, o0 = slot0 + 128 + lane * 16, o1 = o0 + 4096, o2 = slot0 + 128 + 5120 + lane * 8;
    ALD4(r.hdr, pd, oh, 0);
    ALD16(r.a[0], pd, o0, 0); ALD16(r.a[1], pd, o0, 1024); ALD16(r.a[2], pd, o0, 2048); ALD16(r.a[3], pd, o0, 3072);
    ALD16(r.a[4], pd, o1, 0); ALD8(r.p1, pd, o2, 0);
}
#define UNIT_LANDED(r, n) asm volatile("s_waitcnt vmcnt(0)" : "+v"(r.a[0]), "+v"(r.a[1]), "+v"(r.a[2]), "+v"(r.a[3]), "+v"(r.a[4]), "+v"(r.p1), "+v"(r.hdr), "+v"(n) :: "memory")
#define UNIT_PIN(r) asm volatile("" : "+v"(r.a[0]), "+v"(r.a[1]), "+v"(r.a[2]), "+v"(r.a[3]), "+v"(r.a[4]), "+v"(r.p1), "+v"(r.hdr))
__device__ __forceinline__ void unit_unpack(const UnitRegs& r, rsrc_t wsr, rsrc_t pr, LAS unsigned char* smem, unsigned esc, unsigned slot0, unsigned dst0, int lane) {
    const unsigned dst = dst0 + (lane & 15) * 64 + (((lane >> 4) * 16) ^ (((lane >> 3) & 1) << 5));
    const unsigned h0 = rfl(r.hdr);
    if ((h0 >> 8) & 1u) {
        u32x4 b[3];
#pragma unroll
        for (int c = 0; c < 3; ++c) b[c] = bld16<SYS>(pr, slot0 + 128 + lane * 16 + (5 + c) * 1024);
#pragma unroll
        for (int c = 0; c < 5; ++c) bst16<SC1>(wsr, dst + c * 1024, r.a[c]);
#pragma unroll
        for (int c = 0; c < 3; ++c) bst16<SC1>(wsr, dst + (5 + c) * 1024, b[c]);
        return;
    }
    unsigned ev = r.hdr;
    if ((h0 >> 16) > 124u) { const unsigned e2 = bld4<SYS>(pr, slot0 + 128 + 5632 + (lane & 31) * 4); ev = lane < 32 ? ev : e2; }
    *(LAS unsigned*)(smem + esc + lane * 4) = ev;
    const unsigned tq = (h0 & 0xffu) * 0x01010101u;
    unsigned e[16], m[16], ec = 0;
#pragma unroll
    for (int q = 0; q < 16; ++q) {
        const unsigned d = ((r.a[4][q >> 2] >> (2 * (q & 3))) & 0x03030303u) | (((r.p1[q >> 3] >> (q & 7)) & 0x01010101u) << 2);
        m[q] = (d + 0x01010101u) & 0x08080808u; e[q] = tq - d;
        ec += (unsigned)__builtin_popcount(m[q]);
    }
    apply_escapes<16>(e, m, smem, esc + 4u + wave_scan(ec) - ec);
#pragma unroll
    for (int c = 0; c < 8; ++c) {
        u32x4 o;
#pragma unroll
        for (int h = 0; h < 2; ++h) {
            const int q = 2 * c + h; const unsigned sm = r.a[q >> 2][q & 3];
            const unsigned lo = bfi(0x7f7f7f7fu, sm, e[q] << 7), hi = bfi(0x7f7f7f7fu, e[q] >> 1, sm);
            o[2 * h] = __builtin_amdgcn_perm(hi, lo, 0x05010400u); o[2 * h + 1] = __builtin_amdgcn_perm(hi, lo, 0x07030602u);
        }
        bst16<SC1>(wsr, dst + c * 1024, o);
    }
}

__device__ __forceinline__ unsigned strip_out_off(int r, int s, int lane) {
    const int tile = s >> 3, tm = tile >> 3, tn = tile & 7;
    const int row = (r >> 2) * RROWS + tm * 256 + (s & 7) * 32 + (lane >> 5);
    const int col = (r & 3) * RCOLS + tn * 256 + (lane & 31) * 8;
    return ((unsigned)row * ND + col) * 2u;
}

struct StripRegs { u32x4 mk; u32x4 lo[8]; unsigned p2[8], exc; };
__device__ __forceinline__ void strip_issue(StripRegs& r, u32x4 pd, unsigned s, unsigned info, int lane) {
    if (info >> 30) return;
    const unsigned T = info & 0x3fffu, slot = OFF_EO + s * SSLOT;
    const unsigned om = slot + lane * 16;
    ALD16(r.mk, pd, om, 0);
    { const unsigned oe = (unsigned)lane * 4u < ((info >> 22) & 255u) ? slot + lane * 4 : OOB; ALD4(r.exc, pd, oe, 3072); }
#define STRIP_ROUND(k) if ((unsigned)(k) * 1024u < T) { const bool act = (unsigned)((k) * 1024 + lane * 16) < T; \
        const unsigned ol = act ? om + 3328 + ((k) >> 2) * 4096 : OOB, o2 = act ? slot + lane * 4 : OOB; \
        ALD16(r.lo[k], pd, ol, ((k) & 3) * 1024); ALD4(r.p2[k], pd, o2, 1024 + (k) * 256); }
    STRIP_ROUND(0) STRIP_ROUND(1) STRIP_ROUND(2) STRIP_ROUND(3) STRIP_ROUND(4) STRIP_ROUND(5) STRIP_ROUND(6) STRIP_ROUND(7)
}
#define STRIP_LANDED(r, fn, n, cnt) do { \
    asm volatile("s_waitcnt vmcnt(" #cnt ")" : "+v"(r.lo[0]), "+v"(r.lo[1]), "+v"(r.lo[2]), "+v"(r.lo[3]), "+v"(r.lo[4]), "+v"(r.lo[5]), "+v"(r.lo[6]), "+v"(r.lo[7]), \
        "+v"(r.mk), "+v"(r.exc), "+v"(fn), "+v"(n), "+v"(r.p2[0]), "+v"(r.p2[1]), "+v"(r.p2[2]), "+v"(r.p2[3]), "+v"(r.p2[4]), "+v"(r.p2[5]), "+v"(r.p2[6]), "+v"(r.p2[7]) :: "memory"); } while (0)
__device__ __forceinline__ void strip_stage1(const StripRegs& r, LAS unsigned char* smem, unsigned lbase, unsigned esc, unsigned info, int lane) {
    if (info >> 30) return;
    const unsigned T = info & 0x3fffu;
    const unsigned tq = ((info >> 14) & 0xffu) * 0x01010101u;
    *(LAS unsigned*)(smem + esc + lane * 4) = r.exc;
    unsigned ec = 0;
#pragma unroll
    for (int k = 0; k < 8; ++k) {
        if ((unsigned)k * 1024u < T) {
            const unsigned t = r.p2[k] & (r.p2[k] >> 1);
#pragma unroll
            for (int i = 0; i < 4; ++i) ec += (unsigned)__builtin_popcount((t >> (2 * i)) & (r.lo[k][i] >> 7) & 0x01010101u);
        }
    }
    unsigned eo = esc + wave_scan(ec) - ec;
#pragma unroll
    for (int k = 0; k < 8; ++k) {
        if ((unsigned)k * 1024u < T) {
            unsigned e[4], m[4];
#pragma unroll
            for (int i = 0; i < 4; ++i) {
                const unsigned d = ((r.p2[k] >> (2 * i)) & 0x03030303u) | ((r.lo[k][i] >> 5) & 0x04040404u);
                m[i] = (d + 0x01010101u) & 0x08080808u; e[i] = tq - d;
            }
            eo = apply_escapes<4>(e, m, smem, eo);
            unsigned lo[4];
#pragma unroll
            for (int i = 0; i < 4; ++i) { lo[i] = bfi(0x7f7f7f7fu, r.lo[k][i], e[i] << 7); e[i] = (e[i] >> 1) & 0x7f7f7f7fu; }
            *(LAS u32x4*)(smem + lbase + k * 2048 + lane * 32) = (u32x4){lo[0], e[0], lo[1], e[1]};
            *(LAS u32x4*)(smem + lbase + k * 2048 + lane * 32 + 16) = (u32x4){lo[2], e[2], lo[3], e[3]};
        }
    }
}
__device__ __forceinline__ void strip_stage2(u32x4 mk, rsrc_t outr, rsrc_t por, LAS unsigned char* smem, unsigned lbase, int peer, unsigned s, unsigned info, int lane) {
    const unsigned oo = strip_out_off(peer, (int)s, lane);
    if (info >> 31) {
#pragma unroll
        for (int q = 0; q < 16; ++q) bst16<0>(outr, oo + q * 2 * ND * 2, (u32x4){0x7fc07fc0u, 0x7fc07fc0u, 0x7fc07fc0u, 0x7fc07fc0u});
        return;
    }
    if (info >> 30) {
#pragma unroll
        for (int h = 0; h < 4; ++h) {
            u32x4 a[4];
#pragma unroll
            for (int q = 0; q < 4; ++q) a[q] = bld16<SYS>(por, oo + (h * 4 + q) * 2 * ND * 2);
#pragma unroll
            for (int q = 0; q < 4; ++q) bst16<0>(outr, oo + (h * 4 + q) * 2 * ND * 2, a[q]);
        }
        return;
    }
    unsigned c2[8], x[8];
#pragma unroll
    for (int i = 0; i < 8; ++i) {
        const unsigned m16 = mk[i >> 1] >> ((i & 1) * 16);
        c2[i] = (unsigned)__builtin_popcount(m16 & 0xffu) | ((unsigned)__builtin_popcount((m16 >> 8) & 0xffu) << 16);
        x[i] = c2[i];
    }
#define SCAN_STEP(ctrl, rmask) _Pragma("unroll") for (int i = 0; i < 8; ++i) x[i] += (unsigned)__builtin_amdgcn_update_dpp(0, (int)x[i], ctrl, rmask, 0xf, false);
    SCAN_STEP(0x111, 0xf) SCAN_STEP(0x112, 0xf) SCAN_STEP(0x114, 0xf) SCAN_STEP(0x118, 0xf) SCAN_STEP(0x142, 0xa) SCAN_STEP(0x143, 0xc)
    unsigned pos = 0;
#pragma unroll
    for (int bt = 0; bt < 4; ++bt) {
        u32x4 sel[4], d[8]; unsigned off[8];
#pragma unroll
        for (int k = 0; k < 4; ++k) {
            const int i = bt * 2 + (k >> 1), hq = k & 1;
            const unsigned m8 = (mk[bt] >> (k * 8)) & 0xffu;
            sel[k] = *(const LAS u32x4*)(smem + SM_LUT + m8 * 16u);
            const unsigned tot = __builtin_amdgcn_readlane(x[i], 63);
            off[2 * k] = pos + (((x[i] - c2[i]) >> (hq * 16)) & 0xffffu);
            off[2 * k + 1] = off[2 * k] + (unsigned)__builtin_popcount(m8 & 15u);
            pos += (tot >> (hq * 16)) & 0xffffu;
        }
#pragma unroll
        for (int g = 0; g < 8; ++g) {
            const LAS u32x2* pl = (const LAS u32x2*)(smem + lbase + (off[g] & ~3u) * 2u);
            const u32x2 d0 = pl[0], d1 = pl[1];
            d[g] = (u32x4){d0.x, d0.y, d1.x, d1.y};
        }
#pragma unroll
        for (int k = 0; k < 4; ++k) {
            u32x4 o;
#pragma unroll
            for (int g = 0; g < 2; ++g) {
                const u32x4 dd = d[2 * k + g];
                const unsigned l4 = __builtin_amdgcn_alignbyte(dd.z, dd.x, off[2 * k + g]), h4 = __builtin_amdgcn_alignbyte(dd.w, dd.y, off[2 * k + g]);
                o[2 * g] = __builtin_amdgcn_perm(h4, l4, sel[k][2 * g]); o[2 * g + 1] = __builtin_amdgcn_perm(h4, l4, sel[k][2 * g + 1]);
            }
            bst16<0>(outr, oo + (bt * 4 + k) * 2 * ND * 2, o);
        }
    }
}

__device__ __forceinline__ void pack_strip(const Params& p, rsrc_t wsr, rsrc_t outr, LAS unsigned char* smem, int s, int wid, int lane, unsigned seq) {
    const unsigned oo = strip_out_off(p.rank, s, lane);
    const unsigned lbase = (unsigned)wid * 16384u;
    unsigned pos = lbase, mx = 0, sg = 0, esum = 0;
    const unsigned dummy = SM_DUMMY + lane * 2;
    u32x4 mk = {0u, 0u, 0u, 0u};
#pragma unroll
    for (int q2 = 0; q2 < 8; ++q2) {
        u32x4 v[2]; unsigned m8[2];
#pragma unroll
        for (int hq = 0; hq < 2; ++hq) {
            const int q = q2 * 2 + hq, row = wid * 32 + 2 * q + (lane >> 5);
            v[hq] = *(const LAS u32x4*)(smem + row * 512 + (((lane & 31) ^ (row & 7)) * 16));
            bst16<SYS>(outr, oo + q * 2 * ND * 2, v[hq]);
            unsigned m = 0;
#pragma unroll
            for (int k = 0; k < 4; ++k) {
                mx = pkmax(mx, v[hq][k]); sg |= v[hq][k];
                esum = __builtin_amdgcn_udot4((v[hq][k] >> 7) & 0x00ff00ffu, 0x00010001u, esum, false);
                const unsigned t = pkmin(v[hq][k], 0x00010001u);
                m |= ((t | (t >> 15)) & 3u) << (2 * k);
            }
            m8[hq] = m;
        }
        mk[q2 >> 1] |= (m8[0] | (m8[1] << 8)) << ((q2 & 1) * 16);
        const unsigned c2 = (unsigned)__builtin_popcount(m8[0]) | ((unsigned)__builtin_popcount(m8[1]) << 16);
        const unsigned inc = wave_scan(c2), exl = inc - c2, tot = __builtin_amdgcn_readlane(inc, 63);
#pragma unroll
        for (int hq = 0; hq < 2; ++hq) {
            unsigned a = pos + ((exl >> (hq * 16)) & 0xffffu) * 2u;
#pragma unroll
            for (int e = 0; e < 8; ++e) {
                const unsigned bit = (m8[hq] >> e) & 1u;
                *(LAS u16*)(smem + (bit ? a : dummy)) = (u16)(v[hq][e >> 1] >> ((e & 1) * 16));
                a += bit * 2u;
            }
            pos += ((tot >> (hq * 16)) & 0xffffu) * 2u;
        }
    }
    const unsigned T = (pos - lbase) >> 1;
    mx = (mx & 0xffffu) > (mx >> 16) ? (mx & 0xffffu) : (mx >> 16);
    const unsigned top = window_top(wave_sum(esum), T, (wave_max(mx) >> 7) & 0xffu);
    bool raw = FORCE_RAW_OUT || __builtin_amdgcn_ballot_w64((sg & 0x80008000u) != 0u) != 0ull;
    const unsigned slot = OFF_EO + (unsigned)s * SSLOT;
    unsigned cnt = 0;
    if (!raw) {
        if (lane < 16 && T + lane < 8192u) *(LAS u16*)(smem + lbase + (T + lane) * 2u) = (u16)(top << 7);
        const unsigned top2 = top * 0x00010001u;
        bst16<SYS>(wsr, slot + lane * 16, mk);
        unsigned em[4] = {0u, 0u, 0u, 0u};
#pragma unroll
        for (int r = 0; r < 8; ++r) {
            if ((unsigned)r * 1024u < T) {
                const bool act = (unsigned)(r * 1024 + lane * 16) < T;
                const u32x4 a = *(const LAS u32x4*)(smem + lbase + r * 2048 + lane * 32), b = *(const LAS u32x4*)(smem + lbase + r * 2048 + lane * 32 + 16);
                const unsigned w8[8] = {a.x, a.y, a.z, a.w, b.x, b.y, b.z, b.w};
                u32x4 lo; unsigned p2 = 0, em16 = 0;
#pragma unroll
                for (int k = 0; k < 4; ++k) {
                    const unsigned da = pkmin(pksub(top2, (w8[2 * k] >> 7) & 0x00ff00ffu), 0x00070007u), db = pkmin(pksub(top2, (w8[2 * k + 1] >> 7) & 0x00ff00ffu), 0x00070007u);
                    const unsigned d = __builtin_amdgcn_perm(db, da, 0x06040200u);
                    lo[k] = bfi(0x7f7f7f7fu, __builtin_amdgcn_perm(w8[2 * k + 1], w8[2 * k], 0x06040200u), d << 5);
                    p2 |= (d & 0x03030303u) << (2 * k);
                    em16 |= (__builtin_amdgcn_udot4((d + 0x01010101u) & 0x08080808u, 0x08040201u, 0u, false) >> 3) << (4 * k);
                }
                if (act) {
                    bst16<SYS>(wsr, slot + 3328 + r * 1024 + lane * 16, lo);
                    bst4<SYS>(wsr, slot + 1024 + r * 256 + lane * 4, p2);
                    em[r >> 1] |= em16 << (16 * (r & 1));
                }
            }
        }
        const unsigned ec = (unsigned)__builtin_popcount(em[0]) + (unsigned)__builtin_popcount(em[1]) + (unsigned)__builtin_popcount(em[2]) + (unsigned)__builtin_popcount(em[3]);
        const unsigned einc = wave_scan(ec);
        cnt = __builtin_amdgcn_readlane(einc, 63);
        if (cnt > 255u) raw = true;
        else if (cnt) {
            LAS unsigned char* ll = smem + SM_ESC + wid * 320;
            unsigned eo = einc - ec;
#pragma unroll
            for (int h = 0; h < 4; ++h) {
                unsigned mb = em[h];
                while (mb) { const unsigned bt = (unsigned)__builtin_ctz(mb); mb &= mb - 1u;
                    const unsigned val = *(const LAS u16*)(smem + lbase + (unsigned)(h * 2) * 2048u + (bt >> 4) * 2048u + lane * 32u + (bt & 15u) * 2u);
                    ll[eo++] = (unsigned char)(val >> 7); }
            }
            const unsigned lv = *(const LAS unsigned*)(ll + lane * 4);
            if ((unsigned)lane * 4u < cnt) bst4<SYS>(wsr, slot + 3072 + lane * 4, lv);
        }
    }
    WAIT_V(0);
    if (lane < NR && lane != p.rank) {
        const unsigned info = raw ? (1u << 30) : (T | (top << 14) | (cnt << 22));
        char* pw = (char*)p.ws_all[lane];
        __hip_atomic_store((gu64*)(pw + OFF_FOUT + ((unsigned)p.rank * 1024u + (unsigned)s) * 8u), ((u64)seq << 32) | info, __ATOMIC_RELAXED, __HIP_MEMORY_SCOPE_SYSTEM);
    }
}

}

__global__ void __launch_bounds__(512, 2) gemm_ar_fused(Params p) {
    __shared__ __attribute__((aligned(16))) unsigned char smem_[SMEM_BYTES];
    LAS unsigned char* smem = (LAS unsigned char*)smem_;
    const int tid = threadIdx.x, wid = rfl(tid >> 6), lane = tid & 63;
    char* ws = (char*)p.ws_all[p.rank];
    const rsrc_t wsr = make_rsrc(ws, WS_USED);
    const rsrc_t outr = make_rsrc(p.out, OUT_BYTES);
    gu32* tmo = (gu32*)(ws + OFF_TMO);
    gu32* seqp = (gu32*)(ws + OFF_SEQ + (unsigned)(blockIdx.x * 8 + wid) * 4u);
    const unsigned seqv = __hip_atomic_load(seqp, __ATOMIC_RELAXED, __HIP_MEMORY_SCOPE_AGENT);
    STAMP(0);
    const int xcd = blockIdx.x & 7, wi = blockIdx.x >> 3;
    const bool is_gemm = wi < 16 || xcd < 2;
    const int cw = (wi - 16) * 6 + xcd - 2;
    const int lq = cw % 7;
    if (!is_gemm) {
        if (tid == 0) *(LAS unsigned*)(smem + SM_TOKEN) = 0u;
        WAIT_L(0); BAR;
    }
    if (!is_gemm && wid >= 8 - PW) {
        Order od{0, 0};
        for (int t = (wid - (8 - PW)) * 96 + cw; t < 4096; t += NENC) {
            int j, g, u; od.locate<128, 32>(t, j, g, u);
            pack_unit(p, ws, wsr, smem + wid * 16384, j, g, u, lane, seqv);
        }
        STAMP(7);
    }
    const unsigned seq = rfl(seqv) + 1u;
    if (lane == 0) __hip_atomic_store(seqp, seq, __ATOMIC_RELAXED, __HIP_MEMORY_SCOPE_AGENT);
    if (!is_gemm) {
        if (wid >= NWV || cw / 7 >= NT) return;
        const int peer = (p.rank + 1 + lq) & 7;
        const rsrc_t pr = make_rsrc(p.ws_all[peer], WS_USED);
        const rsrc_t por = make_rsrc(p.out_all[peer], OUT_BYTES);
#pragma unroll
        for (int i = 0; i < 4; ++i) {
            const unsigned m8 = (unsigned)(i * 64 + lane); u32x4 sel;
#pragma unroll
            for (int g = 0; g < 2; ++g) {
                unsigned rk = 0;
#pragma unroll
                for (int h = 0; h < 2; ++h) {
                    unsigned w = 0;
#pragma unroll
                    for (int e = 0; e < 2; ++e) { const unsigned bit = (m8 >> (g * 4 + h * 2 + e)) & 1u; w |= (bit ? (rk | ((rk + 4u) << 8)) : 0x0c0cu) << (16 * e); rk += bit; }
                    sel[g * 2 + h] = w;
                }
            }
            *(LAS u32x4*)(smem + SM_LUT + m8 * 16u) = sel;
        }
        if (lane < 16) {
            unsigned rk = 4u, sel = 0;
#pragma unroll
            for (int k = 0; k < 4; ++k) { const unsigned bit = ((unsigned)lane >> k) & 1u; sel |= (bit ? rk : (unsigned)k) << (8 * k); rk += bit; }
            *(LAS unsigned*)(smem + SM_LUTE + lane * 8) = sel;
        }
        const unsigned esc = SM_ESC + (unsigned)wid * 320u;
        bool dead = false;
        const u32x4 pd = make_desc(p.ws_all[peer], WS_USED);
        const unsigned one = 1u;
        LAS unsigned* token = (LAS unsigned*)(smem + SM_TOKEN);
#define QNEXT(dst) do { if (lane == 0) asm volatile("global_atomic_add %0, %1, %2, off sc0" : "+v"(dst) : "v"(qp), "v"(one) : "memory"); } while (0)
#define TOKEN_TAKE() do { for (;;) { unsigned old = 1u; \
            if (lane == 0) old = __hip_atomic_exchange(token, 1u, __ATOMIC_RELAXED, __HIP_MEMORY_SCOPE_WORKGROUP); \
            if (rfl(old) == 0u) break; __builtin_amdgcn_s_sleep(1); } } while (0)
#define TOKEN_GIVE() do { if (lane == 0) __hip_atomic_store(token, 0u, __ATOMIC_RELAXED, __HIP_MEMORY_SCOPE_WORKGROUP); } while (0)
        {
            gu32* qp = (gu32*)(ws + OFF_Q + (unsigned)lq * 128u);
            const unsigned qbase = (seq - 1u) * (1536u + NT * NWV);
            Order od{0, 0};
            unsigned okm = 0;
            unsigned nq = 0;
            UnitRegs ur = {};
            QNEXT(nq);
            UNIT_LANDED(ur, nq);
            bool first = true;
            for (;;) {
                const unsigned i = rfl(nq) - qbase;
                if (i >= 1536u) break;
                int j, g, u; od.locate<32, 16>((int)i, j, g, u);
                if (((okm >> (5 * g)) & 31u) <= (unsigned)j) {
                    if (!dead) dead = wait_sys32((gu32*)(ws + OFF_FIN + (unsigned)peer * 512u + (unsigned)j * 32u + (unsigned)g * 4u), seq, tmo);
                    okm = (okm & ~(31u << (5 * g))) | ((unsigned)(j + 1) << (5 * g));
                }
                if (first) STAMP(1);
                first = false;
                const int step = j * 8 + 1 + lq, li = g ? (g - 1) * 16 + u : u;
                const unsigned slot0 = unit_slot(p.rank, j, g, u);
                const unsigned dst0 = g ? OFF_XL + (unsigned)(step * 64 + li) * 8192u : OFF_WL + (unsigned)(step * 32 + li) * 8192u;
                const unsigned fo = (g ? OFF_XF : OFF_WF) + (unsigned)(((li >> 2) * NSTEP + step) * 4 + (li & 3)) * 4u;
                TOKEN_TAKE();
                unit_issue(ur, pd, slot0, lane);
                UNIT_LANDED(ur, nq);
                TOKEN_GIVE();
                QNEXT(nq);
                unit_unpack(ur, wsr, pr, smem, esc, slot0, dst0, lane);
                UNIT_LANDED(ur, nq);
                if (lane == 0) __hip_atomic_store((gu32*)(ws + fo), seq, __ATOMIC_RELAXED, __HIP_MEMORY_SCOPE_AGENT);
            }
            STAMP(3);
        }
        {
            gu32* qp = (gu32*)(ws + OFF_Q + (unsigned)(7 + lq) * 128u);
            const unsigned qbase = (seq - 1u) * (1024u + NT * NWV);
            gu64* fbase = (gu64*)(ws + OFF_FOUT + (unsigned)peer * 8192u);
            const unsigned lbase = (unsigned)wid * 16384u;
            unsigned nq = 0; u64 fn = 0;
            StripRegs sr = {};
            QNEXT(nq);
            asm volatile("s_waitcnt vmcnt(0)" : "+v"(nq) :: "memory");
            unsigned s = rfl(nq) - qbase;
            bool first = true;
            while (s < 1024u) {
                unsigned fh = rfl((unsigned)(fn >> 32)), info = rfl((unsigned)fn);
                if (fh < seq && !dead) {
                    const u64 t0 = __builtin_amdgcn_s_memrealtime(); unsigned n = 0;
                    for (;;) {
                        const u64 f2 = __hip_atomic_load(fbase + s, __ATOMIC_RELAXED, __HIP_MEMORY_SCOPE_SYSTEM);
                        fh = rfl((unsigned)(f2 >> 32)); info = rfl((unsigned)f2);
                        if (fh >= seq) break;
                        if ((++n & 63u) == 0u && give_up(tmo, seq, t0)) { dead = true; break; }
                        __builtin_amdgcn_s_sleep(4);
                    }
                }
                __builtin_amdgcn_fence(__ATOMIC_ACQUIRE, "wavefront");
                if (dead) info = 3u << 30;
                if (first) STAMP(4);
                first = false;
                TOKEN_TAKE();
                strip_issue(sr, pd, s, info, lane);
                QNEXT(nq);
                STRIP_LANDED(sr, fn, nq, 0);
                TOKEN_GIVE();
                const unsigned sn = rfl(nq) - qbase;
                { gu64* fa = fbase + (sn & 1023u); asm volatile("global_load_dwordx2 %0, %1, off sc0 sc1" : "+v"(fn) : "v"(fa) : "memory"); }
                strip_stage1(sr, smem, lbase, esc, info, lane);
                strip_stage2(sr.mk, outr, por, smem, lbase, peer, s, info, lane);
                asm volatile("s_waitcnt vmcnt(16)" : "+v"(fn) :: "memory");
                s = sn;
            }
            STAMP(6);
        }
        return;
    }

    const int hsel = wi >> 4;
    const int tm = (xcd >> 1) * 4 + ((wi & 15) >> 2), tn = (xcd & 1) * 4 + (wi & 3);
    const bool hmode = xcd < 2;
    const int wr = wid >> 2, wc = wid & 3, fr = lane & 15, fq = lane >> 4;
    constexpr int HTB = 16384;
    const char* cA = ws + OFF_XL + (size_t)tm * 32768 + hsel * HTB + tid * 16;
    const size_t AH = hmode ? 0 : HTB;
    const char* cB = ws + OFF_WL + (size_t)tn * 32768 + tid * 16;
    constexpr size_t ASTEP = 64 * 8192, BSTEP = 32 * 8192;
    const unsigned ldsw = (unsigned)wid * 1024u;
    const int aoff = lds_byte(wr * 64 + fr, fq * 8), boff = lds_byte(wc * 32 + fr, fq * 8);
    const unsigned pollo = (lane < 32 ? OFF_XF + (unsigned)(tm * NSTEP + lane) * 16u : OFF_WF + (unsigned)(tn * NSTEP + lane - 32) * 16u);
    unsigned avail = 0;
#define SA(b, h) (((b) * 2 + (h)) * HTB)
#define SB(b, h) ((4 + (b) * 2 + (h)) * HTB)
#define STAGE(bufoff, gptr) do { _Pragma("unroll") for (int _i = 0; _i < 2; ++_i) \
        __builtin_amdgcn_global_load_lds((const unsigned*)((gptr) + _i * 8192), (LAS unsigned*)(smem + (bufoff) + ldsw + _i * 8192), 16, 0, SC1); } while (0)
#define LDA(dst, b, h) do { _Pragma("unroll") for (int m = 0; m < 4; ++m) _Pragma("unroll") for (int k = 0; k < 2; ++k) dst[m][k] = *(const LAS bf16x8*)(smem + SA(b, h) + aoff + m * 2048 + k * 1024); } while (0)
#define LDB(dst, b, h) do { _Pragma("unroll") for (int n = 0; n < 2; ++n) _Pragma("unroll") for (int k = 0; k < 2; ++k) dst[n][k] = *(const LAS bf16x8*)(smem + SB(b, h) + boff + n * 2048 + k * 1024); } while (0)
#define MMA(ai, bj, At, Bt) do { __builtin_amdgcn_s_setprio(1); _Pragma("unroll") for (int m = 0; m < 4; ++m) _Pragma("unroll") for (int n = 0; n < 2; ++n) _Pragma("unroll") for (int k = 0; k < 2; ++k) \
        acc[ai][bj][m][n] = __builtin_amdgcn_mfma_f32_16x16x32_bf16(Bt[n][k], At[m][k], acc[ai][bj][m][n], 0, 0, 0); __builtin_amdgcn_s_setprio(0); } while (0)
#define NEED(s) do { if ((avail & 0xffffu) <= (unsigned)(s)) { \
        if (wid == 0) { const unsigned s0 = avail & 0xffffu; unsigned av; const u64 t0 = __builtin_amdgcn_s_memrealtime(); unsigned n = 0; \
            for (;;) { const u32x4 f = bld16<SC1>(wsr, pollo + s0 * 16u); \
                const bool ok = (s0 + (lane & 31) >= (unsigned)NSTEP) || (f.x >= seq && f.y >= seq && f.z >= seq && f.w >= seq); \
                const u64 mm = __builtin_amdgcn_ballot_w64(ok); const unsigned both = (unsigned)mm & (unsigned)(mm >> 32); \
                av = s0 + (both == 0xffffffffu ? 32u : (unsigned)__builtin_ctz(~both)); \
                if (av > (unsigned)(s)) break; \
                if ((++n & 63u) == 0u && give_up(tmo, seq, t0)) { av = 0x10000u | NSTEP; break; } \
                __builtin_amdgcn_s_sleep(1); } \
            asm volatile("ds_write_b32 %0, %1\n\ts_waitcnt lgkmcnt(0)" :: "v"(smbase + SM_AVAIL), "v"(av) : "memory"); } \
        CFENCE; BAR; CFENCE; \
        unsigned av2; asm volatile("ds_read_b32 %0, %1\n\ts_waitcnt lgkmcnt(0)" : "=v"(av2) : "v"(smbase + SM_AVAIL) : "memory"); \
        avail = rfl(av2); } } while (0)
    const unsigned smbase = (unsigned)(uintptr_t)smem;

    f32x4 acc[2][2][4][2];
#pragma unroll
    for (int a = 0; a < 2; ++a)
#pragma unroll
        for (int b = 0; b < 2; ++b)
#pragma unroll
            for (int m = 0; m < 4; ++m)
#pragma unroll
                for (int n = 0; n < 2; ++n) acc[a][b][m][n] = (f32x4){0.f, 0.f, 0.f, 0.f};
    bf16x8 At[4][2], B0[2][2], B1[2][2];
    NEED(1);
    STAMP(2);
    STAGE(SB(0, 0), cB); STAGE(SA(0, 0), cA); STAGE(SB(0, 1), cB + HTB); STAGE(SA(0, 1), cA + AH);
    if (wr == 1) BAR;
    WAIT_V(4); BAR;
    STAGE(SB(1, 0), cB + BSTEP); STAGE(SA(1, 0), cA + ASTEP); STAGE(SB(1, 1), cB + BSTEP + HTB);
    WAIT_V(6); BAR;
#define KLOOP(FULL) \
    for (int t = 0; t < NSTEP; t += 2) { \
        const bool last = (t == NSTEP - 2); \
        const char* a1 = cA + (size_t)(t + 1) * ASTEP; \
        const char* a2 = last ? cA : cA + (size_t)(t + 2) * ASTEP; const char* b2 = last ? cB : cB + (size_t)(t + 2) * BSTEP; \
        const char* a3 = a2 + ASTEP; const char* b3 = b2 + BSTEP; \
        if (!last) NEED(t + 2); \
        LOOP_STAMPS; \
          \
        LDB(B0, 0, 0); SCHED; LDA(At, 0, 0); STAGE(SA(1, 1), a1 + (FULL ? HTB : 0)); \
        WAIT_L(8); BAR; WAIT_L(0); MMA(0, 0, At, B0); BAR; SCHED; \
          \
        LDB(B1, 0, 1); STAGE(SB(0, 0), b2); \
        BAR; WAIT_L(0); MMA(0, 1, At, B1); BAR; \
          \
        if (FULL) LDA(At, 0, 1); STAGE(SA(0, 0), a2); \
        BAR; WAIT_L(0); if (FULL) MMA(1, 0, At, B0); BAR; SCHED; \
          \
        STAGE(SB(0, 1), b2 + HTB); \
        WAIT_V(6); BAR; if (FULL) MMA(1, 1, At, B1); BAR; \
        if (!last) NEED(t + 3); \
          \
        LDB(B0, 1, 0); SCHED; LDA(At, 1, 0); STAGE(SA(0, 1), a2 + (FULL ? HTB : 0)); \
        WAIT_L(8); BAR; WAIT_L(0); MMA(0, 0, At, B0); BAR; SCHED; \
          \
        LDB(B1, 1, 1); STAGE(SB(1, 0), b3); \
        BAR; WAIT_L(0); MMA(0, 1, At, B1); BAR; \
          \
        if (FULL) LDA(At, 1, 1); STAGE(SA(1, 0), a3); \
        BAR; WAIT_L(0); if (FULL) MMA(1, 0, At, B0); BAR; SCHED; \
          \
        STAGE(SB(1, 1), b3 + HTB); \
        WAIT_V(6); BAR; if (FULL) MMA(1, 1, At, B1); BAR; \
    }
#define LOOP_STAMPS do { } while (0)
    if (hmode) { KLOOP(false) } else { KLOOP(true) }
    WAIT_V(0);
    if (wr == 0) BAR;
    BAR;
    STAMP(3);
    {
        int tz = threadIdx.x; asm volatile("" : "+v"(tz));
        const int lane2 = tz & 63, fr2 = lane2 & 15, fq2 = lane2 >> 4;
        const bool dead = (avail >> 16) != 0u || rfl(__hip_atomic_load(tmo, __ATOMIC_RELAXED, __HIP_MEMORY_SCOPE_AGENT)) == seq;
#pragma unroll
        for (int ai = 0; ai < 2; ++ai)
#pragma unroll
            for (int m = 0; m < 4; ++m) {
                const int row = ai * 128 + wr * 64 + m * 16 + fr2;
#pragma unroll
                for (int bj = 0; bj < 2; ++bj) {
                    f32x4 v0 = acc[ai][bj][m][0], v1 = acc[ai][bj][m][1];
#pragma unroll
                    for (int k = 0; k < 4; ++k) { v0[k] = v0[k] <= 0.f ? 0.f : v0[k]; v1[k] = v1[k] <= 0.f ? 0.f : v1[k]; }
                    u32x4 w; w.x = cvt_pk_bf16(v0[0], v0[1]); w.y = cvt_pk_bf16(v0[2], v0[3]); w.z = cvt_pk_bf16(v1[0], v1[1]); w.w = cvt_pk_bf16(v1[2], v1[3]);
                    if (dead) w = (u32x4){0x7fc07fc0u, 0x7fc07fc0u, 0x7fc07fc0u, 0x7fc07fc0u};
                    const int chunk = bj * 16 + wc * 4 + fq2;
                    *(LAS u32x4*)(smem + row * 512 + ((chunk ^ (row & 7)) * 16)) = w;
                }
            }
        __syncthreads();
        STAMP(4);
        if (!hmode || wid < 4) pack_strip(p, wsr, outr, smem, (tm * 8 + tn) * 8 + hsel * 4 + wid, wid, lane2, seq);
        STAMP(5);
    }
}

extern "C" void kernel_launch(int rank, int nranks, ncclComm_t comm,
                              hipStream_t stream,
                              void* const* d_in, const int* in_sizes, int n_in,
                              void* d_out, int out_size,
                              const int* in_dtypes, int out_dtype,
                              void* const* d_out_all,
                              void* const* d_ws_all, size_t ws_bytes) {
    (void)comm; (void)in_dtypes; (void)out_dtype;
    if (nranks != NR || n_in != 2 || (unsigned)rank >= (unsigned)NR || ws_bytes < WS_USED || in_sizes[0] != ND * KS || in_sizes[1] != KS * ND || out_size != ND * ND) {
        fprintf(stderr, "kernel_launch: unexpected problem shape\n");
        return;
    }
    Params p;
    p.x = (const u16*)d_in[0]; p.w = (const u16*)d_in[1]; p.out = (u16*)d_out;
    p.out_all = d_out_all; p.ws_all = d_ws_all; p.rank = rank;
    hipLaunchKernelGGL(gemm_ar_fused, dim3(NWG), dim3(512), 0, stream, p);
    HIP_CHECK(hipGetLastError());
}
```

```cpp
#include <hip/hip_runtime.h>
#include <rccl/rccl.h>
#include <cstdio>
#include <cstdint>

#define HIP_CHECK(expr) do { hipError_t _e = (expr); if (_e != hipSuccess) { \
    fprintf(stderr, "HIP error %s at %s:%d\n", hipGetErrorString(_e), __FILE__, __LINE__); \
    return; } } while (0)

#define LAS __attribute__((address_space(3)))
#define GAS __attribute__((address_space(1)))
typedef unsigned short u16;
typedef unsigned long long u64;
typedef short bf16x8 __attribute__((ext_vector_type(8)));
typedef short s16x4 __attribute__((ext_vector_type(4)));
typedef float f32x4 __attribute__((ext_vector_type(4)));
typedef unsigned u32x4 __attribute__((ext_vector_type(4)));
typedef unsigned u32x2 __attribute__((ext_vector_type(2)));
typedef u16 u16x2 __attribute__((ext_vector_type(2)));
typedef GAS unsigned gu32;
typedef GAS u64 gu64;

#define FORCE_RAW_IN 0
#define FORCE_RAW_OUT 0
#define NT 12
#define NL1 4
#define PDELAY 256
#define PSLEEP 32
constexpr int NWV = 4;
#define LAG1 0
#define LAG2 4
#define LAG3 8

namespace {

constexpr int NR = 8;
constexpr int ND = 8192;
constexpr int KS = 1024;
constexpr int RROWS = 4096, RCOLS = 2048;
constexpr int NSTEP = 128;
constexpr int NWG = 256;
constexpr int NPD = 84 * 4;
constexpr int NENC = NPD + 512;
constexpr int NONCE = 256 + 512 + 96;
static_assert(NT == 12, "wave roles");
__device__ __forceinline__ constexpr int lag(int k) { return k == 0 ? 0 : k == 1 ? LAG1 : k == 2 ? LAG2 : LAG3; }
constexpr unsigned long long DEADLINE = 40000000ull;

constexpr unsigned OFF_TMO = 0x80;
constexpr unsigned OFF_Q = 0x400;
constexpr unsigned OFF_ECNT = 0x1000;
constexpr unsigned OFF_FIN = 0x8000;
constexpr unsigned OFF_SEQ = 0xa000;
constexpr unsigned OFF_XF = 0x10000;
constexpr unsigned OFF_WF = 0x18000;
constexpr unsigned OFF_FOUT = 0x20000;
constexpr unsigned USLOT = 8192 + 1024;
constexpr unsigned OFF_EX = 1u << 20;
constexpr unsigned OFF_EW = OFF_EX + (18u << 20);
constexpr unsigned OFF_XL = OFF_EW + (18u << 20);
constexpr unsigned OFF_WL = OFF_XL + (64u << 20);
constexpr unsigned SSLOT = 11264;
constexpr unsigned OFF_EO = OFF_WL + (32u << 20);
constexpr unsigned WS_USED = OFF_EO + (13u << 20);
static_assert(2048u * USLOT <= (18u << 20), "unit area");
static_assert(1024u * SSLOT <= (13u << 20), "strip area");
constexpr unsigned OUT_BYTES = (unsigned)ND * ND * 2u;

constexpr int SM_TOKEN = 131072 + 32, SM_DUMMY = 131072 + 64, SM_LUT = 131072 + 256, SM_LUTE = SM_LUT + 4096, SM_SCR = SM_LUTE + 128;
constexpr int SM_AVAIL = SM_SCR + 24576 + 64;
constexpr int SMEM_BYTES = SM_AVAIL + 16;
constexpr unsigned CAP2 = 2304, CAP3 = 512, CAP4 = 240;

struct Params {
    const u16* x; const u16* w; u16* out;
    void* const* out_all; void* const* ws_all;
    int rank;
};

typedef __amdgpu_buffer_rsrc_t rsrc_t;
__device__ __forceinline__ rsrc_t make_rsrc(const void* base, unsigned bytes) {
    const u64 b = (u64)base;
    const unsigned lo = __builtin_amdgcn_readfirstlane((unsigned)b), hi = __builtin_amdgcn_readfirstlane((unsigned)(b >> 32));
    return __builtin_amdgcn_make_buffer_rsrc((void*)(((u64)hi << 32) | lo), 0, bytes, 0x00020000);
}
constexpr int SC1 = 16, SYS = 17;
template <int AUX> __device__ __forceinline__ u32x4 bld16(rsrc_t r, unsigned off) { return __builtin_amdgcn_raw_buffer_load_b128(r, off, 0, AUX); }
template <int AUX> __device__ __forceinline__ unsigned bld4(rsrc_t r, unsigned off) { return __builtin_amdgcn_raw_buffer_load_b32(r, off, 0, AUX); }
template <int AUX> __device__ __forceinline__ unsigned bld2(rsrc_t r, unsigned off) { return __builtin_amdgcn_raw_buffer_load_b16(r, off, 0, AUX); }
template <int AUX> __device__ __forceinline__ void bst16(rsrc_t r, unsigned off, u32x4 v) { __builtin_amdgcn_raw_buffer_store_b128(v, r, off, 0, AUX); }
template <int AUX> __device__ __forceinline__ void bst8(rsrc_t r, unsigned off, u32x2 v) { __builtin_amdgcn_raw_buffer_store_b64(v, r, off, 0, AUX); }
template <int AUX> __device__ __forceinline__ void bst4(rsrc_t r, unsigned off, unsigned v) { __builtin_amdgcn_raw_buffer_store_b32(v, r, off, 0, AUX); }
template <int AUX> __device__ __forceinline__ void bst2(rsrc_t r, unsigned off, unsigned v) { __builtin_amdgcn_raw_buffer_store_b16((u16)v, r, off, 0, AUX); }

__device__ __forceinline__ u32x4 make_desc(const void* base, unsigned bytes) {
    const u64 b = (u64)base;
    return (u32x4){(unsigned)__builtin_amdgcn_readfirstlane((unsigned)b), (unsigned)__builtin_amdgcn_readfirstlane((unsigned)(b >> 32)), bytes, 0x00020000u};
}
#define ALD16(dst, desc, off, imm) asm volatile("buffer_load_dwordx4 %0, %1, %2, 0 offen offset:%3 sc0 sc1" : "+v"(dst) : "v"(off), "s"(desc), "i"(imm) : "memory")
#define ALD8(dst, desc, off, imm) asm volatile("buffer_load_dwordx2 %0, %1, %2, 0 offen offset:%3 sc0 sc1" : "+v"(dst) : "v"(off), "s"(desc), "i"(imm) : "memory")
#define ALD4(dst, desc, off, imm) asm volatile("buffer_load_dword %0, %1, %2, 0 offen offset:%3 sc0 sc1" : "+v"(dst) : "v"(off), "s"(desc), "i"(imm) : "memory")
#define ALD2(dst, desc, off, imm) asm volatile("buffer_load_ushort %0, %1, %2, 0 offen offset:%3 sc0 sc1" : "+v"(dst) : "v"(off), "s"(desc), "i"(imm) : "memory")
constexpr unsigned OOB = 0xfffff000u;

#define WAIT_V(n) asm volatile("s_waitcnt vmcnt(" #n ")" ::: "memory")
#define WAIT_L(n) asm volatile("s_waitcnt lgkmcnt(" #n ")" ::: "memory")
#define BAR __builtin_amdgcn_s_barrier()
#define SCHED __builtin_amdgcn_sched_barrier(0)
#define CFENCE asm volatile("" ::: "memory")
#define STAMP(k) do { } while (0)

__device__ __forceinline__ unsigned rfl(unsigned v) { return __builtin_amdgcn_readfirstlane(v); }
__device__ __forceinline__ unsigned pkmax(unsigned a, unsigned b) {
    u16x2 x = __builtin_bit_cast(u16x2, a), y = __builtin_bit_cast(u16x2, b);
    return __builtin_bit_cast(unsigned, __builtin_elementwise_max(x, y));
}
__device__ __forceinline__ unsigned pkmin(unsigned a, unsigned b) {
    u16x2 x = __builtin_bit_cast(u16x2, a), y = __builtin_bit_cast(u16x2, b);
    return __builtin_bit_cast(unsigned, __builtin_elementwise_min(x, y));
}
__device__ __forceinline__ unsigned wave_max(unsigned v) {
#pragma unroll
    for (int o = 32; o >= 1; o >>= 1) { const unsigned t = (unsigned)__shfl_xor((int)v, o); v = v > t ? v : t; }
    return rfl(v);
}
__device__ __forceinline__ unsigned cvt_pk_bf16(float lo, float hi) { unsigned r; asm volatile("v_cvt_pk_bf16_f32 %0, %1, %2" : "=v"(r) : "v"(lo), "v"(hi)); return r; }
__device__ __forceinline__ int lds_byte(int r, int c) { const int st = (r >> 4) * 2 + (c >> 5), rr = r & 15, cc = c & 31, ob = rr * 64 + cc * 2; return st * 1024 + (ob ^ (((ob >> 9) & 1) << 5)); }

__device__ __forceinline__ unsigned wave_scan(unsigned x) {
    x += (unsigned)__builtin_amdgcn_update_dpp(0, (int)x, 0x111, 0xf, 0xf, false);
    x += (unsigned)__builtin_amdgcn_update_dpp(0, (int)x, 0x112, 0xf, 0xf, false);
    x += (unsigned)__builtin_amdgcn_update_dpp(0, (int)x, 0x114, 0xf, 0xf, false);
    x += (unsigned)__builtin_amdgcn_update_dpp(0, (int)x, 0x118, 0xf, 0xf, false);
    x += (unsigned)__builtin_amdgcn_update_dpp(0, (int)x, 0x142, 0xa, 0xf, false);
    x += (unsigned)__builtin_amdgcn_update_dpp(0, (int)x, 0x143, 0xc, 0xf, false);
    return x;
}
__device__ __forceinline__ unsigned wave_sum(unsigned v) { return __builtin_amdgcn_readlane(wave_scan(v), 63); }
__device__ __forceinline__ unsigned pksub(unsigned a, unsigned b) {
    u16x2 x = __builtin_bit_cast(u16x2, a), y = __builtin_bit_cast(u16x2, b);
    return __builtin_bit_cast(unsigned, (u16x2)(x - y));
}
__device__ __forceinline__ unsigned bfi(unsigned mask, unsigned a, unsigned b) { return (a & mask) | (b & ~mask); }
__device__ __forceinline__ unsigned choose_window(unsigned esum, unsigned n, unsigned emax) {
    if (!n) return 21u;
    const unsigned num = esum * 32u + n * 63u, den = n * 32u;
    unsigned t = num / den; const unsigned alt = (num - t * den) * 5u > den * 2u ? 1u : 0u;
    t = t > emax ? emax : t; t = t < 21u ? 21u : (t > 253u ? 253u : t);
    return t | (alt << 9);
}
template <int NQ> __device__ __forceinline__ unsigned apply_escapes(unsigned (&e)[NQ], const unsigned (&m)[NQ], LAS unsigned char* smem, unsigned off) {
    unsigned sel[NQ], o[NQ]; u32x2 w[NQ];
#pragma unroll
    for (int q = 0; q < NQ; ++q) {
        sel[q] = *(const LAS unsigned*)(smem + SM_LUTE + __builtin_amdgcn_udot4(m[q], 0x08040201u, 0u, false));
        o[q] = off; off += (unsigned)__builtin_popcount(m[q]);
        const LAS unsigned* pw = (const LAS unsigned*)(smem + (o[q] & ~3u));
        w[q] = (u32x2){pw[0], pw[1]};
    }
#pragma unroll
    for (int q = 0; q < NQ; ++q) e[q] = __builtin_amdgcn_perm(__builtin_amdgcn_alignbyte(w[q].y, w[q].x, o[q]), e[q], sel[q]);
    return off;
}
__device__ __forceinline__ unsigned append_flagged(LAS unsigned char* smem, unsigned e, unsigned m, unsigned off, unsigned dummy) {
#pragma unroll
    for (int k = 0; k < 4; ++k) {
        const unsigned bit = (m >> (8 * k + 2)) & 1u;
        smem[bit ? off : dummy] = (unsigned char)(e >> (8 * k));
        off += bit;
    }
    return off;
}
__device__ __forceinline__ void decode_levels(LAS unsigned char* smem, unsigned h0, unsigned h1, unsigned o2, unsigned l3, unsigned l2, int lane) {
    const unsigned t = h0 & 0xffu, alt = (h0 >> 9) & 1u, n2 = h0 >> 16, n3 = h1 & 0xffffu;
    const unsigned o3 = o2 + ((n2 + 15u) >> 4) * 4u, o4 = o3 + ((n3 + 7u) >> 3) * 4u;
    const unsigned b3q = (t - 6u + alt) * 0x01010101u, spq = (t + 1u + alt) * 0x01010101u;
    unsigned eo = o4;
    for (unsigned r = 0; r * 256u < n3; ++r) {
        const unsigned g = r * 64u + lane;
        unsigned c = *(const LAS u16*)(smem + o3 + g * 2u); c = g * 4u < n3 ? c : 0u;
        const unsigned z = __builtin_amdgcn_perm(0u, c, 0x01010000u);
        const unsigned d = (z & 0x000f000fu) | ((z >> 4) & 0x0f000f00u);
        const unsigned is15 = (d + 0x01010101u) & 0x10101010u, is14 = ((d + 0x02020202u) & 0x10101010u) ^ is15;
        unsigned e[1], m[1];
        e[0] = bfi((is14 << 4) - (is14 >> 4), spq, b3q - d); m[0] = is15 >> 2;
        const unsigned ec = (unsigned)__builtin_popcount(m[0]), inc = wave_scan(ec);
        apply_escapes<1>(e, m, smem, eo + inc - ec);
        eo += __builtin_amdgcn_readlane(inc, 63);
        *(LAS unsigned*)(smem + l3 + g * 4u) = e[0];
    }
    const unsigned tab = (t - 3u) | ((t - 4u) << 8) | ((alt ? t + 1u : t - 5u) << 16);
    eo = l3;
    for (unsigned r = 0; r * 1024u < n2; ++r) {
        const unsigned g = r * 64u + lane;
        unsigned c = *(const LAS unsigned*)(smem + o2 + g * 4u); c = g * 16u < n2 ? c : 0u;
        unsigned e[4], m[4];
#pragma unroll
        for (int k = 0; k < 4; ++k) { const unsigned d = (c >> (2 * k)) & 0x03030303u; e[k] = __builtin_amdgcn_perm(0u, tab, d); m[k] = (d + 0x01010101u) & 0x04040404u; }
        const unsigned ec = (unsigned)__builtin_popcount(c & (c >> 1) & 0x55555555u), inc = wave_scan(ec);
        apply_escapes<4>(e, m, smem, eo + inc - ec);
        eo += __builtin_amdgcn_readlane(inc, 63);
        if (g * 16u < n2) *(LAS u32x4*)(smem + l2 + g * 16u) = (u32x4){e[0], e[1], e[2], e[3]};
    }
}
__device__ __forceinline__ unsigned encode_levels(LAS unsigned char* smem, rsrc_t wsr, unsigned win, unsigned n2, unsigned l2, unsigned l3, unsigned l4, unsigned g2, unsigned room, unsigned dummy, int lane) {
    const unsigned t = win & 0xffu, alt = (win >> 9) & 1u;
    if (lane < 16) smem[l2 + n2 + lane] = (unsigned char)(t - 3u);
    const unsigned t3 = (t - 3u) * 0x00010001u, sc = (alt ? t + 1u : t - 5u) * 0x00010001u;
    unsigned n3 = 0;
    for (unsigned r = 0; r * 1024u < n2; ++r) {
        const unsigned g = r * 64u + lane; const bool act = g * 16u < n2;
        const u32x4 e = *(const LAS u32x4*)(smem + l2 + (act ? g : 0u) * 16u);
        unsigned c = 0, m[4], ec = 0;
#pragma unroll
        for (int k = 0; k < 4; ++k) {
            unsigned dd[2];
#pragma unroll
            for (int h = 0; h < 2; ++h) {
                const unsigned x = (e[k] >> (8 * h)) & 0x00ff00ffu;
                const unsigned a = pkmin(pksub(t3, x), 0x00020002u);
                dd[h] = a + ((a >> 1) & pkmin(x ^ sc, 0x00010001u));
            }
            const unsigned d = dd[0] | (dd[1] << 8);
            c |= d << (2 * k); m[k] = act ? (d + 0x01010101u) & 0x04040404u : 0u; ec += (unsigned)__builtin_popcount(m[k]);
        }
        const unsigned inc = wave_scan(ec), tot = __builtin_amdgcn_readlane(inc, 63);
        if (n3 + tot > CAP3) return 0xffffffffu;
        if (act) bst4<SYS>(wsr, g2 + g * 4u, c);
        unsigned eo = l3 + n3 + inc - ec;
#pragma unroll
        for (int k = 0; k < 4; ++k) eo = append_flagged(smem, e[k], m[k], eo, dummy);
        n3 += tot;
    }
    const unsigned b3 = t - 6u + alt;
    if (lane < 8) smem[l3 + n3 + lane] = (unsigned char)b3;
    const unsigned g3 = g2 + ((n2 + 15u) >> 4) * 4u, g4 = g3 + ((n3 + 7u) >> 3) * 4u;
    if (g4 - g2 > room) return 0xffffffffu;
    const unsigned b32 = b3 * 0x00010001u, sp = (t + 1u + alt) * 0x00010001u;
    unsigned n4 = 0;
    for (unsigned r = 0; r * 256u < n3; ++r) {
        const unsigned g = r * 64u + lane; const bool act = g * 4u < ((n3 + 7u) & ~7u);
        const unsigned e = *(const LAS unsigned*)(smem + l3 + (act ? g : 0u) * 4u);
        unsigned dd[2];
#pragma unroll
        for (int h = 0; h < 2; ++h) {
            const unsigned x = (e >> (8 * h)) & 0x00ff00ffu;
            const unsigned a = pkmin(pksub(b32, x), 0x000e000eu);
            dd[h] = a + (((a + 0x00020002u) >> 4) & pkmin(x ^ sp, 0x00010001u));
        }
        const unsigned d = dd[0] | (dd[1] << 8);
        const unsigned m = act ? ((d + 0x01010101u) & 0x10101010u) >> 2 : 0u;
        const unsigned ec = (unsigned)__builtin_popcount(m), inc = wave_scan(ec), tot = __builtin_amdgcn_readlane(inc, 63);
        if (n4 + tot > CAP4) return 0xffffffffu;
        const unsigned y = d | (d >> 4);
        if (act) bst2<SYS>(wsr, g3 + g * 2u, __builtin_amdgcn_perm(0u, y, 0x0c0c0200u));
        if (tot) append_flagged(smem, e, m, l4 + n4 + inc - ec, dummy);
        n4 += tot;
    }
    if (g4 - g2 + n4 > room) return 0xffffffffu;
    if (n4) { const unsigned lv = *(const LAS unsigned*)(smem + l4 + lane * 4); if ((unsigned)lane * 4u < n4) bst4<SYS>(wsr, g4 + lane * 4, lv); }
    return n3 | (n4 << 16);
}
__device__ __forceinline__ unsigned side_bytes(unsigned n2, unsigned h1) { return ((n2 + 15u) >> 4) * 4u + (((h1 & 0xffffu) + 7u) >> 3) * 4u + (h1 >> 16); }
struct Order {
    int ph, start;
    template <int NWU, int NXU> __device__ __forceinline__ void locate(int i, int& j, int& g, int& u) {
        for (;;) {
            int c = (ph < 16) ? NWU : 0;
#pragma unroll
            for (int k = 0; k < 4; ++k) c += ((unsigned)(ph - lag(k)) < 16u) ? NXU : 0;
            if (i < start + c) break;
            start += c; ++ph;
        }
        u = i - start; g = 0; j = ph;
        if (ph < 16) { if (u < NWU) return; u -= NWU; }
#pragma unroll
        for (int k = 0; k < 4; ++k) {
            if ((unsigned)(ph - lag(k)) < 16u) { if (u < NXU) { g = 1 + k; j = ph - lag(k); return; } u -= NXU; }
        }
    }
};

__device__ __forceinline__ bool give_up(gu32* tmo, unsigned seq, u64 t0) {
    bool dead = rfl(__hip_atomic_load(tmo, __ATOMIC_RELAXED, __HIP_MEMORY_SCOPE_AGENT)) == seq;
    if (!dead && __builtin_amdgcn_s_memrealtime() - t0 > DEADLINE) {
        __hip_atomic_store(tmo, seq, __ATOMIC_RELAXED, __HIP_MEMORY_SCOPE_AGENT);
        dead = true;
    }
    return dead;
}
__device__ __forceinline__ void pack_unit(const Params& p, char* ws, rsrc_t wsr, LAS unsigned char* smem, unsigned lwo, int j, int g, int u, int lane, unsigned seqv) {
    const bool isx = g != 0; const int ub = isx ? (u >> 4) * 64 + (g - 1) * 16 + (u & 15) : u;
    const int ra = p.rank >> 2, rb = p.rank & 3;
    LAS unsigned char* lw = smem + lwo;
    u32x4 v[8];
    if (isx) {
        const u16* src = p.x + (size_t)(ub * 64 + (lane & 15)) * KS + j * 64 + (lane >> 4) * 8;
#pragma unroll
        for (int c = 0; c < 8; ++c) v[c] = *(const u32x4*)(src + (size_t)(c >> 1) * 16 * KS + (c & 1) * 32);
    } else {
        constexpr int P = 144;
        const u16* src = p.w + (size_t)(j * 64 + (lane >> 3)) * ND + ub * 64 + (lane & 7) * 8;
        u32x4 tmp[8];
#pragma unroll
        for (int i = 0; i < 8; ++i) tmp[i] = *(const u32x4*)(src + (size_t)i * 8 * ND);
#pragma unroll
        for (int i = 0; i < 8; ++i) *(LAS u32x4*)(lw + (i * 8 + (lane >> 3)) * P + (lane & 7) * 16) = tmp[i];
        const int q = (lane >> 2) & 3, pp = lane & 3;
#pragma unroll
        for (int c = 0; c < 8; ++c) {
            const int k0 = (c & 1) * 32 + (lane >> 4) * 8, rbk = c >> 1;
            const int col = (rbk >> 1) * 32 + 8 * pp + 4 * (rbk & 1);
            const s16x4 a = __builtin_amdgcn_ds_read_tr16_b64_v4i16((LAS s16x4*)(lw + (k0 + q) * P + col * 2));
            const s16x4 b = __builtin_amdgcn_ds_read_tr16_b64_v4i16((LAS s16x4*)(lw + (k0 + 4 + q) * P + col * 2));
            const u32x2 a2 = __builtin_bit_cast(u32x2, a), b2 = __builtin_bit_cast(u32x2, b);
            v[c] = (u32x4){a2.x, a2.y, b2.x, b2.y};
        }
    }
    const unsigned loff = (lane & 15) * 64 + (((lane >> 4) * 16) ^ (((lane >> 3) & 1) << 5));
    const int cls = isx ? (ub >> 6) : (ub >> 5);
    const bool own = isx ? (cls == ra) : (cls == rb);
    const int li = isx ? (ub & 63) : (ub & 31);
    if (own) {
        const unsigned dst = (isx ? OFF_XL + (unsigned)(j * 8 * 64 + li) * 8192u : OFF_WL + (unsigned)(j * 8 * 32 + li) * 8192u) + loff;
#pragma unroll
        for (int c = 0; c < 8; ++c) bst16<SC1>(wsr, dst + c * 1024, v[c]);
    }
    unsigned e4[16], sm[16], mx = 0, esum = 0;
#pragma unroll
    for (int q = 0; q < 16; ++q) {
        const unsigned a = v[q >> 1][(q & 1) * 2], b = v[q >> 1][(q & 1) * 2 + 1];
        const unsigned lo = __builtin_amdgcn_perm(b, a, 0x06040200u), hi = __builtin_amdgcn_perm(b, a, 0x07050301u);
        e4[q] = bfi(0x01010101u, lo >> 7, hi << 1);
        sm[q] = bfi(0x7f7f7f7fu, lo, hi);
        esum = __builtin_amdgcn_udot4(e4[q], 0x01010101u, esum, false);
        mx = pkmax(mx, a & 0x7fff7fffu); mx = pkmax(mx, b & 0x7fff7fffu);
    }
    mx = (mx & 0xffffu) > (mx >> 16) ? (mx & 0xffffu) : (mx >> 16);
    const unsigned win = choose_window(wave_sum(esum), 4096u, wave_max(mx) >> 7);
    const unsigned top2 = (win & 0xffu) * 0x00010001u;
    unsigned p2[4] = {0u, 0u, 0u, 0u}, m[16], ec = 0;
#pragma unroll
    for (int q = 0; q < 16; ++q) {
        const unsigned da = pkmin(pksub(top2, e4[q] & 0x00ff00ffu), 0x00030003u), db = pkmin(pksub(top2, (e4[q] >> 8) & 0x00ff00ffu), 0x00030003u);
        const unsigned d = da | (db << 8);
        p2[q >> 2] |= d << (2 * (q & 3));
        m[q] = (d + 0x01010101u) & 0x04040404u; ec += (unsigned)__builtin_popcount(m[q]);
    }
    const unsigned einc = wave_scan(ec), n2 = __builtin_amdgcn_readlane(einc, 63);
    const unsigned slot0 = (isx ? OFF_EX : OFF_EW) + (unsigned)(j * 128 + ub) * USLOT;
    const unsigned pay = slot0 + 1024 + lane * 16;
    unsigned h1 = 0xffffffffu;
    if (!FORCE_RAW_IN && n2 <= CAP2) {
        const unsigned l2 = lwo + 10240, l3 = l2 + CAP2 + 16, l4 = l3 + CAP3 + 16, dummy = SM_DUMMY + lane * 2;
        unsigned eo = l2 + einc - ec;
#pragma unroll
        for (int q = 0; q < 16; ++q) eo = append_flagged(smem, e4[q], m[q], eo, dummy);
        h1 = encode_levels(smem, wsr, win, n2, l2, l3, l4, slot0 + 8, 1016u, dummy, lane);
    }
    if (h1 == 0xffffffffu) {
#pragma unroll
        for (int c = 0; c < 8; ++c) bst16<SYS>(wsr, pay + c * 1024, v[c]);
        if (lane == 0) bst4<SYS>(wsr, slot0, 1u << 8);
    } else {
#pragma unroll
        for (int i = 0; i < 4; ++i) bst16<SYS>(wsr, pay + i * 1024, (u32x4){sm[4 * i], sm[4 * i + 1], sm[4 * i + 2], sm[4 * i + 3]});
        bst16<SYS>(wsr, pay + 4096, (u32x4){p2[0], p2[1], p2[2], p2[3]});
        if (lane == 0) bst8<SYS>(wsr, slot0, (u32x2){win | (n2 << 16), h1});
    }
    WAIT_V(0);
    const unsigned seq = rfl(seqv) + 1u;
    if (own && lane == 0) {
        const unsigned f = (isx ? OFF_XF : OFF_WF) + (unsigned)(((li >> 2) * NSTEP + j * 8) * 4 + (li & 3)) * 4u;
        __hip_atomic_store((gu32*)(ws + f), seq, __ATOMIC_RELAXED, __HIP_MEMORY_SCOPE_AGENT);
    }
    unsigned ticket = 0;
    if (lane == 0) ticket = __hip_atomic_fetch_add((gu32*)(ws + OFF_ECNT + (unsigned)(j * 12 + (isx ? cls * 4 + g - 1 : 8 + cls)) * 128u), 1u, __ATOMIC_RELAXED, __HIP_MEMORY_SCOPE_AGENT);
    ticket = rfl(ticket);
    const unsigned per = isx ? 16u : 32u;
    if ((ticket & (per - 1u)) == per - 1u) {
        const bool want = lane < NR && lane != p.rank && (isx ? ((lane >> 2) == cls) : ((lane & 3) == cls));
        if (want) {
            char* pw = (char*)p.ws_all[lane];
            __hip_atomic_store((gu32*)(pw + OFF_FIN + (unsigned)p.rank * 512u + (unsigned)j * 32u + (unsigned)g * 4u), seq, __ATOMIC_RELAXED, __HIP_MEMORY_SCOPE_SYSTEM);
        }
    }
}

struct UnitRegs { u32x4 a[5]; u32x4 sd; };
__device__ __forceinline__ unsigned unit_slot(int rank, int j, int g, int u) {
    const int ub = g ? (rank >> 2) * 64 + (g - 1) * 16 + u : (rank & 3) * 32 + u;
    return (g ? OFF_EX : OFF_EW) + (unsigned)(j * 128 + ub) * USLOT;
}
__device__ __forceinline__ void unit_issue(UnitRegs& r, u32x4 pd, unsigned slot0, int lane) {
    const unsigned oh = lane < 24 ? slot0 + lane * 16 : OOB, o0 = slot0 + 1024 + lane * 16, o1 = o0 + 4096;
    ALD16(r.sd, pd, oh, 0);
    ALD16(r.a[0], pd, o0, 0); ALD16(r.a[1], pd, o0, 1024); ALD16(r.a[2], pd, o0, 2048); ALD16(r.a[3], pd, o0, 3072);
    ALD16(r.a[4], pd, o1, 0);
}
#define UNIT_LANDED(r, n) asm volatile("s_waitcnt vmcnt(0)" : "+v"(r.a[0]), "+v"(r.a[1]), "+v"(r.a[2]), "+v"(r.a[3]), "+v"(r.a[4]), "+v"(r.sd), "+v"(n) :: "memory")
__device__ __forceinline__ void unit_unpack(const UnitRegs& r, rsrc_t wsr, rsrc_t pr, LAS unsigned char* smem, unsigned scr, unsigned slot0, unsigned dst0, int lane) {
    const unsigned dst = dst0 + (lane & 15) * 64 + (((lane >> 4) * 16) ^ (((lane >> 3) & 1) << 5));
    const unsigned h0 = rfl(r.sd.x), h1 = rfl(r.sd.y);
    if ((h0 >> 8) & 1u) {
        u32x4 b[3];
#pragma unroll
        for (int c = 0; c < 3; ++c) b[c] = bld16<SYS>(pr, slot0 + 1024 + lane * 16 + (5 + c) * 1024);
#pragma unroll
        for (int c = 0; c < 5; ++c) bst16<SC1>(wsr, dst + c * 1024, r.a[c]);
#pragma unroll
        for (int c = 0; c < 3; ++c) bst16<SC1>(wsr, dst + (5 + c) * 1024, b[c]);
        return;
    }
    u32x4 sd = r.sd;
    const unsigned len = 8u + side_bytes(h0 >> 16, h1);
    if (len > 384u) { if (lane >= 24 && (unsigned)lane * 16u < len) sd = bld16<SYS>(pr, slot0 + lane * 16); }
    *(LAS u32x4*)(smem + scr + lane * 16) = sd;
    const unsigned l3 = scr + 2048, l2 = l3 + CAP3 + 256;
    decode_levels(smem, h0, h1, scr + 8u, l3, l2, lane);
    const unsigned tq = (h0 & 0xffu) * 0x01010101u;
    unsigned e[16], m[16], ec = 0;
#pragma unroll
    for (int q = 0; q < 16; ++q) {
        const unsigned d = (r.a[4][q >> 2] >> (2 * (q & 3))) & 0x03030303u;
        m[q] = (d + 0x01010101u) & 0x04040404u; e[q] = tq - d;
    }
#pragma unroll
    for (int k = 0; k < 4; ++k) ec += (unsigned)__builtin_popcount(r.a[4][k] & (r.a[4][k] >> 1) & 0x55555555u);
    apply_escapes<16>(e, m, smem, l2 + wave_scan(ec) - ec);
#pragma unroll
    for (int c = 0; c < 8; ++c) {
        u32x4 o;
#pragma unroll
        for (int h = 0; h < 2; ++h) {
            const int q = 2 * c + h; const unsigned sm = r.a[q >> 2][q & 3];
            const unsigned lo = bfi(0x7f7f7f7fu, sm, e[q] << 7), hi = bfi(0x7f7f7f7fu, e[q] >> 1, sm);
            o[2 * h] = __builtin_amdgcn_perm(hi, lo, 0x05010400u); o[2 * h + 1] = __builtin_amdgcn_perm(hi, lo, 0x07030602u);
        }
        bst16<SC1>(wsr, dst + c * 1024, o);
    }
}

__device__ __forceinline__ unsigned strip_out_off(int r, int s, int lane) {
    const int tile = s >> 3, tm = tile >> 3, tn = tile & 7;
    const int row = (r >> 2) * RROWS + tm * 256 + (s & 7) * 32 + (lane >> 5);
    const int col = (r & 3) * RCOLS + tn * 256 + (lane & 31) * 8;
    return ((unsigned)row * ND + col) * 2u;
}

struct StripRegs { u32x4 mk; u32x4 lo[8]; u32x4 sd[2]; };
__device__ __forceinline__ void strip_issue(StripRegs& r, u32x4 pd, unsigned s, unsigned info, int lane) {
    if (info >> 30) return;
    const unsigned T = info & 0x3fffu, sl = (info >> 14) & 0xffu, slot = OFF_EO + s * SSLOT;
    const unsigned om = slot + lane * 16;
    ALD16(r.mk, pd, om, 0);
    { const unsigned o = (unsigned)lane < sl ? om : OOB; ALD16(r.sd[0], pd, o, 1024); }
    if (sl > 64u) { const unsigned o = (unsigned)lane + 64u < sl ? om : OOB; ALD16(r.sd[1], pd, o, 2048); }
#define STRIP_ROUND(k) if ((unsigned)(k) * 1024u < T) { const bool act = (unsigned)((k) * 1024 + lane * 16) < T; \
        const unsigned ol = act ? om + 3072 + ((k) >> 2) * 4096 : OOB; ALD16(r.lo[k], pd, ol, ((k) & 3) * 1024); }
    STRIP_ROUND(0) STRIP_ROUND(1) STRIP_ROUND(2) STRIP_ROUND(3) STRIP_ROUND(4) STRIP_ROUND(5) STRIP_ROUND(6) STRIP_ROUND(7)
}
#define STRIP_LANDED(r, fn, n, cnt) do { \
    asm volatile("s_waitcnt vmcnt(" #cnt ")" : "+v"(r.lo[0]), "+v"(r.lo[1]), "+v"(r.lo[2]), "+v"(r.lo[3]), "+v"(r.lo[4]), "+v"(r.lo[5]), "+v"(r.lo[6]), "+v"(r.lo[7]), \
        "+v"(r.mk), "+v"(r.sd[0]), "+v"(r.sd[1]), "+v"(fn), "+v"(n) :: "memory"); } while (0)
__device__ __forceinline__ void strip_stage1(const StripRegs& r, LAS unsigned char* smem, unsigned lbase, unsigned scr, unsigned info, int lane) {
    if (info >> 30) return;
    const unsigned T = info & 0x3fffu;
    const unsigned h0 = rfl(r.sd[0].x), h1 = rfl(r.sd[0].y);
    *(LAS u32x4*)(smem + scr + lane * 16) = r.sd[0];
    *(LAS u32x4*)(smem + scr + 1024 + lane * 16) = r.sd[1];
    const unsigned l3 = scr + 2048, l2 = l3 + CAP3 + 256;
    decode_levels(smem, h0, h1, scr + 8u + ((T + 31u) >> 5) * 4u, l3, l2, lane);
    const unsigned tq = (h0 & 0xffu) * 0x01010101u;
    unsigned x1[8], ec = 0;
#pragma unroll
    for (int k = 0; k < 8; ++k) {
        if ((unsigned)k * 1024u < T) {
            const unsigned z = __builtin_amdgcn_perm(0u, (unsigned)*(const LAS u16*)(smem + scr + 8 + k * 128 + lane * 2), 0x01010000u);
            x1[k] = (z & 0x000f000fu) | ((z >> 4) & 0x0f000f00u);
#pragma unroll
            for (int i = 0; i < 4; ++i) ec += (unsigned)__builtin_popcount((x1[k] >> i) & (r.lo[k][i] >> 7) & 0x01010101u);
        }
    }
    unsigned eo = l2 + wave_scan(ec) - ec;
    unsigned e[8][4];
#pragma unroll
    for (int k = 0; k < 8; ++k) {
        if ((unsigned)k * 1024u < T) {
            unsigned m[4];
#pragma unroll
            for (int i = 0; i < 4; ++i) {
                const unsigned d = ((x1[k] >> i) & 0x01010101u) | ((r.lo[k][i] >> 6) & 0x02020202u);
                m[i] = (d + 0x01010101u) & 0x04040404u; e[k][i] = tq - d;
            }
            eo = apply_escapes<4>(e[k], m, smem, eo);
        }
    }
#pragma unroll
    for (int k = 0; k < 8; ++k) {
        if ((unsigned)k * 1024u < T) {
            unsigned lo[4], hi[4];
#pragma unroll
            for (int i = 0; i < 4; ++i) { lo[i] = bfi(0x7f7f7f7fu, r.lo[k][i], e[k][i] << 7); hi[i] = (e[k][i] >> 1) & 0x7f7f7f7fu; }
            *(LAS u32x4*)(smem + lbase + k * 2048 + lane * 32) = (u32x4){lo[0], hi[0], lo[1], hi[1]};
            *(LAS u32x4*)(smem + lbase + k * 2048 + lane * 32 + 16) = (u32x4){lo[2], hi[2], lo[3], hi[3]};
        }
    }
}
__device__ __forceinline__ void strip_stage2(u32x4 mk, rsrc_t outr, rsrc_t por, LAS unsigned char* smem, unsigned lbase, int peer, unsigned s, unsigned info, int lane) {
    const unsigned oo = strip_out_off(peer, (int)s, lane);
    if (info >> 31) {
#pragma unroll
        for (int q = 0; q < 16; ++q) bst16<0>(outr, oo + q * 2 * ND * 2, (u32x4){0x7fc07fc0u, 0x7fc07fc0u, 0x7fc07fc0u, 0x7fc07fc0u});
        return;
    }
    if (info >> 30) {
#pragma unroll
        for (int h = 0; h < 4; ++h) {
            u32x4 a[4];
#pragma unroll
            for (int q = 0; q < 4; ++q) a[q] = bld16<SYS>(por, oo + (h * 4 + q) * 2 * ND * 2);
#pragma unroll
            for (int q = 0; q < 4; ++q) bst16<0>(outr, oo + (h * 4 + q) * 2 * ND * 2, a[q]);
        }
        return;
    }
    unsigned c2[8], x[8];
#pragma unroll
    for (int i = 0; i < 8; ++i) {
        const unsigned m16 = mk[i >> 1] >> ((i & 1) * 16);
        c2[i] = (unsigned)__builtin_popcount(m16 & 0xffu) | ((unsigned)__builtin_popcount((m16 >> 8) & 0xffu) << 16);
        x[i] = c2[i];
    }
#define SCAN_STEP(ctrl, rmask) _Pragma("unroll") for (int i = 0; i < 8; ++i) x[i] += (unsigned)__builtin_amdgcn_update_dpp(0, (int)x[i], ctrl, rmask, 0xf, false);
    SCAN_STEP(0x111, 0xf) SCAN_STEP(0x112, 0xf) SCAN_STEP(0x114, 0xf) SCAN_STEP(0x118, 0xf) SCAN_STEP(0x142, 0xa) SCAN_STEP(0x143, 0xc)
    unsigned pos = 0;
#pragma unroll
    for (int bt = 0; bt < 4; ++bt) {
        u32x4 sel[4], d[8]; unsigned off[8];
#pragma unroll
        for (int k = 0; k < 4; ++k) {
            const int i = bt * 2 + (k >> 1), hq = k & 1;
            const unsigned m8 = (mk[bt] >> (k * 8)) & 0xffu;
            sel[k] = *(const LAS u32x4*)(smem + SM_LUT + m8 * 16u);
            const unsigned tot = __builtin_amdgcn_readlane(x[i], 63);
            off[2 * k] = pos + (((x[i] - c2[i]) >> (hq * 16)) & 0xffffu);
            off[2 * k + 1] = off[2 * k] + (unsigned)__builtin_popcount(m8 & 15u);
            pos += (tot >> (hq * 16)) & 0xffffu;
        }
#pragma unroll
        for (int g = 0; g < 8; ++g) {
            const LAS u32x2* pl = (const LAS u32x2*)(smem + lbase + (off[g] & ~3u) * 2u);
            const u32x2 d0 = pl[0], d1 = pl[1];
            d[g] = (u32x4){d0.x, d0.y, d1.x, d1.y};
        }
#pragma unroll
        for (int k = 0; k < 4; ++k) {
            u32x4 o;
#pragma unroll
            for (int g = 0; g < 2; ++g) {
                const u32x4 dd = d[2 * k + g];
                const unsigned l4 = __builtin_amdgcn_alignbyte(dd.z, dd.x, off[2 * k + g]), h4 = __builtin_amdgcn_alignbyte(dd.w, dd.y, off[2 * k + g]);
                o[2 * g] = __builtin_amdgcn_perm(h4, l4, sel[k][2 * g]); o[2 * g + 1] = __builtin_amdgcn_perm(h4, l4, sel[k][2 * g + 1]);
            }
            bst16<0>(outr, oo + (bt * 4 + k) * 2 * ND * 2, o);
        }
    }
}

__device__ __forceinline__ void pack_strip(const Params& p, rsrc_t wsr, rsrc_t outr, LAS unsigned char* smem, int s, int wid, int lane, unsigned seq) {
    const unsigned oo = strip_out_off(p.rank, s, lane);
    const unsigned lbase = (unsigned)wid * 16384u;
    unsigned pos = lbase, mx = 0, sg = 0, esum = 0;
    const unsigned dummy = SM_DUMMY + lane * 2;
    u32x4 mk = {0u, 0u, 0u, 0u};
#pragma unroll
    for (int q2 = 0; q2 < 8; ++q2) {
        u32x4 v[2]; unsigned m8[2];
#pragma unroll
        for (int hq = 0; hq < 2; ++hq) {
            const int q = q2 * 2 + hq, row = wid * 32 + 2 * q + (lane >> 5);
            v[hq] = *(const LAS u32x4*)(smem + row * 512 + (((lane & 31) ^ (row & 7)) * 16));
            bst16<SYS>(outr, oo + q * 2 * ND * 2, v[hq]);
            unsigned m = 0;
#pragma unroll
            for (int k = 0; k < 4; ++k) {
                mx = pkmax(mx, v[hq][k]); sg |= v[hq][k];
                esum = __builtin_amdgcn_udot4((v[hq][k] >> 7) & 0x00ff00ffu, 0x00010001u, esum, false);
                const unsigned t = pkmin(v[hq][k], 0x00010001u);
                m |= ((t | (t >> 15)) & 3u) << (2 * k);
            }
            m8[hq] = m;
        }
        mk[q2 >> 1] |= (m8[0] | (m8[1] << 8)) << ((q2 & 1) * 16);
        const unsigned c2 = (unsigned)__builtin_popcount(m8[0]) | ((unsigned)__builtin_popcount(m8[1]) << 16);
        const unsigned inc = wave_scan(c2), exl = inc - c2, tot = __builtin_amdgcn_readlane(inc, 63);
#pragma unroll
        for (int hq = 0; hq < 2; ++hq) {
            unsigned a = pos + ((exl >> (hq * 16)) & 0xffffu) * 2u;
#pragma unroll
            for (int e = 0; e < 8; ++e) {
                const unsigned bit = (m8[hq] >> e) & 1u;
                *(LAS u16*)(smem + (bit ? a : dummy)) = (u16)(v[hq][e >> 1] >> ((e & 1) * 16));
                a += bit * 2u;
            }
            pos += ((tot >> (hq * 16)) & 0xffffu) * 2u;
        }
    }
    const unsigned T = (pos - lbase) >> 1;
    mx = (mx & 0xffffu) > (mx >> 16) ? (mx & 0xffffu) : (mx >> 16);
    const unsigned win = choose_window(wave_sum(esum), T, (wave_max(mx) >> 7) & 0xffu);
    bool raw = FORCE_RAW_OUT || __builtin_amdgcn_ballot_w64((sg & 0x80008000u) != 0u) != 0ull;
    const unsigned slot = OFF_EO + (unsigned)s * SSLOT;
    unsigned sl = 0;
    if (!raw) {
        const unsigned top = win & 0xffu, top2 = top * 0x00010001u;
        if (lane < 16 && T + lane < 8192u) *(LAS u16*)(smem + lbase + (T + lane) * 2u) = (u16)(top << 7);
        bst16<SYS>(wsr, slot + lane * 16, mk);
        unsigned e4[8][4], m[8][4], ec = 0;
#pragma unroll
        for (int r = 0; r < 8; ++r) {
            if ((unsigned)r * 1024u < T) {
                const bool act = (unsigned)(r * 1024 + lane * 16) < T;
                const u32x4 a = *(const LAS u32x4*)(smem + lbase + r * 2048 + lane * 32), b = *(const LAS u32x4*)(smem + lbase + r * 2048 + lane * 32 + 16);
                const unsigned w8[8] = {a.x, a.y, a.z, a.w, b.x, b.y, b.z, b.w};
                u32x4 lo; unsigned x1 = 0;
#pragma unroll
                for (int k = 0; k < 4; ++k) {
                    const unsigned ea = (w8[2 * k] >> 7) & 0x00ff00ffu, eb = (w8[2 * k + 1] >> 7) & 0x00ff00ffu;
                    const unsigned d = __builtin_amdgcn_perm(pkmin(pksub(top2, eb), 0x00030003u), pkmin(pksub(top2, ea), 0x00030003u), 0x06040200u);
                    e4[r][k] = __builtin_amdgcn_perm(eb, ea, 0x06040200u);
                    lo[k] = bfi(0x7f7f7f7fu, __builtin_amdgcn_perm(w8[2 * k + 1], w8[2 * k], 0x06040200u), d << 6);
                    x1 |= (d & 0x01010101u) << k;
                    m[r][k] = act ? (d + 0x01010101u) & 0x04040404u : 0u; ec += (unsigned)__builtin_popcount(m[r][k]);
                }
                if (act) {
                    bst16<SYS>(wsr, slot + 3072 + r * 1024 + lane * 16, lo);
                    bst2<SYS>(wsr, slot + 1024 + 8 + r * 128 + lane * 2, __builtin_amdgcn_perm(0u, x1 | (x1 >> 4), 0x0c0c0200u));
                }
            }
        }
        const unsigned einc = wave_scan(ec), n2 = __builtin_amdgcn_readlane(einc, 63);
        unsigned h1 = 0xffffffffu;
        const unsigned p1b = ((T + 31u) >> 5) * 4u;
        if (n2 <= CAP2 - 16u) {
            const unsigned l2 = SM_SCR + (unsigned)wid * 3072u, l3 = l2 + CAP2, l4 = l3 + CAP3 + 8;
            unsigned eo = l2 + einc - ec;
#pragma unroll
            for (int r = 0; r < 8; ++r) {
                if ((unsigned)r * 1024u < T) {
#pragma unroll
                    for (int k = 0; k < 4; ++k) eo = append_flagged(smem, e4[r][k], m[r][k], eo, dummy);
                }
            }
            h1 = encode_levels(smem, wsr, win, n2, l2, l3, l4, slot + 1024 + 8 + p1b, 2040u - p1b, dummy, lane);
        }
        if (h1 == 0xffffffffu) raw = true;
        else {
            if (lane == 0) bst8<SYS>(wsr, slot + 1024, (u32x2){win | (n2 << 16), h1});
            sl = (8u + p1b + side_bytes(n2, h1) + 15u) >> 4;
        }
    }
    WAIT_V(0);
    if (lane < NR && lane != p.rank) {
        const unsigned info = raw ? (1u << 30) : (T | (sl << 14));
        char* pw = (char*)p.ws_all[lane];
        __hip_atomic_store((gu64*)(pw + OFF_FOUT + ((unsigned)p.rank * 1024u + (unsigned)s) * 8u), ((u64)seq << 32) | info, __ATOMIC_RELAXED, __HIP_MEMORY_SCOPE_SYSTEM);
    }
}

}

__global__ void __launch_bounds__(512, 2) gemm_ar_fused(Params p) {
    __shared__ __attribute__((aligned(16))) unsigned char smem_[SMEM_BYTES];
    LAS unsigned char* smem = (LAS unsigned char*)smem_;
    const int tid = threadIdx.x, wid = rfl(tid >> 6), lane = tid & 63;
    char* ws = (char*)p.ws_all[p.rank];
    const rsrc_t wsr = make_rsrc(ws, WS_USED);
    const rsrc_t outr = make_rsrc(p.out, OUT_BYTES);
    gu32* tmo = (gu32*)(ws + OFF_TMO);
    gu32* seqp = (gu32*)(ws + OFF_SEQ + (unsigned)(blockIdx.x * 8 + wid) * 4u);
    const unsigned seqv = __hip_atomic_load(seqp, __ATOMIC_RELAXED, __HIP_MEMORY_SCOPE_AGENT);
    STAMP(0);
    const int xcd = blockIdx.x & 7, wi = blockIdx.x >> 3;
    const int nsplit = xcd < 2 ? 16 : (xcd == 2 ? 8 : (xcd == 3 ? 4 : 0));
    const bool is_gemm = wi < 16 + nsplit;
    const int cw = xcd >= 4 ? (wi - 16) * 4 + xcd - 4 : (xcd == 3 ? 64 + wi - 20 : 76 + wi - 24);
    const bool is_link = !is_gemm && wid < NWV;
    if (!is_gemm) {
        if (tid == 0) { *(LAS unsigned*)(smem + SM_TOKEN) = 0u; *(LAS unsigned*)(smem + SM_TOKEN + 4) = 0xffffffffu; }
        WAIT_L(0); BAR;
    }
    if (!is_link) {
        Order od{0, 0};
        const int e = !is_gemm ? (wid - NWV) * 84 + cw
                    : xcd >= 4 ? NPD + (xcd >= 6 ? 0 : 256) + wid * 32 + wi * 2 + (xcd & 1)
                    : xcd < 2 ? NENC + 256 + wid * 64 + wi * 2 + xcd
                    : wi < 16 ? NENC + wid * 32 + wi * 2 + (xcd & 1) : NENC + 768 + wid * 12 + (xcd == 2 ? wi - 16 : wi - 8);
        if (e >= PDELAY) __builtin_amdgcn_s_sleep(PSLEEP);
        for (int t = e; t < 4096; t = (t == e ? NENC + NONCE + e : t + NENC)) {
            int j, g, u; od.locate<128, 32>(t, j, g, u);
            pack_unit(p, ws, wsr, smem, (unsigned)wid * 16384u, j, g, u, lane, seqv);
            if (e >= NENC) break;
        }
        STAMP(7);
        if (is_gemm) __syncthreads();
    }
    const unsigned seq = rfl(seqv) + 1u;
    if (lane == 0) __hip_atomic_store(seqp, seq, __ATOMIC_RELAXED, __HIP_MEMORY_SCOPE_AGENT);
    if (!is_gemm) {
        if (!is_link) return;
        const int lq = cw % 7;
        const int peer = (p.rank + 1 + lq) & 7;
        const rsrc_t pr = make_rsrc(p.ws_all[peer], WS_USED);
        const rsrc_t por = make_rsrc(p.out_all[peer], OUT_BYTES);
#pragma unroll
        for (int i = 0; i < 4; ++i) {
            const unsigned m8 = (unsigned)(i * 64 + lane); u32x4 sel;
#pragma unroll
            for (int g = 0; g < 2; ++g) {
                unsigned rk = 0;
#pragma unroll
                for (int h = 0; h < 2; ++h) {
                    unsigned w = 0;
#pragma unroll
                    for (int e = 0; e < 2; ++e) { const unsigned bit = (m8 >> (g * 4 + h * 2 + e)) & 1u; w |= (bit ? (rk | ((rk + 4u) << 8)) : 0x0c0cu) << (16 * e); rk += bit; }
                    sel[g * 2 + h] = w;
                }
            }
            *(LAS u32x4*)(smem + SM_LUT + m8 * 16u) = sel;
        }
        if (lane < 16) {
            unsigned rk = 4u, sel = 0;
#pragma unroll
            for (int k = 0; k < 4; ++k) { const unsigned bit = ((unsigned)lane >> k) & 1u; sel |= (bit ? rk : (unsigned)k) << (8 * k); rk += bit; }
            *(LAS unsigned*)(smem + SM_LUTE + lane * 4) = sel;
        }
        const unsigned esc = (unsigned)wid * 16384u + 8192u;
        bool dead = false;
        const u32x4 pd = make_desc(p.ws_all[peer], WS_USED);
        const unsigned one = 1u;
        LAS unsigned* token = (LAS unsigned*)(smem + SM_TOKEN);
#define QNEXT(dst) do { if (lane == 0) asm volatile("global_atomic_add %0, %1, %2, off sc0" : "+v"(dst) : "v"(qp), "v"(one) : "memory"); } while (0)
#define TOKEN_TAKE() do { for (;;) { unsigned old = 1u; \
            if (lane == 0) old = __hip_atomic_exchange(token, 1u, __ATOMIC_RELAXED, __HIP_MEMORY_SCOPE_WORKGROUP); \
            if (rfl(old) == 0u) break; __builtin_amdgcn_s_sleep(1); } } while (0)
#define TOKEN_GIVE() do { if (lane == 0) __hip_atomic_store(token, 0u, __ATOMIC_RELAXED, __HIP_MEMORY_SCOPE_WORKGROUP); } while (0)
        {
            gu32* qp = (gu32*)(ws + OFF_Q + (unsigned)lq * 128u);
            const unsigned qbase = (seq - 1u) * (1536u + NT);
            LAS unsigned* utok = token + 1;
            gu64* finp = (gu64*)(ws + OFF_FIN + (unsigned)peer * 512u + (unsigned)lane * 8u);
            Order od{0, 0};
            u64 rdy0 = 0, rdy1 = 0, fv = 0;
            unsigned nq = 0;
            UnitRegs ur = {};
            if (wid == 0) {
                QNEXT(nq);
                UNIT_LANDED(ur, nq);
                if (lane == 0) __hip_atomic_store(utok, nq - qbase, __ATOMIC_RELAXED, __HIP_MEMORY_SCOPE_WORKGROUP);
            }
            unsigned pfo = 0;
#define PUBLISH(n) do { if (pfo) { WAIT_V(n); if (lane == 0) __hip_atomic_store((gu32*)(ws + pfo), seq, __ATOMIC_RELAXED, __HIP_MEMORY_SCOPE_AGENT); pfo = 0; } } while (0)
            for (;;) {
                unsigned i;
                for (;;) {
                    unsigned old = 0xffffffffu;
                    if (lane == 0) old = __hip_atomic_exchange(utok, 0xffffffffu, __ATOMIC_RELAXED, __HIP_MEMORY_SCOPE_WORKGROUP);
                    i = rfl(old);
                    if (i != 0xffffffffu) break;
                    if (pfo) PUBLISH(0); else __builtin_amdgcn_s_sleep(1);
                }
                if (i >= 1536u) { if (lane == 0) __hip_atomic_store(utok, i, __ATOMIC_RELAXED, __HIP_MEMORY_SCOPE_WORKGROUP); break; }
                QNEXT(nq);
                int j, g, u; od.locate<32, 16>((int)i, j, g, u);
                const int rb = j * 4 + (g >> 1);
                if (!((((g & 1) ? rdy1 : rdy0) >> rb) & 1ull) && !dead) {
                    PUBLISH(0);
                    const bool was0 = (rdy0 | rdy1) == 0ull;
                    const u64 t0 = __builtin_amdgcn_s_memrealtime(); unsigned n = 0;
                    for (;;) {
                        const u64 f2 = __hip_atomic_load(finp, __ATOMIC_RELAXED, __HIP_MEMORY_SCOPE_SYSTEM);
                        rdy0 = __builtin_amdgcn_ballot_w64((unsigned)f2 >= seq); rdy1 = __builtin_amdgcn_ballot_w64((unsigned)(f2 >> 32) >= seq);
                        if ((((g & 1) ? rdy1 : rdy0) >> rb) & 1ull) break;
                        if ((++n & 63u) == 0u && give_up(tmo, seq, t0)) { dead = true; break; }
                        __builtin_amdgcn_s_sleep(2);
                    }
                    __builtin_amdgcn_fence(__ATOMIC_ACQUIRE, "wavefront");
                    if (was0) STAMP(1);
                }
                const int step = j * 8 + 1 + lq, li = g ? (g - 1) * 16 + u : u;
                const unsigned slot0 = unit_slot(p.rank, j, g, u);
                const unsigned dst0 = g ? OFF_XL + (unsigned)(step * 64 + li) * 8192u : OFF_WL + (unsigned)(step * 32 + li) * 8192u;
                const bool refresh = rdy0 != 0x7777777777777777ull || rdy1 != 0x3333333333333333ull;
                for (;;) {
                    unsigned old = 1u;
                    if (lane == 0) old = __hip_atomic_exchange(token, 1u, __ATOMIC_RELAXED, __HIP_MEMORY_SCOPE_WORKGROUP);
                    if (rfl(old) == 0u) break;
                    if (pfo) PUBLISH(0); else __builtin_amdgcn_s_sleep(1);
                }
                unit_issue(ur, pd, slot0, lane);
                asm volatile("s_waitcnt vmcnt(6)" : "+v"(nq) :: "memory");
                if (lane == 0) __hip_atomic_store(utok, nq - qbase, __ATOMIC_RELAXED, __HIP_MEMORY_SCOPE_WORKGROUP);
                UNIT_LANDED(ur, nq);
                TOKEN_GIVE();
                PUBLISH(0);
                if (refresh) asm volatile("global_load_dwordx2 %0, %1, off sc0 sc1" : "+v"(fv) : "v"(finp) : "memory");
                pfo = (g ? OFF_XF : OFF_WF) + (unsigned)(((li >> 2) * NSTEP + step) * 4 + (li & 3)) * 4u;
                unit_unpack(ur, wsr, pr, smem, esc, slot0, dst0, lane);
                if (refresh) {
                    asm volatile("s_waitcnt vmcnt(8)" : "+v"(fv) :: "memory");
                    rdy0 = __builtin_amdgcn_ballot_w64((unsigned)fv >= seq); rdy1 = __builtin_amdgcn_ballot_w64((unsigned)(fv >> 32) >= seq);
                }
            }
            PUBLISH(0);
            STAMP(3);
        }
        {
            gu32* qp = (gu32*)(ws + OFF_Q + (unsigned)(7 + lq) * 128u);
            const unsigned qbase = (seq - 1u) * (1024u + NT * NWV);
            gu64* fbase = (gu64*)(ws + OFF_FOUT + (unsigned)peer * 8192u);
            const unsigned lbase = (unsigned)wid * 16384u;
            unsigned nq = 0; u64 fn = 0;
            StripRegs sr = {};
            QNEXT(nq);
            asm volatile("s_waitcnt vmcnt(0)" : "+v"(nq) :: "memory");
            unsigned s = rfl(nq) - qbase;
            bool first = true;
            while (s < 1024u) {
                unsigned fh = rfl((unsigned)(fn >> 32)), info = rfl((unsigned)fn);
                if (fh < seq && !dead) {
                    const u64 t0 = __builtin_amdgcn_s_memrealtime(); unsigned n = 0;
                    for (;;) {
                        const u64 f2 = __hip_atomic_load(fbase + s, __ATOMIC_RELAXED, __HIP_MEMORY_SCOPE_SYSTEM);
                        fh = rfl((unsigned)(f2 >> 32)); info = rfl((unsigned)f2);
                        if (fh >= seq) break;
                        if ((++n & 63u) == 0u && give_up(tmo, seq, t0)) { dead = true; break; }
                        __builtin_amdgcn_s_sleep(4);
                    }
                }
                __builtin_amdgcn_fence(__ATOMIC_ACQUIRE, "wavefront");
                if (dead) info = 3u << 30;
                if (first) STAMP(4);
                first = false;
                TOKEN_TAKE();
                strip_issue(sr, pd, s, info, lane);
                QNEXT(nq);
                STRIP_LANDED(sr, fn, nq, 0);
                TOKEN_GIVE();
                const unsigned sn = rfl(nq) - qbase;
                { gu64* fa = fbase + (sn & 1023u); asm volatile("global_load_dwordx2 %0, %1, off sc0 sc1" : "+v"(fn) : "v"(fa) : "memory"); }
                strip_stage1(sr, smem, lbase, esc, info, lane);
                strip_stage2(sr.mk, outr, por, smem, lbase, peer, s, info, lane);
                asm volatile("s_waitcnt vmcnt(16)" : "+v"(fn) :: "memory");
                s = sn;
            }
            STAMP(6);
        }
        return;
    }

    const bool hmode = (wi & 15) < nsplit;
    const int hsel = wi >> 4;
    const int tm = (xcd >> 1) * 4 + ((wi & 15) >> 2), tn = (xcd & 1) * 4 + (wi & 3);
    const int wr = wid >> 2, wc = wid & 3, fr = lane & 15, fq = lane >> 4;
    constexpr int HTB = 16384;
    const char* cA = ws + OFF_XL + (size_t)tm * 32768 + hsel * HTB;
    const char* cB = ws + OFF_WL + (size_t)tn * 32768;
    const unsigned voff[2] = {(unsigned)tid * 16u, (unsigned)tid * 16u + 8192u};
    constexpr size_t ASTEP = 64 * 8192, BSTEP = 32 * 8192;
    const unsigned ldsw = (unsigned)wid * 1024u;
    const int aoff = lds_byte(wr * 64 + fr, fq * 8), boff = lds_byte(wc * 32 + fr, fq * 8);
    const unsigned pollo = lane < 32 ? OFF_XF + (unsigned)(tm * NSTEP) * 16u : OFF_WF + (unsigned)(tn * NSTEP) * 16u;
    const unsigned nl = hmode ? 1u : (xcd < 4 ? (unsigned)NL1 : 16u);
    auto smap = [nl](unsigned n) -> unsigned {
        if (n < nl) return n * 8u;
        const unsigned m = n - nl, full = (16u - nl) * 8u;
        if (m < full) return (m & 7u) == 7u ? (nl + (m >> 3)) * 8u : (m & ~7u) + 1u + (m & 7u);
        const unsigned r = m - full; return (16u - nl + r / 7u) * 8u + 1u + r % 7u;
    };
    const unsigned stab0 = smap((unsigned)lane), stab1 = smap((unsigned)lane + 64u);
#define SMAP(n) smap(n)
#define STAB(n) ((unsigned)__builtin_amdgcn_readlane((n) & 64 ? stab1 : stab0, (n) & 63))
    unsigned avail = 0;
#define SA(b, h) (((b) * 2 + (h)) * HTB)
#define SB(b, h) ((4 + (b) * 2 + (h)) * HTB)
#define STAGE(bufoff, gptr) do { _Pragma("unroll") for (int _i = 0; _i < 2; ++_i) \
        __builtin_amdgcn_global_load_lds((const unsigned*)((gptr) + voff[_i]), (LAS unsigned*)(smem + (bufoff) + ldsw + _i * 8192), 16, 0, SC1); } while (0)
#define LDA(dst, b, h) do { _Pragma("unroll") for (int m = 0; m < 4; ++m) _Pragma("unroll") for (int k = 0; k < 2; ++k) dst[m][k] = *(const LAS bf16x8*)(smem + SA(b, h) + aoff + m * 2048 + k * 1024); } while (0)
#define LDB(dst, b, h) do { _Pragma("unroll") for (int n = 0; n < 2; ++n) _Pragma("unroll") for (int k = 0; k < 2; ++k) dst[n][k] = *(const LAS bf16x8*)(smem + SB(b, h) + boff + n * 2048 + k * 1024); } while (0)
#define MMA(ai, bj, At, Bt) do { __builtin_amdgcn_s_setprio(1); _Pragma("unroll") for (int m = 0; m < 4; ++m) _Pragma("unroll") for (int n = 0; n < 2; ++n) _Pragma("unroll") for (int k = 0; k < 2; ++k) \
        acc[ai][bj][m][n] = __builtin_amdgcn_mfma_f32_16x16x32_bf16(Bt[n][k], At[m][k], acc[ai][bj][m][n], 0, 0, 0); __builtin_amdgcn_s_setprio(0); } while (0)
#define NEED(s) do { if (__builtin_expect((avail & 0xffffu) <= (unsigned)(s), 0)) { \
        if (wid == 0) { const unsigned s0 = avail & 0xffffu; unsigned av; const u64 t0 = __builtin_amdgcn_s_memrealtime(); unsigned n = 0; \
            const unsigned pn = (s0 + (lane & 31)) & 127u, po = pollo + SMAP(pn) * 16u; \
            for (;;) { const u32x4 f = bld16<SC1>(wsr, po); \
                const bool ok = (s0 + (lane & 31) >= (unsigned)NSTEP) || (f.x >= seq && f.y >= seq && f.z >= seq && f.w >= seq); \
                const u64 mm = __builtin_amdgcn_ballot_w64(ok); const unsigned both = (unsigned)mm & (unsigned)(mm >> 32); \
                av = s0 + (both == 0xffffffffu ? 32u : (unsigned)__builtin_ctz(~both)); \
                if (av > (unsigned)(s)) break; \
                if ((++n & 63u) == 0u && give_up(tmo, seq, t0)) { av = 0x10000u | NSTEP; break; } \
                __builtin_amdgcn_s_sleep(1); } \
            asm volatile("ds_write_b32 %0, %1\n\ts_waitcnt lgkmcnt(0)" :: "v"(smbase + SM_AVAIL), "v"(av) : "memory"); } \
        CFENCE; BAR; CFENCE; \
        unsigned av2; asm volatile("ds_read_b32 %0, %1\n\ts_waitcnt lgkmcnt(0)" : "=v"(av2) : "v"(smbase + SM_AVAIL) : "memory"); \
        avail = rfl(av2); } } while (0)
    const unsigned smbase = (unsigned)(uintptr_t)smem;

    f32x4 acc[2][2][4][2];
#pragma unroll
    for (int a = 0; a < 2; ++a)
#pragma unroll
        for (int b = 0; b < 2; ++b)
#pragma unroll
            for (int m = 0; m < 4; ++m)
#pragma unroll
                for (int n = 0; n < 2; ++n) acc[a][b][m][n] = (f32x4){0.f, 0.f, 0.f, 0.f};
#define LOOP_STAMPS do { } while (0)
    if (hmode) {
        bf16x8 fa[2][4][2], fb[2][2][2][2];
#define HSTAGE(so, step) do { const char* ga = cA + (size_t)(step) * ASTEP; const char* gb = cB + (size_t)(step) * BSTEP; \
            STAGE((so), ga); STAGE((so) + HTB, gb); STAGE((so) + 2 * HTB, gb + HTB); } while (0)
#define HREAD(set, so) do { const LAS unsigned char* pa = smem + (so) + aoff; const LAS unsigned char* pb = smem + (so) + HTB + boff; \
            _Pragma("unroll") for (int h = 0; h < 2; ++h) _Pragma("unroll") for (int n = 0; n < 2; ++n) _Pragma("unroll") for (int k = 0; k < 2; ++k) \
                fb[set][h][n][k] = *(const LAS bf16x8*)(pb + h * HTB + n * 2048 + k * 1024); \
            _Pragma("unroll") for (int m = 0; m < 4; ++m) _Pragma("unroll") for (int k = 0; k < 2; ++k) fa[set][m][k] = *(const LAS bf16x8*)(pa + m * 2048 + k * 1024); } while (0)
#define HSUB(t, X, Y) do { \
            if ((t) + 3 < NSTEP) NEED((t) + 3); \
            WAIT_V(6); BAR; \
            const unsigned sn = so == 98304u ? 0u : so + 49152u; \
            HSTAGE(so, (t) + 3 < NSTEP ? (t) + 3 : 0); \
            HREAD(Y, sn); SCHED; \
            MMA(0, 0, fa[X], fb[X][0]); MMA(0, 1, fa[X], fb[X][1]); SCHED; \
            __builtin_amdgcn_s_waitcnt(0xC07F);   SCHED; so = sn; } while (0)
        NEED(2);
        STAMP(2);
        HSTAGE(0, 0); HSTAGE(49152, 1); HSTAGE(98304, 2);
        WAIT_V(12); BAR;
        unsigned so = 0;
        HREAD(0, 0u); __builtin_amdgcn_s_waitcnt(0xC07F); SCHED;
        for (int t = 0; t < NSTEP; t += 2) {
            LOOP_STAMPS;
            HSUB(t, 0, 1);
            HSUB(t + 1, 1, 0);
        }
    } else {
        bf16x8 At[4][2], B0[2][2], B1[2][2];
        NEED(1);
        STAMP(2);
        STAGE(SB(0, 0), cB); STAGE(SA(0, 0), cA); STAGE(SB(0, 1), cB + HTB); STAGE(SA(0, 1), cA + HTB);
        if (wr == 1) BAR;
        WAIT_V(4); BAR;
        { const size_t s1 = STAB(1); STAGE(SB(1, 0), cB + s1 * BSTEP); STAGE(SA(1, 0), cA + s1 * ASTEP); STAGE(SB(1, 1), cB + s1 * BSTEP + HTB); }
        WAIT_V(6); BAR;
        unsigned sp = STAB(1);
        for (int t = 0; t < NSTEP; t += 2) {
            const bool last = (t == NSTEP - 2);
            const unsigned s1 = sp, s2 = STAB((t + 2) & 127), s3 = STAB((t + 3) & 127);
            sp = s3;
            const char* a1 = cA + s1 * ASTEP;
            const char* a2 = cA + s2 * ASTEP; const char* b2 = cB + s2 * BSTEP;
            const char* a3 = cA + s3 * ASTEP; const char* b3 = cB + s3 * BSTEP;
            if (!last) NEED(t + 3);
            LOOP_STAMPS;
            LDB(B0, 0, 0); SCHED; LDA(At, 0, 0); STAGE(SA(1, 1), a1 + HTB);
            WAIT_L(8); BAR; WAIT_L(0); MMA(0, 0, At, B0); BAR; SCHED;
            LDB(B1, 0, 1); STAGE(SB(0, 0), b2);
            BAR; WAIT_L(0); MMA(0, 1, At, B1); BAR;
            LDA(At, 0, 1); STAGE(SA(0, 0), a2);
            BAR; WAIT_L(0); MMA(1, 0, At, B0); BAR; SCHED;
            STAGE(SB(0, 1), b2 + HTB);
            WAIT_V(6); BAR; MMA(1, 1, At, B1); BAR;
            LDB(B0, 1, 0); SCHED; LDA(At, 1, 0); STAGE(SA(0, 1), a2 + HTB);
            WAIT_L(8); BAR; WAIT_L(0); MMA(0, 0, At, B0); BAR; SCHED;
            LDB(B1, 1, 1); STAGE(SB(1, 0), b3);
            BAR; WAIT_L(0); MMA(0, 1, At, B1); BAR;
            LDA(At, 1, 1); STAGE(SA(1, 0), a3);
            BAR; WAIT_L(0); MMA(1, 0, At, B0); BAR; SCHED;
            STAGE(SB(1, 1), b3 + HTB);
            WAIT_V(6); BAR; MMA(1, 1, At, B1); BAR;
        }
        if (wr == 0) BAR;
    }
    const unsigned tmov = __hip_atomic_load(tmo, __ATOMIC_RELAXED, __HIP_MEMORY_SCOPE_AGENT);
    WAIT_V(0);
    BAR;
    STAMP(3);
    {
        int tz = threadIdx.x; asm volatile("" : "+v"(tz));
        const int lane2 = tz & 63, fr2 = lane2 & 15, fq2 = lane2 >> 4;
        const bool dead = (avail >> 16) != 0u || rfl(tmov) == seq;
#pragma unroll
        for (int ai = 0; ai < 2; ++ai)
            if (ai == 0 || !hmode)
#pragma unroll
            for (int m = 0; m < 4; ++m) {
                const int row = ai * 128 + wr * 64 + m * 16 + fr2;
#pragma unroll
                for (int bj = 0; bj < 2; ++bj) {
                    f32x4 v0 = acc[ai][bj][m][0], v1 = acc[ai][bj][m][1];
#pragma unroll
                    for (int k = 0; k < 4; ++k) { v0[k] = v0[k] <= 0.f ? 0.f : v0[k]; v1[k] = v1[k] <= 0.f ? 0.f : v1[k]; }
                    u32x4 w; w.x = cvt_pk_bf16(v0[0], v0[1]); w.y = cvt_pk_bf16(v0[2], v0[3]); w.z = cvt_pk_bf16(v1[0], v1[1]); w.w = cvt_pk_bf16(v1[2], v1[3]);
                    if (dead) w = (u32x4){0x7fc07fc0u, 0x7fc07fc0u, 0x7fc07fc0u, 0x7fc07fc0u};
                    const int chunk = bj * 16 + wc * 4 + fq2;
                    *(LAS u32x4*)(smem + row * 512 + ((chunk ^ (row & 7)) * 16)) = w;
                }
            }
        __syncthreads();
        STAMP(4);
        if (!hmode || wid < 4) pack_strip(p, wsr, outr, smem, (tm * 8 + tn) * 8 + hsel * 4 + wid, wid, lane2, seq);
        STAMP(5);
    }
}

extern "C" void kernel_launch(int rank, int nranks, ncclComm_t comm,
                              hipStream_t stream,
                              void* const* d_in, const int* in_sizes, int n_in,
                              void* d_out, int out_size,
                              const int* in_dtypes, int out_dtype,
                              void* const* d_out_all,
                              void* const* d_ws_all, size_t ws_bytes) {
    (void)comm; (void)in_dtypes; (void)out_dtype;
    if (nranks != NR || n_in != 2 || (unsigned)rank >= (unsigned)NR || ws_bytes < WS_USED || in_sizes[0] != ND * KS || in_sizes[1] != KS * ND || out_size != ND * ND) {
        fprintf(stderr, "kernel_launch: unexpected problem shape\n");
        return;
    }
    Params p;
    p.x = (const u16*)d_in[0]; p.w = (const u16*)d_in[1]; p.out = (u16*)d_out;
    p.out_all = d_out_all; p.ws_all = d_ws_all; p.rank = rank;
    hipLaunchKernelGGL(gemm_ar_fused, dim3(NWG), dim3(512), 0, stream, p);
    HIP_CHECK(hipGetLastError());
}
```

```cpp
#include <hip/hip_runtime.h>
#include <rccl/rccl.h>
#include <cstdio>
#include <cstdint>

#define HIP_CHECK(expr) do { hipError_t _e = (expr); if (_e != hipSuccess) { \
    fprintf(stderr, "HIP error %s at %s:%d\n", hipGetErrorString(_e), __FILE__, __LINE__); \
    return; } } while (0)

#define LAS __attribute__((address_space(3)))
#define GAS __attribute__((address_space(1)))
typedef unsigned short u16;
typedef unsigned long long u64;
typedef short bf16x8 __attribute__((ext_vector_type(8)));
typedef short s16x4 __attribute__((ext_vector_type(4)));
typedef float f32x4 __attribute__((ext_vector_type(4)));
typedef unsigned u32x4 __attribute__((ext_vector_type(4)));
typedef unsigned u32x2 __attribute__((ext_vector_type(2)));
typedef u16 u16x2 __attribute__((ext_vector_type(2)));
typedef GAS unsigned gu32;
typedef GAS u64 gu64;

#define FORCE_RAW_IN 0
#define FORCE_RAW_OUT 0
#define NT 12
#define NPUSH 0
#define OAUX 18
#define NRAW 1
#define KH1 12
#define KH2 20
#define KH3 32
#define NL1 4
#define PDELAY 336
#define PSLEEP 64
constexpr int NWV = 4;
#define LAG1 2
#define LAG2 7
#define LAG3 11

namespace {

constexpr int NR = 8;
constexpr int ND = 8192;
constexpr int KS = 1024;
constexpr int RROWS = 4096, RCOLS = 2048;
constexpr int NSTEP = 128;
constexpr int NWG = 256;
constexpr int NPD = 84 * 4;
constexpr int NENC = NPD + 512;
constexpr int NONCE = 256 + 512 + 96;
static_assert(NT == 12, "wave roles");
static_assert(NPUSH <= 2 || LAG1 == 0, "pushed groups lead the stream");
constexpr unsigned NSKIP = NPUSH ? 16u + 16u * NPUSH : 0u;
__device__ __forceinline__ constexpr int lag(int k) { return k == 0 ? 0 : k == 1 ? LAG1 : k == 2 ? LAG2 : LAG3; }
constexpr unsigned long long DEADLINE = 40000000ull;

constexpr unsigned OFF_TMO = 0x80;
constexpr unsigned OFF_Q = 0x400;
constexpr unsigned OFF_ECNT = 0x1000;
constexpr unsigned OFF_FIN = 0x8000;
constexpr unsigned OFF_SEQ = 0xa000;
constexpr unsigned OFF_PF = 0xc000;
constexpr unsigned OFF_XF = 0x10000;
constexpr unsigned OFF_WF = 0x18000;
constexpr unsigned OFF_FOUT = 0x20000;
constexpr unsigned USLOT = 8192 + 1024;
constexpr unsigned OFF_EX = 1u << 20;
constexpr unsigned OFF_EW = OFF_EX + (18u << 20);
constexpr unsigned OFF_XL = OFF_EW + (18u << 20);
constexpr unsigned OFF_WL = OFF_XL + (64u << 20);
constexpr unsigned SSLOT = 11264;
constexpr unsigned OFF_EO = OFF_WL + (32u << 20);
constexpr unsigned OFF_PART = OFF_EO + (13u << 20);
constexpr unsigned WS_USED = OFF_PART + (32u << 20);
static_assert(2048u * USLOT <= (18u << 20), "unit area");
static_assert(1024u * SSLOT <= (13u << 20), "strip area");
constexpr unsigned OUT_BYTES = (unsigned)ND * ND * 2u;

constexpr int SM_TOKEN = 131072 + 32, SM_DUMMY = 131072 + 64, SM_LUT = 131072 + 256, SM_LUTE = SM_LUT + 4096, SM_SCR = SM_LUTE + 128;
constexpr int SM_AVAIL = SM_SCR + 24576 + 64;
constexpr int SMEM_BYTES = SM_AVAIL + 16;
constexpr unsigned CAP2 = 2304, CAP3 = 512, CAP4 = 240;

struct Params {
    const u16* x; const u16* w; u16* out;
    void* const* out_all; void* const* ws_all;
    int rank;
};

typedef __amdgpu_buffer_rsrc_t rsrc_t;
__device__ __forceinline__ rsrc_t make_rsrc(const void* base, unsigned bytes) {
    const u64 b = (u64)base;
    const unsigned lo = __builtin_amdgcn_readfirstlane((unsigned)b), hi = __builtin_amdgcn_readfirstlane((unsigned)(b >> 32));
    return __builtin_amdgcn_make_buffer_rsrc((void*)(((u64)hi << 32) | lo), 0, bytes, 0x00020000);
}
constexpr int SC1 = 16, SYS = 17;
template <int AUX> __device__ __forceinline__ u32x4 bld16(rsrc_t r, unsigned off) { return __builtin_amdgcn_raw_buffer_load_b128(r, off, 0, AUX); }
template <int AUX> __device__ __forceinline__ unsigned bld4(rsrc_t r, unsigned off) { return __builtin_amdgcn_raw_buffer_load_b32(r, off, 0, AUX); }
template <int AUX> __device__ __forceinline__ unsigned bld2(rsrc_t r, unsigned off) { return __builtin_amdgcn_raw_buffer_load_b16(r, off, 0, AUX); }
template <int AUX> __device__ __forceinline__ void bst16(rsrc_t r, unsigned off, u32x4 v) { __builtin_amdgcn_raw_buffer_store_b128(v, r, off, 0, AUX); }
template <int AUX> __device__ __forceinline__ void bst8(rsrc_t r, unsigned off, u32x2 v) { __builtin_amdgcn_raw_buffer_store_b64(v, r, off, 0, AUX); }
template <int AUX> __device__ __forceinline__ void bst4(rsrc_t r, unsigned off, unsigned v) { __builtin_amdgcn_raw_buffer_store_b32(v, r, off, 0, AUX); }
template <int AUX> __device__ __forceinline__ void bst2(rsrc_t r, unsigned off, unsigned v) { __builtin_amdgcn_raw_buffer_store_b16((u16)v, r, off, 0, AUX); }

__device__ __forceinline__ u32x4 make_desc(const void* base, unsigned bytes) {
    const u64 b = (u64)base;
    return (u32x4){(unsigned)__builtin_amdgcn_readfirstlane((unsigned)b), (unsigned)__builtin_amdgcn_readfirstlane((unsigned)(b >> 32)), bytes, 0x00020000u};
}
#define ALD16(dst, desc, off, imm) asm volatile("buffer_load_dwordx4 %0, %1, %2, 0 offen offset:%3 sc0 sc1" : "+v"(dst) : "v"(off), "s"(desc), "i"(imm) : "memory")
#define ALD8(dst, desc, off, imm) asm volatile("buffer_load_dwordx2 %0, %1, %2, 0 offen offset:%3 sc0 sc1" : "+v"(dst) : "v"(off), "s"(desc), "i"(imm) : "memory")
#define ALD4(dst, desc, off, imm) asm volatile("buffer_load_dword %0, %1, %2, 0 offen offset:%3 sc0 sc1" : "+v"(dst) : "v"(off), "s"(desc), "i"(imm) : "memory")
#define ALD2(dst, desc, off, imm) asm volatile("buffer_load_ushort %0, %1, %2, 0 offen offset:%3 sc0 sc1" : "+v"(dst) : "v"(off), "s"(desc), "i"(imm) : "memory")
constexpr unsigned OOB = 0xfffff000u;

#define WAIT_V(n) asm volatile("s_waitcnt vmcnt(" #n ")" ::: "memory")
#define WAIT_L(n) asm volatile("s_waitcnt lgkmcnt(" #n ")" ::: "memory")
#define BAR __builtin_amdgcn_s_barrier()
#define SCHED __builtin_amdgcn_sched_barrier(0)
#define CFENCE asm volatile("" ::: "memory")
#define STAMP(k) do { } while (0)

__device__ __forceinline__ unsigned rfl(unsigned v) { return __builtin_amdgcn_readfirstlane(v); }
__device__ __forceinline__ unsigned pkmax(unsigned a, unsigned b) {
    u16x2 x = __builtin_bit_cast(u16x2, a), y = __builtin_bit_cast(u16x2, b);
    return __builtin_bit_cast(unsigned, __builtin_elementwise_max(x, y));
}
__device__ __forceinline__ unsigned pkmin(unsigned a, unsigned b) {
    u16x2 x = __builtin_bit_cast(u16x2, a), y = __builtin_bit_cast(u16x2, b);
    return __builtin_bit_cast(unsigned, __builtin_elementwise_min(x, y));
}
__device__ __forceinline__ unsigned wave_max(unsigned v) {
#define MAX_STEP(ctrl, rmask) { const unsigned t = (unsigned)__builtin_amdgcn_update_dpp(0, (int)v, ctrl, rmask, 0xf, false); v = v > t ? v : t; }
    MAX_STEP(0x111, 0xf) MAX_STEP(0x112, 0xf) MAX_STEP(0x114, 0xf) MAX_STEP(0x118, 0xf) MAX_STEP(0x142, 0xa) MAX_STEP(0x143, 0xc)
    return __builtin_amdgcn_readlane(v, 63);
}
__device__ __forceinline__ unsigned pkmin1(unsigned a) {
    unsigned r; asm("v_pk_min_u16 %0, %1, 1 op_sel_hi:[1,0]" : "=v"(r) : "v"(a)); return r;
}
__device__ __forceinline__ unsigned cvt_pk_bf16(float lo, float hi) { unsigned r; asm volatile("v_cvt_pk_bf16_f32 %0, %1, %2" : "=v"(r) : "v"(lo), "v"(hi)); return r; }
__device__ __forceinline__ int lds_byte(int r, int c) { const int st = (r >> 4) * 2 + (c >> 5), rr = r & 15, cc = c & 31, ob = rr * 64 + cc * 2; return st * 1024 + (ob ^ (((ob >> 9) & 1) << 5)); }

__device__ __forceinline__ unsigned wave_scan(unsigned x) {
    x += (unsigned)__builtin_amdgcn_update_dpp(0, (int)x, 0x111, 0xf, 0xf, false);
    x += (unsigned)__builtin_amdgcn_update_dpp(0, (int)x, 0x112, 0xf, 0xf, false);
    x += (unsigned)__builtin_amdgcn_update_dpp(0, (int)x, 0x114, 0xf, 0xf, false);
    x += (unsigned)__builtin_amdgcn_update_dpp(0, (int)x, 0x118, 0xf, 0xf, false);
    x += (unsigned)__builtin_amdgcn_update_dpp(0, (int)x, 0x142, 0xa, 0xf, false);
    x += (unsigned)__builtin_amdgcn_update_dpp(0, (int)x, 0x143, 0xc, 0xf, false);
    return x;
}
__device__ __forceinline__ unsigned wave_sum(unsigned v) { return __builtin_amdgcn_readlane(wave_scan(v), 63); }
__device__ __forceinline__ unsigned pksub(unsigned a, unsigned b) {
    u16x2 x = __builtin_bit_cast(u16x2, a), y = __builtin_bit_cast(u16x2, b);
    return __builtin_bit_cast(unsigned, (u16x2)(x - y));
}
__device__ __forceinline__ unsigned bfi(unsigned mask, unsigned a, unsigned b) { return (a & mask) | (b & ~mask); }
__device__ __forceinline__ unsigned choose_window(unsigned esum, unsigned n, unsigned emax) {
    if (!n) return 21u;
    const unsigned num = esum * 32u + n * 63u, den = n * 32u;
    unsigned t = num / den; const unsigned alt = (num - t * den) * 5u > den * 2u ? 1u : 0u;
    t = t > emax ? emax : t; t = t < 21u ? 21u : (t > 253u ? 253u : t);
    return t | (alt << 9);
}
template <int NQ> __device__ __forceinline__ unsigned apply_escapes(unsigned (&e)[NQ], const unsigned (&m)[NQ], LAS unsigned char* smem, unsigned off) {
    unsigned sel[NQ], o[NQ]; u32x2 w[NQ];
#pragma unroll
    for (int q = 0; q < NQ; ++q) {
        sel[q] = *(const LAS unsigned*)(smem + SM_LUTE + __builtin_amdgcn_udot4(m[q], 0x08040201u, 0u, false));
        o[q] = off; off += (unsigned)__builtin_popcount(m[q]);
        const LAS unsigned* pw = (const LAS unsigned*)(smem + (o[q] & ~3u));
        w[q] = (u32x2){pw[0], pw[1]};
    }
#pragma unroll
    for (int q = 0; q < NQ; ++q) e[q] = __builtin_amdgcn_perm(__builtin_amdgcn_alignbyte(w[q].y, w[q].x, o[q]), e[q], sel[q]);
    return off;
}
__device__ __forceinline__ unsigned append_flagged(LAS unsigned char* smem, unsigned e, unsigned m, unsigned off, unsigned dummy) {
#pragma unroll
    for (int k = 0; k < 4; ++k) {
        const unsigned bit = (m >> (8 * k + 2)) & 1u;
        smem[bit ? off : dummy] = (unsigned char)(e >> (8 * k));
        off += bit;
    }
    return off;
}
__device__ __forceinline__ void decode_levels(LAS unsigned char* smem, unsigned h0, unsigned h1, unsigned o2, unsigned l3, unsigned l2, int lane) {
    const unsigned t = h0 & 0xffu, alt = (h0 >> 9) & 1u, n2 = h0 >> 16, n3 = h1 & 0xffffu;
    const unsigned o3 = o2 + ((n2 + 15u) >> 4) * 4u, o4 = o3 + ((n3 + 7u) >> 3) * 4u;
    const unsigned b3q = (t - 6u + alt) * 0x01010101u, spq = (t + 1u + alt) * 0x01010101u;
    unsigned eo = o4;
    for (unsigned r = 0; r * 256u < n3; ++r) {
        const unsigned g = r * 64u + lane;
        unsigned c = *(const LAS u16*)(smem + o3 + g * 2u); c = g * 4u < n3 ? c : 0u;
        const unsigned z = __builtin_amdgcn_perm(0u, c, 0x01010000u);
        const unsigned d = (z & 0x000f000fu) | ((z >> 4) & 0x0f000f00u);
        const unsigned is15 = (d + 0x01010101u) & 0x10101010u, is14 = ((d + 0x02020202u) & 0x10101010u) ^ is15;
        unsigned e[1], m[1];
        e[0] = bfi((is14 << 4) - (is14 >> 4), spq, b3q - d); m[0] = is15 >> 2;
        const unsigned ec = (unsigned)__builtin_popcount(m[0]), inc = wave_scan(ec);
        apply_escapes<1>(e, m, smem, eo + inc - ec);
        eo += __builtin_amdgcn_readlane(inc, 63);
        *(LAS unsigned*)(smem + l3 + g * 4u) = e[0];
    }
    const unsigned tab = (t - 3u) | ((t - 4u) << 8) | ((alt ? t + 1u : t - 5u) << 16);
    eo = l3;
    for (unsigned r = 0; r * 1024u < n2; ++r) {
        const unsigned g = r * 64u + lane;
        unsigned c = *(const LAS unsigned*)(smem + o2 + g * 4u); c = g * 16u < n2 ? c : 0u;
        unsigned e[4], m[4];
#pragma unroll
        for (int k = 0; k < 4; ++k) { const unsigned d = (c >> (2 * k)) & 0x03030303u; e[k] = __builtin_amdgcn_perm(0u, tab, d); m[k] = (d + 0x01010101u) & 0x04040404u; }
        const unsigned ec = (unsigned)__builtin_popcount(c & (c >> 1) & 0x55555555u), inc = wave_scan(ec);
        apply_escapes<4>(e, m, smem, eo + inc - ec);
        eo += __builtin_amdgcn_readlane(inc, 63);
        if (g * 16u < n2) *(LAS u32x4*)(smem + l2 + g * 16u) = (u32x4){e[0], e[1], e[2], e[3]};
    }
}
__device__ __forceinline__ unsigned encode_levels(LAS unsigned char* smem, rsrc_t wsr, unsigned win, unsigned n2, unsigned l2, unsigned l3, unsigned l4, unsigned g2, unsigned room, unsigned dummy, int lane) {
    const unsigned t = win & 0xffu, alt = (win >> 9) & 1u;
    if (lane < 16) smem[l2 + n2 + lane] = (unsigned char)(t - 3u);
    const unsigned t3 = (t - 3u) * 0x00010001u, sc = (alt ? t + 1u : t - 5u) * 0x00010001u;
    unsigned n3 = 0;
    for (unsigned r = 0; r * 1024u < n2; ++r) {
        const unsigned g = r * 64u + lane; const bool act = g * 16u < n2;
        const u32x4 e = *(const LAS u32x4*)(smem + l2 + (act ? g : 0u) * 16u);
        unsigned c = 0, m[4], ec = 0;
#pragma unroll
        for (int k = 0; k < 4; ++k) {
            unsigned dd[2];
#pragma unroll
            for (int h = 0; h < 2; ++h) {
                const unsigned x = (e[k] >> (8 * h)) & 0x00ff00ffu;
                const unsigned a = pkmin(pksub(t3, x), 0x00020002u);
                dd[h] = a + ((a >> 1) & pkmin1(x ^ sc));
            }
            const unsigned d = dd[0] | (dd[1] << 8);
            c |= d << (2 * k); m[k] = act ? (d + 0x01010101u) & 0x04040404u : 0u; ec += (unsigned)__builtin_popcount(m[k]);
        }
        const unsigned inc = wave_scan(ec), tot = __builtin_amdgcn_readlane(inc, 63);
        if (n3 + tot > CAP3) return 0xffffffffu;
        if (act) bst4<SYS>(wsr, g2 + g * 4u, c);
        unsigned eo = l3 + n3 + inc - ec;
#pragma unroll
        for (int k = 0; k < 4; ++k) eo = append_flagged(smem, e[k], m[k], eo, dummy);
        n3 += tot;
    }
    const unsigned b3 = t - 6u + alt;
    if (lane < 8) smem[l3 + n3 + lane] = (unsigned char)b3;
    const unsigned g3 = g2 + ((n2 + 15u) >> 4) * 4u, g4 = g3 + ((n3 + 7u) >> 3) * 4u;
    if (g4 - g2 > room) return 0xffffffffu;
    const unsigned b32 = b3 * 0x00010001u, sp = (t + 1u + alt) * 0x00010001u;
    unsigned n4 = 0;
    for (unsigned r = 0; r * 256u < n3; ++r) {
        const unsigned g = r * 64u + lane; const bool act = g * 4u < ((n3 + 7u) & ~7u);
        const unsigned e = *(const LAS unsigned*)(smem + l3 + (act ? g : 0u) * 4u);
        unsigned dd[2];
#pragma unroll
        for (int h = 0; h < 2; ++h) {
            const unsigned x = (e >> (8 * h)) & 0x00ff00ffu;
            const unsigned a = pkmin(pksub(b32, x), 0x000e000eu);
            dd[h] = a + (((a + 0x00020002u) >> 4) & pkmin1(x ^ sp));
        }
        const unsigned d = dd[0] | (dd[1] << 8);
        const unsigned m = act ? ((d + 0x01010101u) & 0x10101010u) >> 2 : 0u;
        const unsigned ec = (unsigned)__builtin_popcount(m), inc = wave_scan(ec), tot = __builtin_amdgcn_readlane(inc, 63);
        if (n4 + tot > CAP4) return 0xffffffffu;
        const unsigned y = d | (d >> 4);
        if (act) bst2<SYS>(wsr, g3 + g * 2u, __builtin_amdgcn_perm(0u, y, 0x0c0c0200u));
        if (tot) append_flagged(smem, e, m, l4 + n4 + inc - ec, dummy);
        n4 += tot;
    }
    if (g4 - g2 + n4 > room) return 0xffffffffu;
    if (n4) { const unsigned lv = *(const LAS unsigned*)(smem + l4 + lane * 4); if ((unsigned)lane * 4u < n4) bst4<SYS>(wsr, g4 + lane * 4, lv); }
    return n3 | (n4 << 16);
}
__device__ __forceinline__ unsigned side_bytes(unsigned n2, unsigned h1) { return ((n2 + 15u) >> 4) * 4u + (((h1 & 0xffffu) + 7u) >> 3) * 4u + (h1 >> 16); }
struct Order {
    int ph, start;
    template <int NWU, int NXU> __device__ __forceinline__ void locate(int i, int& j, int& g, int& u) {
        for (;;) {
            int c = (ph < 16) ? NWU : 0;
#pragma unroll
            for (int k = 0; k < 4; ++k) c += ((unsigned)(ph - lag(k)) < 16u) ? NXU : 0;
            if (i < start + c) break;
            start += c; ++ph;
        }
        u = i - start; g = 0; j = ph;
        if (ph < 16) { if (u < NWU) return; u -= NWU; }
#pragma unroll
        for (int k = 0; k < 4; ++k) {
            if ((unsigned)(ph - lag(k)) < 16u) { if (u < NXU) { g = 1 + k; j = ph - lag(k); return; } u -= NXU; }
        }
    }
};

__device__ __forceinline__ bool give_up(gu32* tmo, unsigned seq, u64 t0) {
    bool dead = rfl(__hip_atomic_load(tmo, __ATOMIC_RELAXED, __HIP_MEMORY_SCOPE_AGENT)) == seq;
    if (!dead && __builtin_amdgcn_s_memrealtime() - t0 > DEADLINE) {
        __hip_atomic_store(tmo, seq, __ATOMIC_RELAXED, __HIP_MEMORY_SCOPE_AGENT);
        dead = true;
    }
    return dead;
}
__device__ __forceinline__ void pack_unit(const Params& p, char* ws, rsrc_t wsr, LAS unsigned char* smem, unsigned lwo, int j, int g, int u, int lane, unsigned seqv) {
    const bool isx = g != 0; const int ub = isx ? (u >> 4) * 64 + (g - 1) * 16 + (u & 15) : u;
    const int ra = p.rank >> 2, rb = p.rank & 3;
    LAS unsigned char* lw = smem + lwo;
    u32x4 v[8];
    if (isx) {
        const u16* src = p.x + (size_t)(ub * 64 + (lane & 15)) * KS + j * 64 + (lane >> 4) * 8;
#pragma unroll
        for (int c = 0; c < 8; ++c) v[c] = *(const u32x4*)(src + (size_t)(c >> 1) * 16 * KS + (c & 1) * 32);
    } else {
        constexpr int P = 144;
        const u16* src = p.w + (size_t)(j * 64 + (lane >> 3)) * ND + ub * 64 + (lane & 7) * 8;
        u32x4 tmp[8];
#pragma unroll
        for (int i = 0; i < 8; ++i) tmp[i] = *(const u32x4*)(src + (size_t)i * 8 * ND);
#pragma unroll
        for (int i = 0; i < 8; ++i) *(LAS u32x4*)(lw + (i * 8 + (lane >> 3)) * P + (lane & 7) * 16) = tmp[i];
        const int q = (lane >> 2) & 3, pp = lane & 3;
#pragma unroll
        for (int c = 0; c < 8; ++c) {
            const int k0 = (c & 1) * 32 + (lane >> 4) * 8, rbk = c >> 1;
            const int col = (rbk >> 1) * 32 + 8 * pp + 4 * (rbk & 1);
            const s16x4 a = __builtin_amdgcn_ds_read_tr16_b64_v4i16((LAS s16x4*)(lw + (k0 + q) * P + col * 2));
            const s16x4 b = __builtin_amdgcn_ds_read_tr16_b64_v4i16((LAS s16x4*)(lw + (k0 + 4 + q) * P + col * 2));
            const u32x2 a2 = __builtin_bit_cast(u32x2, a), b2 = __builtin_bit_cast(u32x2, b);
            v[c] = (u32x4){a2.x, a2.y, b2.x, b2.y};
        }
    }
    const unsigned loff = (lane & 15) * 64 + (((lane >> 4) * 16) ^ (((lane >> 3) & 1) << 5));
    const int cls = isx ? (ub >> 6) : (ub >> 5);
    const bool own = isx ? (cls == ra) : (cls == rb);
    const int li = isx ? (ub & 63) : (ub & 31);
    if (own) {
        const unsigned dst = (isx ? OFF_XL + (unsigned)(j * 8 * 64 + li) * 8192u : OFF_WL + (unsigned)(j * 8 * 32 + li) * 8192u) + loff;
#pragma unroll
        for (int c = 0; c < 8; ++c) bst16<SC1>(wsr, dst + c * 1024, v[c]);
    }
    if (j == 0 && g < NPUSH) {
        for (int r = 0; r < NR; ++r) {
            if (r == p.rank || (isx ? (r >> 2) : (r & 3)) != cls) continue;
            const rsrc_t rr = make_rsrc(p.ws_all[r], WS_USED);
            const int step = 1 + ((p.rank - r - 1) & 7);
            const unsigned dst = (isx ? OFF_XL + (unsigned)(step * 64 + li) * 8192u : OFF_WL + (unsigned)(step * 32 + li) * 8192u) + loff;
#pragma unroll
            for (int c = 0; c < 8; ++c) bst16<SYS>(rr, dst + c * 1024, v[c]);
        }
        WAIT_V(0);
        const unsigned seq = rfl(seqv) + 1u;
        if (lane < NR && (isx ? (lane >> 2) : (lane & 3)) == cls) {
            const int step = lane == p.rank ? 0 : 1 + ((p.rank - lane - 1) & 7);
            char* pw = (char*)p.ws_all[lane];
            __hip_atomic_store((gu32*)(pw + (isx ? OFF_XF : OFF_WF) + (unsigned)(((li >> 2) * NSTEP + step) * 4 + (li & 3)) * 4u), seq, __ATOMIC_RELAXED, __HIP_MEMORY_SCOPE_SYSTEM);
        }
        return;
    }
    const unsigned slot0 = (isx ? OFF_EX : OFF_EW) + (unsigned)(j * 128 + ub) * USLOT;
    const unsigned pay = slot0 + 1024 + lane * 16;
    if (j == 0 && g < NRAW) {
#pragma unroll
        for (int c = 0; c < 8; ++c) bst16<SYS>(wsr, pay + c * 1024, v[c]);
        if (lane == 0) bst4<SYS>(wsr, slot0, 1u << 8);
    } else {
    unsigned e4[16], sm[16], mx = 0, esum = 0;
#pragma unroll
    for (int q = 0; q < 16; ++q) {
        const unsigned a = v[q >> 1][(q & 1) * 2], b = v[q >> 1][(q & 1) * 2 + 1];
        const unsigned lo = __builtin_amdgcn_perm(b, a, 0x06040200u), hi = __builtin_amdgcn_perm(b, a, 0x07050301u);
        e4[q] = bfi(0x01010101u, lo >> 7, hi << 1);
        sm[q] = bfi(0x7f7f7f7fu, lo, hi);
        esum = __builtin_amdgcn_udot4(e4[q], 0x01010101u, esum, false);
        mx = pkmax(mx, a & 0x7fff7fffu); mx = pkmax(mx, b & 0x7fff7fffu);
    }
    mx = (mx & 0xffffu) > (mx >> 16) ? (mx & 0xffffu) : (mx >> 16);
    const unsigned win = choose_window(wave_sum(esum), 4096u, wave_max(mx) >> 7);
    const unsigned top2 = (win & 0xffu) * 0x00010001u;
    unsigned p2[4] = {0u, 0u, 0u, 0u}, m[16], ec = 0;
#pragma unroll
    for (int q = 0; q < 16; ++q) {
        const unsigned da = pkmin(pksub(top2, e4[q] & 0x00ff00ffu), 0x00030003u), db = pkmin(pksub(top2, (e4[q] >> 8) & 0x00ff00ffu), 0x00030003u);
        const unsigned d = da | (db << 8);
        p2[q >> 2] |= d << (2 * (q & 3));
        m[q] = (d + 0x01010101u) & 0x04040404u; ec += (unsigned)__builtin_popcount(m[q]);
    }
    const unsigned einc = wave_scan(ec), n2 = __builtin_amdgcn_readlane(einc, 63);
    unsigned h1 = 0xffffffffu;
    if (!FORCE_RAW_IN && n2 <= CAP2) {
        const unsigned l2 = lwo + 10240, l3 = l2 + CAP2 + 16, l4 = l3 + CAP3 + 16, dummy = SM_DUMMY + lane * 2;
        unsigned eo = l2 + einc - ec;
#pragma unroll
        for (int q = 0; q < 16; ++q) eo = append_flagged(smem, e4[q], m[q], eo, dummy);
        h1 = encode_levels(smem, wsr, win, n2, l2, l3, l4, slot0 + 8, 1016u, dummy, lane);
    }
    if (h1 == 0xffffffffu) {
#pragma unroll
        for (int c = 0; c < 8; ++c) bst16<SYS>(wsr, pay + c * 1024, v[c]);
        if (lane == 0) bst4<SYS>(wsr, slot0, 1u << 8);
    } else {
#pragma unroll
        for (int i = 0; i < 4; ++i) bst16<SYS>(wsr, pay + i * 1024, (u32x4){sm[4 * i], sm[4 * i + 1], sm[4 * i + 2], sm[4 * i + 3]});
        bst16<SYS>(wsr, pay + 4096, (u32x4){p2[0], p2[1], p2[2], p2[3]});
        if (lane == 0) bst8<SYS>(wsr, slot0, (u32x2){win | (n2 << 16), h1});
    }
    }
    WAIT_V(0);
    const unsigned seq = rfl(seqv) + 1u;
    if (own && lane == 0) {
        const unsigned f = (isx ? OFF_XF : OFF_WF) + (unsigned)(((li >> 2) * NSTEP + j * 8) * 4 + (li & 3)) * 4u;
        __hip_atomic_store((gu32*)(ws + f), seq, __ATOMIC_RELAXED, __HIP_MEMORY_SCOPE_AGENT);
    }
    unsigned ticket = 0;
    if (lane == 0) ticket = __hip_atomic_fetch_add((gu32*)(ws + OFF_ECNT + (unsigned)(j * 12 + (isx ? cls * 4 + g - 1 : 8 + cls)) * 128u), 1u, __ATOMIC_RELAXED, __HIP_MEMORY_SCOPE_AGENT);
    ticket = rfl(ticket);
    const unsigned per = isx ? 16u : 32u;
    if ((ticket & (per - 1u)) == per - 1u) {
        const bool want = lane < NR && lane != p.rank && (isx ? ((lane >> 2) == cls) : ((lane & 3) == cls));
        if (want) {
            char* pw = (char*)p.ws_all[lane];
            __hip_atomic_store((gu32*)(pw + OFF_FIN + (unsigned)p.rank * 512u + (unsigned)j * 32u + (unsigned)g * 4u), seq, __ATOMIC_RELAXED, __HIP_MEMORY_SCOPE_SYSTEM);
        }
    }
}

struct UnitRegs { u32x4 a[5]; u32x4 sd; };
__device__ __forceinline__ unsigned unit_slot(int rank, int j, int g, int u) {
    const int ub = g ? (rank >> 2) * 64 + (g - 1) * 16 + u : (rank & 3) * 32 + u;
    return (g ? OFF_EX : OFF_EW) + (unsigned)(j * 128 + ub) * USLOT;
}
__device__ __forceinline__ void unit_issue(UnitRegs& r, u32x4 pd, unsigned slot0, int lane) {
    const unsigned oh = lane < 24 ? slot0 + lane * 16 : OOB, o0 = slot0 + 1024 + lane * 16, o1 = o0 + 4096;
    ALD16(r.sd, pd, oh, 0);
    ALD16(r.a[0], pd, o0, 0); ALD16(r.a[1], pd, o0, 1024); ALD16(r.a[2], pd, o0, 2048); ALD16(r.a[3], pd, o0, 3072);
    ALD16(r.a[4], pd, o1, 0);
}
#define UNIT_LANDED(r, n) asm volatile("s_waitcnt vmcnt(0)" : "+v"(r.a[0]), "+v"(r.a[1]), "+v"(r.a[2]), "+v"(r.a[3]), "+v"(r.a[4]), "+v"(r.sd), "+v"(n) :: "memory")
__device__ __forceinline__ void unit_unpack(const UnitRegs& r, rsrc_t wsr, rsrc_t pr, LAS unsigned char* smem, unsigned scr, unsigned slot0, unsigned dst0, int lane) {
    const unsigned dst = dst0 + (lane & 15) * 64 + (((lane >> 4) * 16) ^ (((lane >> 3) & 1) << 5));
    const unsigned h0 = rfl(r.sd.x), h1 = rfl(r.sd.y);
    if ((h0 >> 8) & 1u) {
        u32x4 b[3];
#pragma unroll
        for (int c = 0; c < 3; ++c) b[c] = bld16<SYS>(pr, slot0 + 1024 + lane * 16 + (5 + c) * 1024);
#pragma unroll
        for (int c = 0; c < 5; ++c) bst16<SC1>(wsr, dst + c * 1024, r.a[c]);
#pragma unroll
        for (int c = 0; c < 3; ++c) bst16<SC1>(wsr, dst + (5 + c) * 1024, b[c]);
        return;
    }
    u32x4 sd = r.sd;
    const unsigned len = 8u + side_bytes(h0 >> 16, h1);
    if (len > 384u) { if (lane >= 24 && (unsigned)lane * 16u < len) sd = bld16<SYS>(pr, slot0 + lane * 16); }
    *(LAS u32x4*)(smem + scr + lane * 16) = sd;
    const unsigned l3 = scr + 2048, l2 = l3 + CAP3 + 256;
    decode_levels(smem, h0, h1, scr + 8u, l3, l2, lane);
    const unsigned tq = (h0 & 0xffu) * 0x01010101u;
    unsigned e[16], m[16], ec = 0;
#pragma unroll
    for (int q = 0; q < 16; ++q) {
        const unsigned d = (r.a[4][q >> 2] >> (2 * (q & 3))) & 0x03030303u;
        m[q] = (d + 0x01010101u) & 0x04040404u; e[q] = tq - d;
    }
#pragma unroll
    for (int k = 0; k < 4; ++k) ec += (unsigned)__builtin_popcount(r.a[4][k] & (r.a[4][k] >> 1) & 0x55555555u);
    apply_escapes<16>(e, m, smem, l2 + wave_scan(ec) - ec);
#pragma unroll
    for (int c = 0; c < 8; ++c) {
        u32x4 o;
#pragma unroll
        for (int h = 0; h < 2; ++h) {
            const int q = 2 * c + h; const unsigned sm = r.a[q >> 2][q & 3];
            const unsigned lo = bfi(0x7f7f7f7fu, sm, e[q] << 7), hi = bfi(0x7f7f7f7fu, e[q] >> 1, sm);
            o[2 * h] = __builtin_amdgcn_perm(hi, lo, 0x05010400u); o[2 * h + 1] = __builtin_amdgcn_perm(hi, lo, 0x07030602u);
        }
        bst16<SC1>(wsr, dst + c * 1024, o);
    }
}

__device__ __forceinline__ unsigned strip_out_off(int r, int s, int lane) {
    const int tile = s >> 3, tm = tile >> 3, tn = tile & 7;
    const int row = (r >> 2) * RROWS + tm * 256 + (s & 7) * 32 + (lane >> 5);
    const int col = (r & 3) * RCOLS + tn * 256 + (lane & 31) * 8;
    return ((unsigned)row * ND + col) * 2u;
}

struct StripRegs { u32x4 mk; u32x4 lo[8]; u32x4 sd[2]; };
__device__ __forceinline__ void strip_issue(StripRegs& r, u32x4 pd, unsigned s, unsigned info, int lane) {
    if (info >> 30) return;
    const unsigned T = info & 0x3fffu, sl = (info >> 14) & 0xffu, slot = OFF_EO + s * SSLOT;
    const unsigned om = slot + lane * 16;
    ALD16(r.mk, pd, om, 0);
    { const unsigned o = (unsigned)lane < sl ? om : OOB; ALD16(r.sd[0], pd, o, 1024); }
    if (sl > 64u) { const unsigned o = (unsigned)lane + 64u < sl ? om : OOB; ALD16(r.sd[1], pd, o, 2048); }
#define STRIP_ROUND(k) if ((unsigned)(k) * 1024u < T) { const bool act = (unsigned)((k) * 1024 + lane * 16) < T; \
        const unsigned ol = act ? om + 3072 + ((k) >> 2) * 4096 : OOB; ALD16(r.lo[k], pd, ol, ((k) & 3) * 1024); }
    STRIP_ROUND(0) STRIP_ROUND(1) STRIP_ROUND(2) STRIP_ROUND(3) STRIP_ROUND(4) STRIP_ROUND(5) STRIP_ROUND(6) STRIP_ROUND(7)
}
#define STRIP_LANDED(r, fn, n, cnt) do { \
    asm volatile("s_waitcnt vmcnt(" #cnt ")" : "+v"(r.lo[0]), "+v"(r.lo[1]), "+v"(r.lo[2]), "+v"(r.lo[3]), "+v"(r.lo[4]), "+v"(r.lo[5]), "+v"(r.lo[6]), "+v"(r.lo[7]), \
        "+v"(r.mk), "+v"(r.sd[0]), "+v"(r.sd[1]), "+v"(fn), "+v"(n) :: "memory"); } while (0)
__device__ __forceinline__ void strip_stage1(const StripRegs& r, LAS unsigned char* smem, unsigned lbase, unsigned scr, unsigned info, int lane) {
    if (info >> 30) return;
    const unsigned T = info & 0x3fffu;
    const unsigned h0 = rfl(r.sd[0].x), h1 = rfl(r.sd[0].y);
    *(LAS u32x4*)(smem + scr + lane * 16) = r.sd[0];
    *(LAS u32x4*)(smem + scr + 1024 + lane * 16) = r.sd[1];
    const unsigned l3 = scr + 2048, l2 = l3 + CAP3 + 256;
    decode_levels(smem, h0, h1, scr + 8u + ((T + 31u) >> 5) * 4u, l3, l2, lane);
    const unsigned tq = (h0 & 0xffu) * 0x01010101u;
    unsigned x1[8], ec = 0;
#pragma unroll
    for (int k = 0; k < 8; ++k) {
        if ((unsigned)k * 1024u < T) {
            const unsigned z = __builtin_amdgcn_perm(0u, (unsigned)*(const LAS u16*)(smem + scr + 8 + k * 128 + lane * 2), 0x01010000u);
            x1[k] = (z & 0x000f000fu) | ((z >> 4) & 0x0f000f00u);
#pragma unroll
            for (int i = 0; i < 4; ++i) ec += (unsigned)__builtin_popcount((x1[k] >> i) & (r.lo[k][i] >> 7) & 0x01010101u);
        }
    }
    unsigned eo = l2 + wave_scan(ec) - ec;
    unsigned e[8][4];
#pragma unroll
    for (int k = 0; k < 8; ++k) {
        if ((unsigned)k * 1024u < T) {
            unsigned m[4];
#pragma unroll
            for (int i = 0; i < 4; ++i) {
                const unsigned d = ((x1[k] >> i) & 0x01010101u) | ((r.lo[k][i] >> 6) & 0x02020202u);
                m[i] = (d + 0x01010101u) & 0x04040404u; e[k][i] = tq - d;
            }
            eo = apply_escapes<4>(e[k], m, smem, eo);
        }
    }
#pragma unroll
    for (int k = 0; k < 8; ++k) {
        if ((unsigned)k * 1024u < T) {
            unsigned lo[4], hi[4];
#pragma unroll
            for (int i = 0; i < 4; ++i) { lo[i] = bfi(0x7f7f7f7fu, r.lo[k][i], e[k][i] << 7); hi[i] = (e[k][i] >> 1) & 0x7f7f7f7fu; }
            *(LAS u32x4*)(smem + lbase + k * 2048 + lane * 32) = (u32x4){lo[0], hi[0], lo[1], hi[1]};
            *(LAS u32x4*)(smem + lbase + k * 2048 + lane * 32 + 16) = (u32x4){lo[2], hi[2], lo[3], hi[3]};
        }
    }
}
__device__ __forceinline__ void strip_stage2(u32x4 mk, rsrc_t outr, rsrc_t por, LAS unsigned char* smem, unsigned lbase, int peer, unsigned s, unsigned info, int lane) {
    const unsigned oo = strip_out_off(peer, (int)s, lane);
    if (info >> 31) {
#pragma unroll
        for (int q = 0; q < 16; ++q) bst16<OAUX>(outr, oo + q * 2 * ND * 2, (u32x4){0x7fc07fc0u, 0x7fc07fc0u, 0x7fc07fc0u, 0x7fc07fc0u});
        return;
    }
    if (info >> 30) {
#pragma unroll
        for (int h = 0; h < 4; ++h) {
            u32x4 a[4];
#pragma unroll
            for (int q = 0; q < 4; ++q) a[q] = bld16<SYS>(por, oo + (h * 4 + q) * 2 * ND * 2);
#pragma unroll
            for (int q = 0; q < 4; ++q) bst16<OAUX>(outr, oo + (h * 4 + q) * 2 * ND * 2, a[q]);
        }
        return;
    }
    unsigned c2[8], x[8];
#pragma unroll
    for (int i = 0; i < 8; ++i) {
        const unsigned m16 = mk[i >> 1] >> ((i & 1) * 16);
        c2[i] = (unsigned)__builtin_popcount(m16 & 0xffu) | ((unsigned)__builtin_popcount((m16 >> 8) & 0xffu) << 16);
        x[i] = c2[i];
    }
#define SCAN_STEP(ctrl, rmask) _Pragma("unroll") for (int i = 0; i < 8; ++i) x[i] += (unsigned)__builtin_amdgcn_update_dpp(0, (int)x[i], ctrl, rmask, 0xf, false);
    SCAN_STEP(0x111, 0xf) SCAN_STEP(0x112, 0xf) SCAN_STEP(0x114, 0xf) SCAN_STEP(0x118, 0xf) SCAN_STEP(0x142, 0xa) SCAN_STEP(0x143, 0xc)
    unsigned pos = 0;
#pragma unroll
    for (int bt = 0; bt < 4; ++bt) {
        u32x4 sel[4], d[8]; unsigned off[8];
#pragma unroll
        for (int k = 0; k < 4; ++k) {
            const int i = bt * 2 + (k >> 1), hq = k & 1;
            const unsigned m8 = (mk[bt] >> (k * 8)) & 0xffu;
            sel[k] = *(const LAS u32x4*)(smem + SM_LUT + m8 * 16u);
            const unsigned tot = __builtin_amdgcn_readlane(x[i], 63);
            off[2 * k] = pos + (((x[i] - c2[i]) >> (hq * 16)) & 0xffffu);
            off[2 * k + 1] = off[2 * k] + (unsigned)__builtin_popcount(m8 & 15u);
            pos += (tot >> (hq * 16)) & 0xffffu;
        }
#pragma unroll
        for (int g = 0; g < 8; ++g) {
            const LAS u32x2* pl = (const LAS u32x2*)(smem + lbase + (off[g] & ~3u) * 2u);
            const u32x2 d0 = pl[0], d1 = pl[1];
            d[g] = (u32x4){d0.x, d0.y, d1.x, d1.y};
        }
#pragma unroll
        for (int k = 0; k < 4; ++k) {
            u32x4 o;
#pragma unroll
            for (int g = 0; g < 2; ++g) {
                const u32x4 dd = d[2 * k + g];
                const unsigned l4 = __builtin_amdgcn_alignbyte(dd.z, dd.x, off[2 * k + g]), h4 = __builtin_amdgcn_alignbyte(dd.w, dd.y, off[2 * k + g]);
                o[2 * g] = __builtin_amdgcn_perm(h4, l4, sel[k][2 * g]); o[2 * g + 1] = __builtin_amdgcn_perm(h4, l4, sel[k][2 * g + 1]);
            }
            bst16<OAUX>(outr, oo + (bt * 4 + k) * 2 * ND * 2, o);
        }
    }
}

__device__ __forceinline__ void pack_strip(const Params& p, rsrc_t wsr, rsrc_t outr, LAS unsigned char* smem, int s, int wid, int lane, unsigned seq) {
    const unsigned oo = strip_out_off(p.rank, s, lane);
    const unsigned lbase = (unsigned)wid * 16384u;
    unsigned pos = lbase, mx = 0, sg = 0, esum = 0, ts = 0;
    const unsigned dummy = SM_DUMMY + lane * 2;
    u32x4 mk = {0u, 0u, 0u, 0u};
#pragma unroll
    for (int q2 = 0; q2 < 8; ++q2) {
        u32x4 v[2]; unsigned m8[2];
#pragma unroll
        for (int hq = 0; hq < 2; ++hq) {
            const int q = q2 * 2 + hq, row = wid * 32 + 2 * q + (lane >> 5);
            v[hq] = *(const LAS u32x4*)(smem + row * 512 + (((lane & 31) ^ (row & 7)) * 16));
            bst16<SYS>(outr, oo + q * 2 * ND * 2, v[hq]);
            unsigned m = 0;
#pragma unroll
            for (int k = 0; k < 4; ++k) {
                sg |= v[hq][k];
                if (q2 < 2) {
                    mx = pkmax(mx, v[hq][k]);
                    esum = __builtin_amdgcn_udot4((v[hq][k] >> 7) & 0x00ff00ffu, 0x00010001u, esum, false);
                }
                const unsigned t = pkmin1(v[hq][k]);
                m |= ((t | (t >> 15)) & 3u) << (2 * k);
            }
            m8[hq] = m;
        }
        mk[q2 >> 1] |= (m8[0] | (m8[1] << 8)) << ((q2 & 1) * 16);
        const unsigned c2 = (unsigned)__builtin_popcount(m8[0]) | ((unsigned)__builtin_popcount(m8[1]) << 16);
        const unsigned inc = wave_scan(c2), exl = inc - c2, tot = __builtin_amdgcn_readlane(inc, 63);
#pragma unroll
        for (int hq = 0; hq < 2; ++hq) {
            unsigned a = pos + ((exl >> (hq * 16)) & 0xffffu) * 2u;
#pragma unroll
            for (int e = 0; e < 8; ++e) {
                const unsigned bit = (m8[hq] >> e) & 1u;
                *(LAS u16*)(smem + (bit ? a : dummy)) = (u16)(v[hq][e >> 1] >> ((e & 1) * 16));
                a += bit * 2u;
            }
            pos += ((tot >> (hq * 16)) & 0xffffu) * 2u;
        }
        if (q2 == 1) ts = (pos - lbase) >> 1;
    }
    const unsigned T = (pos - lbase) >> 1;
    STAMP(13);
    mx = (mx & 0xffffu) > (mx >> 16) ? (mx & 0xffffu) : (mx >> 16);
    const unsigned win = choose_window(wave_sum(esum), ts, (wave_max(mx) >> 7) & 0xffu);
    bool raw = FORCE_RAW_OUT || __builtin_amdgcn_ballot_w64((sg & 0x80008000u) != 0u) != 0ull;
    const unsigned slot = OFF_EO + (unsigned)s * SSLOT;
    unsigned sl = 0;
    if (!raw) {
        const unsigned top = win & 0xffu, top2 = top * 0x00010001u;
        if (lane < 16 && T + lane < 8192u) *(LAS u16*)(smem + lbase + (T + lane) * 2u) = (u16)(top << 7);
        bst16<SYS>(wsr, slot + lane * 16, mk);
        unsigned e4[8][4], m[8][4], ec = 0;
#pragma unroll
        for (int r = 0; r < 8; ++r) {
            if ((unsigned)r * 1024u < T) {
                const bool act = (unsigned)(r * 1024 + lane * 16) < T;
                const u32x4 a = *(const LAS u32x4*)(smem + lbase + r * 2048 + lane * 32), b = *(const LAS u32x4*)(smem + lbase + r * 2048 + lane * 32 + 16);
                const unsigned w8[8] = {a.x, a.y, a.z, a.w, b.x, b.y, b.z, b.w};
                u32x4 lo; unsigned x1 = 0;
#pragma unroll
                for (int k = 0; k < 4; ++k) {
                    const unsigned ea = (w8[2 * k] >> 7) & 0x00ff00ffu, eb = (w8[2 * k + 1] >> 7) & 0x00ff00ffu;
                    const unsigned d = __builtin_amdgcn_perm(pkmin(pksub(top2, eb), 0x00030003u), pkmin(pksub(top2, ea), 0x00030003u), 0x06040200u);
                    e4[r][k] = __builtin_amdgcn_perm(eb, ea, 0x06040200u);
                    lo[k] = bfi(0x7f7f7f7fu, __builtin_amdgcn_perm(w8[2 * k + 1], w8[2 * k], 0x06040200u), d << 6);
                    x1 |= (d & 0x01010101u) << k;
                    m[r][k] = act ? (d + 0x01010101u) & 0x04040404u : 0u; ec += (unsigned)__builtin_popcount(m[r][k]);
                }
                if (act) {
                    bst16<SYS>(wsr, slot + 3072 + r * 1024 + lane * 16, lo);
                    bst2<SYS>(wsr, slot + 1024 + 8 + r * 128 + lane * 2, __builtin_amdgcn_perm(0u, x1 | (x1 >> 4), 0x0c0c0200u));
                }
            }
        }
        STAMP(14);
        const unsigned einc = wave_scan(ec), n2 = __builtin_amdgcn_readlane(einc, 63);
        unsigned h1 = 0xffffffffu;
        const unsigned p1b = ((T + 31u) >> 5) * 4u;
        if (n2 <= CAP2 - 16u) {
            const unsigned l2 = SM_SCR + (unsigned)wid * 3072u, l3 = l2 + CAP2, l4 = l3 + CAP3 + 8;
            unsigned eo = l2 + einc - ec;
#pragma unroll
            for (int r = 0; r < 8; ++r) {
                if ((unsigned)r * 1024u < T) {
#pragma unroll
                    for (int k = 0; k < 4; ++k) eo = append_flagged(smem, e4[r][k], m[r][k], eo, dummy);
                }
            }
            h1 = encode_levels(smem, wsr, win, n2, l2, l3, l4, slot + 1024 + 8 + p1b, 2040u - p1b, dummy, lane);
        }
        if (h1 == 0xffffffffu) raw = true;
        else {
            if (lane == 0) bst8<SYS>(wsr, slot + 1024, (u32x2){win | (n2 << 16), h1});
            sl = (8u + p1b + side_bytes(n2, h1) + 15u) >> 4;
        }
    }
    STAMP(15);
    WAIT_V(0);
    if (lane < NR && lane != p.rank) {
        const unsigned info = raw ? (1u << 30) : (T | (sl << 14));
        char* pw = (char*)p.ws_all[lane];
        __hip_atomic_store((gu64*)(pw + OFF_FOUT + ((unsigned)p.rank * 1024u + (unsigned)s) * 8u), ((u64)seq << 32) | info, __ATOMIC_RELAXED, __HIP_MEMORY_SCOPE_SYSTEM);
    }
}

}

__global__ void __launch_bounds__(512, 2) gemm_ar_fused(Params p) {
    __shared__ __attribute__((aligned(16))) unsigned char smem_[SMEM_BYTES];
    LAS unsigned char* smem = (LAS unsigned char*)smem_;
    const int tid = threadIdx.x, wid = rfl(tid >> 6), lane = tid & 63;
    char* ws = (char*)p.ws_all[p.rank];
    const rsrc_t wsr = make_rsrc(ws, WS_USED);
    const rsrc_t outr = make_rsrc(p.out, OUT_BYTES);
    gu32* tmo = (gu32*)(ws + OFF_TMO);
    gu32* seqp = (gu32*)(ws + OFF_SEQ + (unsigned)(blockIdx.x * 8 + wid) * 4u);
    const unsigned seqv = __hip_atomic_load(seqp, __ATOMIC_RELAXED, __HIP_MEMORY_SCOPE_AGENT);
    STAMP(0);
    const int xcd = blockIdx.x & 7, wi = blockIdx.x >> 3;
    const int nsplit = xcd < 2 ? 16 : (xcd == 2 ? 8 : (xcd == 3 ? 4 : 0));
    const bool is_gemm = wi < 16 + nsplit;
    const int cw = xcd >= 4 ? (wi - 16) * 4 + xcd - 4 : (xcd == 3 ? 64 + wi - 20 : 76 + wi - 24);
    const bool is_link = !is_gemm && wid < NWV;
    if (!is_gemm) {
        if (tid == 0) { *(LAS unsigned*)(smem + SM_TOKEN) = 0u; *(LAS unsigned*)(smem + SM_TOKEN + 4) = 0xffffffffu; }
        WAIT_L(0); BAR;
    }
    if (!is_link) {
        Order od{0, 0};
        const int e = !is_gemm ? (wid - NWV) * 84 + cw
                    : xcd >= 4 ? NPD + (xcd >= 6 ? 0 : 256) + wid * 32 + wi * 2 + (xcd & 1)
                    : xcd < 2 ? NENC + 256 + wid * 64 + wi * 2 + xcd
                    : wi < 16 ? NENC + wid * 32 + wi * 2 + (xcd & 1) : NENC + 768 + wid * 12 + (xcd == 2 ? wi - 16 : wi - 8);
        if (e >= PDELAY) __builtin_amdgcn_s_sleep(PSLEEP);
        for (int t = e; t < 4096; t = (t == e ? NENC + NONCE + e : t + NENC)) {
            int j, g, u; od.locate<128, 32>(t, j, g, u);
            pack_unit(p, ws, wsr, smem, (unsigned)wid * 16384u, j, g, u, lane, seqv);
            if (e >= NENC) break;
        }
        STAMP(7);
        if (is_gemm) __syncthreads();
    }
    const unsigned seq = rfl(seqv) + 1u;
    if (lane == 0) __hip_atomic_store(seqp, seq, __ATOMIC_RELAXED, __HIP_MEMORY_SCOPE_AGENT);
    if (!is_gemm) {
        if (!is_link) return;
        const int lq = cw % 7;
        const int peer = (p.rank + 1 + lq) & 7;
        const rsrc_t pr = make_rsrc(p.ws_all[peer], WS_USED);
        const rsrc_t por = make_rsrc(p.out_all[peer], OUT_BYTES);
#pragma unroll
        for (int i = 0; i < 4; ++i) {
            const unsigned m8 = (unsigned)(i * 64 + lane); u32x4 sel;
#pragma unroll
            for (int g = 0; g < 2; ++g) {
                unsigned rk = 0;
#pragma unroll
                for (int h = 0; h < 2; ++h) {
                    unsigned w = 0;
#pragma unroll
                    for (int e = 0; e < 2; ++e) { const unsigned bit = (m8 >> (g * 4 + h * 2 + e)) & 1u; w |= (bit ? (rk | ((rk + 4u) << 8)) : 0x0c0cu) << (16 * e); rk += bit; }
                    sel[g * 2 + h] = w;
                }
            }
            *(LAS u32x4*)(smem + SM_LUT + m8 * 16u) = sel;
        }
        if (lane < 16) {
            unsigned rk = 4u, sel = 0;
#pragma unroll
            for (int k = 0; k < 4; ++k) { const unsigned bit = ((unsigned)lane >> k) & 1u; sel |= (bit ? rk : (unsigned)k) << (8 * k); rk += bit; }
            *(LAS unsigned*)(smem + SM_LUTE + lane * 4) = sel;
        }
        const unsigned esc = (unsigned)wid * 16384u + 8192u;
        bool dead = false;
        const u32x4 pd = make_desc(p.ws_all[peer], WS_USED);
        const unsigned one = 1u;
        LAS unsigned* token = (LAS unsigned*)(smem + SM_TOKEN);
#define QNEXT(dst) do { if (lane == 0) asm volatile("global_atomic_add %0, %1, %2, off sc0" : "+v"(dst) : "v"(qp), "v"(one) : "memory"); } while (0)
#define TOKEN_TAKE() do { for (;;) { unsigned old = 1u; \
            if (lane == 0) old = __hip_atomic_exchange(token, 1u, __ATOMIC_RELAXED, __HIP_MEMORY_SCOPE_WORKGROUP); \
            if (rfl(old) == 0u) break; __builtin_amdgcn_s_sleep(1); } } while (0)
#define TOKEN_GIVE() do { if (lane == 0) __hip_atomic_store(token, 0u, __ATOMIC_RELAXED, __HIP_MEMORY_SCOPE_WORKGROUP); } while (0)
        {
            gu32* qp = (gu32*)(ws + OFF_Q + (unsigned)lq * 128u);
            const unsigned qbase = (seq - 1u) * (1536u - NSKIP + NT) - NSKIP;
            LAS unsigned* utok = token + 1;
            gu64* finp = (gu64*)(ws + OFF_FIN + (unsigned)peer * 512u + (unsigned)lane * 8u);
            Order od{0, 0};
            u64 rdy0 = 0, rdy1 = 0, fv = 0;
            unsigned nq = 0;
            UnitRegs ur = {};
            if (wid == 0) {
                QNEXT(nq);
                UNIT_LANDED(ur, nq);
                if (lane == 0) __hip_atomic_store(utok, nq - qbase, __ATOMIC_RELAXED, __HIP_MEMORY_SCOPE_WORKGROUP);
            }
            unsigned pfo = 0;
#define PUBLISH(n) do { if (pfo) { WAIT_V(n); if (lane == 0) __hip_atomic_store((gu32*)(ws + pfo), seq, __ATOMIC_RELAXED, __HIP_MEMORY_SCOPE_AGENT); pfo = 0; } } while (0)
            for (;;) {
                unsigned i;
                for (;;) {
                    unsigned old = 0xffffffffu;
                    if (lane == 0) old = __hip_atomic_exchange(utok, 0xffffffffu, __ATOMIC_RELAXED, __HIP_MEMORY_SCOPE_WORKGROUP);
                    i = rfl(old);
                    if (i != 0xffffffffu) break;
                    if (pfo) PUBLISH(0); else __builtin_amdgcn_s_sleep(1);
                }
                if (i >= 1536u) { if (lane == 0) __hip_atomic_store(utok, i, __ATOMIC_RELAXED, __HIP_MEMORY_SCOPE_WORKGROUP); break; }
                QNEXT(nq);
                int j, g, u; od.locate<32, 16>((int)i, j, g, u);
                const int rb = j * 4 + (g >> 1);
                if (!((((g & 1) ? rdy1 : rdy0) >> rb) & 1ull) && !dead) {
                    PUBLISH(0);
                    const bool was0 = (rdy0 | rdy1) == 0ull;
                    const u64 t0 = __builtin_amdgcn_s_memrealtime(); unsigned n = 0;
                    for (;;) {
                        const u64 f2 = __hip_atomic_load(finp, __ATOMIC_RELAXED, __HIP_MEMORY_SCOPE_SYSTEM);
                        rdy0 = __builtin_amdgcn_ballot_w64((unsigned)f2 >= seq); rdy1 = __builtin_amdgcn_ballot_w64((unsigned)(f2 >> 32) >= seq);
                        if ((((g & 1) ? rdy1 : rdy0) >> rb) & 1ull) break;
                        if ((++n & 63u) == 0u && give_up(tmo, seq, t0)) { dead = true; break; }
                        __builtin_amdgcn_s_sleep(2);
                    }
                    __builtin_amdgcn_fence(__ATOMIC_ACQUIRE, "wavefront");
                    if (was0) STAMP(1);
                }
                const int step = j * 8 + 1 + lq, li = g ? (g - 1) * 16 + u : u;
                const unsigned slot0 = unit_slot(p.rank, j, g, u);
                const unsigned dst0 = g ? OFF_XL + (unsigned)(step * 64 + li) * 8192u : OFF_WL + (unsigned)(step * 32 + li) * 8192u;
                const bool refresh = (rdy0 | 3ull) != 0x7777777777777777ull || (rdy1 | 1ull) != 0x3333333333333333ull;
                for (;;) {
                    unsigned old = 1u;
                    if (lane == 0) old = __hip_atomic_exchange(token, 1u, __ATOMIC_RELAXED, __HIP_MEMORY_SCOPE_WORKGROUP);
                    if (rfl(old) == 0u) break;
                    if (pfo) PUBLISH(0); else __builtin_amdgcn_s_sleep(1);
                }
                unit_issue(ur, pd, slot0, lane);
                asm volatile("s_waitcnt vmcnt(6)" : "+v"(nq) :: "memory");
                if (lane == 0) __hip_atomic_store(utok, nq - qbase, __ATOMIC_RELAXED, __HIP_MEMORY_SCOPE_WORKGROUP);
                UNIT_LANDED(ur, nq);
                TOKEN_GIVE();
                PUBLISH(0);
                if (refresh) asm volatile("global_load_dwordx2 %0, %1, off sc0 sc1" : "+v"(fv) : "v"(finp) : "memory");
                pfo = (g ? OFF_XF : OFF_WF) + (unsigned)(((li >> 2) * NSTEP + step) * 4 + (li & 3)) * 4u;
                unit_unpack(ur, wsr, pr, smem, esc, slot0, dst0, lane);
                if (refresh) {
                    asm volatile("s_waitcnt vmcnt(8)" : "+v"(fv) :: "memory");
                    rdy0 = __builtin_amdgcn_ballot_w64((unsigned)fv >= seq); rdy1 = __builtin_amdgcn_ballot_w64((unsigned)(fv >> 32) >= seq);
                }
            }
            PUBLISH(0);
            STAMP(3);
        }
        {
            gu32* qp = (gu32*)(ws + OFF_Q + (unsigned)(7 + lq) * 128u);
            const unsigned qbase = (seq - 1u) * (1024u + NT * NWV);
            gu64* fbase = (gu64*)(ws + OFF_FOUT + (unsigned)peer * 8192u);
            const unsigned lbase = (unsigned)wid * 16384u;
            unsigned nq = 0; u64 fn = 0;
            StripRegs sr = {};
            QNEXT(nq);
            asm volatile("s_waitcnt vmcnt(0)" : "+v"(nq) :: "memory");
            unsigned s = rfl(nq) - qbase;
            bool first = true;
            while (s < 1024u) {
                unsigned fh = rfl((unsigned)(fn >> 32)), info = rfl((unsigned)fn);
                if (fh < seq && !dead) {
                    const u64 t0 = __builtin_amdgcn_s_memrealtime(); unsigned n = 0;
                    for (;;) {
                        const u64 f2 = __hip_atomic_load(fbase + s, __ATOMIC_RELAXED, __HIP_MEMORY_SCOPE_SYSTEM);
                        fh = rfl((unsigned)(f2 >> 32)); info = rfl((unsigned)f2);
                        if (fh >= seq) break;
                        if ((++n & 63u) == 0u && give_up(tmo, seq, t0)) { dead = true; break; }
                        __builtin_amdgcn_s_sleep(4);
                    }
                }
                __builtin_amdgcn_fence(__ATOMIC_ACQUIRE, "wavefront");
                if (dead) info = 3u << 30;
                if (first) STAMP(4);
                first = false;
                TOKEN_TAKE();
                strip_issue(sr, pd, s, info, lane);
                QNEXT(nq);
                STRIP_LANDED(sr, fn, nq, 0);
                TOKEN_GIVE();
                const unsigned sn = rfl(nq) - qbase;
                { gu64* fa = fbase + (sn & 1023u); asm volatile("global_load_dwordx2 %0, %1, off sc0 sc1" : "+v"(fn) : "v"(fa) : "memory"); }
                strip_stage1(sr, smem, lbase, esc, info, lane);
                strip_stage2(sr.mk, outr, por, smem, lbase, peer, s, info, lane);
                asm volatile("s_waitcnt vmcnt(16)" : "+v"(fn) :: "memory");
                s = sn;
            }
            STAMP(6);
        }
        return;
    }

    const bool split = (wi & 15) < nsplit;
    const int wr = wid >> 2, wc = wid & 3;
    constexpr int HTB = 16384;
    constexpr size_t ASTEP = 64 * 8192, BSTEP = 32 * 8192;
    const unsigned ldsw = (unsigned)wid * 1024u;
    const unsigned smbase = (unsigned)(uintptr_t)smem;
#define SA(b, h) (((b) * 2 + (h)) * HTB)
#define SB(b, h) ((4 + (b) * 2 + (h)) * HTB)
#define STAGE(bufoff, gptr) do { _Pragma("unroll") for (int _i = 0; _i < 2; ++_i) \
        __builtin_amdgcn_global_load_lds((const unsigned*)((gptr) + voff[_i]), (LAS unsigned*)(smem + (bufoff) + ldsw + _i * 8192), 16, 0, SYS); } while (0)
#define LDA(dst, b, h) do { _Pragma("unroll") for (int m = 0; m < 4; ++m) _Pragma("unroll") for (int k = 0; k < 2; ++k) dst[m][k] = *(const LAS bf16x8*)(smem + SA(b, h) + aoff + m * 2048 + k * 1024); } while (0)
#define LDB(dst, b, h) do { _Pragma("unroll") for (int n = 0; n < 2; ++n) _Pragma("unroll") for (int k = 0; k < 2; ++k) dst[n][k] = *(const LAS bf16x8*)(smem + SB(b, h) + boff + n * 2048 + k * 1024); } while (0)
#define MMA(ai, bj, At, Bt) do { __builtin_amdgcn_s_setprio(1); _Pragma("unroll") for (int m = 0; m < 4; ++m) _Pragma("unroll") for (int n = 0; n < 2; ++n) _Pragma("unroll") for (int k = 0; k < 2; ++k) \
        acc[ai][bj][m][n] = __builtin_amdgcn_mfma_f32_16x16x32_bf16(Bt[n][k], At[m][k], acc[ai][bj][m][n], 0, 0, 0); __builtin_amdgcn_s_setprio(0); } while (0)
#define ZERO_ACC(a) _Pragma("unroll") for (int b = 0; b < 2; ++b) _Pragma("unroll") for (int m = 0; m < 4; ++m) _Pragma("unroll") for (int n = 0; n < 2; ++n) acc[a][b][m][n] = (f32x4){0.f, 0.f, 0.f, 0.f};
#define NEED(s) do { if (__builtin_expect((avail & 0xffffu) <= (unsigned)(s), 0)) { \
        if (wid == 0) { const unsigned s0 = avail & 0xffffu; unsigned av; const u64 t0 = __builtin_amdgcn_s_memrealtime(); unsigned n = 0; \
            const unsigned pn = (s0 + (lane & 31)) & 127u, po = pollo + SMAP(pn) * 16u; \
            for (;;) { const u32x4 f = bld16<SYS>(wsr, po); \
                const bool ok = (s0 + (lane & 31) >= (unsigned)te) || (f.x >= seq && f.y >= seq && f.z >= seq && f.w >= seq); \
                const u64 mm = __builtin_amdgcn_ballot_w64(ok); const unsigned both = (unsigned)mm & (unsigned)(mm >> 32); \
                av = s0 + (both == 0xffffffffu ? 32u : (unsigned)__builtin_ctz(~both)); \
                if (av > (unsigned)(s)) break; \
                if ((++n & 63u) == 0u && give_up(tmo, seq, t0)) { av = 0x10000u | NSTEP; break; } \
                __builtin_amdgcn_s_sleep(1); } \
            asm volatile("ds_write_b32 %0, %1\n\ts_waitcnt lgkmcnt(0)" :: "v"(smbase + SM_AVAIL), "v"(av) : "memory"); } \
        CFENCE; BAR; CFENCE; \
        unsigned av2; asm volatile("ds_read_b32 %0, %1\n\ts_waitcnt lgkmcnt(0)" : "=v"(av2) : "v"(smbase + SM_AVAIL) : "memory"); \
        avail = rfl(av2); } } while (0)
#define LOOP_STAMPS do { } while (0)
#define OPAQUE_U(dst, src) unsigned dst##_v = (unsigned)(src); asm volatile("" : "+v"(dst##_v)); const unsigned dst = rfl(dst##_v);
    for (int job = 0; job < 2; ++job) {
        int tid = threadIdx.x; asm volatile("" : "+v"(tid));
        const int lane = tid & 63, fr = lane & 15, fq = lane >> 4;
        const bool pair = split && job == 0;
        const int hsel = wi >> 4;
        const int atile = 44 + (xcd < 2 ? xcd * 32 + wi : (xcd == 2 ? 64 : 80) + (wi & 15) + nsplit * hsel);
        if (job && atile >= 128) return;
        const int tm = job ? atile >> 3 : (xcd >> 1) * 4 + ((wi & 15) >> 2), tn = job ? atile & 7 : (xcd & 1) * 4 + (wi & 3);
        const char* cA = ws + OFF_XL + (size_t)tm * 32768;
        const char* cB = ws + OFF_WL + (size_t)tn * 32768;
        const unsigned voff[2] = {(unsigned)tid * 16u, (unsigned)tid * 16u + 8192u};
        const int aoff = lds_byte(wr * 64 + fr, fq * 8), boff = lds_byte(wc * 32 + fr, fq * 8);
        const unsigned pollo = lane < 32 ? OFF_XF + (unsigned)(tm * NSTEP) * 16u : OFF_WF + (unsigned)(tn * NSTEP) * 16u;
        const int kh = tm < 8 ? KH1 : (tm < 12 ? KH2 : KH3);
        const int tb = job ? NSTEP - kh : 0, te = pair ? 64 : (job ? NSTEP : NSTEP - kh);
        const unsigned nl = tm < 8 ? (unsigned)NL1 : 16u;
        auto smap = [nl, pair, hsel](unsigned n) -> unsigned {
            if (pair) return (((n & ~3u) << 1) | (unsigned)(hsel << 2) | (n & 3u)) & 127u;
            if (n < nl) return n * 8u;
            const unsigned m = n - nl, full = (16u - nl) * 8u;
            if (m < full) return (m & 7u) == 7u ? (nl + (m >> 3)) * 8u : (m & ~7u) + 1u + (m & 7u);
            const unsigned r = m - full; return (16u - nl + r / 7u) * 8u + 1u + r % 7u;
        };
        const unsigned stab0 = smap((unsigned)lane), stab1 = smap((unsigned)lane + 64u);
#define SMAP(n) smap(n)
#define STAB(n) ((unsigned)__builtin_amdgcn_readlane((n) & 64 ? stab1 : stab0, (n) & 63))
        unsigned avail = (unsigned)tb;
        f32x4 acc[2][2][4][2];
        ZERO_ACC(0) ZERO_ACC(1)
        bf16x8 At[4][2], B0[2][2], B1[2][2];
        NEED(tb + 1);
        if (!job) STAMP(2);
        { const size_t s0 = STAB(tb); STAGE(SB(0, 0), cB + s0 * BSTEP); STAGE(SA(0, 0), cA + s0 * ASTEP); STAGE(SB(0, 1), cB + s0 * BSTEP + HTB); STAGE(SA(0, 1), cA + s0 * ASTEP + HTB); }
        if (wr == 1) BAR;
        WAIT_V(4); BAR;
        unsigned sp = STAB(tb + 1);
        { const size_t s1 = sp; STAGE(SB(1, 0), cB + s1 * BSTEP); STAGE(SA(1, 0), cA + s1 * ASTEP); STAGE(SB(1, 1), cB + s1 * BSTEP + HTB); }
        WAIT_V(6); BAR;
        for (int t = tb; t < te; t += 2) {
            const bool last = (t == te - 2);
            const int tn2 = last ? tb : t + 2;
            const unsigned s1 = sp, s2 = STAB(tn2), s3 = STAB(tn2 + 1);
            sp = s3;
            const char* a1 = cA + s1 * ASTEP;
            const char* a2 = cA + s2 * ASTEP; const char* b2 = cB + s2 * BSTEP;
            const char* a3 = cA + s3 * ASTEP; const char* b3 = cB + s3 * BSTEP;
            if (!last) NEED(t + 3);
            LOOP_STAMPS;
            LDB(B0, 0, 0); SCHED; LDA(At, 0, 0); STAGE(SA(1, 1), a1 + HTB);
            WAIT_L(8); BAR; WAIT_L(0); MMA(0, 0, At, B0); BAR; SCHED;
            LDB(B1, 0, 1); STAGE(SB(0, 0), b2);
            BAR; WAIT_L(0); MMA(0, 1, At, B1); BAR;
            LDA(At, 0, 1); STAGE(SA(0, 0), a2);
            BAR; WAIT_L(0); MMA(1, 0, At, B0); BAR; SCHED;
            STAGE(SB(0, 1), b2 + HTB);
            WAIT_V(6); BAR; MMA(1, 1, At, B1); BAR;
            LDB(B0, 1, 0); SCHED; LDA(At, 1, 0); STAGE(SA(0, 1), a2 + HTB);
            WAIT_L(8); BAR; WAIT_L(0); MMA(0, 0, At, B0); BAR; SCHED;
            LDB(B1, 1, 1); STAGE(SB(1, 0), b3);
            BAR; WAIT_L(0); MMA(0, 1, At, B1); BAR;
            LDA(At, 1, 1); STAGE(SA(1, 0), a3);
            BAR; WAIT_L(0); MMA(1, 0, At, B0); BAR; SCHED;
            STAGE(SB(1, 1), b3 + HTB);
            WAIT_V(6); BAR; MMA(1, 1, At, B1); BAR;
        }
        if (wr == 0) BAR;
        if (!job) STAMP(12);
        int tz = threadIdx.x; asm volatile("" : "+v"(tz));
        const int lane2 = tz & 63;
        const unsigned po = OFF_PART + (unsigned)(tm * 8 + tn) * 262144u + (unsigned)wid * 32768u + (unsigned)lane2 * 16u;
        gu32* pfl = (gu32*)(ws + OFF_PF + (unsigned)((tm * 8 + tn) * 16 + wid) * 4u);
        const unsigned give = job ? 3u : (pair ? (hsel ? 1u : 2u) : 0u), take = job ? 0u : (pair ? (hsel ? 2u : 1u) : 3u);
#pragma unroll
        for (int h = 0; h < 2; ++h) {
            OPAQUE_U(g1, (give >> h) & 1u)
            if (g1) {
#pragma unroll
                for (int q = 0; q < 16; ++q) bst16<SC1>(wsr, po + (h * 16 + q) * 1024, __builtin_bit_cast(u32x4, acc[h][(q >> 3) & 1][(q >> 1) & 3][q & 1]));
                WAIT_V(0);
                if (lane2 == 0) __hip_atomic_store(pfl + h * 8, seq, __ATOMIC_RELAXED, __HIP_MEMORY_SCOPE_AGENT);
            }
        }
        if (job) { STAMP(11); return; }
#pragma unroll
        for (int h = 0; h < 2; ++h) {
            OPAQUE_U(t1, (take >> h) & 1u)
            if (t1) {
                if (rfl(__hip_atomic_load(pfl + h * 8, __ATOMIC_RELAXED, __HIP_MEMORY_SCOPE_AGENT)) < seq) {
                    const u64 t0 = __builtin_amdgcn_s_memrealtime(); unsigned n = 0;
                    while (rfl(__hip_atomic_load(pfl + h * 8, __ATOMIC_RELAXED, __HIP_MEMORY_SCOPE_AGENT)) < seq) {
                        if ((++n & 63u) == 0u && give_up(tmo, seq, t0)) break;
                        __builtin_amdgcn_s_sleep(1);
                    }
                }
                __builtin_amdgcn_fence(__ATOMIC_ACQUIRE, "wavefront");
                u32x4 pv[16];
#pragma unroll
                for (int q = 0; q < 16; ++q) pv[q] = bld16<SC1>(wsr, po + (h * 16 + q) * 1024);
#pragma unroll
                for (int q = 0; q < 16; ++q) { acc[h][(q >> 3) & 1][(q >> 1) & 3][q & 1] += __builtin_bit_cast(f32x4, pv[q]);
                    asm volatile("" : "+v"(acc[h][(q >> 3) & 1][(q >> 1) & 3][q & 1])); }
                CFENCE; SCHED;
            }
        }
        const unsigned tmov = __hip_atomic_load(tmo, __ATOMIC_RELAXED, __HIP_MEMORY_SCOPE_AGENT);
        WAIT_V(0); BAR;
        STAMP(3);
        const int fr2 = lane2 & 15, fq2 = lane2 >> 4;
        const bool dead = (avail >> 16) != 0u || rfl(tmov) == seq;
#pragma unroll
        for (int ai = 0; ai < 2; ++ai) {
            OPAQUE_U(t1, (take >> ai) & 1u)
            if (t1) {
#pragma unroll
                for (int m = 0; m < 4; ++m) {
                    const int row = ai * 128 + wr * 64 + m * 16 + fr2;
#pragma unroll
                    for (int bj = 0; bj < 2; ++bj) {
                        f32x4 v0 = acc[ai][bj][m][0], v1 = acc[ai][bj][m][1];
#pragma unroll
                        for (int k = 0; k < 4; ++k) { v0[k] = v0[k] <= 0.f ? 0.f : v0[k]; v1[k] = v1[k] <= 0.f ? 0.f : v1[k]; }
                        u32x4 w; w.x = cvt_pk_bf16(v0[0], v0[1]); w.y = cvt_pk_bf16(v0[2], v0[3]); w.z = cvt_pk_bf16(v1[0], v1[1]); w.w = cvt_pk_bf16(v1[2], v1[3]);
                        if (dead) w = (u32x4){0x7fc07fc0u, 0x7fc07fc0u, 0x7fc07fc0u, 0x7fc07fc0u};
                        const int chunk = bj * 16 + wc * 4 + fq2;
                        *(LAS u32x4*)(smem + row * 512 + ((chunk ^ (row & 7)) * 16)) = w;
                    }
                }
            }
        }
        __syncthreads();
        STAMP(4);
        if ((take >> wr) & 1u) pack_strip(p, wsr, outr, smem, (tm * 8 + tn) * 8 + wid, wid, lane2, seq);
        STAMP(5);
        if (!split) return;
        __syncthreads();
    }
}

extern "C" void kernel_launch(int rank, int nranks, ncclComm_t comm,
                              hipStream_t stream,
                              void* const* d_in, const int* in_sizes, int n_in,
                              void* d_out, int out_size,
                              const int* in_dtypes, int out_dtype,
                              void* const* d_out_all,
                              void* const* d_ws_all, size_t ws_bytes) {
    (void)comm; (void)in_dtypes; (void)out_dtype;
    if (nranks != NR || n_in != 2 || (unsigned)rank >= (unsigned)NR || ws_bytes < WS_USED || in_sizes[0] != ND * KS || in_sizes[1] != KS * ND || out_size != ND * ND) {
        fprintf(stderr, "kernel_launch: unexpected problem shape\n");
        return;
    }
    Params p;
    p.x = (const u16*)d_in[0]; p.w = (const u16*)d_in[1]; p.out = (u16*)d_out;
    p.out_all = d_out_all; p.ws_all = d_ws_all; p.rank = rank;
    hipLaunchKernelGGL(gemm_ar_fused, dim3(NWG), dim3(512), 0, stream, p);
    HIP_CHECK(hipGetLastError());
}
```

```cpp
#include <hip/hip_runtime.h>
#include <rccl/rccl.h>
#include <cstdio>
#include <cstdint>

#define HIP_CHECK(expr) do { hipError_t _e = (expr); if (_e != hipSuccess) { \
    fprintf(stderr, "HIP error %s at %s:%d\n", hipGetErrorString(_e), __FILE__, __LINE__); \
    return; } } while (0)

#define LAS __attribute__((address_space(3)))
#define GAS __attribute__((address_space(1)))
typedef unsigned short u16;
typedef unsigned long long u64;
typedef short bf16x8 __attribute__((ext_vector_type(8)));
typedef short s16x4 __attribute__((ext_vector_type(4)));
typedef float f32x4 __attribute__((ext_vector_type(4)));
typedef unsigned u32x4 __attribute__((ext_vector_type(4)));
typedef unsigned u32x2 __attribute__((ext_vector_type(2)));
typedef u16 u16x2 __attribute__((ext_vector_type(2)));
typedef GAS unsigned gu32;
typedef GAS u64 gu64;

#define FORCE_RAW_IN 0
#define FORCE_RAW_OUT 0
#define NT 12
#define OAUX 18
#define KH1 12
#define KH2 24
#define KH3 40
#define NL1 4
#define PDELAY 336
#define PSLEEP 64
constexpr int NWV = 4;
#define NLW 6
#define LAG1 0
#define LAG2 9
#define LAG3 13

namespace {

constexpr int NR = 8;
constexpr int ND = 8192;
constexpr int KS = 1024;
constexpr int RROWS = 4096, RCOLS = 2048;
constexpr int NSTEP = 128;
constexpr int NWG = 256;
constexpr int NPD = 84 * 4;
constexpr int NENC = NPD + 512;
constexpr int NONCE = 256 + 512 + 96;
static_assert(NT == 12, "wave roles");
__device__ __forceinline__ constexpr int lag(int k) { return k == 0 ? 0 : k == 1 ? LAG1 : k == 2 ? LAG2 : LAG3; }
constexpr unsigned long long DEADLINE = 40000000ull;

constexpr unsigned OFF_TMO = 0x80;
constexpr unsigned OFF_Q = 0x400;
constexpr unsigned OFF_ECNT = 0x1000;
constexpr unsigned OFF_FIN = 0x8000;
constexpr unsigned OFF_SEQ = 0xa000;
constexpr unsigned OFF_PF = 0xc000;
constexpr unsigned OFF_XF = 0x10000;
constexpr unsigned OFF_WF = 0x18000;
constexpr unsigned OFF_FOUT = 0x20000;
constexpr unsigned USLOT = 8192 + 1024;
constexpr unsigned OFF_EX = 1u << 20;
constexpr unsigned OFF_EW = OFF_EX + (18u << 20);
constexpr unsigned OFF_XL = OFF_EW + (18u << 20);
constexpr unsigned OFF_WL = OFF_XL + (64u << 20);
constexpr unsigned SSLOT = 11264;
constexpr unsigned OFF_EO = OFF_WL + (32u << 20);
constexpr unsigned OFF_PART = OFF_EO + (13u << 20);
constexpr unsigned WS_USED = OFF_PART + (32u << 20);
static_assert(2048u * USLOT <= (18u << 20), "unit area");
static_assert(1024u * SSLOT <= (13u << 20), "strip area");
constexpr unsigned OUT_BYTES = (unsigned)ND * ND * 2u;

constexpr int SM_TOKEN = 131072 + 32, SM_DUMMY = 131072 + 64, SM_LUT = 131072 + 256, SM_LUTE = SM_LUT + 4096, SM_SCR = SM_LUTE + 128;
constexpr int SM_AVAIL = SM_SCR + 24576 + 64;
constexpr int SMEM_BYTES = SM_AVAIL + 16;
constexpr unsigned CAP2 = 2304, CAP3 = 512, CAP4 = 240;

struct Params {
    const u16* x; const u16* w; u16* out;
    void* const* out_all; void* const* ws_all;
    int rank;
};

typedef __amdgpu_buffer_rsrc_t rsrc_t;
__device__ __forceinline__ rsrc_t make_rsrc(const void* base, unsigned bytes) {
    const u64 b = (u64)base;
    const unsigned lo = __builtin_amdgcn_readfirstlane((unsigned)b), hi = __builtin_amdgcn_readfirstlane((unsigned)(b >> 32));
    return __builtin_amdgcn_make_buffer_rsrc((void*)(((u64)hi << 32) | lo), 0, bytes, 0x00020000);
}
constexpr int SC1 = 16, SYS = 17;
template <int AUX> __device__ __forceinline__ u32x4 bld16(rsrc_t r, unsigned off) { return __builtin_amdgcn_raw_buffer_load_b128(r, off, 0, AUX); }
template <int AUX> __device__ __forceinline__ unsigned bld4(rsrc_t r, unsigned off) { return __builtin_amdgcn_raw_buffer_load_b32(r, off, 0, AUX); }
template <int AUX> __device__ __forceinline__ unsigned bld2(rsrc_t r, unsigned off) { return __builtin_amdgcn_raw_buffer_load_b16(r, off, 0, AUX); }
template <int AUX> __device__ __forceinline__ void bst16(rsrc_t r, unsigned off, u32x4 v) { __builtin_amdgcn_raw_buffer_store_b128(v, r, off, 0, AUX); }
template <int AUX> __device__ __forceinline__ void bst8(rsrc_t r, unsigned off, u32x2 v) { __builtin_amdgcn_raw_buffer_store_b64(v, r, off, 0, AUX); }
template <int AUX> __device__ __forceinline__ void bst4(rsrc_t r, unsigned off, unsigned v) { __builtin_amdgcn_raw_buffer_store_b32(v, r, off, 0, AUX); }
template <int AUX> __device__ __forceinline__ void bst2(rsrc_t r, unsigned off, unsigned v) { __builtin_amdgcn_raw_buffer_store_b16((u16)v, r, off, 0, AUX); }

__device__ __forceinline__ u32x4 make_desc(const void* base, unsigned bytes) {
    const u64 b = (u64)base;
    return (u32x4){(unsigned)__builtin_amdgcn_readfirstlane((unsigned)b), (unsigned)__builtin_amdgcn_readfirstlane((unsigned)(b >> 32)), bytes, 0x00020000u};
}
#define ALD16(dst, desc, off, imm) asm volatile("buffer_load_dwordx4 %0, %1, %2, 0 offen offset:%3 sc0 sc1" : "+v"(dst) : "v"(off), "s"(desc), "i"(imm) : "memory")
#define ALD8(dst, desc, off, imm) asm volatile("buffer_load_dwordx2 %0, %1, %2, 0 offen offset:%3 sc0 sc1" : "+v"(dst) : "v"(off), "s"(desc), "i"(imm) : "memory")
#define ALD4(dst, desc, off, imm) asm volatile("buffer_load_dword %0, %1, %2, 0 offen offset:%3 sc0 sc1" : "+v"(dst) : "v"(off), "s"(desc), "i"(imm) : "memory")
#define ALD2(dst, desc, off, imm) asm volatile("buffer_load_ushort %0, %1, %2, 0 offen offset:%3 sc0 sc1" : "+v"(dst) : "v"(off), "s"(desc), "i"(imm) : "memory")
constexpr unsigned OOB = 0xfffff000u;

#define WAIT_V(n) asm volatile("s_waitcnt vmcnt(" #n ")" ::: "memory")
#define WAIT_L(n) asm volatile("s_waitcnt lgkmcnt(" #n ")" ::: "memory")
#define BAR __builtin_amdgcn_s_barrier()
#define SCHED __builtin_amdgcn_sched_barrier(0)
#define CFENCE asm volatile("" ::: "memory")
#define STAMP(k) do { } while (0)

__device__ __forceinline__ unsigned rfl(unsigned v) { return __builtin_amdgcn_readfirstlane(v); }
__device__ __forceinline__ unsigned pkmax(unsigned a, unsigned b) {
    u16x2 x = __builtin_bit_cast(u16x2, a), y = __builtin_bit_cast(u16x2, b);
    return __builtin_bit_cast(unsigned, __builtin_elementwise_max(x, y));
}
__device__ __forceinline__ unsigned pkmin(unsigned a, unsigned b) {
    u16x2 x = __builtin_bit_cast(u16x2, a), y = __builtin_bit_cast(u16x2, b);
    return __builtin_bit_cast(unsigned, __builtin_elementwise_min(x, y));
}
__device__ __forceinline__ unsigned wave_max(unsigned v) {
#define MAX_STEP(ctrl, rmask) { const unsigned t = (unsigned)__builtin_amdgcn_update_dpp(0, (int)v, ctrl, rmask, 0xf, false); v = v > t ? v : t; }
    MAX_STEP(0x111, 0xf) MAX_STEP(0x112, 0xf) MAX_STEP(0x114, 0xf) MAX_STEP(0x118, 0xf) MAX_STEP(0x142, 0xa) MAX_STEP(0x143, 0xc)
    return __builtin_amdgcn_readlane(v, 63);
}
__device__ __forceinline__ unsigned pkmin1(unsigned a) {
    unsigned r; asm("v_pk_min_u16 %0, %1, 1 op_sel_hi:[1,0]" : "=v"(r) : "v"(a)); return r;
}
__device__ __forceinline__ unsigned cvt_pk_bf16(float lo, float hi) { unsigned r; asm volatile("v_cvt_pk_bf16_f32 %0, %1, %2" : "=v"(r) : "v"(lo), "v"(hi)); return r; }
__device__ __forceinline__ int lds_byte(int r, int c) { const int st = (r >> 4) * 2 + (c >> 5), rr = r & 15, cc = c & 31, ob = rr * 64 + cc * 2; return st * 1024 + (ob ^ (((ob >> 9) & 1) << 5)); }

__device__ __forceinline__ unsigned wave_scan(unsigned x) {
    x += (unsigned)__builtin_amdgcn_update_dpp(0, (int)x, 0x111, 0xf, 0xf, false);
    x += (unsigned)__builtin_amdgcn_update_dpp(0, (int)x, 0x112, 0xf, 0xf, false);
    x += (unsigned)__builtin_amdgcn_update_dpp(0, (int)x, 0x114, 0xf, 0xf, false);
    x += (unsigned)__builtin_amdgcn_update_dpp(0, (int)x, 0x118, 0xf, 0xf, false);
    x += (unsigned)__builtin_amdgcn_update_dpp(0, (int)x, 0x142, 0xa, 0xf, false);
    x += (unsigned)__builtin_amdgcn_update_dpp(0, (int)x, 0x143, 0xc, 0xf, false);
    return x;
}
__device__ __forceinline__ unsigned wave_sum(unsigned v) { return __builtin_amdgcn_readlane(wave_scan(v), 63); }
__device__ __forceinline__ unsigned pksub(unsigned a, unsigned b) {
    u16x2 x = __builtin_bit_cast(u16x2, a), y = __builtin_bit_cast(u16x2, b);
    return __builtin_bit_cast(unsigned, (u16x2)(x - y));
}
__device__ __forceinline__ unsigned bfi(unsigned mask, unsigned a, unsigned b) { return (a & mask) | (b & ~mask); }
__device__ __forceinline__ unsigned choose_window(unsigned esum, unsigned n, unsigned emax) {
    if (!n) return 21u;
    const unsigned num = esum * 32u + n * 63u, den = n * 32u;
    unsigned t = num / den; const unsigned alt = (num - t * den) * 5u > den * 2u ? 1u : 0u;
    t = t > emax ? emax : t; t = t < 21u ? 21u : (t > 253u ? 253u : t);
    return t | (alt << 9);
}
template <int NQ> __device__ __forceinline__ unsigned apply_escapes(unsigned (&e)[NQ], const unsigned (&m)[NQ], LAS unsigned char* smem, unsigned off) {
    unsigned sel[NQ], o[NQ]; u32x2 w[NQ];
#pragma unroll
    for (int q = 0; q < NQ; ++q) {
        sel[q] = *(const LAS unsigned*)(smem + SM_LUTE + __builtin_amdgcn_udot4(m[q], 0x08040201u, 0u, false));
        o[q] = off; off += (unsigned)__builtin_popcount(m[q]);
        const LAS unsigned* pw = (const LAS unsigned*)(smem + (o[q] & ~3u));
        w[q] = (u32x2){pw[0], pw[1]};
    }
#pragma unroll
    for (int q = 0; q < NQ; ++q) e[q] = __builtin_amdgcn_perm(__builtin_amdgcn_alignbyte(w[q].y, w[q].x, o[q]), e[q], sel[q]);
    return off;
}
__device__ __forceinline__ unsigned append_flagged(LAS unsigned char* smem, unsigned e, unsigned m, unsigned off, unsigned dummy) {
#pragma unroll
    for (int k = 0; k < 4; ++k) {
        const unsigned bit = (m >> (8 * k + 2)) & 1u;
        smem[bit ? off : dummy] = (unsigned char)(e >> (8 * k));
        off += bit;
    }
    return off;
}
__device__ __forceinline__ void decode_levels(LAS unsigned char* smem, unsigned h0, unsigned h1, unsigned o2, unsigned l3, unsigned l2, int lane) {
    const unsigned t = h0 & 0xffu, alt = (h0 >> 9) & 1u, n2 = h0 >> 16, n3 = h1 & 0xffffu;
    const unsigned o3 = o2 + ((n2 + 15u) >> 4) * 4u, o4 = o3 + ((n3 + 7u) >> 3) * 4u;
    const unsigned b3q = (t - 6u + alt) * 0x01010101u, spq = (t + 1u + alt) * 0x01010101u;
    unsigned eo = o4;
    for (unsigned r = 0; r * 256u < n3; ++r) {
        const unsigned g = r * 64u + lane;
        unsigned c = *(const LAS u16*)(smem + o3 + g * 2u); c = g * 4u < n3 ? c : 0u;
        const unsigned z = __builtin_amdgcn_perm(0u, c, 0x01010000u);
        const unsigned d = (z & 0x000f000fu) | ((z >> 4) & 0x0f000f00u);
        const unsigned is15 = (d + 0x01010101u) & 0x10101010u, is14 = ((d + 0x02020202u) & 0x10101010u) ^ is15;
        unsigned e[1], m[1];
        e[0] = bfi((is14 << 4) - (is14 >> 4), spq, b3q - d); m[0] = is15 >> 2;
        const unsigned ec = (unsigned)__builtin_popcount(m[0]), inc = wave_scan(ec);
        apply_escapes<1>(e, m, smem, eo + inc - ec);
        eo += __builtin_amdgcn_readlane(inc, 63);
        *(LAS unsigned*)(smem + l3 + g * 4u) = e[0];
    }
    const unsigned tab = (t - 3u) | ((t - 4u) << 8) | ((alt ? t + 1u : t - 5u) << 16);
    eo = l3;
    for (unsigned r = 0; r * 1024u < n2; ++r) {
        const unsigned g = r * 64u + lane;
        unsigned c = *(const LAS unsigned*)(smem + o2 + g * 4u); c = g * 16u < n2 ? c : 0u;
        unsigned e[4], m[4];
#pragma unroll
        for (int k = 0; k < 4; ++k) { const unsigned d = (c >> (2 * k)) & 0x03030303u; e[k] = __builtin_amdgcn_perm(0u, tab, d); m[k] = (d + 0x01010101u) & 0x04040404u; }
        const unsigned ec = (unsigned)__builtin_popcount(c & (c >> 1) & 0x55555555u), inc = wave_scan(ec);
        apply_escapes<4>(e, m, smem, eo + inc - ec);
        eo += __builtin_amdgcn_readlane(inc, 63);
        if (g * 16u < n2) *(LAS u32x4*)(smem + l2 + g * 16u) = (u32x4){e[0], e[1], e[2], e[3]};
    }
}
__device__ __forceinline__ unsigned encode_levels(LAS unsigned char* smem, rsrc_t wsr, unsigned win, unsigned n2, unsigned l2, unsigned l3, unsigned l4, unsigned g2, unsigned room, unsigned dummy, int lane) {
    const unsigned t = win & 0xffu, alt = (win >> 9) & 1u;
    if (lane < 16) smem[l2 + n2 + lane] = (unsigned char)(t - 3u);
    const unsigned t3 = (t - 3u) * 0x00010001u, sc = (alt ? t + 1u : t - 5u) * 0x00010001u;
    unsigned n3 = 0;
    for (unsigned r = 0; r * 1024u < n2; ++r) {
        const unsigned g = r * 64u + lane; const bool act = g * 16u < n2;
        const u32x4 e = *(const LAS u32x4*)(smem + l2 + (act ? g : 0u) * 16u);
        unsigned c = 0, m[4], ec = 0;
#pragma unroll
        for (int k = 0; k < 4; ++k) {
            unsigned dd[2];
#pragma unroll
            for (int h = 0; h < 2; ++h) {
                const unsigned x = (e[k] >> (8 * h)) & 0x00ff00ffu;
                const unsigned a = pkmin(pksub(t3, x), 0x00020002u);
                dd[h] = a + ((a >> 1) & pkmin1(x ^ sc));
            }
            const unsigned d = dd[0] | (dd[1] << 8);
            c |= d << (2 * k); m[k] = act ? (d + 0x01010101u) & 0x04040404u : 0u; ec += (unsigned)__builtin_popcount(m[k]);
        }
        const unsigned inc = wave_scan(ec), tot = __builtin_amdgcn_readlane(inc, 63);
        if (n3 + tot > CAP3) return 0xffffffffu;
        if (act) bst4<SYS>(wsr, g2 + g * 4u, c);
        unsigned eo = l3 + n3 + inc - ec;
#pragma unroll
        for (int k = 0; k < 4; ++k) eo = append_flagged(smem, e[k], m[k], eo, dummy);
        n3 += tot;
    }
    const unsigned b3 = t - 6u + alt;
    if (lane < 8) smem[l3 + n3 + lane] = (unsigned char)b3;
    const unsigned g3 = g2 + ((n2 + 15u) >> 4) * 4u, g4 = g3 + ((n3 + 7u) >> 3) * 4u;
    if (g4 - g2 > room) return 0xffffffffu;
    const unsigned b32 = b3 * 0x00010001u, sp = (t + 1u + alt) * 0x00010001u;
    unsigned n4 = 0;
    for (unsigned r = 0; r * 256u < n3; ++r) {
        const unsigned g = r * 64u + lane; const bool act = g * 4u < ((n3 + 7u) & ~7u);
        const unsigned e = *(const LAS unsigned*)(smem + l3 + (act ? g : 0u) * 4u);
        unsigned dd[2];
#pragma unroll
        for (int h = 0; h < 2; ++h) {
            const unsigned x = (e >> (8 * h)) & 0x00ff00ffu;
            const unsigned a = pkmin(pksub(b32, x), 0x000e000eu);
            dd[h] = a + (((a + 0x00020002u) >> 4) & pkmin1(x ^ sp));
        }
        const unsigned d = dd[0] | (dd[1] << 8);
        const unsigned m = act ? ((d + 0x01010101u) & 0x10101010u) >> 2 : 0u;
        const unsigned ec = (unsigned)__builtin_popcount(m), inc = wave_scan(ec), tot = __builtin_amdgcn_readlane(inc, 63);
        if (n4 + tot > CAP4) return 0xffffffffu;
        const unsigned y = d | (d >> 4);
        if (act) bst2<SYS>(wsr, g3 + g * 2u, __builtin_amdgcn_perm(0u, y, 0x0c0c0200u));
        if (tot) append_flagged(smem, e, m, l4 + n4 + inc - ec, dummy);
        n4 += tot;
    }
    if (g4 - g2 + n4 > room) return 0xffffffffu;
    if (n4) { const unsigned lv = *(const LAS unsigned*)(smem + l4 + lane * 4); if ((unsigned)lane * 4u < n4) bst4<SYS>(wsr, g4 + lane * 4, lv); }
    return n3 | (n4 << 16);
}
__device__ __forceinline__ unsigned side_bytes(unsigned n2, unsigned h1) { return ((n2 + 15u) >> 4) * 4u + (((h1 & 0xffffu) + 7u) >> 3) * 4u + (h1 >> 16); }
struct Order {
    int ph, start;
    template <int NWU, int NXU> __device__ __forceinline__ void locate(int i, int& j, int& g, int& u) {
        for (;;) {
            int c = (ph < 16) ? NWU : 0;
#pragma unroll
            for (int k = 0; k < 4; ++k) c += ((unsigned)(ph - lag(k)) < 16u) ? NXU : 0;
            if (i < start + c) break;
            start += c; ++ph;
        }
        u = i - start; g = 0; j = ph;
        if (ph < 16) { if (u < NWU) return; u -= NWU; }
#pragma unroll
        for (int k = 0; k < 4; ++k) {
            if ((unsigned)(ph - lag(k)) < 16u) { if (u < NXU) { g = 1 + k; j = ph - lag(k); return; } u -= NXU; }
        }
    }
};

__device__ __forceinline__ bool give_up(gu32* tmo, unsigned seq, u64 t0) {
    bool dead = rfl(__hip_atomic_load(tmo, __ATOMIC_RELAXED, __HIP_MEMORY_SCOPE_AGENT)) == seq;
    if (!dead && __builtin_amdgcn_s_memrealtime() - t0 > DEADLINE) {
        __hip_atomic_store(tmo, seq, __ATOMIC_RELAXED, __HIP_MEMORY_SCOPE_AGENT);
        dead = true;
    }
    return dead;
}
__device__ __forceinline__ unsigned early_word(unsigned u) { return (u / 3u) * 8u + 5u + u % 3u; }

__device__ __forceinline__ void pack_unit(const Params& p, char* ws, rsrc_t wsr, LAS unsigned char* smem, unsigned lwo, int j, int g, int u, int lane, unsigned seqv) {
    const bool isx = g != 0; const int ub = isx ? (u >> 4) * 64 + (g - 1) * 16 + (u & 15) : u;
    const int ra = p.rank >> 2, rb = p.rank & 3;
    LAS unsigned char* lw = smem + lwo;
    u32x4 v[8];
    if (isx) {
        const u16* src = p.x + (size_t)(ub * 64 + (lane & 15)) * KS + j * 64 + (lane >> 4) * 8;
#pragma unroll
        for (int c = 0; c < 8; ++c) v[c] = *(const u32x4*)(src + (size_t)(c >> 1) * 16 * KS + (c & 1) * 32);
    } else {
        constexpr int P = 144;
        const u16* src = p.w + (size_t)(j * 64 + (lane >> 3)) * ND + ub * 64 + (lane & 7) * 8;
        u32x4 tmp[8];
#pragma unroll
        for (int i = 0; i < 8; ++i) tmp[i] = *(const u32x4*)(src + (size_t)i * 8 * ND);
#pragma unroll
        for (int i = 0; i < 8; ++i) *(LAS u32x4*)(lw + (i * 8 + (lane >> 3)) * P + (lane & 7) * 16) = tmp[i];
        const int q = (lane >> 2) & 3, pp = lane & 3;
#pragma unroll
        for (int c = 0; c < 8; ++c) {
            const int k0 = (c & 1) * 32 + (lane >> 4) * 8, rbk = c >> 1;
            const int col = (rbk >> 1) * 32 + 8 * pp + 4 * (rbk & 1);
            const s16x4 a = __builtin_amdgcn_ds_read_tr16_b64_v4i16((LAS s16x4*)(lw + (k0 + q) * P + col * 2));
            const s16x4 b = __builtin_amdgcn_ds_read_tr16_b64_v4i16((LAS s16x4*)(lw + (k0 + 4 + q) * P + col * 2));
            const u32x2 a2 = __builtin_bit_cast(u32x2, a), b2 = __builtin_bit_cast(u32x2, b);
            v[c] = (u32x4){a2.x, a2.y, b2.x, b2.y};
        }
    }
    const unsigned loff = (lane & 15) * 64 + (((lane >> 4) * 16) ^ (((lane >> 3) & 1) << 5));
    const int cls = isx ? (ub >> 6) : (ub >> 5);
    const bool own = isx ? (cls == ra) : (cls == rb);
    const int li = isx ? (ub & 63) : (ub & 31);
    if (own) {
        const unsigned dst = (isx ? OFF_XL + (unsigned)(j * 8 * 64 + li) * 8192u : OFF_WL + (unsigned)(j * 8 * 32 + li) * 8192u) + loff;
#pragma unroll
        for (int c = 0; c < 8; ++c) bst16<SC1>(wsr, dst + c * 1024, v[c]);
    }
    const unsigned slot0 = (isx ? OFF_EX : OFF_EW) + (unsigned)(j * 128 + ub) * USLOT;
    const unsigned pay = slot0 + 1024 + lane * 16;
    const bool early = j == 0 && g == 0;
    if (early) {
#pragma unroll
        for (int c = 0; c < 8; ++c) bst16<SYS>(wsr, pay + c * 1024, v[c]);
        if (lane == 0) bst4<SYS>(wsr, slot0, 1u << 8);
    } else {
    unsigned e4[16], sm[16], mx = 0, esum = 0;
#pragma unroll
    for (int q = 0; q < 16; ++q) {
        const unsigned a = v[q >> 1][(q & 1) * 2], b = v[q >> 1][(q & 1) * 2 + 1];
        const unsigned lo = __builtin_amdgcn_perm(b, a, 0x06040200u), hi = __builtin_amdgcn_perm(b, a, 0x07050301u);
        e4[q] = bfi(0x01010101u, lo >> 7, hi << 1);
        sm[q] = bfi(0x7f7f7f7fu, lo, hi);
        esum = __builtin_amdgcn_udot4(e4[q], 0x01010101u, esum, false);
        mx = pkmax(mx, a & 0x7fff7fffu); mx = pkmax(mx, b & 0x7fff7fffu);
    }
    mx = (mx & 0xffffu) > (mx >> 16) ? (mx & 0xffffu) : (mx >> 16);
    const unsigned win = choose_window(wave_sum(esum), 4096u, wave_max(mx) >> 7);
    const unsigned top2 = (win & 0xffu) * 0x00010001u;
    unsigned p2[4] = {0u, 0u, 0u, 0u}, m[16], ec = 0;
#pragma unroll
    for (int q = 0; q < 16; ++q) {
        const unsigned da = pkmin(pksub(top2, e4[q] & 0x00ff00ffu), 0x00030003u), db = pkmin(pksub(top2, (e4[q] >> 8) & 0x00ff00ffu), 0x00030003u);
        const unsigned d = da | (db << 8);
        p2[q >> 2] |= d << (2 * (q & 3));
        m[q] = (d + 0x01010101u) & 0x04040404u; ec += (unsigned)__builtin_popcount(m[q]);
    }
    const unsigned einc = wave_scan(ec), n2 = __builtin_amdgcn_readlane(einc, 63);
    unsigned h1 = 0xffffffffu;
    if (!FORCE_RAW_IN && n2 <= CAP2) {
        const unsigned l2 = lwo + 10240, l3 = l2 + CAP2 + 16, l4 = l3 + CAP3 + 16, dummy = SM_DUMMY + lane * 2;
        unsigned eo = l2 + einc - ec;
#pragma unroll
        for (int q = 0; q < 16; ++q) eo = append_flagged(smem, e4[q], m[q], eo, dummy);
        h1 = encode_levels(smem, wsr, win, n2, l2, l3, l4, slot0 + 8, 1016u, dummy, lane);
    }
    if (h1 == 0xffffffffu) {
#pragma unroll
        for (int c = 0; c < 8; ++c) bst16<SYS>(wsr, pay + c * 1024, v[c]);
        if (lane == 0) bst4<SYS>(wsr, slot0, 1u << 8);
    } else {
#pragma unroll
        for (int i = 0; i < 4; ++i) bst16<SYS>(wsr, pay + i * 1024, (u32x4){sm[4 * i], sm[4 * i + 1], sm[4 * i + 2], sm[4 * i + 3]});
        bst16<SYS>(wsr, pay + 4096, (u32x4){p2[0], p2[1], p2[2], p2[3]});
        if (lane == 0) bst8<SYS>(wsr, slot0, (u32x2){win | (n2 << 16), h1});
    }
    }
    WAIT_V(0);
    const unsigned seq = rfl(seqv) + 1u;
    if (own && lane == 0) {
        const unsigned f = (isx ? OFF_XF : OFF_WF) + (unsigned)(((li >> 2) * NSTEP + j * 8) * 4 + (li & 3)) * 4u;
        __hip_atomic_store((gu32*)(ws + f), seq, __ATOMIC_RELAXED, __HIP_MEMORY_SCOPE_AGENT);
    }
    const unsigned per = isx ? 16u : 32u;
    unsigned ticket = per - 1u;
    if (!early) {
        if (lane == 0) ticket = __hip_atomic_fetch_add((gu32*)(ws + OFF_ECNT + (unsigned)(j * 12 + (isx ? cls * 4 + g - 1 : 8 + cls)) * 128u), 1u, __ATOMIC_RELAXED, __HIP_MEMORY_SCOPE_AGENT);
        ticket = rfl(ticket);
    }
    if ((ticket & (per - 1u)) == per - 1u) {
        const bool want = lane < NR && lane != p.rank && (isx ? ((lane >> 2) == cls) : ((lane & 3) == cls));
        if (want) {
            char* pw = (char*)p.ws_all[lane];
            const unsigned word = early ? early_word((unsigned)li) : (unsigned)(j * 8 + g);
            __hip_atomic_store((gu32*)(pw + OFF_FIN + (unsigned)p.rank * 512u + word * 4u), seq, __ATOMIC_RELAXED, __HIP_MEMORY_SCOPE_SYSTEM);
        }
    }
}

struct UnitRegs { u32x4 a[5]; u32x4 sd; };
__device__ __forceinline__ unsigned unit_slot(int rank, int j, int g, int u) {
    const int ub = g ? (rank >> 2) * 64 + (g - 1) * 16 + u : (rank & 3) * 32 + u;
    return (g ? OFF_EX : OFF_EW) + (unsigned)(j * 128 + ub) * USLOT;
}
__device__ __forceinline__ void unit_issue(UnitRegs& r, u32x4 pd, unsigned slot0, int lane) {
    const unsigned oh = lane < 24 ? slot0 + lane * 16 : OOB, o0 = slot0 + 1024 + lane * 16, o1 = o0 + 4096;
    ALD16(r.sd, pd, oh, 0);
    ALD16(r.a[0], pd, o0, 0); ALD16(r.a[1], pd, o0, 1024); ALD16(r.a[2], pd, o0, 2048); ALD16(r.a[3], pd, o0, 3072);
    ALD16(r.a[4], pd, o1, 0);
}
#define UNIT_LANDED(r, n) asm volatile("s_waitcnt vmcnt(0)" : "+v"(r.a[0]), "+v"(r.a[1]), "+v"(r.a[2]), "+v"(r.a[3]), "+v"(r.a[4]), "+v"(r.sd), "+v"(n) :: "memory")
__device__ __forceinline__ void unit_unpack(const UnitRegs& r, rsrc_t wsr, rsrc_t pr, LAS unsigned char* smem, unsigned scr, unsigned slot0, unsigned dst0, int lane) {
    const unsigned dst = dst0 + (lane & 15) * 64 + (((lane >> 4) * 16) ^ (((lane >> 3) & 1) << 5));
    const unsigned h0 = rfl(r.sd.x), h1 = rfl(r.sd.y);
    if ((h0 >> 8) & 1u) {
        u32x4 b[3];
#pragma unroll
        for (int c = 0; c < 3; ++c) b[c] = bld16<SYS>(pr, slot0 + 1024 + lane * 16 + (5 + c) * 1024);
#pragma unroll
        for (int c = 0; c < 5; ++c) bst16<SC1>(wsr, dst + c * 1024, r.a[c]);
#pragma unroll
        for (int c = 0; c < 3; ++c) bst16<SC1>(wsr, dst + (5 + c) * 1024, b[c]);
        return;
    }
    u32x4 sd = r.sd;
    const unsigned len = 8u + side_bytes(h0 >> 16, h1);
    if (len > 384u) { if (lane >= 24 && (unsigned)lane * 16u < len) sd = bld16<SYS>(pr, slot0 + lane * 16); }
    *(LAS u32x4*)(smem + scr + lane * 16) = sd;
    const unsigned l3 = scr + 2048, l2 = l3 + CAP3 + 256;
    decode_levels(smem, h0, h1, scr + 8u, l3, l2, lane);
    const unsigned tq = (h0 & 0xffu) * 0x01010101u;
    unsigned e[16], m[16], ec = 0;
#pragma unroll
    for (int q = 0; q < 16; ++q) {
        const unsigned d = (r.a[4][q >> 2] >> (2 * (q & 3))) & 0x03030303u;
        m[q] = (d + 0x01010101u) & 0x04040404u; e[q] = tq - d;
    }
#pragma unroll
    for (int k = 0; k < 4; ++k) ec += (unsigned)__builtin_popcount(r.a[4][k] & (r.a[4][k] >> 1) & 0x55555555u);
    apply_escapes<16>(e, m, smem, l2 + wave_scan(ec) - ec);
#pragma unroll
    for (int c = 0; c < 8; ++c) {
        u32x4 o;
#pragma unroll
        for (int h = 0; h < 2; ++h) {
            const int q = 2 * c + h; const unsigned sm = r.a[q >> 2][q & 3];
            const unsigned lo = bfi(0x7f7f7f7fu, sm, e[q] << 7), hi = bfi(0x7f7f7f7fu, e[q] >> 1, sm);
            o[2 * h] = __builtin_amdgcn_perm(hi, lo, 0x05010400u); o[2 * h + 1] = __builtin_amdgcn_perm(hi, lo, 0x07030602u);
        }
        bst16<SC1>(wsr, dst + c * 1024, o);
    }
}

__device__ __forceinline__ unsigned strip_out_off(int r, int s, int lane) {
    const int tile = s >> 3, tm = tile >> 3, tn = tile & 7;
    const int row = (r >> 2) * RROWS + tm * 256 + (s & 7) * 32 + (lane >> 5);
    const int col = (r & 3) * RCOLS + tn * 256 + (lane & 31) * 8;
    return ((unsigned)row * ND + col) * 2u;
}

struct StripRegs { u32x4 mk; u32x4 lo[8]; u32x4 sd[2]; };
__device__ __forceinline__ void strip_issue(StripRegs& r, u32x4 pd, unsigned s, unsigned info, int lane) {
    if (info >> 30) return;
    const unsigned T = info & 0x3fffu, sl = (info >> 14) & 0xffu, slot = OFF_EO + s * SSLOT;
    const unsigned om = slot + lane * 16;
    ALD16(r.mk, pd, om, 0);
    { const unsigned o = (unsigned)lane < sl ? om : OOB; ALD16(r.sd[0], pd, o, 1024); }
    if (sl > 64u) { const unsigned o = (unsigned)lane + 64u < sl ? om : OOB; ALD16(r.sd[1], pd, o, 2048); }
#define STRIP_ROUND(k) if ((unsigned)(k) * 1024u < T) { const bool act = (unsigned)((k) * 1024 + lane * 16) < T; \
        const unsigned ol = act ? om + 3072 + ((k) >> 2) * 4096 : OOB; ALD16(r.lo[k], pd, ol, ((k) & 3) * 1024); }
    STRIP_ROUND(0) STRIP_ROUND(1) STRIP_ROUND(2) STRIP_ROUND(3) STRIP_ROUND(4) STRIP_ROUND(5) STRIP_ROUND(6) STRIP_ROUND(7)
}
#define STRIP_LANDED(r, fn, n, cnt) do { \
    asm volatile("s_waitcnt vmcnt(" #cnt ")" : "+v"(r.lo[0]), "+v"(r.lo[1]), "+v"(r.lo[2]), "+v"(r.lo[3]), "+v"(r.lo[4]), "+v"(r.lo[5]), "+v"(r.lo[6]), "+v"(r.lo[7]), \
        "+v"(r.mk), "+v"(r.sd[0]), "+v"(r.sd[1]), "+v"(fn), "+v"(n) :: "memory"); } while (0)
__device__ __forceinline__ void strip_stage1(const StripRegs& r, LAS unsigned char* smem, unsigned lbase, unsigned scr, unsigned info, int lane) {
    if (info >> 30) return;
    const unsigned T = info & 0x3fffu;
    const unsigned h0 = rfl(r.sd[0].x), h1 = rfl(r.sd[0].y);
    *(LAS u32x4*)(smem + scr + lane * 16) = r.sd[0];
    *(LAS u32x4*)(smem + scr + 1024 + lane * 16) = r.sd[1];
    const unsigned l3 = scr + 2048, l2 = l3 + CAP3 + 256;
    decode_levels(smem, h0, h1, scr + 8u + ((T + 31u) >> 5) * 4u, l3, l2, lane);
    const unsigned tq = (h0 & 0xffu) * 0x01010101u;
    unsigned x1[8], ec = 0;
#pragma unroll
    for (int k = 0; k < 8; ++k) {
        if ((unsigned)k * 1024u < T) {
            const unsigned z = __builtin_amdgcn_perm(0u, (unsigned)*(const LAS u16*)(smem + scr + 8 + k * 128 + lane * 2), 0x01010000u);
            x1[k] = (z & 0x000f000fu) | ((z >> 4) & 0x0f000f00u);
#pragma unroll
            for (int i = 0; i < 4; ++i) ec += (unsigned)__builtin_popcount((x1[k] >> i) & (r.lo[k][i] >> 7) & 0x01010101u);
        }
    }
    unsigned eo = l2 + wave_scan(ec) - ec;
    unsigned e[8][4];
#pragma unroll
    for (int k = 0; k < 8; ++k) {
        if ((unsigned)k * 1024u < T) {
            unsigned m[4];
#pragma unroll
            for (int i = 0; i < 4; ++i) {
                const unsigned d = ((x1[k] >> i) & 0x01010101u) | ((r.lo[k][i] >> 6) & 0x02020202u);
                m[i] = (d + 0x01010101u) & 0x04040404u; e[k][i] = tq - d;
            }
            eo = apply_escapes<4>(e[k], m, smem, eo);
        }
    }
#pragma unroll
    for (int k = 0; k < 8; ++k) {
        if ((unsigned)k * 1024u < T) {
            unsigned lo[4], hi[4];
#pragma unroll
            for (int i = 0; i < 4; ++i) { lo[i] = bfi(0x7f7f7f7fu, r.lo[k][i], e[k][i] << 7); hi[i] = (e[k][i] >> 1) & 0x7f7f7f7fu; }
            *(LAS u32x4*)(smem + lbase + k * 2048 + lane * 32) = (u32x4){lo[0], hi[0], lo[1], hi[1]};
            *(LAS u32x4*)(smem + lbase + k * 2048 + lane * 32 + 16) = (u32x4){lo[2], hi[2], lo[3], hi[3]};
        }
    }
}
__device__ __forceinline__ void strip_stage2(u32x4 mk, rsrc_t outr, rsrc_t por, LAS unsigned char* smem, unsigned lbase, int peer, unsigned s, unsigned info, int lane) {
    const unsigned oo = strip_out_off(peer, (int)s, lane);
    if (info >> 31) {
#pragma unroll
        for (int q = 0; q < 16; ++q) bst16<OAUX>(outr, oo + q * 2 * ND * 2, (u32x4){0x7fc07fc0u, 0x7fc07fc0u, 0x7fc07fc0u, 0x7fc07fc0u});
        return;
    }
    if (info >> 30) {
#pragma unroll
        for (int h = 0; h < 4; ++h) {
            u32x4 a[4];
#pragma unroll
            for (int q = 0; q < 4; ++q) a[q] = bld16<SYS>(por, oo + (h * 4 + q) * 2 * ND * 2);
#pragma unroll
            for (int q = 0; q < 4; ++q) bst16<OAUX>(outr, oo + (h * 4 + q) * 2 * ND * 2, a[q]);
        }
        return;
    }
    unsigned c2[8], x[8];
#pragma unroll
    for (int i = 0; i < 8; ++i) {
        const unsigned m16 = mk[i >> 1] >> ((i & 1) * 16);
        c2[i] = (unsigned)__builtin_popcount(m16 & 0xffu) | ((unsigned)__builtin_popcount((m16 >> 8) & 0xffu) << 16);
        x[i] = c2[i];
    }
#define SCAN_STEP(ctrl, rmask) _Pragma("unroll") for (int i = 0; i < 8; ++i) x[i] += (unsigned)__builtin_amdgcn_update_dpp(0, (int)x[i], ctrl, rmask, 0xf, false);
    SCAN_STEP(0x111, 0xf) SCAN_STEP(0x112, 0xf) SCAN_STEP(0x114, 0xf) SCAN_STEP(0x118, 0xf) SCAN_STEP(0x142, 0xa) SCAN_STEP(0x143, 0xc)
    unsigned pos = 0;
#pragma unroll
    for (int bt = 0; bt < 4; ++bt) {
        u32x4 sel[4], d[8]; unsigned off[8];
#pragma unroll
        for (int k = 0; k < 4; ++k) {
            const int i = bt * 2 + (k >> 1), hq = k & 1;
            const unsigned m8 = (mk[bt] >> (k * 8)) & 0xffu;
            sel[k] = *(const LAS u32x4*)(smem + SM_LUT + m8 * 16u);
            const unsigned tot = __builtin_amdgcn_readlane(x[i], 63);
            off[2 * k] = pos + (((x[i] - c2[i]) >> (hq * 16)) & 0xffffu);
            off[2 * k + 1] = off[2 * k] + (unsigned)__builtin_popcount(m8 & 15u);
            pos += (tot >> (hq * 16)) & 0xffffu;
        }
#pragma unroll
        for (int g = 0; g < 8; ++g) {
            const LAS u32x2* pl = (const LAS u32x2*)(smem + lbase + (off[g] & ~3u) * 2u);
            const u32x2 d0 = pl[0], d1 = pl[1];
            d[g] = (u32x4){d0.x, d0.y, d1.x, d1.y};
        }
#pragma unroll
        for (int k = 0; k < 4; ++k) {
            u32x4 o;
#pragma unroll
            for (int g = 0; g < 2; ++g) {
                const u32x4 dd = d[2 * k + g];
                const unsigned l4 = __builtin_amdgcn_alignbyte(dd.z, dd.x, off[2 * k + g]), h4 = __builtin_amdgcn_alignbyte(dd.w, dd.y, off[2 * k + g]);
                o[2 * g] = __builtin_amdgcn_perm(h4, l4, sel[k][2 * g]); o[2 * g + 1] = __builtin_amdgcn_perm(h4, l4, sel[k][2 * g + 1]);
            }
            bst16<OAUX>(outr, oo + (bt * 4 + k) * 2 * ND * 2, o);
        }
    }
}

__device__ __forceinline__ void pack_strip(const Params& p, rsrc_t wsr, rsrc_t outr, LAS unsigned char* smem, int s, int wid, int lane, unsigned seq) {
    const unsigned oo = strip_out_off(p.rank, s, lane);
    const unsigned lbase = (unsigned)wid * 16384u;
    unsigned pos = lbase, mx = 0, sg = 0, esum = 0, ts = 0;
    const unsigned dummy = SM_DUMMY + lane * 2;
    u32x4 mk = {0u, 0u, 0u, 0u};
#pragma unroll
    for (int q2 = 0; q2 < 8; ++q2) {
        u32x4 v[2]; unsigned m8[2];
#pragma unroll
        for (int hq = 0; hq < 2; ++hq) {
            const int q = q2 * 2 + hq, row = wid * 32 + 2 * q + (lane >> 5);
            v[hq] = *(const LAS u32x4*)(smem + row * 512 + (((lane & 31) ^ (row & 7)) * 16));
            bst16<SYS>(outr, oo + q * 2 * ND * 2, v[hq]);
            unsigned m = 0;
#pragma unroll
            for (int k = 0; k < 4; ++k) {
                sg |= v[hq][k];
                if (q2 < 2) {
                    mx = pkmax(mx, v[hq][k]);
                    esum = __builtin_amdgcn_udot4((v[hq][k] >> 7) & 0x00ff00ffu, 0x00010001u, esum, false);
                }
                const unsigned t = pkmin1(v[hq][k]);
                m |= ((t | (t >> 15)) & 3u) << (2 * k);
            }
            m8[hq] = m;
        }
        mk[q2 >> 1] |= (m8[0] | (m8[1] << 8)) << ((q2 & 1) * 16);
        const unsigned c2 = (unsigned)__builtin_popcount(m8[0]) | ((unsigned)__builtin_popcount(m8[1]) << 16);
        const unsigned inc = wave_scan(c2), exl = inc - c2, tot = __builtin_amdgcn_readlane(inc, 63);
#pragma unroll
        for (int hq = 0; hq < 2; ++hq) {
            unsigned a = pos + ((exl >> (hq * 16)) & 0xffffu) * 2u;
#pragma unroll
            for (int e = 0; e < 8; ++e) {
                const unsigned bit = (m8[hq] >> e) & 1u;
                *(LAS u16*)(smem + (bit ? a : dummy)) = (u16)(v[hq][e >> 1] >> ((e & 1) * 16));
                a += bit * 2u;
            }
            pos += ((tot >> (hq * 16)) & 0xffffu) * 2u;
        }
        if (q2 == 1) ts = (pos - lbase) >> 1;
    }
    const unsigned T = (pos - lbase) >> 1;
    STAMP(13);
    mx = (mx & 0xffffu) > (mx >> 16) ? (mx & 0xffffu) : (mx >> 16);
    const unsigned win = choose_window(wave_sum(esum), ts, (wave_max(mx) >> 7) & 0xffu);
    bool raw = FORCE_RAW_OUT || __builtin_amdgcn_ballot_w64((sg & 0x80008000u) != 0u) != 0ull;
    const unsigned slot = OFF_EO + (unsigned)s * SSLOT;
    unsigned sl = 0;
    if (!raw) {
        const unsigned top = win & 0xffu, top2 = top * 0x00010001u;
        if (lane < 16 && T + lane < 8192u) *(LAS u16*)(smem + lbase + (T + lane) * 2u) = (u16)(top << 7);
        bst16<SYS>(wsr, slot + lane * 16, mk);
        unsigned e4[8][4], m[8][4], ec = 0;
#pragma unroll
        for (int r = 0; r < 8; ++r) {
            if ((unsigned)r * 1024u < T) {
                const bool act = (unsigned)(r * 1024 + lane * 16) < T;
                const u32x4 a = *(const LAS u32x4*)(smem + lbase + r * 2048 + lane * 32), b = *(const LAS u32x4*)(smem + lbase + r * 2048 + lane * 32 + 16);
                const unsigned w8[8] = {a.x, a.y, a.z, a.w, b.x, b.y, b.z, b.w};
                u32x4 lo; unsigned x1 = 0;
#pragma unroll
                for (int k = 0; k < 4; ++k) {
                    const unsigned ea = (w8[2 * k] >> 7) & 0x00ff00ffu, eb = (w8[2 * k + 1] >> 7) & 0x00ff00ffu;
                    const unsigned d = __builtin_amdgcn_perm(pkmin(pksub(top2, eb), 0x00030003u), pkmin(pksub(top2, ea), 0x00030003u), 0x06040200u);
                    e4[r][k] = __builtin_amdgcn_perm(eb, ea, 0x06040200u);
                    lo[k] = bfi(0x7f7f7f7fu, __builtin_amdgcn_perm(w8[2 * k + 1], w8[2 * k], 0x06040200u), d << 6);
                    x1 |= (d & 0x01010101u) << k;
                    m[r][k] = act ? (d + 0x01010101u) & 0x04040404u : 0u; ec += (unsigned)__builtin_popcount(m[r][k]);
                }
                if (act) {
                    bst16<SYS>(wsr, slot + 3072 + r * 1024 + lane * 16, lo);
                    bst2<SYS>(wsr, slot + 1024 + 8 + r * 128 + lane * 2, __builtin_amdgcn_perm(0u, x1 | (x1 >> 4), 0x0c0c0200u));
                }
            }
        }
        STAMP(14);
        const unsigned einc = wave_scan(ec), n2 = __builtin_amdgcn_readlane(einc, 63);
        unsigned h1 = 0xffffffffu;
        const unsigned p1b = ((T + 31u) >> 5) * 4u;
        if (n2 <= CAP2 - 16u) {
            const unsigned l2 = SM_SCR + (unsigned)wid * 3072u, l3 = l2 + CAP2, l4 = l3 + CAP3 + 8;
            unsigned eo = l2 + einc - ec;
#pragma unroll
            for (int r = 0; r < 8; ++r) {
                if ((unsigned)r * 1024u < T) {
#pragma unroll
                    for (int k = 0; k < 4; ++k) eo = append_flagged(smem, e4[r][k], m[r][k], eo, dummy);
                }
            }
            h1 = encode_levels(smem, wsr, win, n2, l2, l3, l4, slot + 1024 + 8 + p1b, 2040u - p1b, dummy, lane);
        }
        if (h1 == 0xffffffffu) raw = true;
        else {
            if (lane == 0) bst8<SYS>(wsr, slot + 1024, (u32x2){win | (n2 << 16), h1});
            sl = (8u + p1b + side_bytes(n2, h1) + 15u) >> 4;
        }
    }
    STAMP(15);
    WAIT_V(0);
    if (lane < NR && lane != p.rank) {
        const unsigned info = raw ? (1u << 30) : (T | (sl << 14));
        char* pw = (char*)p.ws_all[lane];
        __hip_atomic_store((gu64*)(pw + OFF_FOUT + ((unsigned)p.rank * 1024u + (unsigned)s) * 8u), ((u64)seq << 32) | info, __ATOMIC_RELAXED, __HIP_MEMORY_SCOPE_SYSTEM);
    }
}

}

__global__ void __launch_bounds__(512, 2) gemm_ar_fused(Params p) {
    __shared__ __attribute__((aligned(16))) unsigned char smem_[SMEM_BYTES];
    LAS unsigned char* smem = (LAS unsigned char*)smem_;
    const int tid = threadIdx.x, wid = rfl(tid >> 6), lane = tid & 63;
    char* ws = (char*)p.ws_all[p.rank];
    const rsrc_t wsr = make_rsrc(ws, WS_USED);
    const rsrc_t outr = make_rsrc(p.out, OUT_BYTES);
    gu32* tmo = (gu32*)(ws + OFF_TMO);
    gu32* seqp = (gu32*)(ws + OFF_SEQ + (unsigned)(blockIdx.x * 8 + wid) * 4u);
    const unsigned seqv = __hip_atomic_load(seqp, __ATOMIC_RELAXED, __HIP_MEMORY_SCOPE_AGENT);
    STAMP(0);
    const int xcd = blockIdx.x & 7, wi = blockIdx.x >> 3;
    const int nsplit = xcd < 2 ? 16 : (xcd == 2 ? 8 : (xcd == 3 ? 4 : 0));
    const bool is_gemm = wi < 16 + nsplit;
    const int cw = xcd >= 4 ? (wi - 16) * 4 + xcd - 4 : (xcd == 3 ? 64 + wi - 20 : 76 + wi - 24);
    const bool is_link = !is_gemm && wid < NWV;
    if (!is_gemm) {
        if (tid == 0) { *(LAS unsigned*)(smem + SM_TOKEN) = 0u; *(LAS unsigned*)(smem + SM_TOKEN + 4) = 0xffffffffu; }
        WAIT_L(0); BAR;
    }
    if (!is_link) {
        Order od{0, 0};
        const int e = !is_gemm ? (wid - NWV) * 84 + cw
                    : xcd >= 4 ? NPD + (xcd >= 6 ? 0 : 256) + wid * 32 + wi * 2 + (xcd & 1)
                    : xcd < 2 ? NENC + 256 + wid * 64 + wi * 2 + xcd
                    : wi < 16 ? NENC + wid * 32 + wi * 2 + (xcd & 1) : NENC + 768 + wid * 12 + (xcd == 2 ? wi - 16 : wi - 8);
        if (e >= PDELAY) __builtin_amdgcn_s_sleep(PSLEEP);
        for (int t = e; t < 4096; t = (t == e ? NENC + NONCE + e : t + NENC)) {
            int j, g, u; od.locate<128, 32>(t, j, g, u);
            pack_unit(p, ws, wsr, smem, (unsigned)wid * 16384u, j, g, u, lane, seqv);
            if (e >= NENC) break;
        }
        STAMP(7);
        if (is_gemm) __syncthreads();
    }
    const unsigned seq = rfl(seqv) + 1u;
    if (lane == 0) __hip_atomic_store(seqp, seq, __ATOMIC_RELAXED, __HIP_MEMORY_SCOPE_AGENT);
    if (!is_gemm) {
        if (wid >= NLW) return;
        const int lq = cw % 7;
        const int peer = (p.rank + 1 + lq) & 7;
        const rsrc_t pr = make_rsrc(p.ws_all[peer], WS_USED);
        const rsrc_t por = make_rsrc(p.out_all[peer], OUT_BYTES);
#pragma unroll
        for (int i = 0; i < 4; ++i) {
            const unsigned m8 = (unsigned)(i * 64 + lane); u32x4 sel;
#pragma unroll
            for (int g = 0; g < 2; ++g) {
                unsigned rk = 0;
#pragma unroll
                for (int h = 0; h < 2; ++h) {
                    unsigned w = 0;
#pragma unroll
                    for (int e = 0; e < 2; ++e) { const unsigned bit = (m8 >> (g * 4 + h * 2 + e)) & 1u; w |= (bit ? (rk | ((rk + 4u) << 8)) : 0x0c0cu) << (16 * e); rk += bit; }
                    sel[g * 2 + h] = w;
                }
            }
            *(LAS u32x4*)(smem + SM_LUT + m8 * 16u) = sel;
        }
        if (lane < 16) {
            unsigned rk = 4u, sel = 0;
#pragma unroll
            for (int k = 0; k < 4; ++k) { const unsigned bit = ((unsigned)lane >> k) & 1u; sel |= (bit ? rk : (unsigned)k) << (8 * k); rk += bit; }
            *(LAS unsigned*)(smem + SM_LUTE + lane * 4) = sel;
        }
        const unsigned esc = (unsigned)wid * 16384u + 8192u;
        bool dead = false;
        const u32x4 pd = make_desc(p.ws_all[peer], WS_USED);
        const unsigned one = 1u;
        LAS unsigned* token = (LAS unsigned*)(smem + SM_TOKEN);
#define QNEXT(dst) do { if (lane == 0) asm volatile("global_atomic_add %0, %1, %2, off sc0" : "+v"(dst) : "v"(qp), "v"(one) : "memory"); } while (0)
#define TOKEN_TAKE() do { for (;;) { unsigned old = 1u; \
            if (lane == 0) old = __hip_atomic_exchange(token, 1u, __ATOMIC_RELAXED, __HIP_MEMORY_SCOPE_WORKGROUP); \
            if (rfl(old) == 0u) break; __builtin_amdgcn_s_sleep(1); } } while (0)
#define TOKEN_GIVE() do { if (lane == 0) __hip_atomic_store(token, 0u, __ATOMIC_RELAXED, __HIP_MEMORY_SCOPE_WORKGROUP); } while (0)
        {
            gu32* qp = (gu32*)(ws + OFF_Q + (unsigned)lq * 128u);
            const unsigned qbase = (seq - 1u) * (1536u + NT);
            LAS unsigned* utok = token + 1;
            gu64* finp = (gu64*)(ws + OFF_FIN + (unsigned)peer * 512u + (unsigned)lane * 8u);
            Order od{0, 0};
            u64 rdy0 = 0, rdy1 = 0, fv = 0;
            unsigned nq = 0;
            UnitRegs ur = {};
            if (wid == 0) {
                QNEXT(nq);
                UNIT_LANDED(ur, nq);
                if (lane == 0) __hip_atomic_store(utok, nq - qbase, __ATOMIC_RELAXED, __HIP_MEMORY_SCOPE_WORKGROUP);
            }
            unsigned pfo = 0;
#define PUBLISH(n) do { if (pfo) { WAIT_V(n); if (lane == 0) __hip_atomic_store((gu32*)(ws + pfo), seq, __ATOMIC_RELAXED, __HIP_MEMORY_SCOPE_AGENT); pfo = 0; } } while (0)
            for (;;) {
                unsigned i;
                for (;;) {
                    unsigned old = 0xffffffffu;
                    if (lane == 0) old = __hip_atomic_exchange(utok, 0xffffffffu, __ATOMIC_RELAXED, __HIP_MEMORY_SCOPE_WORKGROUP);
                    i = rfl(old);
                    if (i != 0xffffffffu) break;
                    if (pfo) PUBLISH(0); else __builtin_amdgcn_s_sleep(1);
                }
                if (i >= 1536u) { if (lane == 0) __hip_atomic_store(utok, i, __ATOMIC_RELAXED, __HIP_MEMORY_SCOPE_WORKGROUP); break; }
                QNEXT(nq);
                int j, g, u; od.locate<32, 16>((int)i, j, g, u);
                const unsigned word = i < 32u ? early_word(i) : (unsigned)(j * 8 + g), rb = word >> 1;
                if (!((((word & 1u) ? rdy1 : rdy0) >> rb) & 1ull) && !dead) {
                    PUBLISH(0);
                    const bool was0 = (rdy0 | rdy1) == 0ull;
                    const u64 t0 = __builtin_amdgcn_s_memrealtime(); unsigned n = 0;
                    for (;;) {
                        const u64 f2 = __hip_atomic_load(finp, __ATOMIC_RELAXED, __HIP_MEMORY_SCOPE_SYSTEM);
                        rdy0 = __builtin_amdgcn_ballot_w64((unsigned)f2 >= seq); rdy1 = __builtin_amdgcn_ballot_w64((unsigned)(f2 >> 32) >= seq);
                        if ((((word & 1u) ? rdy1 : rdy0) >> rb) & 1ull) break;
                        if ((++n & 63u) == 0u && give_up(tmo, seq, t0)) { dead = true; break; }
                        __builtin_amdgcn_s_sleep(2);
                    }
                    __builtin_amdgcn_fence(__ATOMIC_ACQUIRE, "wavefront");
                    if (was0) STAMP(1);
                }
                const int step = j * 8 + 1 + lq, li = g ? (g - 1) * 16 + u : u;
                const unsigned slot0 = unit_slot(p.rank, j, g, u);
                const unsigned dst0 = g ? OFF_XL + (unsigned)(step * 64 + li) * 8192u : OFF_WL + (unsigned)(step * 32 + li) * 8192u;
                const bool refresh = (~rdy0 & 0x7777777777777776ull) != 0ull || (~rdy1 & 0x3333333333333333ull) != 0ull;
                for (;;) {
                    unsigned old = 1u;
                    if (lane == 0) old = __hip_atomic_exchange(token, 1u, __ATOMIC_RELAXED, __HIP_MEMORY_SCOPE_WORKGROUP);
                    if (rfl(old) == 0u) break;
                    if (pfo) PUBLISH(0); else __builtin_amdgcn_s_sleep(1);
                }
                unit_issue(ur, pd, slot0, lane);
                asm volatile("s_waitcnt vmcnt(6)" : "+v"(nq) :: "memory");
                if (lane == 0) __hip_atomic_store(utok, nq - qbase, __ATOMIC_RELAXED, __HIP_MEMORY_SCOPE_WORKGROUP);
                UNIT_LANDED(ur, nq);
                TOKEN_GIVE();
                PUBLISH(0);
                if (refresh) asm volatile("global_load_dwordx2 %0, %1, off sc0 sc1" : "+v"(fv) : "v"(finp) : "memory");
                pfo = (g ? OFF_XF : OFF_WF) + (unsigned)(((li >> 2) * NSTEP + step) * 4 + (li & 3)) * 4u;
                unit_unpack(ur, wsr, pr, smem, esc, slot0, dst0, lane);
                if (refresh) {
                    asm volatile("s_waitcnt vmcnt(8)" : "+v"(fv) :: "memory");
                    rdy0 = __builtin_amdgcn_ballot_w64((unsigned)fv >= seq); rdy1 = __builtin_amdgcn_ballot_w64((unsigned)(fv >> 32) >= seq);
                }
            }
            PUBLISH(0);
            STAMP(3);
        }
        {
            gu32* qp = (gu32*)(ws + OFF_Q + (unsigned)(7 + lq) * 128u);
            const unsigned qbase = (seq - 1u) * (1024u + NT * NLW);
            gu64* fbase = (gu64*)(ws + OFF_FOUT + (unsigned)peer * 8192u);
            const unsigned lbase = (unsigned)wid * 16384u;
            unsigned nq = 0; u64 fn = 0;
            StripRegs sr = {};
            QNEXT(nq);
            asm volatile("s_waitcnt vmcnt(0)" : "+v"(nq) :: "memory");
            unsigned s = rfl(nq) - qbase;
            bool first = true;
            while (s < 1024u) {
                unsigned fh = rfl((unsigned)(fn >> 32)), info = rfl((unsigned)fn);
                if (fh < seq && !dead) {
                    const u64 t0 = __builtin_amdgcn_s_memrealtime(); unsigned n = 0;
                    for (;;) {
                        const u64 f2 = __hip_atomic_load(fbase + s, __ATOMIC_RELAXED, __HIP_MEMORY_SCOPE_SYSTEM);
                        fh = rfl((unsigned)(f2 >> 32)); info = rfl((unsigned)f2);
                        if (fh >= seq) break;
                        if ((++n & 63u) == 0u && give_up(tmo, seq, t0)) { dead = true; break; }
                        __builtin_amdgcn_s_sleep(4);
                    }
                }
                __builtin_amdgcn_fence(__ATOMIC_ACQUIRE, "wavefront");
                if (dead) info = 3u << 30;
                if (first) STAMP(4);
                first = false;
                TOKEN_TAKE();
                strip_issue(sr, pd, s, info, lane);
                QNEXT(nq);
                STRIP_LANDED(sr, fn, nq, 0);
                TOKEN_GIVE();
                const unsigned sn = rfl(nq) - qbase;
                { gu64* fa = fbase + (sn & 1023u); asm volatile("global_load_dwordx2 %0, %1, off sc0 sc1" : "+v"(fn) : "v"(fa) : "memory"); }
                strip_stage1(sr, smem, lbase, esc, info, lane);
                strip_stage2(sr.mk, outr, por, smem, lbase, peer, s, info, lane);
                asm volatile("s_waitcnt vmcnt(16)" : "+v"(fn) :: "memory");
                s = sn;
            }
            STAMP(6);
        }
        return;
    }

    const bool split = (wi & 15) < nsplit;
    const int wr = wid >> 2, wc = wid & 3;
    constexpr int HTB = 16384;
    constexpr size_t ASTEP = 64 * 8192, BSTEP = 32 * 8192;
    const unsigned ldsw = (unsigned)wid * 1024u;
    const unsigned smbase = (unsigned)(uintptr_t)smem;
#define SA(b, h) (((b) * 2 + (h)) * HTB)
#define SB(b, h) ((4 + (b) * 2 + (h)) * HTB)
#define STAGE(bufoff, gptr) do { _Pragma("unroll") for (int _i = 0; _i < 2; ++_i) \
        __builtin_amdgcn_global_load_lds((const unsigned*)((gptr) + voff[_i]), (LAS unsigned*)(smem + (bufoff) + ldsw + _i * 8192), 16, 0, SYS); } while (0)
#define LDA(dst, b, h) do { _Pragma("unroll") for (int m = 0; m < 4; ++m) _Pragma("unroll") for (int k = 0; k < 2; ++k) dst[m][k] = *(const LAS bf16x8*)(smem + SA(b, h) + aoff + m * 2048 + k * 1024); } while (0)
#define LDB(dst, b, h) do { _Pragma("unroll") for (int n = 0; n < 2; ++n) _Pragma("unroll") for (int k = 0; k < 2; ++k) dst[n][k] = *(const LAS bf16x8*)(smem + SB(b, h) + boff + n * 2048 + k * 1024); } while (0)
#define MMA(ai, bj, At, Bt) do { __builtin_amdgcn_s_setprio(1); _Pragma("unroll") for (int m = 0; m < 4; ++m) _Pragma("unroll") for (int n = 0; n < 2; ++n) _Pragma("unroll") for (int k = 0; k < 2; ++k) \
        acc[ai][bj][m][n] = __builtin_amdgcn_mfma_f32_16x16x32_bf16(Bt[n][k], At[m][k], acc[ai][bj][m][n], 0, 0, 0); __builtin_amdgcn_s_setprio(0); } while (0)
#define ZERO_ACC(a) _Pragma("unroll") for (int b = 0; b < 2; ++b) _Pragma("unroll") for (int m = 0; m < 4; ++m) _Pragma("unroll") for (int n = 0; n < 2; ++n) acc[a][b][m][n] = (f32x4){0.f, 0.f, 0.f, 0.f};
#define NEED(s) do { if (__builtin_expect((avail & 0xffffu) <= (unsigned)(s), 0)) { \
        if (wid == 0) { const unsigned s0 = avail & 0xffffu; unsigned av; const u64 t0 = __builtin_amdgcn_s_memrealtime(); unsigned n = 0; \
            const unsigned pn = (s0 + (lane & 31)) & 127u, po = pollo + SMAP(pn) * 16u; \
            for (;;) { const u32x4 f = bld16<SYS>(wsr, po); \
                const bool ok = (s0 + (lane & 31) >= (unsigned)te) || (f.x >= seq && f.y >= seq && f.z >= seq && f.w >= seq); \
                const u64 mm = __builtin_amdgcn_ballot_w64(ok); const unsigned both = (unsigned)mm & (unsigned)(mm >> 32); \
                av = s0 + (both == 0xffffffffu ? 32u : (unsigned)__builtin_ctz(~both)); \
                if (av > (unsigned)(s)) break; \
                if ((++n & 63u) == 0u && give_up(tmo, seq, t0)) { av = 0x10000u | NSTEP; break; } \
                __builtin_amdgcn_s_sleep(1); } \
            asm volatile("ds_write_b32 %0, %1\n\ts_waitcnt lgkmcnt(0)" :: "v"(smbase + SM_AVAIL), "v"(av) : "memory"); } \
        CFENCE; BAR; CFENCE; \
        unsigned av2; asm volatile("ds_read_b32 %0, %1\n\ts_waitcnt lgkmcnt(0)" : "=v"(av2) : "v"(smbase + SM_AVAIL) : "memory"); \
        avail = rfl(av2); } } while (0)
#define LOOP_STAMPS do { } while (0)
#define OPAQUE_U(dst, src) unsigned dst##_v = (unsigned)(src); asm volatile("" : "+v"(dst##_v)); const unsigned dst = rfl(dst##_v);
    for (int job = 0; job < 2; ++job) {
        int tid = threadIdx.x; asm volatile("" : "+v"(tid));
        const int lane = tid & 63, fr = lane & 15, fq = lane >> 4;
        const bool pair = split && job == 0;
        const int hsel = wi >> 4;
        const int atile = 44 + (xcd < 2 ? xcd * 32 + wi : (xcd == 2 ? 64 : 80) + (wi & 15) + nsplit * hsel);
        if (job && atile >= 128) return;
        const int tm = job ? atile >> 3 : (xcd >> 1) * 4 + ((wi & 15) >> 2), tn = job ? atile & 7 : (xcd & 1) * 4 + (wi & 3);
        const char* cA = ws + OFF_XL + (size_t)tm * 32768;
        const char* cB = ws + OFF_WL + (size_t)tn * 32768;
        const unsigned voff[2] = {(unsigned)tid * 16u, (unsigned)tid * 16u + 8192u};
        const int aoff = lds_byte(wr * 64 + fr, fq * 8), boff = lds_byte(wc * 32 + fr, fq * 8);
        const unsigned pollo = lane < 32 ? OFF_XF + (unsigned)(tm * NSTEP) * 16u : OFF_WF + (unsigned)(tn * NSTEP) * 16u;
        const int kh = tm < 8 ? KH1 : (tm < 12 ? KH2 : KH3);
        const int tb = job ? NSTEP - kh : 0, te = pair ? 64 : (job ? NSTEP : NSTEP - kh);
        const unsigned nl = tm < 8 ? (unsigned)NL1 : 16u;
        auto smap = [nl, pair, hsel](unsigned n) -> unsigned {
            if (pair) return (((n & ~3u) << 1) | (unsigned)(hsel << 2) | (n & 3u)) & 127u;
            if (n < nl) return n * 8u;
            const unsigned m = n - nl, full = (16u - nl) * 8u;
            if (m < full) return (m & 7u) == 7u ? (nl + (m >> 3)) * 8u : (m & ~7u) + 1u + (m & 7u);
            const unsigned r = m - full; return (16u - nl + r / 7u) * 8u + 1u + r % 7u;
        };
        const unsigned stab0 = smap((unsigned)lane), stab1 = smap((unsigned)lane + 64u);
#define SMAP(n) smap(n)
#define STAB(n) ((unsigned)__builtin_amdgcn_readlane((n) & 64 ? stab1 : stab0, (n) & 63))
        unsigned avail = (unsigned)tb;
        f32x4 acc[2][2][4][2];
        ZERO_ACC(0) ZERO_ACC(1)
        bf16x8 At[4][2], B0[2][2], B1[2][2];
        NEED(tb + 1);
        if (!job) STAMP(2);
        { const size_t s0 = STAB(tb); STAGE(SB(0, 0), cB + s0 * BSTEP); STAGE(SA(0, 0), cA + s0 * ASTEP); STAGE(SB(0, 1), cB + s0 * BSTEP + HTB); STAGE(SA(0, 1), cA + s0 * ASTEP + HTB); }
        if (wr == 1) BAR;
        WAIT_V(4); BAR;
        unsigned sp = STAB(tb + 1);
        { const size_t s1 = sp; STAGE(SB(1, 0), cB + s1 * BSTEP); STAGE(SA(1, 0), cA + s1 * ASTEP); STAGE(SB(1, 1), cB + s1 * BSTEP + HTB); }
        WAIT_V(6); BAR;
        for (int t = tb; t < te; t += 2) {
            const bool last = (t == te - 2);
            const int tn2 = last ? tb : t + 2;
            const unsigned s1 = sp, s2 = STAB(tn2), s3 = STAB(tn2 + 1);
            sp = s3;
            const char* a1 = cA + s1 * ASTEP;
            const char* a2 = cA + s2 * ASTEP; const char* b2 = cB + s2 * BSTEP;
            const char* a3 = cA + s3 * ASTEP; const char* b3 = cB + s3 * BSTEP;
            if (!last) NEED(t + 3);
            LOOP_STAMPS;
            LDB(B0, 0, 0); SCHED; LDA(At, 0, 0); STAGE(SA(1, 1), a1 + HTB);
            WAIT_L(8); BAR; WAIT_L(0); MMA(0, 0, At, B0); BAR; SCHED;
            LDB(B1, 0, 1); STAGE(SB(0, 0), b2);
            BAR; WAIT_L(0); MMA(0, 1, At, B1); BAR;
            LDA(At, 0, 1); STAGE(SA(0, 0), a2);
            BAR; WAIT_L(0); MMA(1, 0, At, B0); BAR; SCHED;
            STAGE(SB(0, 1), b2 + HTB);
            WAIT_V(6); BAR; MMA(1, 1, At, B1); BAR;
            LDB(B0, 1, 0); SCHED; LDA(At, 1, 0); STAGE(SA(0, 1), a2 + HTB);
            WAIT_L(8); BAR; WAIT_L(0); MMA(0, 0, At, B0); BAR; SCHED;
            LDB(B1, 1, 1); STAGE(SB(1, 0), b3);
            BAR; WAIT_L(0); MMA(0, 1, At, B1); BAR;
            LDA(At, 1, 1); STAGE(SA(1, 0), a3);
            BAR; WAIT_L(0); MMA(1, 0, At, B0); BAR; SCHED;
            STAGE(SB(1, 1), b3 + HTB);
            WAIT_V(6); BAR; MMA(1, 1, At, B1); BAR;
        }
        if (wr == 0) BAR;
        if (!job) STAMP(12);
        int tz = threadIdx.x; asm volatile("" : "+v"(tz));
        const int lane2 = tz & 63;
        const unsigned po = OFF_PART + (unsigned)(tm * 8 + tn) * 262144u + (unsigned)wid * 32768u + (unsigned)lane2 * 16u;
        gu32* pfl = (gu32*)(ws + OFF_PF + (unsigned)((tm * 8 + tn) * 16 + wid) * 4u);
        const unsigned give = job ? 3u : (pair ? (hsel ? 1u : 2u) : 0u), take = job ? 0u : (pair ? (hsel ? 2u : 1u) : 3u);
#pragma unroll
        for (int h = 0; h < 2; ++h) {
            OPAQUE_U(g1, (give >> h) & 1u)
            if (g1) {
#pragma unroll
                for (int q = 0; q < 16; ++q) bst16<SC1>(wsr, po + (h * 16 + q) * 1024, __builtin_bit_cast(u32x4, acc[h][(q >> 3) & 1][(q >> 1) & 3][q & 1]));
                WAIT_V(0);
                if (lane2 == 0) __hip_atomic_store(pfl + h * 8, seq, __ATOMIC_RELAXED, __HIP_MEMORY_SCOPE_AGENT);
            }
        }
        if (job) { STAMP(11); return; }
#pragma unroll
        for (int h = 0; h < 2; ++h) {
            OPAQUE_U(t1, (take >> h) & 1u)
            if (t1) {
                if (rfl(__hip_atomic_load(pfl + h * 8, __ATOMIC_RELAXED, __HIP_MEMORY_SCOPE_AGENT)) < seq) {
                    const u64 t0 = __builtin_amdgcn_s_memrealtime(); unsigned n = 0;
                    while (rfl(__hip_atomic_load(pfl + h * 8, __ATOMIC_RELAXED, __HIP_MEMORY_SCOPE_AGENT)) < seq) {
                        if ((++n & 63u) == 0u && give_up(tmo, seq, t0)) break;
                        __builtin_amdgcn_s_sleep(1);
                    }
                }
                __builtin_amdgcn_fence(__ATOMIC_ACQUIRE, "wavefront");
                u32x4 pv[16];
#pragma unroll
                for (int q = 0; q < 16; ++q) pv[q] = bld16<SC1>(wsr, po + (h * 16 + q) * 1024);
#pragma unroll
                for (int q = 0; q < 16; ++q) { acc[h][(q >> 3) & 1][(q >> 1) & 3][q & 1] += __builtin_bit_cast(f32x4, pv[q]);
                    asm volatile("" : "+v"(acc[h][(q >> 3) & 1][(q >> 1) & 3][q & 1])); }
                CFENCE; SCHED;
            }
        }
        const unsigned tmov = __hip_atomic_load(tmo, __ATOMIC_RELAXED, __HIP_MEMORY_SCOPE_AGENT);
        WAIT_V(0); BAR;
        STAMP(3);
        const int fr2 = lane2 & 15, fq2 = lane2 >> 4;
        const bool dead = (avail >> 16) != 0u || rfl(tmov) == seq;
#pragma unroll
        for (int ai = 0; ai < 2; ++ai) {
            OPAQUE_U(t1, (take >> ai) & 1u)
            if (t1) {
#pragma unroll
                for (int m = 0; m < 4; ++m) {
                    const int row = ai * 128 + wr * 64 + m * 16 + fr2;
#pragma unroll
                    for (int bj = 0; bj < 2; ++bj) {
                        f32x4 v0 = acc[ai][bj][m][0], v1 = acc[ai][bj][m][1];
#pragma unroll
                        for (int k = 0; k < 4; ++k) { v0[k] = v0[k] <= 0.f ? 0.f : v0[k]; v1[k] = v1[k] <= 0.f ? 0.f : v1[k]; }
                        u32x4 w; w.x = cvt_pk_bf16(v0[0], v0[1]); w.y = cvt_pk_bf16(v0[2], v0[3]); w.z = cvt_pk_bf16(v1[0], v1[1]); w.w = cvt_pk_bf16(v1[2], v1[3]);
                        if (dead) w = (u32x4){0x7fc07fc0u, 0x7fc07fc0u, 0x7fc07fc0u, 0x7fc07fc0u};
                        const int chunk = bj * 16 + wc * 4 + fq2;
                        *(LAS u32x4*)(smem + row * 512 + ((chunk ^ (row & 7)) * 16)) = w;
                    }
                }
            }
        }
        __syncthreads();
        STAMP(4);
        if ((take >> wr) & 1u) pack_strip(p, wsr, outr, smem, (tm * 8 + tn) * 8 + wid, wid, lane2, seq);
        STAMP(5);
        if (!split) return;
        __syncthreads();
    }
}

extern "C" void kernel_launch(int rank, int nranks, ncclComm_t comm,
                              hipStream_t stream,
                              void* const* d_in, const int* in_sizes, int n_in,
                              void* d_out, int out_size,
                              const int* in_dtypes, int out_dtype,
                              void* const* d_out_all,
                              void* const* d_ws_all, size_t ws_bytes) {
    (void)comm; (void)in_dtypes; (void)out_dtype;
    if (nranks != NR || n_in != 2 || (unsigned)rank >= (unsigned)NR || ws_bytes < WS_USED || in_sizes[0] != ND * KS || in_sizes[1] != KS * ND || out_size != ND * ND) {
        fprintf(stderr, "kernel_launch: unexpected problem shape\n");
        return;
    }
    Params p;
    p.x = (const u16*)d_in[0]; p.w = (const u16*)d_in[1]; p.out = (u16*)d_out;
    p.out_all = d_out_all; p.ws_all = d_ws_all; p.rank = rank;
    hipLaunchKernelGGL(gemm_ar_fused, dim3(NWG), dim3(512), 0, stream, p);
    HIP_CHECK(hipGetLastError());
}
```

```cpp
#include <hip/hip_runtime.h>
#include <rccl/rccl.h>
#include <cstdio>
#include <cstdint>

#define HIP_CHECK(expr) do { hipError_t _e = (expr); if (_e != hipSuccess) { \
    fprintf(stderr, "HIP error %s at %s:%d\n", hipGetErrorString(_e), __FILE__, __LINE__); \
    return; } } while (0)

#define LAS __attribute__((address_space(3)))
#define GAS __attribute__((address_space(1)))
typedef unsigned short u16;
typedef unsigned long long u64;
typedef short bf16x8 __attribute__((ext_vector_type(8)));
typedef short s16x4 __attribute__((ext_vector_type(4)));
typedef float f32x4 __attribute__((ext_vector_type(4)));
typedef unsigned u32x4 __attribute__((ext_vector_type(4)));
typedef unsigned u32x2 __attribute__((ext_vector_type(2)));
typedef u16 u16x2 __attribute__((ext_vector_type(2)));
typedef GAS unsigned gu32;
typedef GAS u64 gu64;

#define FORCE_RAW_IN 0
#define FORCE_RAW_OUT 0
#define NT 12
constexpr int NWV = 4;
#define NLW 6
#define WS_AT_END 1
#define OAUX 18
#define LAG1 0
#define LAG2 9
#define LAG3 13
#define KH1 12
#define KH2 24
#define KH3 40
#define NL1 4
#define PDELAY 336
#define PSLEEP 64

namespace {

constexpr int NR = 8;
constexpr int ND = 8192;
constexpr int KS = 1024;
constexpr int RROWS = 4096, RCOLS = 2048;
constexpr int NSTEP = 128;
constexpr int NWG = 256;
constexpr int NPD = 84 * 4;
constexpr int NENC = NPD + 512;
constexpr int NONCE = 256 + 512 + 96;
static_assert(KH1 % 2 == 0 && KH2 % 2 == 0 && KH3 % 2 == 0 && KH1 >= 2 && NLW >= NWV && NLW <= 8, "knobs");
__device__ __forceinline__ constexpr int lag(int k) { return k == 0 ? 0 : k == 1 ? LAG1 : k == 2 ? LAG2 : LAG3; }
constexpr unsigned long long DEADLINE = 40000000ull;

constexpr unsigned USLOT = 8192 + 1024;
constexpr unsigned SSLOT = 11264;
constexpr unsigned OFF_PART = 0;
constexpr unsigned OFF_EO = OFF_PART + (32u << 20);
constexpr unsigned OFF_XL = OFF_EO + (12u << 20);
constexpr unsigned OFF_WL = OFF_XL + (64u << 20);
constexpr unsigned OFF_EX = OFF_WL + (32u << 20);
constexpr unsigned OFF_EW = OFF_EX + (18u << 20);
constexpr unsigned OFF_C = OFF_EW + (18u << 20);
constexpr unsigned WS_USED = OFF_C + (1u << 20);
constexpr unsigned OFF_TMO = OFF_C + 0x80;
constexpr unsigned OFF_Q = OFF_C + 0x400;
constexpr unsigned OFF_ECNT = OFF_C + 0x1000;
constexpr unsigned OFF_FIN = OFF_C + 0x8000;
constexpr unsigned OFF_SEQ = OFF_C + 0xa000;
constexpr unsigned OFF_PF = OFF_C + 0xc000;
constexpr unsigned OFF_XF = OFF_C + 0x10000;
constexpr unsigned OFF_WF = OFF_C + 0x18000;
constexpr unsigned OFF_FOUT = OFF_C + 0x20000;
static_assert(2048u * USLOT <= (18u << 20), "unit area");
static_assert(1024u * SSLOT <= (12u << 20), "strip area");
constexpr unsigned OUT_BYTES = (unsigned)ND * ND * 2u;

constexpr int SM_TOKEN = 131072 + 32, SM_DUMMY = 131072 + 64, SM_LUT = 131072 + 256, SM_LUTE = SM_LUT + 4096, SM_SCR = SM_LUTE + 128;
constexpr int SM_AVAIL = SM_SCR + 24576 + 64;
constexpr int SMEM_BYTES = SM_AVAIL + 16;
constexpr unsigned CAP2 = 2304, CAP3 = 512, CAP4 = 240;

struct Params {
    const u16* x; const u16* w; u16* out;
    void* const* out_all; void* const* ws_all;
    int rank;
    size_t ws_off;
};

typedef __amdgpu_buffer_rsrc_t rsrc_t;
__device__ __forceinline__ rsrc_t make_rsrc(const void* base, unsigned bytes) {
    const u64 b = (u64)base;
    const unsigned lo = __builtin_amdgcn_readfirstlane((unsigned)b), hi = __builtin_amdgcn_readfirstlane((unsigned)(b >> 32));
    return __builtin_amdgcn_make_buffer_rsrc((void*)(((u64)hi << 32) | lo), 0, bytes, 0x00020000);
}
constexpr int SC1 = 16, SYS = 17;
template <int AUX> __device__ __forceinline__ u32x4 bld16(rsrc_t r, unsigned off) { return __builtin_amdgcn_raw_buffer_load_b128(r, off, 0, AUX); }
template <int AUX> __device__ __forceinline__ unsigned bld4(rsrc_t r, unsigned off) { return __builtin_amdgcn_raw_buffer_load_b32(r, off, 0, AUX); }
template <int AUX> __device__ __forceinline__ unsigned bld2(rsrc_t r, unsigned off) { return __builtin_amdgcn_raw_buffer_load_b16(r, off, 0, AUX); }
template <int AUX> __device__ __forceinline__ void bst16(rsrc_t r, unsigned off, u32x4 v) { __builtin_amdgcn_raw_buffer_store_b128(v, r, off, 0, AUX); }
template <int AUX> __device__ __forceinline__ void bst8(rsrc_t r, unsigned off, u32x2 v) { __builtin_amdgcn_raw_buffer_store_b64(v, r, off, 0, AUX); }
template <int AUX> __device__ __forceinline__ void bst4(rsrc_t r, unsigned off, unsigned v) { __builtin_amdgcn_raw_buffer_store_b32(v, r, off, 0, AUX); }
template <int AUX> __device__ __forceinline__ void bst2(rsrc_t r, unsigned off, unsigned v) { __builtin_amdgcn_raw_buffer_store_b16((u16)v, r, off, 0, AUX); }

__device__ __forceinline__ u32x4 make_desc(const void* base, unsigned bytes) {
    const u64 b = (u64)base;
    return (u32x4){(unsigned)__builtin_amdgcn_readfirstlane((unsigned)b), (unsigned)__builtin_amdgcn_readfirstlane((unsigned)(b >> 32)), bytes, 0x00020000u};
}
#define ALD16(dst, desc, off, imm) asm volatile("buffer_load_dwordx4 %0, %1, %2, 0 offen offset:%3 sc0 sc1" : "+v"(dst) : "v"(off), "s"(desc), "i"(imm) : "memory")
#define ALD8(dst, desc, off, imm) asm volatile("buffer_load_dwordx2 %0, %1, %2, 0 offen offset:%3 sc0 sc1" : "+v"(dst) : "v"(off), "s"(desc), "i"(imm) : "memory")
#define ALD4(dst, desc, off, imm) asm volatile("buffer_load_dword %0, %1, %2, 0 offen offset:%3 sc0 sc1" : "+v"(dst) : "v"(off), "s"(desc), "i"(imm) : "memory")
#define ALD2(dst, desc, off, imm) asm volatile("buffer_load_ushort %0, %1, %2, 0 offen offset:%3 sc0 sc1" : "+v"(dst) : "v"(off), "s"(desc), "i"(imm) : "memory")
constexpr unsigned OOB = 0xfffff000u;

#define WAIT_V(n) asm volatile("s_waitcnt vmcnt(" #n ")" ::: "memory")
#define WAIT_L(n) asm volatile("s_waitcnt lgkmcnt(" #n ")" ::: "memory")
#define BAR __builtin_amdgcn_s_barrier()
#define SCHED __builtin_amdgcn_sched_barrier(0)
#define CFENCE asm volatile("" ::: "memory")
constexpr size_t WS_TOTAL = WS_USED;
#define STAMP(k) do { } while (0)

__device__ __forceinline__ unsigned rfl(unsigned v) { return __builtin_amdgcn_readfirstlane(v); }
__device__ __forceinline__ unsigned pkmax(unsigned a, unsigned b) {
    u16x2 x = __builtin_bit_cast(u16x2, a), y = __builtin_bit_cast(u16x2, b);
    return __builtin_bit_cast(unsigned, __builtin_elementwise_max(x, y));
}
__device__ __forceinline__ unsigned pkmin(unsigned a, unsigned b) {
    u16x2 x = __builtin_bit_cast(u16x2, a), y = __builtin_bit_cast(u16x2, b);
    return __builtin_bit_cast(unsigned, __builtin_elementwise_min(x, y));
}
__device__ __forceinline__ unsigned wave_max(unsigned v) {
#define MAX_STEP(ctrl, rmask) { const unsigned t = (unsigned)__builtin_amdgcn_update_dpp(0, (int)v, ctrl, rmask, 0xf, false); v = v > t ? v : t; }
    MAX_STEP(0x111, 0xf) MAX_STEP(0x112, 0xf) MAX_STEP(0x114, 0xf) MAX_STEP(0x118, 0xf) MAX_STEP(0x142, 0xa) MAX_STEP(0x143, 0xc)
    return __builtin_amdgcn_readlane(v, 63);
}
__device__ __forceinline__ unsigned pkmin1(unsigned a) {
    unsigned r; asm("v_pk_min_u16 %0, %1, 1 op_sel_hi:[1,0]" : "=v"(r) : "v"(a)); return r;
}
__device__ __forceinline__ unsigned cvt_pk_bf16(float lo, float hi) { unsigned r; asm volatile("v_cvt_pk_bf16_f32 %0, %1, %2" : "=v"(r) : "v"(lo), "v"(hi)); return r; }
__device__ __forceinline__ int lds_byte(int r, int c) { const int st = (r >> 4) * 2 + (c >> 5), rr = r & 15, cc = c & 31, ob = rr * 64 + cc * 2; return st * 1024 + (ob ^ (((ob >> 9) & 1) << 5)); }

__device__ __forceinline__ unsigned wave_scan(unsigned x) {
    x += (unsigned)__builtin_amdgcn_update_dpp(0, (int)x, 0x111, 0xf, 0xf, false);
    x += (unsigned)__builtin_amdgcn_update_dpp(0, (int)x, 0x112, 0xf, 0xf, false);
    x += (unsigned)__builtin_amdgcn_update_dpp(0, (int)x, 0x114, 0xf, 0xf, false);
    x += (unsigned)__builtin_amdgcn_update_dpp(0, (int)x, 0x118, 0xf, 0xf, false);
    x += (unsigned)__builtin_amdgcn_update_dpp(0, (int)x, 0x142, 0xa, 0xf, false);
    x += (unsigned)__builtin_amdgcn_update_dpp(0, (int)x, 0x143, 0xc, 0xf, false);
    return x;
}
__device__ __forceinline__ unsigned wave_sum(unsigned v) { return __builtin_amdgcn_readlane(wave_scan(v), 63); }
__device__ __forceinline__ unsigned pksub(unsigned a, unsigned b) {
    u16x2 x = __builtin_bit_cast(u16x2, a), y = __builtin_bit_cast(u16x2, b);
    return __builtin_bit_cast(unsigned, (u16x2)(x - y));
}
__device__ __forceinline__ unsigned bfi(unsigned mask, unsigned a, unsigned b) { return (a & mask) | (b & ~mask); }
__device__ __forceinline__ unsigned choose_window(unsigned esum, unsigned n, unsigned emax) {
    if (!n) return 21u;
    const unsigned num = esum * 32u + n * 63u, den = n * 32u;
    unsigned t = num / den; const unsigned alt = (num - t * den) * 5u > den * 2u ? 1u : 0u;
    t = t > emax ? emax : t; t = t < 21u ? 21u : (t > 253u ? 253u : t);
    return t | (alt << 9);
}
template <int NQ> __device__ __forceinline__ unsigned apply_escapes(unsigned (&e)[NQ], const unsigned (&m)[NQ], LAS unsigned char* smem, unsigned off) {
    unsigned sel[NQ], o[NQ]; u32x2 w[NQ];
#pragma unroll
    for (int q = 0; q < NQ; ++q) {
        sel[q] = *(const LAS unsigned*)(smem + SM_LUTE + __builtin_amdgcn_udot4(m[q], 0x08040201u, 0u, false));
        o[q] = off; off += (unsigned)__builtin_popcount(m[q]);
        const LAS unsigned* pw = (const LAS unsigned*)(smem + (o[q] & ~3u));
        w[q] = (u32x2){pw[0], pw[1]};
    }
#pragma unroll
    for (int q = 0; q < NQ; ++q) e[q] = __builtin_amdgcn_perm(__builtin_amdgcn_alignbyte(w[q].y, w[q].x, o[q]), e[q], sel[q]);
    return off;
}
__device__ __forceinline__ unsigned append_flagged(LAS unsigned char* smem, unsigned e, unsigned m, unsigned off, unsigned dummy) {
#pragma unroll
    for (int k = 0; k < 4; ++k) {
        const unsigned bit = (m >> (8 * k + 2)) & 1u;
        smem[bit ? off : dummy] = (unsigned char)(e >> (8 * k));
        off += bit;
    }
    return off;
}
__device__ __forceinline__ void decode_levels(LAS unsigned char* smem, unsigned h0, unsigned h1, unsigned o2, unsigned l3, unsigned l2, int lane) {
    const unsigned t = h0 & 0xffu, alt = (h0 >> 9) & 1u, n2 = h0 >> 16, n3 = h1 & 0xffffu;
    const unsigned o3 = o2 + ((n2 + 15u) >> 4) * 4u, o4 = o3 + ((n3 + 7u) >> 3) * 4u;
    const unsigned b3q = (t - 6u + alt) * 0x01010101u, spq = (t + 1u + alt) * 0x01010101u;
    unsigned eo = o4;
    for (unsigned r = 0; r * 256u < n3; ++r) {
        const unsigned g = r * 64u + lane;
        unsigned c = *(const LAS u16*)(smem + o3 + g * 2u); c = g * 4u < n3 ? c : 0u;
        const unsigned z = __builtin_amdgcn_perm(0u, c, 0x01010000u);
        const unsigned d = (z & 0x000f000fu) | ((z >> 4) & 0x0f000f00u);
        const unsigned is15 = (d + 0x01010101u) & 0x10101010u, is14 = ((d + 0x02020202u) & 0x10101010u) ^ is15;
        unsigned e[1], m[1];
        e[0] = bfi((is14 << 4) - (is14 >> 4), spq, b3q - d); m[0] = is15 >> 2;
        const unsigned ec = (unsigned)__builtin_popcount(m[0]), inc = wave_scan(ec);
        apply_escapes<1>(e, m, smem, eo + inc - ec);
        eo += __builtin_amdgcn_readlane(inc, 63);
        *(LAS unsigned*)(smem + l3 + g * 4u) = e[0];
    }
    const unsigned tab = (t - 3u) | ((t - 4u) << 8) | ((alt ? t + 1u : t - 5u) << 16);
    eo = l3;
    for (unsigned r = 0; r * 1024u < n2; ++r) {
        const unsigned g = r * 64u + lane;
        unsigned c = *(const LAS unsigned*)(smem + o2 + g * 4u); c = g * 16u < n2 ? c : 0u;
        unsigned e[4], m[4];
#pragma unroll
        for (int k = 0; k < 4; ++k) { const unsigned d = (c >> (2 * k)) & 0x03030303u; e[k] = __builtin_amdgcn_perm(0u, tab, d); m[k] = (d + 0x01010101u) & 0x04040404u; }
        const unsigned ec = (unsigned)__builtin_popcount(c & (c >> 1) & 0x55555555u), inc = wave_scan(ec);
        apply_escapes<4>(e, m, smem, eo + inc - ec);
        eo += __builtin_amdgcn_readlane(inc, 63);
        if (g * 16u < n2) *(LAS u32x4*)(smem + l2 + g * 16u) = (u32x4){e[0], e[1], e[2], e[3]};
    }
}
__device__ __forceinline__ unsigned encode_levels(LAS unsigned char* smem, rsrc_t wsr, unsigned win, unsigned n2, unsigned l2, unsigned l3, unsigned l4, unsigned g2, unsigned room, unsigned dummy, int lane) {
    const unsigned t = win & 0xffu, alt = (win >> 9) & 1u;
    if (lane < 16) smem[l2 + n2 + lane] = (unsigned char)(t - 3u);
    const unsigned t3 = (t - 3u) * 0x00010001u, sc = (alt ? t + 1u : t - 5u) * 0x00010001u;
    unsigned n3 = 0;
    for (unsigned r = 0; r * 1024u < n2; ++r) {
        const unsigned g = r * 64u + lane; const bool act = g * 16u < n2;
        const u32x4 e = *(const LAS u32x4*)(smem + l2 + (act ? g : 0u) * 16u);
        unsigned c = 0, m[4], ec = 0;
#pragma unroll
        for (int k = 0; k < 4; ++k) {
            unsigned dd[2];
#pragma unroll
            for (int h = 0; h < 2; ++h) {
                const unsigned x = (e[k] >> (8 * h)) & 0x00ff00ffu;
                const unsigned a = pkmin(pksub(t3, x), 0x00020002u);
                dd[h] = a + ((a >> 1) & pkmin1(x ^ sc));
            }
            const unsigned d = dd[0] | (dd[1] << 8);
            c |= d << (2 * k); m[k] = act ? (d + 0x01010101u) & 0x04040404u : 0u; ec += (unsigned)__builtin_popcount(m[k]);
        }
        const unsigned inc = wave_scan(ec), tot = __builtin_amdgcn_readlane(inc, 63);
        if (n3 + tot > CAP3) return 0xffffffffu;
        if (act) bst4<SYS>(wsr, g2 + g * 4u, c);
        unsigned eo = l3 + n3 + inc - ec;
#pragma unroll
        for (int k = 0; k < 4; ++k) eo = append_flagged(smem, e[k], m[k], eo, dummy);
        n3 += tot;
    }
    const unsigned b3 = t - 6u + alt;
    if (lane < 8) smem[l3 + n3 + lane] = (unsigned char)b3;
    const unsigned g3 = g2 + ((n2 + 15u) >> 4) * 4u, g4 = g3 + ((n3 + 7u) >> 3) * 4u;
    if (g4 - g2 > room) return 0xffffffffu;
    const unsigned b32 = b3 * 0x00010001u, sp = (t + 1u + alt) * 0x00010001u;
    unsigned n4 = 0;
    for (unsigned r = 0; r * 256u < n3; ++r) {
        const unsigned g = r * 64u + lane; const bool act = g * 4u < ((n3 + 7u) & ~7u);
        const unsigned e = *(const LAS unsigned*)(smem + l3 + (act ? g : 0u) * 4u);
        unsigned dd[2];
#pragma unroll
        for (int h = 0; h < 2; ++h) {
            const unsigned x = (e >> (8 * h)) & 0x00ff00ffu;
            const unsigned a = pkmin(pksub(b32, x), 0x000e000eu);
            dd[h] = a + (((a + 0x00020002u) >> 4) & pkmin1(x ^ sp));
        }
        const unsigned d = dd[0] | (dd[1] << 8);
        const unsigned m = act ? ((d + 0x01010101u) & 0x10101010u) >> 2 : 0u;
        const unsigned ec = (unsigned)__builtin_popcount(m), inc = wave_scan(ec), tot = __builtin_amdgcn_readlane(inc, 63);
        if (n4 + tot > CAP4) return 0xffffffffu;
        const unsigned y = d | (d >> 4);
        if (act) bst2<SYS>(wsr, g3 + g * 2u, __builtin_amdgcn_perm(0u, y, 0x0c0c0200u));
        if (tot) append_flagged(smem, e, m, l4 + n4 + inc - ec, dummy);
        n4 += tot;
    }
    if (g4 - g2 + n4 > room) return 0xffffffffu;
    if (n4) { const unsigned lv = *(const LAS unsigned*)(smem + l4 + lane * 4); if ((unsigned)lane * 4u < n4) bst4<SYS>(wsr, g4 + lane * 4, lv); }
    return n3 | (n4 << 16);
}
__device__ __forceinline__ unsigned side_bytes(unsigned n2, unsigned h1) { return ((n2 + 15u) >> 4) * 4u + (((h1 & 0xffffu) + 7u) >> 3) * 4u + (h1 >> 16); }
struct Order {
    int ph, start;
    template <int NWU, int NXU> __device__ __forceinline__ void locate(int i, int& j, int& g, int& u) {
        for (;;) {
            int c = (ph < 16) ? NWU : 0;
#pragma unroll
            for (int k = 0; k < 4; ++k) c += ((unsigned)(ph - lag(k)) < 16u) ? NXU : 0;
            if (i < start + c) break;
            start += c; ++ph;
        }
        u = i - start; g = 0; j = ph;
        if (ph < 16) { if (u < NWU) return; u -= NWU; }
#pragma unroll
        for (int k = 0; k < 4; ++k) {
            if ((unsigned)(ph - lag(k)) < 16u) { if (u < NXU) { g = 1 + k; j = ph - lag(k); return; } u -= NXU; }
        }
    }
};

__device__ __forceinline__ bool give_up(gu32* tmo, unsigned seq, u64 t0) {
    bool dead = rfl(__hip_atomic_load(tmo, __ATOMIC_RELAXED, __HIP_MEMORY_SCOPE_AGENT)) == seq;
    if (!dead && __builtin_amdgcn_s_memrealtime() - t0 > DEADLINE) {
        __hip_atomic_store(tmo, seq, __ATOMIC_RELAXED, __HIP_MEMORY_SCOPE_AGENT);
        dead = true;
    }
    return dead;
}
__device__ __forceinline__ unsigned early_word(unsigned u) { return (u / 3u) * 8u + 5u + u % 3u; }

__device__ __forceinline__ void pack_unit(const Params& p, char* ws, rsrc_t wsr, LAS unsigned char* smem, unsigned lwo, int j, int g, int u, int lane, unsigned seqv) {
    const bool isx = g != 0; const int ub = isx ? (u >> 4) * 64 + (g - 1) * 16 + (u & 15) : u;
    const int ra = p.rank >> 2, rb = p.rank & 3;
    LAS unsigned char* lw = smem + lwo;
    u32x4 v[8];
    if (isx) {
        const u16* src = p.x + (size_t)(ub * 64 + (lane & 15)) * KS + j * 64 + (lane >> 4) * 8;
#pragma unroll
        for (int c = 0; c < 8; ++c) v[c] = *(const u32x4*)(src + (size_t)(c >> 1) * 16 * KS + (c & 1) * 32);
    } else {
        constexpr int P = 144;
        const u16* src = p.w + (size_t)(j * 64 + (lane >> 3)) * ND + ub * 64 + (lane & 7) * 8;
        u32x4 tmp[8];
#pragma unroll
        for (int i = 0; i < 8; ++i) tmp[i] = *(const u32x4*)(src + (size_t)i * 8 * ND);
#pragma unroll
        for (int i = 0; i < 8; ++i) *(LAS u32x4*)(lw + (i * 8 + (lane >> 3)) * P + (lane & 7) * 16) = tmp[i];
        const int q = (lane >> 2) & 3, pp = lane & 3;
#pragma unroll
        for (int c = 0; c < 8; ++c) {
            const int k0 = (c & 1) * 32 + (lane >> 4) * 8, rbk = c >> 1;
            const int col = (rbk >> 1) * 32 + 8 * pp + 4 * (rbk & 1);
            const s16x4 a = __builtin_amdgcn_ds_read_tr16_b64_v4i16((LAS s16x4*)(lw + (k0 + q) * P + col * 2));
            const s16x4 b = __builtin_amdgcn_ds_read_tr16_b64_v4i16((LAS s16x4*)(lw + (k0 + 4 + q) * P + col * 2));
            const u32x2 a2 = __builtin_bit_cast(u32x2, a), b2 = __builtin_bit_cast(u32x2, b);
            v[c] = (u32x4){a2.x, a2.y, b2.x, b2.y};
        }
    }
    const unsigned loff = (lane & 15) * 64 + (((lane >> 4) * 16) ^ (((lane >> 3) & 1) << 5));
    const int cls = isx ? (ub >> 6) : (ub >> 5);
    const bool own = isx ? (cls == ra) : (cls == rb);
    const int li = isx ? (ub & 63) : (ub & 31);
    if (own) {
        const unsigned dst = (isx ? OFF_XL + (unsigned)(j * 8 * 64 + li) * 8192u : OFF_WL + (unsigned)(j * 8 * 32 + li) * 8192u) + loff;
#pragma unroll
        for (int c = 0; c < 8; ++c) bst16<SC1>(wsr, dst + c * 1024, v[c]);
    }
    const unsigned slot0 = (isx ? OFF_EX : OFF_EW) + (unsigned)(j * 128 + ub) * USLOT;
    const unsigned pay = slot0 + 1024 + lane * 16;
    const bool early = j == 0 && g == 0;
    if (early) {
#pragma unroll
        for (int c = 0; c < 8; ++c) bst16<SYS>(wsr, pay + c * 1024, v[c]);
        if (lane == 0) bst4<SYS>(wsr, slot0, 1u << 8);
    } else {
    unsigned e4[16], sm[16], mx = 0, esum = 0;
#pragma unroll
    for (int q = 0; q < 16; ++q) {
        const unsigned a = v[q >> 1][(q & 1) * 2], b = v[q >> 1][(q & 1) * 2 + 1];
        const unsigned lo = __builtin_amdgcn_perm(b, a, 0x06040200u), hi = __builtin_amdgcn_perm(b, a, 0x07050301u);
        e4[q] = bfi(0x01010101u, lo >> 7, hi << 1);
        sm[q] = bfi(0x7f7f7f7fu, lo, hi);
        esum = __builtin_amdgcn_udot4(e4[q], 0x01010101u, esum, false);
        mx = pkmax(mx, a & 0x7fff7fffu); mx = pkmax(mx, b & 0x7fff7fffu);
    }
    mx = (mx & 0xffffu) > (mx >> 16) ? (mx & 0xffffu) : (mx >> 16);
    const unsigned win = choose_window(wave_sum(esum), 4096u, wave_max(mx) >> 7);
    const unsigned top2 = (win & 0xffu) * 0x00010001u;
    unsigned p2[4] = {0u, 0u, 0u, 0u}, m[16], ec = 0;
#pragma unroll
    for (int q = 0; q < 16; ++q) {
        const unsigned da = pkmin(pksub(top2, e4[q] & 0x00ff00ffu), 0x00030003u), db = pkmin(pksub(top2, (e4[q] >> 8) & 0x00ff00ffu), 0x00030003u);
        const unsigned d = da | (db << 8);
        p2[q >> 2] |= d << (2 * (q & 3));
        m[q] = (d + 0x01010101u) & 0x04040404u; ec += (unsigned)__builtin_popcount(m[q]);
    }
    const unsigned einc = wave_scan(ec), n2 = __builtin_amdgcn_readlane(einc, 63);
    unsigned h1 = 0xffffffffu;
    if (!FORCE_RAW_IN && n2 <= CAP2) {
        const unsigned l2 = lwo + 10240, l3 = l2 + CAP2 + 16, l4 = l3 + CAP3 + 16, dummy = SM_DUMMY + lane * 2;
        unsigned eo = l2 + einc - ec;
#pragma unroll
        for (int q = 0; q < 16; ++q) eo = append_flagged(smem, e4[q], m[q], eo, dummy);
        h1 = encode_levels(smem, wsr, win, n2, l2, l3, l4, slot0 + 8, 1016u, dummy, lane);
    }
    if (h1 == 0xffffffffu) {
#pragma unroll
        for (int c = 0; c < 8; ++c) bst16<SYS>(wsr, pay + c * 1024, v[c]);
        if (lane == 0) bst4<SYS>(wsr, slot0, 1u << 8);
    } else {
#pragma unroll
        for (int i = 0; i < 4; ++i) bst16<SYS>(wsr, pay + i * 1024, (u32x4){sm[4 * i], sm[4 * i + 1], sm[4 * i + 2], sm[4 * i + 3]});
        bst16<SYS>(wsr, pay + 4096, (u32x4){p2[0], p2[1], p2[2], p2[3]});
        if (lane == 0) bst8<SYS>(wsr, slot0, (u32x2){win | (n2 << 16), h1});
    }
    }
    WAIT_V(0);
    const unsigned seq = rfl(seqv) + 1u;
    if (own && lane == 0) {
        const unsigned f = (isx ? OFF_XF : OFF_WF) + (unsigned)(((li >> 2) * NSTEP + j * 8) * 4 + (li & 3)) * 4u;
        __hip_atomic_store((gu32*)(ws + f), seq, __ATOMIC_RELAXED, __HIP_MEMORY_SCOPE_AGENT);
    }
    const unsigned per = isx ? 16u : 32u;
    unsigned ticket = per - 1u;
    if (!early) {
        if (lane == 0) ticket = __hip_atomic_fetch_add((gu32*)(ws + OFF_ECNT + (unsigned)(j * 12 + (isx ? cls * 4 + g - 1 : 8 + cls)) * 128u), 1u, __ATOMIC_RELAXED, __HIP_MEMORY_SCOPE_AGENT);
        ticket = rfl(ticket);
    }
    if ((ticket & (per - 1u)) == per - 1u) {
        const bool want = lane < NR && lane != p.rank && (isx ? ((lane >> 2) == cls) : ((lane & 3) == cls));
        if (want) {
            char* pw = ((char*)p.ws_all[lane] + p.ws_off);
            const unsigned word = early ? early_word((unsigned)li) : (unsigned)(j * 8 + g);
            __hip_atomic_store((gu32*)(pw + OFF_FIN + (unsigned)p.rank * 512u + word * 4u), seq, __ATOMIC_RELAXED, __HIP_MEMORY_SCOPE_SYSTEM);
        }
    }
}

struct UnitRegs { u32x4 a[5]; u32x4 sd; };
__device__ __forceinline__ unsigned unit_slot(int rank, int j, int g, int u) {
    const int ub = g ? (rank >> 2) * 64 + (g - 1) * 16 + u : (rank & 3) * 32 + u;
    return (g ? OFF_EX : OFF_EW) + (unsigned)(j * 128 + ub) * USLOT;
}
__device__ __forceinline__ void unit_issue(UnitRegs& r, u32x4 pd, unsigned slot0, int lane) {
    const unsigned oh = lane < 24 ? slot0 + lane * 16 : OOB, o0 = slot0 + 1024 + lane * 16, o1 = o0 + 4096;
    ALD16(r.sd, pd, oh, 0);
    ALD16(r.a[0], pd, o0, 0); ALD16(r.a[1], pd, o0, 1024); ALD16(r.a[2], pd, o0, 2048); ALD16(r.a[3], pd, o0, 3072);
    ALD16(r.a[4], pd, o1, 0);
}
#define UNIT_LANDED(r, n) asm volatile("s_waitcnt vmcnt(0)" : "+v"(r.a[0]), "+v"(r.a[1]), "+v"(r.a[2]), "+v"(r.a[3]), "+v"(r.a[4]), "+v"(r.sd), "+v"(n) :: "memory")
__device__ __forceinline__ void unit_unpack(const UnitRegs& r, rsrc_t wsr, rsrc_t pr, LAS unsigned char* smem, unsigned scr, unsigned slot0, unsigned dst0, int lane) {
    const unsigned dst = dst0 + (lane & 15) * 64 + (((lane >> 4) * 16) ^ (((lane >> 3) & 1) << 5));
    const unsigned h0 = rfl(r.sd.x), h1 = rfl(r.sd.y);
    if ((h0 >> 8) & 1u) {
        u32x4 b[3];
#pragma unroll
        for (int c = 0; c < 3; ++c) b[c] = bld16<SYS>(pr, slot0 + 1024 + lane * 16 + (5 + c) * 1024);
#pragma unroll
        for (int c = 0; c < 5; ++c) bst16<SC1>(wsr, dst + c * 1024, r.a[c]);
#pragma unroll
        for (int c = 0; c < 3; ++c) bst16<SC1>(wsr, dst + (5 + c) * 1024, b[c]);
        return;
    }
    u32x4 sd = r.sd;
    const unsigned len = 8u + side_bytes(h0 >> 16, h1);
    if (len > 384u) { if (lane >= 24 && (unsigned)lane * 16u < len) sd = bld16<SYS>(pr, slot0 + lane * 16); }
    *(LAS u32x4*)(smem + scr + lane * 16) = sd;
    const unsigned l3 = scr + 2048, l2 = l3 + CAP3 + 256;
    decode_levels(smem, h0, h1, scr + 8u, l3, l2, lane);
    const unsigned tq = (h0 & 0xffu) * 0x01010101u;
    unsigned e[16], m[16], ec = 0;
#pragma unroll
    for (int q = 0; q < 16; ++q) {
        const unsigned d = (r.a[4][q >> 2] >> (2 * (q & 3))) & 0x03030303u;
        m[q] = (d + 0x01010101u) & 0x04040404u; e[q] = tq - d;
    }
#pragma unroll
    for (int k = 0; k < 4; ++k) ec += (unsigned)__builtin_popcount(r.a[4][k] & (r.a[4][k] >> 1) & 0x55555555u);
    apply_escapes<16>(e, m, smem, l2 + wave_scan(ec) - ec);
#pragma unroll
    for (int c = 0; c < 8; ++c) {
        u32x4 o;
#pragma unroll
        for (int h = 0; h < 2; ++h) {
            const int q = 2 * c + h; const unsigned sm = r.a[q >> 2][q & 3];
            const unsigned lo = bfi(0x7f7f7f7fu, sm, e[q] << 7), hi = bfi(0x7f7f7f7fu, e[q] >> 1, sm);
            o[2 * h] = __builtin_amdgcn_perm(hi, lo, 0x05010400u); o[2 * h + 1] = __builtin_amdgcn_perm(hi, lo, 0x07030602u);
        }
        bst16<SC1>(wsr, dst + c * 1024, o);
    }
}

__device__ __forceinline__ unsigned strip_out_off(int r, int s, int lane) {
    const int tile = s >> 3, tm = tile >> 3, tn = tile & 7;
    const int row = (r >> 2) * RROWS + tm * 256 + (s & 7) * 32 + (lane >> 5);
    const int col = (r & 3) * RCOLS + tn * 256 + (lane & 31) * 8;
    return ((unsigned)row * ND + col) * 2u;
}

struct StripRegs { u32x4 mk; u32x4 lo[8]; u32x4 sd[2]; };
__device__ __forceinline__ void strip_issue(StripRegs& r, u32x4 pd, unsigned s, unsigned info, int lane) {
    if (info >> 30) return;
    const unsigned T = info & 0x3fffu, sl = (info >> 14) & 0xffu, slot = OFF_EO + s * SSLOT;
    const unsigned om = slot + lane * 16;
    ALD16(r.mk, pd, om, 0);
    { const unsigned o = (unsigned)lane < sl ? om : OOB; ALD16(r.sd[0], pd, o, 1024); }
    if (sl > 64u) { const unsigned o = (unsigned)lane + 64u < sl ? om : OOB; ALD16(r.sd[1], pd, o, 2048); }
#define STRIP_ROUND(k) if ((unsigned)(k) * 1024u < T) { const bool act = (unsigned)((k) * 1024 + lane * 16) < T; \
        const unsigned ol = act ? om + 3072 + ((k) >> 2) * 4096 : OOB; ALD16(r.lo[k], pd, ol, ((k) & 3) * 1024); }
    STRIP_ROUND(0) STRIP_ROUND(1) STRIP_ROUND(2) STRIP_ROUND(3) STRIP_ROUND(4) STRIP_ROUND(5) STRIP_ROUND(6) STRIP_ROUND(7)
}
#define STRIP_LANDED(r, fn, n, cnt) do { \
    asm volatile("s_waitcnt vmcnt(" #cnt ")" : "+v"(r.lo[0]), "+v"(r.lo[1]), "+v"(r.lo[2]), "+v"(r.lo[3]), "+v"(r.lo[4]), "+v"(r.lo[5]), "+v"(r.lo[6]), "+v"(r.lo[7]), \
        "+v"(r.mk), "+v"(r.sd[0]), "+v"(r.sd[1]), "+v"(fn), "+v"(n) :: "memory"); } while (0)
__device__ __forceinline__ void strip_stage1(const StripRegs& r, LAS unsigned char* smem, unsigned lbase, unsigned scr, unsigned info, int lane) {
    if (info >> 30) return;
    const unsigned T = info & 0x3fffu;
    const unsigned h0 = rfl(r.sd[0].x), h1 = rfl(r.sd[0].y);
    *(LAS u32x4*)(smem + scr + lane * 16) = r.sd[0];
    *(LAS u32x4*)(smem + scr + 1024 + lane * 16) = r.sd[1];
    const unsigned l3 = scr + 2048, l2 = l3 + CAP3 + 256;
    decode_levels(smem, h0, h1, scr + 8u + ((T + 31u) >> 5) * 4u, l3, l2, lane);
    const unsigned tq = (h0 & 0xffu) * 0x01010101u;
    unsigned x1[8], ec = 0;
#pragma unroll
    for (int k = 0; k < 8; ++k) {
        if ((unsigned)k * 1024u < T) {
            const unsigned z = __builtin_amdgcn_perm(0u, (unsigned)*(const LAS u16*)(smem + scr + 8 + k * 128 + lane * 2), 0x01010000u);
            x1[k] = (z & 0x000f000fu) | ((z >> 4) & 0x0f000f00u);
#pragma unroll
            for (int i = 0; i < 4; ++i) ec += (unsigned)__builtin_popcount((x1[k] >> i) & (r.lo[k][i] >> 7) & 0x01010101u);
        }
    }
    unsigned eo = l2 + wave_scan(ec) - ec;
    unsigned e[8][4];
#pragma unroll
    for (int k = 0; k < 8; ++k) {
        if ((unsigned)k * 1024u < T) {
            unsigned m[4];
#pragma unroll
            for (int i = 0; i < 4; ++i) {
                const unsigned d = ((x1[k] >> i) & 0x01010101u) | ((r.lo[k][i] >> 6) & 0x02020202u);
                m[i] = (d + 0x01010101u) & 0x04040404u; e[k][i] = tq - d;
            }
            eo = apply_escapes<4>(e[k], m, smem, eo);
        }
    }
#pragma unroll
    for (int k = 0; k < 8; ++k) {
        if ((unsigned)k * 1024u < T) {
            unsigned lo[4], hi[4];
#pragma unroll
            for (int i = 0; i < 4; ++i) { lo[i] = bfi(0x7f7f7f7fu, r.lo[k][i], e[k][i] << 7); hi[i] = (e[k][i] >> 1) & 0x7f7f7f7fu; }
            *(LAS u32x4*)(smem + lbase + k * 2048 + lane * 32) = (u32x4){lo[0], hi[0], lo[1], hi[1]};
            *(LAS u32x4*)(smem + lbase + k * 2048 + lane * 32 + 16) = (u32x4){lo[2], hi[2], lo[3], hi[3]};
        }
    }
}
__device__ __forceinline__ void strip_stage2(u32x4 mk, rsrc_t outr, rsrc_t por, LAS unsigned char* smem, unsigned lbase, int peer, unsigned s, unsigned info, int lane) {
    const unsigned oo = strip_out_off(peer, (int)s, lane);
    if (info >> 31) {
#pragma unroll
        for (int q = 0; q < 16; ++q) bst16<OAUX>(outr, oo + q * 2 * ND * 2, (u32x4){0x7fc07fc0u, 0x7fc07fc0u, 0x7fc07fc0u, 0x7fc07fc0u});
        return;
    }
    if (info >> 30) {
#pragma unroll
        for (int h = 0; h < 4; ++h) {
            u32x4 a[4];
#pragma unroll
            for (int q = 0; q < 4; ++q) a[q] = bld16<SYS>(por, oo + (h * 4 + q) * 2 * ND * 2);
#pragma unroll
            for (int q = 0; q < 4; ++q) bst16<OAUX>(outr, oo + (h * 4 + q) * 2 * ND * 2, a[q]);
        }
        return;
    }
    unsigned c2[8], x[8];
#pragma unroll
    for (int i = 0; i < 8; ++i) {
        const unsigned m16 = mk[i >> 1] >> ((i & 1) * 16);
        c2[i] = (unsigned)__builtin_popcount(m16 & 0xffu) | ((unsigned)__builtin_popcount((m16 >> 8) & 0xffu) << 16);
        x[i] = c2[i];
    }
#define SCAN_STEP(ctrl, rmask) _Pragma("unroll") for (int i = 0; i < 8; ++i) x[i] += (unsigned)__builtin_amdgcn_update_dpp(0, (int)x[i], ctrl, rmask, 0xf, false);
    SCAN_STEP(0x111, 0xf) SCAN_STEP(0x112, 0xf) SCAN_STEP(0x114, 0xf) SCAN_STEP(0x118, 0xf) SCAN_STEP(0x142, 0xa) SCAN_STEP(0x143, 0xc)
    unsigned pos = 0;
#pragma unroll
    for (int bt = 0; bt < 4; ++bt) {
        u32x4 sel[4], d[8]; unsigned off[8];
#pragma unroll
        for (int k = 0; k < 4; ++k) {
            const int i = bt * 2 + (k >> 1), hq = k & 1;
            const unsigned m8 = (mk[bt] >> (k * 8)) & 0xffu;
            sel[k] = *(const LAS u32x4*)(smem + SM_LUT + m8 * 16u);
            const unsigned tot = __builtin_amdgcn_readlane(x[i], 63);
            off[2 * k] = pos + (((x[i] - c2[i]) >> (hq * 16)) & 0xffffu);
            off[2 * k + 1] = off[2 * k] + (unsigned)__builtin_popcount(m8 & 15u);
            pos += (tot >> (hq * 16)) & 0xffffu;
        }
#pragma unroll
        for (int g = 0; g < 8; ++g) {
            const LAS u32x2* pl = (const LAS u32x2*)(smem + lbase + (off[g] & ~3u) * 2u);
            const u32x2 d0 = pl[0], d1 = pl[1];
            d[g] = (u32x4){d0.x, d0.y, d1.x, d1.y};
        }
#pragma unroll
        for (int k = 0; k < 4; ++k) {
            u32x4 o;
#pragma unroll
            for (int g = 0; g < 2; ++g) {
                const u32x4 dd = d[2 * k + g];
                const unsigned l4 = __builtin_amdgcn_alignbyte(dd.z, dd.x, off[2 * k + g]), h4 = __builtin_amdgcn_alignbyte(dd.w, dd.y, off[2 * k + g]);
                o[2 * g] = __builtin_amdgcn_perm(h4, l4, sel[k][2 * g]); o[2 * g + 1] = __builtin_amdgcn_perm(h4, l4, sel[k][2 * g + 1]);
            }
            bst16<OAUX>(outr, oo + (bt * 4 + k) * 2 * ND * 2, o);
        }
    }
}

__device__ __forceinline__ void pack_strip(const Params& p, rsrc_t wsr, rsrc_t outr, LAS unsigned char* smem, int s, int wid, int lane, unsigned seq) {
    const unsigned oo = strip_out_off(p.rank, s, lane);
    const unsigned lbase = (unsigned)wid * 16384u;
    unsigned pos = lbase, mx = 0, sg = 0, esum = 0, ts = 0;
    const unsigned dummy = SM_DUMMY + lane * 2;
    u32x4 mk = {0u, 0u, 0u, 0u};
#pragma unroll
    for (int q2 = 0; q2 < 8; ++q2) {
        u32x4 v[2]; unsigned m8[2];
#pragma unroll
        for (int hq = 0; hq < 2; ++hq) {
            const int q = q2 * 2 + hq, row = wid * 32 + 2 * q + (lane >> 5);
            v[hq] = *(const LAS u32x4*)(smem + row * 512 + (((lane & 31) ^ (row & 7)) * 16));
            bst16<SYS>(outr, oo + q * 2 * ND * 2, v[hq]);
            unsigned m = 0;
#pragma unroll
            for (int k = 0; k < 4; ++k) {
                sg |= v[hq][k];
                if (q2 < 2) {
                    mx = pkmax(mx, v[hq][k]);
                    esum = __builtin_amdgcn_udot4((v[hq][k] >> 7) & 0x00ff00ffu, 0x00010001u, esum, false);
                }
                const unsigned t = pkmin1(v[hq][k]);
                m |= ((t | (t >> 15)) & 3u) << (2 * k);
            }
            m8[hq] = m;
        }
        mk[q2 >> 1] |= (m8[0] | (m8[1] << 8)) << ((q2 & 1) * 16);
        const unsigned c2 = (unsigned)__builtin_popcount(m8[0]) | ((unsigned)__builtin_popcount(m8[1]) << 16);
        const unsigned inc = wave_scan(c2), exl = inc - c2, tot = __builtin_amdgcn_readlane(inc, 63);
#pragma unroll
        for (int hq = 0; hq < 2; ++hq) {
            unsigned a = pos + ((exl >> (hq * 16)) & 0xffffu) * 2u;
#pragma unroll
            for (int e = 0; e < 8; ++e) {
                const unsigned bit = (m8[hq] >> e) & 1u;
                *(LAS u16*)(smem + (bit ? a : dummy)) = (u16)(v[hq][e >> 1] >> ((e & 1) * 16));
                a += bit * 2u;
            }
            pos += ((tot >> (hq * 16)) & 0xffffu) * 2u;
        }
        if (q2 == 1) ts = (pos - lbase) >> 1;
    }
    const unsigned T = (pos - lbase) >> 1;
    STAMP(13);
    mx = (mx & 0xffffu) > (mx >> 16) ? (mx & 0xffffu) : (mx >> 16);
    const unsigned win = choose_window(wave_sum(esum), ts, (wave_max(mx) >> 7) & 0xffu);
    bool raw = FORCE_RAW_OUT || __builtin_amdgcn_ballot_w64((sg & 0x80008000u) != 0u) != 0ull;
    const unsigned slot = OFF_EO + (unsigned)s * SSLOT;
    unsigned sl = 0;
    if (!raw) {
        const unsigned top = win & 0xffu, top2 = top * 0x00010001u;
        if (lane < 16 && T + lane < 8192u) *(LAS u16*)(smem + lbase + (T + lane) * 2u) = (u16)(top << 7);
        bst16<SYS>(wsr, slot + lane * 16, mk);
        unsigned e4[8][4], m[8][4], ec = 0;
#pragma unroll
        for (int r = 0; r < 8; ++r) {
            if ((unsigned)r * 1024u < T) {
                const bool act = (unsigned)(r * 1024 + lane * 16) < T;
                const u32x4 a = *(const LAS u32x4*)(smem + lbase + r * 2048 + lane * 32), b = *(const LAS u32x4*)(smem + lbase + r * 2048 + lane * 32 + 16);
                const unsigned w8[8] = {a.x, a.y, a.z, a.w, b.x, b.y, b.z, b.w};
                u32x4 lo; unsigned x1 = 0;
#pragma unroll
                for (int k = 0; k < 4; ++k) {
                    const unsigned ea = (w8[2 * k] >> 7) & 0x00ff00ffu, eb = (w8[2 * k + 1] >> 7) & 0x00ff00ffu;
                    const unsigned d = __builtin_amdgcn_perm(pkmin(pksub(top2, eb), 0x00030003u), pkmin(pksub(top2, ea), 0x00030003u), 0x06040200u);
                    e4[r][k] = __builtin_amdgcn_perm(eb, ea, 0x06040200u);
                    lo[k] = bfi(0x7f7f7f7fu, __builtin_amdgcn_perm(w8[2 * k + 1], w8[2 * k], 0x06040200u), d << 6);
                    x1 |= (d & 0x01010101u) << k;
                    m[r][k] = act ? (d + 0x01010101u) & 0x04040404u : 0u; ec += (unsigned)__builtin_popcount(m[r][k]);
                }
                if (act) {
                    bst16<SYS>(wsr, slot + 3072 + r * 1024 + lane * 16, lo);
                    bst2<SYS>(wsr, slot + 1024 + 8 + r * 128 + lane * 2, __builtin_amdgcn_perm(0u, x1 | (x1 >> 4), 0x0c0c0200u));
                }
            }
        }
        STAMP(14);
        const unsigned einc = wave_scan(ec), n2 = __builtin_amdgcn_readlane(einc, 63);
        unsigned h1 = 0xffffffffu;
        const unsigned p1b = ((T + 31u) >> 5) * 4u;
        if (n2 <= CAP2 - 16u) {
            const unsigned l2 = SM_SCR + (unsigned)wid * 3072u, l3 = l2 + CAP2, l4 = l3 + CAP3 + 8;
            unsigned eo = l2 + einc - ec;
#pragma unroll
            for (int r = 0; r < 8; ++r) {
                if ((unsigned)r * 1024u < T) {
#pragma unroll
                    for (int k = 0; k < 4; ++k) eo = append_flagged(smem, e4[r][k], m[r][k], eo, dummy);
                }
            }
            h1 = encode_levels(smem, wsr, win, n2, l2, l3, l4, slot + 1024 + 8 + p1b, 2040u - p1b, dummy, lane);
        }
        if (h1 == 0xffffffffu) raw = true;
        else {
            if (lane == 0) bst8<SYS>(wsr, slot + 1024, (u32x2){win | (n2 << 16), h1});
            sl = (8u + p1b + side_bytes(n2, h1) + 15u) >> 4;
        }
    }
    STAMP(15);
    WAIT_V(0);
    if (lane < NR && lane != p.rank) {
        const unsigned info = raw ? (1u << 30) : (T | (sl << 14));
        char* pw = ((char*)p.ws_all[lane] + p.ws_off);
        __hip_atomic_store((gu64*)(pw + OFF_FOUT + ((unsigned)p.rank * 1024u + (unsigned)s) * 8u), ((u64)seq << 32) | info, __ATOMIC_RELAXED, __HIP_MEMORY_SCOPE_SYSTEM);
    }
}

}

__global__ void __launch_bounds__(512, 2) gemm_ar_fused(Params p) {
    __shared__ __attribute__((aligned(16))) unsigned char smem_[SMEM_BYTES];
    LAS unsigned char* smem = (LAS unsigned char*)smem_;
    const int tid = threadIdx.x, wid = rfl(tid >> 6), lane = tid & 63;
    char* ws = ((char*)p.ws_all[p.rank] + p.ws_off);
    const rsrc_t wsr = make_rsrc(ws, WS_USED);
    const rsrc_t outr = make_rsrc(p.out, OUT_BYTES);
    gu32* tmo = (gu32*)(ws + OFF_TMO);
    gu32* seqp = (gu32*)(ws + OFF_SEQ + (unsigned)(blockIdx.x * 8 + wid) * 4u);
    const unsigned seqv = __hip_atomic_load(seqp, __ATOMIC_RELAXED, __HIP_MEMORY_SCOPE_AGENT);
    STAMP(0);
    const int xcd = blockIdx.x & 7, wi = blockIdx.x >> 3;
    const int nsplit = xcd < 2 ? 16 : (xcd == 2 ? 8 : (xcd == 3 ? 4 : 0));
    const bool is_gemm = wi < 16 + nsplit;
    const int cw = xcd >= 4 ? (wi - 16) * 4 + xcd - 4 : (xcd == 3 ? 64 + wi - 20 : 76 + wi - 24);
    const bool is_link = !is_gemm && wid < NWV;
    if (!is_gemm) {
        if (tid == 0) { *(LAS unsigned*)(smem + SM_TOKEN) = 0u; *(LAS unsigned*)(smem + SM_TOKEN + 4) = 0xffffffffu; }
        WAIT_L(0); BAR;
    }
    if (!is_link) {
        Order od{0, 0};
        const int e = !is_gemm ? (wid - NWV) * 84 + cw
                    : xcd >= 4 ? NPD + (xcd >= 6 ? 0 : 256) + wid * 32 + wi * 2 + (xcd & 1)
                    : xcd < 2 ? NENC + 256 + wid * 64 + wi * 2 + xcd
                    : wi < 16 ? NENC + wid * 32 + wi * 2 + (xcd & 1) : NENC + 768 + wid * 12 + (xcd == 2 ? wi - 16 : wi - 8);
        if (e >= PDELAY) __builtin_amdgcn_s_sleep(PSLEEP);
        for (int t = e; t < 4096; t = (t == e ? NENC + NONCE + e : t + NENC)) {
            int j, g, u; od.locate<128, 32>(t, j, g, u);
            pack_unit(p, ws, wsr, smem, (unsigned)wid * 16384u, j, g, u, lane, seqv);
            if (e >= NENC) break;
        }
        STAMP(7);
        if (is_gemm) __syncthreads();
    }
    const unsigned seq = rfl(seqv) + 1u;
    if (lane == 0) __hip_atomic_store(seqp, seq, __ATOMIC_RELAXED, __HIP_MEMORY_SCOPE_AGENT);
    if (!is_gemm) {
        if (wid >= NLW) return;
        const int lq = cw % 7;
        const int peer = (p.rank + 1 + lq) & 7;
        const rsrc_t pr = make_rsrc((char*)p.ws_all[peer] + p.ws_off, WS_USED);
        const rsrc_t por = make_rsrc(p.out_all[peer], OUT_BYTES);
#pragma unroll
        for (int i = 0; i < 4; ++i) {
            const unsigned m8 = (unsigned)(i * 64 + lane); u32x4 sel;
#pragma unroll
            for (int g = 0; g < 2; ++g) {
                unsigned rk = 0;
#pragma unroll
                for (int h = 0; h < 2; ++h) {
                    unsigned w = 0;
#pragma unroll
                    for (int e = 0; e < 2; ++e) { const unsigned bit = (m8 >> (g * 4 + h * 2 + e)) & 1u; w |= (bit ? (rk | ((rk + 4u) << 8)) : 0x0c0cu) << (16 * e); rk += bit; }
                    sel[g * 2 + h] = w;
                }
            }
            *(LAS u32x4*)(smem + SM_LUT + m8 * 16u) = sel;
        }
        if (lane < 16) {
            unsigned rk = 4u, sel = 0;
#pragma unroll
            for (int k = 0; k < 4; ++k) { const unsigned bit = ((unsigned)lane >> k) & 1u; sel |= (bit ? rk : (unsigned)k) << (8 * k); rk += bit; }
            *(LAS unsigned*)(smem + SM_LUTE + lane * 4) = sel;
        }
        const unsigned esc = (unsigned)wid * 16384u + 8192u;
        bool dead = false;
        const u32x4 pd = make_desc((char*)p.ws_all[peer] + p.ws_off, WS_USED);
        const unsigned one = 1u;
        LAS unsigned* token = (LAS unsigned*)(smem + SM_TOKEN);
#define QNEXT(dst) do { if (lane == 0) asm volatile("global_atomic_add %0, %1, %2, off sc0" : "+v"(dst) : "v"(qp), "v"(one) : "memory"); } while (0)
#define TOKEN_TAKE() do { for (;;) { unsigned old = 1u; \
            if (lane == 0) old = __hip_atomic_exchange(token, 1u, __ATOMIC_RELAXED, __HIP_MEMORY_SCOPE_WORKGROUP); \
            if (rfl(old) == 0u) break; __builtin_amdgcn_s_sleep(1); } } while (0)
#define TOKEN_GIVE() do { if (lane == 0) __hip_atomic_store(token, 0u, __ATOMIC_RELAXED, __HIP_MEMORY_SCOPE_WORKGROUP); } while (0)
        {
            gu32* qp = (gu32*)(ws + OFF_Q + (unsigned)lq * 128u);
            const unsigned qbase = (seq - 1u) * (1536u + NT);
            LAS unsigned* utok = token + 1;
            gu64* finp = (gu64*)(ws + OFF_FIN + (unsigned)peer * 512u + (unsigned)lane * 8u);
            Order od{0, 0};
            u64 rdy0 = 0, rdy1 = 0, fv = 0;
            unsigned nq = 0;
            UnitRegs ur = {};
            if (wid == 0) {
                QNEXT(nq);
                UNIT_LANDED(ur, nq);
                if (lane == 0) __hip_atomic_store(utok, nq - qbase, __ATOMIC_RELAXED, __HIP_MEMORY_SCOPE_WORKGROUP);
            }
            unsigned pfo = 0;
#define PUBLISH(n) do { if (pfo) { WAIT_V(n); if (lane == 0) __hip_atomic_store((gu32*)(ws + pfo), seq, __ATOMIC_RELAXED, __HIP_MEMORY_SCOPE_AGENT); pfo = 0; } } while (0)
            for (;;) {
                unsigned i;
                for (;;) {
                    unsigned old = 0xffffffffu;
                    if (lane == 0) old = __hip_atomic_exchange(utok, 0xffffffffu, __ATOMIC_RELAXED, __HIP_MEMORY_SCOPE_WORKGROUP);
                    i = rfl(old);
                    if (i != 0xffffffffu) break;
                    if (pfo) PUBLISH(0); else __builtin_amdgcn_s_sleep(1);
                }
                if (i >= 1536u) { if (lane == 0) __hip_atomic_store(utok, i, __ATOMIC_RELAXED, __HIP_MEMORY_SCOPE_WORKGROUP); break; }
                QNEXT(nq);
                int j, g, u; od.locate<32, 16>((int)i, j, g, u);
                const unsigned word = i < 32u ? early_word(i) : (unsigned)(j * 8 + g), rb = word >> 1;
                if (!((((word & 1u) ? rdy1 : rdy0) >> rb) & 1ull) && !dead) {
                    PUBLISH(0);
                    const bool was0 = (rdy0 | rdy1) == 0ull;
                    const u64 t0 = __builtin_amdgcn_s_memrealtime(); unsigned n = 0;
                    for (;;) {
                        const u64 f2 = __hip_atomic_load(finp, __ATOMIC_RELAXED, __HIP_MEMORY_SCOPE_SYSTEM);
                        rdy0 = __builtin_amdgcn_ballot_w64((unsigned)f2 >= seq); rdy1 = __builtin_amdgcn_ballot_w64((unsigned)(f2 >> 32) >= seq);
                        if ((((word & 1u) ? rdy1 : rdy0) >> rb) & 1ull) break;
                        if ((++n & 63u) == 0u && give_up(tmo, seq, t0)) { dead = true; break; }
                        __builtin_amdgcn_s_sleep(2);
                    }
                    __builtin_amdgcn_fence(__ATOMIC_ACQUIRE, "wavefront");
                    if (was0) STAMP(1);
                }
                const int step = j * 8 + 1 + lq, li = g ? (g - 1) * 16 + u : u;
                const unsigned slot0 = unit_slot(p.rank, j, g, u);
                const unsigned dst0 = g ? OFF_XL + (unsigned)(step * 64 + li) * 8192u : OFF_WL + (unsigned)(step * 32 + li) * 8192u;
                const bool refresh = (~rdy0 & 0x7777777777777776ull) != 0ull || (~rdy1 & 0x3333333333333333ull) != 0ull;
                for (;;) {
                    unsigned old = 1u;
                    if (lane == 0) old = __hip_atomic_exchange(token, 1u, __ATOMIC_RELAXED, __HIP_MEMORY_SCOPE_WORKGROUP);
                    if (rfl(old) == 0u) break;
                    if (pfo) PUBLISH(0); else __builtin_amdgcn_s_sleep(1);
                }
                unit_issue(ur, pd, slot0, lane);
                asm volatile("s_waitcnt vmcnt(6)" : "+v"(nq) :: "memory");
                if (lane == 0) __hip_atomic_store(utok, nq - qbase, __ATOMIC_RELAXED, __HIP_MEMORY_SCOPE_WORKGROUP);
                UNIT_LANDED(ur, nq);
                TOKEN_GIVE();
                PUBLISH(0);
                if (refresh) asm volatile("global_load_dwordx2 %0, %1, off sc0 sc1" : "+v"(fv) : "v"(finp) : "memory");
                pfo = (g ? OFF_XF : OFF_WF) + (unsigned)(((li >> 2) * NSTEP + step) * 4 + (li & 3)) * 4u;
                unit_unpack(ur, wsr, pr, smem, esc, slot0, dst0, lane);
                if (refresh) {
                    asm volatile("s_waitcnt vmcnt(8)" : "+v"(fv) :: "memory");
                    rdy0 = __builtin_amdgcn_ballot_w64((unsigned)fv >= seq); rdy1 = __builtin_amdgcn_ballot_w64((unsigned)(fv >> 32) >= seq);
                }
            }
            PUBLISH(0);
            STAMP(3);
        }
        {
            gu32* qp = (gu32*)(ws + OFF_Q + (unsigned)(7 + lq) * 128u);
            const unsigned qbase = (seq - 1u) * (1024u + NT * NLW);
            gu64* fbase = (gu64*)(ws + OFF_FOUT + (unsigned)peer * 8192u);
            const unsigned lbase = (unsigned)wid * 16384u;
            unsigned nq = 0; u64 fn = 0;
            StripRegs sr = {};
            QNEXT(nq);
            asm volatile("s_waitcnt vmcnt(0)" : "+v"(nq) :: "memory");
            unsigned s = rfl(nq) - qbase;
            bool first = true;
            while (s < 1024u) {
                unsigned fh = rfl((unsigned)(fn >> 32)), info = rfl((unsigned)fn);
                if (fh < seq && !dead) {
                    const u64 t0 = __builtin_amdgcn_s_memrealtime(); unsigned n = 0;
                    for (;;) {
                        const u64 f2 = __hip_atomic_load(fbase + s, __ATOMIC_RELAXED, __HIP_MEMORY_SCOPE_SYSTEM);
                        fh = rfl((unsigned)(f2 >> 32)); info = rfl((unsigned)f2);
                        if (fh >= seq) break;
                        if ((++n & 63u) == 0u && give_up(tmo, seq, t0)) { dead = true; break; }
                        __builtin_amdgcn_s_sleep(4);
                    }
                }
                __builtin_amdgcn_fence(__ATOMIC_ACQUIRE, "wavefront");
                if (dead) info = 3u << 30;
                if (first) STAMP(4);
                first = false;
                TOKEN_TAKE();
                strip_issue(sr, pd, s, info, lane);
                QNEXT(nq);
                STRIP_LANDED(sr, fn, nq, 0);
                TOKEN_GIVE();
                const unsigned sn = rfl(nq) - qbase;
                { gu64* fa = fbase + (sn & 1023u); asm volatile("global_load_dwordx2 %0, %1, off sc0 sc1" : "+v"(fn) : "v"(fa) : "memory"); }
                strip_stage1(sr, smem, lbase, esc, info, lane);
                strip_stage2(sr.mk, outr, por, smem, lbase, peer, s, info, lane);
                asm volatile("s_waitcnt vmcnt(16)" : "+v"(fn) :: "memory");
                s = sn;
            }
            STAMP(6);
        }
        return;
    }

    const bool split = (wi & 15) < nsplit;
    const int wr = wid >> 2, wc = wid & 3;
    constexpr int HTB = 16384;
    constexpr size_t ASTEP = 64 * 8192, BSTEP = 32 * 8192;
    const unsigned ldsw = (unsigned)wid * 1024u;
    const unsigned smbase = (unsigned)(uintptr_t)smem;
#define SA(b, h) (((b) * 2 + (h)) * HTB)
#define SB(b, h) ((4 + (b) * 2 + (h)) * HTB)
#define STAGE(bufoff, gptr) do { _Pragma("unroll") for (int _i = 0; _i < 2; ++_i) \
        __builtin_amdgcn_global_load_lds((const unsigned*)((gptr) + voff[_i]), (LAS unsigned*)(smem + (bufoff) + ldsw + _i * 8192), 16, 0, SYS); } while (0)
#define LDA(dst, b, h) do { _Pragma("unroll") for (int m = 0; m < 4; ++m) _Pragma("unroll") for (int k = 0; k < 2; ++k) dst[m][k] = *(const LAS bf16x8*)(smem + SA(b, h) + aoff + m * 2048 + k * 1024); } while (0)
#define LDB(dst, b, h) do { _Pragma("unroll") for (int n = 0; n < 2; ++n) _Pragma("unroll") for (int k = 0; k < 2; ++k) dst[n][k] = *(const LAS bf16x8*)(smem + SB(b, h) + boff + n * 2048 + k * 1024); } while (0)
#define MMA(ai, bj, At, Bt) do { __builtin_amdgcn_s_setprio(1); _Pragma("unroll") for (int m = 0; m < 4; ++m) _Pragma("unroll") for (int n = 0; n < 2; ++n) _Pragma("unroll") for (int k = 0; k < 2; ++k) \
        acc[ai][bj][m][n] = __builtin_amdgcn_mfma_f32_16x16x32_bf16(Bt[n][k], At[m][k], acc[ai][bj][m][n], 0, 0, 0); __builtin_amdgcn_s_setprio(0); } while (0)
#define ZERO_ACC(a) _Pragma("unroll") for (int b = 0; b < 2; ++b) _Pragma("unroll") for (int m = 0; m < 4; ++m) _Pragma("unroll") for (int n = 0; n < 2; ++n) acc[a][b][m][n] = (f32x4){0.f, 0.f, 0.f, 0.f};
#define NEED(s) do { if (__builtin_expect((avail & 0xffffu) <= (unsigned)(s), 0)) { \
        if (wid == 0) { const unsigned s0 = avail & 0xffffu; unsigned av; const u64 t0 = __builtin_amdgcn_s_memrealtime(); unsigned n = 0; \
            const unsigned pn = (s0 + (lane & 31)) & 127u, po = pollo + SMAP(pn) * 16u; \
            for (;;) { const u32x4 f = bld16<SYS>(wsr, po); \
                const bool ok = (s0 + (lane & 31) >= (unsigned)te) || (f.x >= seq && f.y >= seq && f.z >= seq && f.w >= seq); \
                const u64 mm = __builtin_amdgcn_ballot_w64(ok); const unsigned both = (unsigned)mm & (unsigned)(mm >> 32); \
                av = s0 + (both == 0xffffffffu ? 32u : (unsigned)__builtin_ctz(~both)); \
                if (av > (unsigned)(s)) break; \
                if ((++n & 63u) == 0u && give_up(tmo, seq, t0)) { av = 0x10000u | NSTEP; break; } \
                __builtin_amdgcn_s_sleep(1); } \
            asm volatile("ds_write_b32 %0, %1\n\ts_waitcnt lgkmcnt(0)" :: "v"(smbase + SM_AVAIL), "v"(av) : "memory"); } \
        CFENCE; BAR; CFENCE; \
        unsigned av2; asm volatile("ds_read_b32 %0, %1\n\ts_waitcnt lgkmcnt(0)" : "=v"(av2) : "v"(smbase + SM_AVAIL) : "memory"); \
        avail = rfl(av2); } } while (0)
#define LOOP_STAMPS do { } while (0)
#define OPAQUE_U(dst, src) unsigned dst##_v = (unsigned)(src); asm volatile("" : "+v"(dst##_v)); const unsigned dst = rfl(dst##_v);
    for (int job = 0; job < 2; ++job) {
        int tid = threadIdx.x; asm volatile("" : "+v"(tid));
        const int lane = tid & 63, fr = lane & 15, fq = lane >> 4;
        const bool pair = split && job == 0;
        const int hsel = wi >> 4;
        const int atile = 44 + (xcd < 2 ? xcd * 32 + wi : (xcd == 2 ? 64 : 80) + (wi & 15) + nsplit * hsel);
        if (job && atile >= 128) return;
        const int tm = job ? atile >> 3 : (xcd >> 1) * 4 + ((wi & 15) >> 2), tn = job ? atile & 7 : (xcd & 1) * 4 + (wi & 3);
        const char* cA = ws + OFF_XL + (size_t)tm * 32768;
        const char* cB = ws + OFF_WL + (size_t)tn * 32768;
        const unsigned voff[2] = {(unsigned)tid * 16u, (unsigned)tid * 16u + 8192u};
        const int aoff = lds_byte(wr * 64 + fr, fq * 8), boff = lds_byte(wc * 32 + fr, fq * 8);
        const unsigned pollo = lane < 32 ? OFF_XF + (unsigned)(tm * NSTEP) * 16u : OFF_WF + (unsigned)(tn * NSTEP) * 16u;
        const int kh = tm < 8 ? KH1 : (tm < 12 ? KH2 : KH3);
        const int tb = job ? NSTEP - kh : 0, te = pair ? 64 : (job ? NSTEP : NSTEP - kh);
        const unsigned nl = tm < 8 ? (unsigned)NL1 : 16u;
        auto smap = [nl, pair, hsel](unsigned n) -> unsigned {
            if (pair) return (((n & ~3u) << 1) | (unsigned)(hsel << 2) | (n & 3u)) & 127u;
            if (n < nl) return n * 8u;
            const unsigned m = n - nl, full = (16u - nl) * 8u;
            if (m < full) return (m & 7u) == 7u ? (nl + (m >> 3)) * 8u : (m & ~7u) + 1u + (m & 7u);
            const unsigned r = m - full; return (16u - nl + r / 7u) * 8u + 1u + r % 7u;
        };
        const unsigned stab0 = smap((unsigned)lane), stab1 = smap((unsigned)lane + 64u);
#define SMAP(n) smap(n)
#define STAB(n) ((unsigned)__builtin_amdgcn_readlane((n) & 64 ? stab1 : stab0, (n) & 63))
        unsigned avail = (unsigned)tb;
        f32x4 acc[2][2][4][2];
        ZERO_ACC(0) ZERO_ACC(1)
        bf16x8 At[4][2], B0[2][2], B1[2][2];
        NEED(tb + 1);
        if (!job) STAMP(2);
        { const size_t s0 = STAB(tb); STAGE(SB(0, 0), cB + s0 * BSTEP); STAGE(SA(0, 0), cA + s0 * ASTEP); STAGE(SB(0, 1), cB + s0 * BSTEP + HTB); STAGE(SA(0, 1), cA + s0 * ASTEP + HTB); }
        if (wr == 1) BAR;
        WAIT_V(4); BAR;
        unsigned sp = STAB(tb + 1);
        { const size_t s1 = sp; STAGE(SB(1, 0), cB + s1 * BSTEP); STAGE(SA(1, 0), cA + s1 * ASTEP); STAGE(SB(1, 1), cB + s1 * BSTEP + HTB); }
        WAIT_V(6); BAR;
        for (int t = tb; t < te; t += 2) {
            const bool last = (t == te - 2);
            const int tn2 = last ? tb : t + 2;
            const unsigned s1 = sp, s2 = STAB(tn2), s3 = STAB(tn2 + 1);
            sp = s3;
            const char* a1 = cA + s1 * ASTEP;
            const char* a2 = cA + s2 * ASTEP; const char* b2 = cB + s2 * BSTEP;
            const char* a3 = cA + s3 * ASTEP; const char* b3 = cB + s3 * BSTEP;
            if (!last) NEED(t + 3);
            LOOP_STAMPS;
            LDB(B0, 0, 0); SCHED; LDA(At, 0, 0); STAGE(SA(1, 1), a1 + HTB);
            WAIT_L(8); BAR; WAIT_L(0); MMA(0, 0, At, B0); BAR; SCHED;
            LDB(B1, 0, 1); STAGE(SB(0, 0), b2);
            BAR; WAIT_L(0); MMA(0, 1, At, B1); BAR;
            LDA(At, 0, 1); STAGE(SA(0, 0), a2);
            BAR; WAIT_L(0); MMA(1, 0, At, B0); BAR; SCHED;
            STAGE(SB(0, 1), b2 + HTB);
            WAIT_V(6); BAR; MMA(1, 1, At, B1); BAR;
            LDB(B0, 1, 0); SCHED; LDA(At, 1, 0); STAGE(SA(0, 1), a2 + HTB);
            WAIT_L(8); BAR; WAIT_L(0); MMA(0, 0, At, B0); BAR; SCHED;
            LDB(B1, 1, 1); STAGE(SB(1, 0), b3);
            BAR; WAIT_L(0); MMA(0, 1, At, B1); BAR;
            LDA(At, 1, 1); STAGE(SA(1, 0), a3);
            BAR; WAIT_L(0); MMA(1, 0, At, B0); BAR; SCHED;
            STAGE(SB(1, 1), b3 + HTB);
            WAIT_V(6); BAR; MMA(1, 1, At, B1); BAR;
        }
        if (wr == 0) BAR;
        if (!job) STAMP(12);
        int tz = threadIdx.x; asm volatile("" : "+v"(tz));
        const int lane2 = tz & 63;
        const unsigned po = OFF_PART + (unsigned)(tm * 8 + tn) * 262144u + (unsigned)wid * 32768u + (unsigned)lane2 * 16u;
        gu32* pfl = (gu32*)(ws + OFF_PF + (unsigned)((tm * 8 + tn) * 16 + wid) * 4u);
        const unsigned give = job ? 3u : (pair ? (hsel ? 1u : 2u) : 0u), take = job ? 0u : (pair ? (hsel ? 2u : 1u) : 3u);
#pragma unroll
        for (int h = 0; h < 2; ++h) {
            OPAQUE_U(g1, (give >> h) & 1u)
            if (g1) {
#pragma unroll
                for (int q = 0; q < 16; ++q) bst16<SC1>(wsr, po + (h * 16 + q) * 1024, __builtin_bit_cast(u32x4, acc[h][(q >> 3) & 1][(q >> 1) & 3][q & 1]));
                WAIT_V(0);
                if (lane2 == 0) __hip_atomic_store(pfl + h * 8, seq, __ATOMIC_RELAXED, __HIP_MEMORY_SCOPE_AGENT);
            }
        }
        if (job) { STAMP(11); return; }
#pragma unroll
        for (int h = 0; h < 2; ++h) {
            OPAQUE_U(t1, (take >> h) & 1u)
            if (t1) {
                if (rfl(__hip_atomic_load(pfl + h * 8, __ATOMIC_RELAXED, __HIP_MEMORY_SCOPE_AGENT)) < seq) {
                    const u64 t0 = __builtin_amdgcn_s_memrealtime(); unsigned n = 0;
                    while (rfl(__hip_atomic_load(pfl + h * 8, __ATOMIC_RELAXED, __HIP_MEMORY_SCOPE_AGENT)) < seq) {
                        if ((++n & 63u) == 0u && give_up(tmo, seq, t0)) break;
                        __builtin_amdgcn_s_sleep(1);
                    }
                }
                __builtin_amdgcn_fence(__ATOMIC_ACQUIRE, "wavefront");
                u32x4 pv[16];
#pragma unroll
                for (int q = 0; q < 16; ++q) pv[q] = bld16<SC1>(wsr, po + (h * 16 + q) * 1024);
#pragma unroll
                for (int q = 0; q < 16; ++q) { acc[h][(q >> 3) & 1][(q >> 1) & 3][q & 1] += __builtin_bit_cast(f32x4, pv[q]);
                    asm volatile("" : "+v"(acc[h][(q >> 3) & 1][(q >> 1) & 3][q & 1])); }
                CFENCE; SCHED;
            }
        }
        const unsigned tmov = __hip_atomic_load(tmo, __ATOMIC_RELAXED, __HIP_MEMORY_SCOPE_AGENT);
        WAIT_V(0); BAR;
        STAMP(3);
        const int fr2 = lane2 & 15, fq2 = lane2 >> 4;
        const bool dead = (avail >> 16) != 0u || rfl(tmov) == seq;
#pragma unroll
        for (int ai = 0; ai < 2; ++ai) {
            OPAQUE_U(t1, (take >> ai) & 1u)
            if (t1) {
#pragma unroll
                for (int m = 0; m < 4; ++m) {
                    const int row = ai * 128 + wr * 64 + m * 16 + fr2;
#pragma unroll
                    for (int bj = 0; bj < 2; ++bj) {
                        f32x4 v0 = acc[ai][bj][m][0], v1 = acc[ai][bj][m][1];
#pragma unroll
                        for (int k = 0; k < 4; ++k) { v0[k] = v0[k] <= 0.f ? 0.f : v0[k]; v1[k] = v1[k] <= 0.f ? 0.f : v1[k]; }
                        u32x4 w; w.x = cvt_pk_bf16(v0[0], v0[1]); w.y = cvt_pk_bf16(v0[2], v0[3]); w.z = cvt_pk_bf16(v1[0], v1[1]); w.w = cvt_pk_bf16(v1[2], v1[3]);
                        if (dead) w = (u32x4){0x7fc07fc0u, 0x7fc07fc0u, 0x7fc07fc0u, 0x7fc07fc0u};
                        const int chunk = bj * 16 + wc * 4 + fq2;
                        *(LAS u32x4*)(smem + row * 512 + ((chunk ^ (row & 7)) * 16)) = w;
                    }
                }
            }
        }
        __syncthreads();
        STAMP(4);
        if ((take >> wr) & 1u) pack_strip(p, wsr, outr, smem, (tm * 8 + tn) * 8 + wid, wid, lane2, seq);
        STAMP(5);
        if (!split) return;
        __syncthreads();
    }
}

extern "C" void kernel_launch(int rank, int nranks, ncclComm_t comm,
                              hipStream_t stream,
                              void* const* d_in, const int* in_sizes, int n_in,
                              void* d_out, int out_size,
                              const int* in_dtypes, int out_dtype,
                              void* const* d_out_all,
                              void* const* d_ws_all, size_t ws_bytes) {
    (void)comm; (void)in_dtypes; (void)out_dtype;
    if (nranks != NR || n_in != 2 || (unsigned)rank >= (unsigned)NR || ws_bytes < WS_TOTAL || in_sizes[0] != ND * KS || in_sizes[1] != KS * ND || out_size != ND * ND) {
        fprintf(stderr, "kernel_launch: unexpected problem shape\n");
        return;
    }
    Params p;
    p.x = (const u16*)d_in[0]; p.w = (const u16*)d_in[1]; p.out = (u16*)d_out;
    p.out_all = d_out_all; p.ws_all = d_ws_all; p.rank = rank;
    p.ws_off = WS_AT_END ? (ws_bytes - WS_TOTAL) & ~(size_t)((2u << 20) - 1u) : 0;
    hipLaunchKernelGGL(gemm_ar_fused, dim3(NWG), dim3(512), 0, stream, p);
    HIP_CHECK(hipGetLastError());
}
```

```cpp
#include <hip/hip_runtime.h>
#include <rccl/rccl.h>
#include <cstdio>
#include <cstdint>

#define HIP_CHECK(expr) do { hipError_t _e = (expr); if (_e != hipSuccess) { \
    fprintf(stderr, "HIP error %s at %s:%d\n", hipGetErrorString(_e), __FILE__, __LINE__); \
    return; } } while (0)

#define LAS __attribute__((address_space(3)))
#define GAS __attribute__((address_space(1)))
typedef unsigned short u16;
typedef unsigned long long u64;
typedef short bf16x8 __attribute__((ext_vector_type(8)));
typedef short s16x4 __attribute__((ext_vector_type(4)));
typedef float f32x4 __attribute__((ext_vector_type(4)));
typedef unsigned u32x4 __attribute__((ext_vector_type(4)));
typedef unsigned u32x2 __attribute__((ext_vector_type(2)));
typedef u16 u16x2 __attribute__((ext_vector_type(2)));
typedef GAS unsigned gu32;
typedef GAS u64 gu64;

#define FORCE_RAW_IN 0
#define FORCE_RAW_OUT 0
#define NT 12
constexpr int NWV = 4;
#define NLW 6
#define EGU 1
#define EGS 2
#define WS_AT_END 1
#define OAUX 18
#define LAG1 0
#define LAG2 7
#define LAG3 14
#define KH1 12
#define KH2 22
#define KH3 42
#define NL1 4
#define PDELAY 336
#define PSLEEP 64

namespace {

constexpr int NR = 8;
constexpr int ND = 8192;
constexpr int KS = 1024;
constexpr int RROWS = 4096, RCOLS = 2048;
constexpr int NSTEP = 128;
constexpr int NWG = 256;
constexpr int NPD = 84 * 4;
constexpr int NENC = NPD + 512;
constexpr int NONCE = 256 + 512 + 96;
static_assert(KH1 % 2 == 0 && KH2 % 2 == 0 && KH3 % 2 == 0 && KH1 >= 2 && NLW >= NWV && NLW <= 8, "knobs");
__device__ __forceinline__ constexpr int lag(int k) { return k == 0 ? 0 : k == 1 ? LAG1 : k == 2 ? LAG2 : LAG3; }
constexpr unsigned long long DEADLINE = 40000000ull;

constexpr unsigned USLOT = 8192 + 1024;
constexpr unsigned SSLOT = 11264;
constexpr unsigned OFF_PART = 0;
constexpr unsigned OFF_EO = OFF_PART + (32u << 20);
constexpr unsigned OFF_XL = OFF_EO + (12u << 20);
constexpr unsigned OFF_WL = OFF_XL + (64u << 20);
constexpr unsigned OFF_EX = OFF_WL + (32u << 20);
constexpr unsigned OFF_EW = OFF_EX + (18u << 20);
constexpr unsigned OFF_C = OFF_EW + (18u << 20);
constexpr unsigned WS_USED = OFF_C + (1u << 20);
constexpr unsigned OFF_TMO = OFF_C + 0x80;
constexpr unsigned OFF_Q = OFF_C + 0x400;
constexpr unsigned OFF_ECNT = OFF_C + 0x1000;
constexpr unsigned OFF_FIN = OFF_C + 0x8000;
constexpr unsigned OFF_SEQ = OFF_C + 0xa000;
constexpr unsigned OFF_PF = OFF_C + 0xc000;
constexpr unsigned OFF_XF = OFF_C + 0x10000;
constexpr unsigned OFF_WF = OFF_C + 0x18000;
constexpr unsigned OFF_FOUT = OFF_C + 0x20000;
static_assert(2048u * USLOT <= (18u << 20), "unit area");
static_assert(1024u * SSLOT <= (12u << 20), "strip area");
constexpr unsigned OUT_BYTES = (unsigned)ND * ND * 2u;

constexpr int SM_TOKEN = 131072 + 32, SM_DUMMY = 131072 + 64, SM_LUT = 131072 + 256, SM_LUTE = SM_LUT + 4096, SM_SCR = SM_LUTE + 128;
constexpr int SM_AVAIL = SM_SCR + 24576 + 64;
constexpr int SMEM_BYTES = SM_AVAIL + 16;
constexpr unsigned CAP2 = 2304, CAP3 = 512, CAP4 = 240;

struct Params {
    const u16* x; const u16* w; u16* out;
    void* const* out_all; void* const* ws_all;
    int rank;
    size_t ws_off;
};

typedef __amdgpu_buffer_rsrc_t rsrc_t;
__device__ __forceinline__ rsrc_t make_rsrc(const void* base, unsigned bytes) {
    const u64 b = (u64)base;
    const unsigned lo = __builtin_amdgcn_readfirstlane((unsigned)b), hi = __builtin_amdgcn_readfirstlane((unsigned)(b >> 32));
    return __builtin_amdgcn_make_buffer_rsrc((void*)(((u64)hi << 32) | lo), 0, bytes, 0x00020000);
}
constexpr int SC1 = 16, SYS = 17;
template <int AUX> __device__ __forceinline__ u32x4 bld16(rsrc_t r, unsigned off) { return __builtin_amdgcn_raw_buffer_load_b128(r, off, 0, AUX); }
template <int AUX> __device__ __forceinline__ unsigned bld4(rsrc_t r, unsigned off) { return __builtin_amdgcn_raw_buffer_load_b32(r, off, 0, AUX); }
template <int AUX> __device__ __forceinline__ unsigned bld2(rsrc_t r, unsigned off) { return __builtin_amdgcn_raw_buffer_load_b16(r, off, 0, AUX); }
template <int AUX> __device__ __forceinline__ void bst16(rsrc_t r, unsigned off, u32x4 v) { __builtin_amdgcn_raw_buffer_store_b128(v, r, off, 0, AUX); }
template <int AUX> __device__ __forceinline__ void bst8(rsrc_t r, unsigned off, u32x2 v) { __builtin_amdgcn_raw_buffer_store_b64(v, r, off, 0, AUX); }
template <int AUX> __device__ __forceinline__ void bst4(rsrc_t r, unsigned off, unsigned v) { __builtin_amdgcn_raw_buffer_store_b32(v, r, off, 0, AUX); }
template <int AUX> __device__ __forceinline__ void bst2(rsrc_t r, unsigned off, unsigned v) { __builtin_amdgcn_raw_buffer_store_b16((u16)v, r, off, 0, AUX); }

__device__ __forceinline__ u32x4 make_desc(const void* base, unsigned bytes) {
    const u64 b = (u64)base;
    return (u32x4){(unsigned)__builtin_amdgcn_readfirstlane((unsigned)b), (unsigned)__builtin_amdgcn_readfirstlane((unsigned)(b >> 32)), bytes, 0x00020000u};
}
#define ALD16(dst, desc, off, imm) asm volatile("buffer_load_dwordx4 %0, %1, %2, 0 offen offset:%3 sc0 sc1" : "+v"(dst) : "v"(off), "s"(desc), "i"(imm) : "memory")
#define ALD8(dst, desc, off, imm) asm volatile("buffer_load_dwordx2 %0, %1, %2, 0 offen offset:%3 sc0 sc1" : "+v"(dst) : "v"(off), "s"(desc), "i"(imm) : "memory")
#define ALD4(dst, desc, off, imm) asm volatile("buffer_load_dword %0, %1, %2, 0 offen offset:%3 sc0 sc1" : "+v"(dst) : "v"(off), "s"(desc), "i"(imm) : "memory")
#define ALD2(dst, desc, off, imm) asm volatile("buffer_load_ushort %0, %1, %2, 0 offen offset:%3 sc0 sc1" : "+v"(dst) : "v"(off), "s"(desc), "i"(imm) : "memory")
constexpr unsigned OOB = 0xfffff000u;

#define WAIT_V(n) asm volatile("s_waitcnt vmcnt(" #n ")" ::: "memory")
#define WAIT_VN(n) asm volatile("s_waitcnt vmcnt(%0)" :: "i"(n) : "memory")
#define WAIT_L(n) asm volatile("s_waitcnt lgkmcnt(" #n ")" ::: "memory")
#define BAR __builtin_amdgcn_s_barrier()
#define SCHED __builtin_amdgcn_sched_barrier(0)
#define CFENCE asm volatile("" ::: "memory")
constexpr size_t WS_TOTAL = WS_USED;
#define STAMP(k) do { } while (0)

__device__ __forceinline__ unsigned rfl(unsigned v) { return __builtin_amdgcn_readfirstlane(v); }
__device__ __forceinline__ unsigned pkmax(unsigned a, unsigned b) {
    u16x2 x = __builtin_bit_cast(u16x2, a), y = __builtin_bit_cast(u16x2, b);
    return __builtin_bit_cast(unsigned, __builtin_elementwise_max(x, y));
}
__device__ __forceinline__ unsigned pkmin(unsigned a, unsigned b) {
    u16x2 x = __builtin_bit_cast(u16x2, a), y = __builtin_bit_cast(u16x2, b);
    return __builtin_bit_cast(unsigned, __builtin_elementwise_min(x, y));
}
__device__ __forceinline__ unsigned wave_max(unsigned v) {
#define MAX_STEP(ctrl, rmask) { const unsigned t = (unsigned)__builtin_amdgcn_update_dpp(0, (int)v, ctrl, rmask, 0xf, false); v = v > t ? v : t; }
    MAX_STEP(0x111, 0xf) MAX_STEP(0x112, 0xf) MAX_STEP(0x114, 0xf) MAX_STEP(0x118, 0xf) MAX_STEP(0x142, 0xa) MAX_STEP(0x143, 0xc)
    return __builtin_amdgcn_readlane(v, 63);
}
__device__ __forceinline__ unsigned pkmin1(unsigned a) {
    unsigned r; asm("v_pk_min_u16 %0, %1, 1 op_sel_hi:[1,0]" : "=v"(r) : "v"(a)); return r;
}
__device__ __forceinline__ unsigned cvt_pk_bf16(float lo, float hi) { unsigned r; asm volatile("v_cvt_pk_bf16_f32 %0, %1, %2" : "=v"(r) : "v"(lo), "v"(hi)); return r; }
__device__ __forceinline__ int lds_byte(int r, int c) { const int st = (r >> 4) * 2 + (c >> 5), rr = r & 15, cc = c & 31, ob = rr * 64 + cc * 2; return st * 1024 + (ob ^ (((ob >> 9) & 1) << 5)); }

__device__ __forceinline__ unsigned wave_scan(unsigned x) {
    x += (unsigned)__builtin_amdgcn_update_dpp(0, (int)x, 0x111, 0xf, 0xf, false);
    x += (unsigned)__builtin_amdgcn_update_dpp(0, (int)x, 0x112, 0xf, 0xf, false);
    x += (unsigned)__builtin_amdgcn_update_dpp(0, (int)x, 0x114, 0xf, 0xf, false);
    x += (unsigned)__builtin_amdgcn_update_dpp(0, (int)x, 0x118, 0xf, 0xf, false);
    x += (unsigned)__builtin_amdgcn_update_dpp(0, (int)x, 0x142, 0xa, 0xf, false);
    x += (unsigned)__builtin_amdgcn_update_dpp(0, (int)x, 0x143, 0xc, 0xf, false);
    return x;
}
__device__ __forceinline__ unsigned wave_sum(unsigned v) { return __builtin_amdgcn_readlane(wave_scan(v), 63); }
__device__ __forceinline__ unsigned pksub(unsigned a, unsigned b) {
    u16x2 x = __builtin_bit_cast(u16x2, a), y = __builtin_bit_cast(u16x2, b);
    return __builtin_bit_cast(unsigned, (u16x2)(x - y));
}
__device__ __forceinline__ unsigned bfi(unsigned mask, unsigned a, unsigned b) { return (a & mask) | (b & ~mask); }
__device__ __forceinline__ unsigned choose_window(unsigned esum, unsigned n, unsigned emax) {
    if (!n) return 21u;
    const unsigned num = esum * 32u + n * 63u, den = n * 32u;
    unsigned t = num / den; const unsigned alt = (num - t * den) * 5u > den * 2u ? 1u : 0u;
    t = t > emax ? emax : t; t = t < 21u ? 21u : (t > 253u ? 253u : t);
    return t | (alt << 9);
}
template <int NQ> __device__ __forceinline__ unsigned apply_escapes(unsigned (&e)[NQ], const unsigned (&m)[NQ], LAS unsigned char* smem, unsigned off) {
    unsigned sel[NQ], o[NQ]; u32x2 w[NQ];
#pragma unroll
    for (int q = 0; q < NQ; ++q) {
        sel[q] = *(const LAS unsigned*)(smem + SM_LUTE + __builtin_amdgcn_udot4(m[q], 0x08040201u, 0u, false));
        o[q] = off; off += (unsigned)__builtin_popcount(m[q]);
        const LAS unsigned* pw = (const LAS unsigned*)(smem + (o[q] & ~3u));
        w[q] = (u32x2){pw[0], pw[1]};
    }
#pragma unroll
    for (int q = 0; q < NQ; ++q) e[q] = __builtin_amdgcn_perm(__builtin_amdgcn_alignbyte(w[q].y, w[q].x, o[q]), e[q], sel[q]);
    return off;
}
__device__ __forceinline__ unsigned append_flagged(LAS unsigned char* smem, unsigned e, unsigned m, unsigned off, unsigned dummy) {
#pragma unroll
    for (int k = 0; k < 4; ++k) {
        const unsigned bit = (m >> (8 * k + 2)) & 1u;
        smem[bit ? off : dummy] = (unsigned char)(e >> (8 * k));
        off += bit;
    }
    return off;
}
__device__ __forceinline__ void decode_levels(LAS unsigned char* smem, unsigned h0, unsigned h1, unsigned o2, unsigned l3, unsigned l2, int lane) {
    const unsigned t = h0 & 0xffu, alt = (h0 >> 9) & 1u, n2 = h0 >> 16, n3 = h1 & 0xffffu;
    const unsigned o3 = o2 + ((n2 + 15u) >> 4) * 4u, o4 = o3 + ((n3 + 7u) >> 3) * 4u;
    const unsigned b3q = (t - 6u + alt) * 0x01010101u, spq = (t + 1u + alt) * 0x01010101u;
    unsigned eo = o4;
    for (unsigned r = 0; r * 256u < n3; ++r) {
        const unsigned g = r * 64u + lane;
        unsigned c = *(const LAS u16*)(smem + o3 + g * 2u); c = g * 4u < n3 ? c : 0u;
        const unsigned z = __builtin_amdgcn_perm(0u, c, 0x01010000u);
        const unsigned d = (z & 0x000f000fu) | ((z >> 4) & 0x0f000f00u);
        const unsigned is15 = (d + 0x01010101u) & 0x10101010u, is14 = ((d + 0x02020202u) & 0x10101010u) ^ is15;
        unsigned e[1], m[1];
        e[0] = bfi((is14 << 4) - (is14 >> 4), spq, b3q - d); m[0] = is15 >> 2;
        const unsigned ec = (unsigned)__builtin_popcount(m[0]), inc = wave_scan(ec);
        apply_escapes<1>(e, m, smem, eo + inc - ec);
        eo += __builtin_amdgcn_readlane(inc, 63);
        *(LAS unsigned*)(smem + l3 + g * 4u) = e[0];
    }
    const unsigned tab = (t - 3u) | ((t - 4u) << 8) | ((alt ? t + 1u : t - 5u) << 16);
    eo = l3;
    for (unsigned r = 0; r * 1024u < n2; ++r) {
        const unsigned g = r * 64u + lane;
        unsigned c = *(const LAS unsigned*)(smem + o2 + g * 4u); c = g * 16u < n2 ? c : 0u;
        unsigned e[4], m[4];
#pragma unroll
        for (int k = 0; k < 4; ++k) { const unsigned d = (c >> (2 * k)) & 0x03030303u; e[k] = __builtin_amdgcn_perm(0u, tab, d); m[k] = (d + 0x01010101u) & 0x04040404u; }
        const unsigned ec = (unsigned)__builtin_popcount(c & (c >> 1) & 0x55555555u), inc = wave_scan(ec);
        apply_escapes<4>(e, m, smem, eo + inc - ec);
        eo += __builtin_amdgcn_readlane(inc, 63);
        if (g * 16u < n2) *(LAS u32x4*)(smem + l2 + g * 16u) = (u32x4){e[0], e[1], e[2], e[3]};
    }
}
__device__ __forceinline__ unsigned encode_levels(LAS unsigned char* smem, rsrc_t wsr, unsigned win, unsigned n2, unsigned l2, unsigned l3, unsigned l4, unsigned g2, unsigned room, unsigned dummy, int lane) {
    const unsigned t = win & 0xffu, alt = (win >> 9) & 1u;
    if (lane < 16) smem[l2 + n2 + lane] = (unsigned char)(t - 3u);
    const unsigned t3 = (t - 3u) * 0x00010001u, sc = (alt ? t + 1u : t - 5u) * 0x00010001u;
    unsigned n3 = 0;
    for (unsigned r = 0; r * 1024u < n2; ++r) {
        const unsigned g = r * 64u + lane; const bool act = g * 16u < n2;
        const u32x4 e = *(const LAS u32x4*)(smem + l2 + (act ? g : 0u) * 16u);
        unsigned c = 0, m[4], ec = 0;
#pragma unroll
        for (int k = 0; k < 4; ++k) {
            unsigned dd[2];
#pragma unroll
            for (int h = 0; h < 2; ++h) {
                const unsigned x = (e[k] >> (8 * h)) & 0x00ff00ffu;
                const unsigned a = pkmin(pksub(t3, x), 0x00020002u);
                dd[h] = a + ((a >> 1) & pkmin1(x ^ sc));
            }
            const unsigned d = dd[0] | (dd[1] << 8);
            c |= d << (2 * k); m[k] = act ? (d + 0x01010101u) & 0x04040404u : 0u; ec += (unsigned)__builtin_popcount(m[k]);
        }
        const unsigned inc = wave_scan(ec), tot = __builtin_amdgcn_readlane(inc, 63);
        if (n3 + tot > CAP3) return 0xffffffffu;
        if (act) bst4<SYS>(wsr, g2 + g * 4u, c);
        unsigned eo = l3 + n3 + inc - ec;
#pragma unroll
        for (int k = 0; k < 4; ++k) eo = append_flagged(smem, e[k], m[k], eo, dummy);
        n3 += tot;
    }
    const unsigned b3 = t - 6u + alt;
    if (lane < 8) smem[l3 + n3 + lane] = (unsigned char)b3;
    const unsigned g3 = g2 + ((n2 + 15u) >> 4) * 4u, g4 = g3 + ((n3 + 7u) >> 3) * 4u;
    if (g4 - g2 > room) return 0xffffffffu;
    const unsigned b32 = b3 * 0x00010001u, sp = (t + 1u + alt) * 0x00010001u;
    unsigned n4 = 0;
    for (unsigned r = 0; r * 256u < n3; ++r) {
        const unsigned g = r * 64u + lane; const bool act = g * 4u < ((n3 + 7u) & ~7u);
        const unsigned e = *(const LAS unsigned*)(smem + l3 + (act ? g : 0u) * 4u);
        unsigned dd[2];
#pragma unroll
        for (int h = 0; h < 2; ++h) {
            const unsigned x = (e >> (8 * h)) & 0x00ff00ffu;
            const unsigned a = pkmin(pksub(b32, x), 0x000e000eu);
            dd[h] = a + (((a + 0x00020002u) >> 4) & pkmin1(x ^ sp));
        }
        const unsigned d = dd[0] | (dd[1] << 8);
        const unsigned m = act ? ((d + 0x01010101u) & 0x10101010u) >> 2 : 0u;
        const unsigned ec = (unsigned)__builtin_popcount(m), inc = wave_scan(ec), tot = __builtin_amdgcn_readlane(inc, 63);
        if (n4 + tot > CAP4) return 0xffffffffu;
        const unsigned y = d | (d >> 4);
        if (act) bst2<SYS>(wsr, g3 + g * 2u, __builtin_amdgcn_perm(0u, y, 0x0c0c0200u));
        if (tot) append_flagged(smem, e, m, l4 + n4 + inc - ec, dummy);
        n4 += tot;
    }
    if (g4 - g2 + n4 > room) return 0xffffffffu;
    if (n4) { const unsigned lv = *(const LAS unsigned*)(smem + l4 + lane * 4); if ((unsigned)lane * 4u < n4) bst4<SYS>(wsr, g4 + lane * 4, lv); }
    return n3 | (n4 << 16);
}
__device__ __forceinline__ unsigned side_bytes(unsigned n2, unsigned h1) { return ((n2 + 15u) >> 4) * 4u + (((h1 & 0xffffu) + 7u) >> 3) * 4u + (h1 >> 16); }
struct Order {
    int ph, start;
    template <int NWU, int NXU> __device__ __forceinline__ void locate(int i, int& j, int& g, int& u) {
        for (;;) {
            int c = (ph < 16) ? NWU : 0;
#pragma unroll
            for (int k = 0; k < 4; ++k) c += ((unsigned)(ph - lag(k)) < 16u) ? NXU : 0;
            if (i < start + c) break;
            start += c; ++ph;
        }
        u = i - start; g = 0; j = ph;
        if (ph < 16) { if (u < NWU) return; u -= NWU; }
#pragma unroll
        for (int k = 0; k < 4; ++k) {
            if ((unsigned)(ph - lag(k)) < 16u) { if (u < NXU) { g = 1 + k; j = ph - lag(k); return; } u -= NXU; }
        }
    }
};

__device__ __forceinline__ bool give_up(gu32* tmo, unsigned seq, u64 t0) {
    bool dead = rfl(__hip_atomic_load(tmo, __ATOMIC_RELAXED, __HIP_MEMORY_SCOPE_AGENT)) == seq;
    if (!dead && __builtin_amdgcn_s_memrealtime() - t0 > DEADLINE) {
        __hip_atomic_store(tmo, seq, __ATOMIC_RELAXED, __HIP_MEMORY_SCOPE_AGENT);
        dead = true;
    }
    return dead;
}
__device__ __forceinline__ unsigned early_word(unsigned u) { return (u / 3u) * 8u + 5u + u % 3u; }

__device__ __forceinline__ void pack_unit(const Params& p, char* ws, rsrc_t wsr, LAS unsigned char* smem, unsigned lwo, int j, int g, int u, int lane, unsigned seqv) {
    const bool isx = g != 0; const int ub = isx ? (u >> 4) * 64 + (g - 1) * 16 + (u & 15) : u;
    const int ra = p.rank >> 2, rb = p.rank & 3;
    LAS unsigned char* lw = smem + lwo;
    u32x4 v[8];
    if (isx) {
        const u16* src = p.x + (size_t)(ub * 64 + (lane & 15)) * KS + j * 64 + (lane >> 4) * 8;
#pragma unroll
        for (int c = 0; c < 8; ++c) v[c] = *(const u32x4*)(src + (size_t)(c >> 1) * 16 * KS + (c & 1) * 32);
    } else {
        constexpr int P = 144;
        const u16* src = p.w + (size_t)(j * 64 + (lane >> 3)) * ND + ub * 64 + (lane & 7) * 8;
        u32x4 tmp[8];
#pragma unroll
        for (int i = 0; i < 8; ++i) tmp[i] = *(const u32x4*)(src + (size_t)i * 8 * ND);
#pragma unroll
        for (int i = 0; i < 8; ++i) *(LAS u32x4*)(lw + (i * 8 + (lane >> 3)) * P + (lane & 7) * 16) = tmp[i];
        const int q = (lane >> 2) & 3, pp = lane & 3;
#pragma unroll
        for (int c = 0; c < 8; ++c) {
            const int k0 = (c & 1) * 32 + (lane >> 4) * 8, rbk = c >> 1;
            const int col = (rbk >> 1) * 32 + 8 * pp + 4 * (rbk & 1);
            const s16x4 a = __builtin_amdgcn_ds_read_tr16_b64_v4i16((LAS s16x4*)(lw + (k0 + q) * P + col * 2));
            const s16x4 b = __builtin_amdgcn_ds_read_tr16_b64_v4i16((LAS s16x4*)(lw + (k0 + 4 + q) * P + col * 2));
            const u32x2 a2 = __builtin_bit_cast(u32x2, a), b2 = __builtin_bit_cast(u32x2, b);
            v[c] = (u32x4){a2.x, a2.y, b2.x, b2.y};
        }
    }
    const unsigned loff = (lane & 15) * 64 + (((lane >> 4) * 16) ^ (((lane >> 3) & 1) << 5));
    const int cls = isx ? (ub >> 6) : (ub >> 5);
    const bool own = isx ? (cls == ra) : (cls == rb);
    const int li = isx ? (ub & 63) : (ub & 31);
    if (own) {
        const unsigned dst = (isx ? OFF_XL + (unsigned)(j * 8 * 64 + li) * 8192u : OFF_WL + (unsigned)(j * 8 * 32 + li) * 8192u) + loff;
#pragma unroll
        for (int c = 0; c < 8; ++c) bst16<SC1>(wsr, dst + c * 1024, v[c]);
    }
    const unsigned slot0 = (isx ? OFF_EX : OFF_EW) + (unsigned)(j * 128 + ub) * USLOT;
    const unsigned pay = slot0 + 1024 + lane * 16;
    const bool early = j == 0 && g == 0;
    if (early) {
#pragma unroll
        for (int c = 0; c < 8; ++c) bst16<SYS>(wsr, pay + c * 1024, v[c]);
        if (lane == 0) bst4<SYS>(wsr, slot0, 1u << 8);
    } else {
    unsigned e4[16], sm[16], mx = 0, esum = 0;
#pragma unroll
    for (int q = 0; q < 16; ++q) {
        const unsigned a = v[q >> 1][(q & 1) * 2], b = v[q >> 1][(q & 1) * 2 + 1];
        const unsigned lo = __builtin_amdgcn_perm(b, a, 0x06040200u), hi = __builtin_amdgcn_perm(b, a, 0x07050301u);
        e4[q] = bfi(0x01010101u, lo >> 7, hi << 1);
        sm[q] = bfi(0x7f7f7f7fu, lo, hi);
        esum = __builtin_amdgcn_udot4(e4[q], 0x01010101u, esum, false);
        mx = pkmax(mx, a & 0x7fff7fffu); mx = pkmax(mx, b & 0x7fff7fffu);
    }
    mx = (mx & 0xffffu) > (mx >> 16) ? (mx & 0xffffu) : (mx >> 16);
    const unsigned win = choose_window(wave_sum(esum), 4096u, wave_max(mx) >> 7);
    const unsigned top2 = (win & 0xffu) * 0x00010001u;
    unsigned p2[4] = {0u, 0u, 0u, 0u}, m[16], ec = 0;
#pragma unroll
    for (int q = 0; q < 16; ++q) {
        const unsigned da = pkmin(pksub(top2, e4[q] & 0x00ff00ffu), 0x00030003u), db = pkmin(pksub(top2, (e4[q] >> 8) & 0x00ff00ffu), 0x00030003u);
        const unsigned d = da | (db << 8);
        p2[q >> 2] |= d << (2 * (q & 3));
        m[q] = (d + 0x01010101u) & 0x04040404u; ec += (unsigned)__builtin_popcount(m[q]);
    }
    const unsigned einc = wave_scan(ec), n2 = __builtin_amdgcn_readlane(einc, 63);
    unsigned h1 = 0xffffffffu;
    if (!FORCE_RAW_IN && n2 <= CAP2) {
        const unsigned l2 = lwo + 10240, l3 = l2 + CAP2 + 16, l4 = l3 + CAP3 + 16, dummy = SM_DUMMY + lane * 2;
        unsigned eo = l2 + einc - ec;
#pragma unroll
        for (int q = 0; q < 16; ++q) eo = append_flagged(smem, e4[q], m[q], eo, dummy);
        h1 = encode_levels(smem, wsr, win, n2, l2, l3, l4, slot0 + 8, 1016u, dummy, lane);
    }
    if (h1 == 0xffffffffu) {
#pragma unroll
        for (int c = 0; c < 8; ++c) bst16<SYS>(wsr, pay + c * 1024, v[c]);
        if (lane == 0) bst4<SYS>(wsr, slot0, 1u << 8);
    } else {
#pragma unroll
        for (int i = 0; i < 4; ++i) bst16<SYS>(wsr, pay + i * 1024, (u32x4){sm[4 * i], sm[4 * i + 1], sm[4 * i + 2], sm[4 * i + 3]});
        bst16<SYS>(wsr, pay + 4096, (u32x4){p2[0], p2[1], p2[2], p2[3]});
        if (lane == 0) bst8<SYS>(wsr, slot0, (u32x2){win | (n2 << 16), h1});
    }
    }
    WAIT_V(0);
    const unsigned seq = rfl(seqv) + 1u;
    if (own && lane == 0) {
        const unsigned f = (isx ? OFF_XF : OFF_WF) + (unsigned)(((li >> 2) * NSTEP + j * 8) * 4 + (li & 3)) * 4u;
        __hip_atomic_store((gu32*)(ws + f), seq, __ATOMIC_RELAXED, __HIP_MEMORY_SCOPE_AGENT);
    }
    const unsigned per = isx ? 16u : 32u;
    unsigned ticket = per - 1u;
    if (!early) {
        if (lane == 0) ticket = __hip_atomic_fetch_add((gu32*)(ws + OFF_ECNT + (unsigned)(j * 12 + (isx ? cls * 4 + g - 1 : 8 + cls)) * 128u), 1u, __ATOMIC_RELAXED, __HIP_MEMORY_SCOPE_AGENT);
        ticket = rfl(ticket);
    }
    if ((ticket & (per - 1u)) == per - 1u) {
        const bool want = lane < NR && lane != p.rank && (isx ? ((lane >> 2) == cls) : ((lane & 3) == cls));
        if (want) {
            char* pw = ((char*)p.ws_all[lane] + p.ws_off);
            const unsigned word = early ? early_word((unsigned)li) : (unsigned)(j * 8 + g);
            __hip_atomic_store((gu32*)(pw + OFF_FIN + (unsigned)p.rank * 512u + word * 4u), seq, __ATOMIC_RELAXED, __HIP_MEMORY_SCOPE_SYSTEM);
        }
    }
}

struct UnitRegs { u32x4 a[5]; u32x4 sd; };
__device__ __forceinline__ unsigned unit_slot(int rank, int j, int g, int u) {
    const int ub = g ? (rank >> 2) * 64 + (g - 1) * 16 + u : (rank & 3) * 32 + u;
    return (g ? OFF_EX : OFF_EW) + (unsigned)(j * 128 + ub) * USLOT;
}
__device__ __forceinline__ void unit_issue(UnitRegs& r, u32x4 pd, unsigned slot0, int lane) {
    const unsigned oh = lane < 24 ? slot0 + lane * 16 : OOB, o0 = slot0 + 1024 + lane * 16, o1 = o0 + 4096;
    ALD16(r.sd, pd, oh, 0);
    ALD16(r.a[0], pd, o0, 0); ALD16(r.a[1], pd, o0, 1024); ALD16(r.a[2], pd, o0, 2048); ALD16(r.a[3], pd, o0, 3072);
    ALD16(r.a[4], pd, o1, 0);
}
#define UNIT_LANDED(r, n) asm volatile("s_waitcnt vmcnt(0)" : "+v"(r.a[0]), "+v"(r.a[1]), "+v"(r.a[2]), "+v"(r.a[3]), "+v"(r.a[4]), "+v"(r.sd), "+v"(n) :: "memory")
__device__ __forceinline__ void unit_unpack(const UnitRegs& r, rsrc_t wsr, rsrc_t pr, LAS unsigned char* smem, unsigned scr, unsigned slot0, unsigned dst0, int lane) {
    const unsigned dst = dst0 + (lane & 15) * 64 + (((lane >> 4) * 16) ^ (((lane >> 3) & 1) << 5));
    const unsigned h0 = rfl(r.sd.x), h1 = rfl(r.sd.y);
    if ((h0 >> 8) & 1u) {
        u32x4 b[3];
#pragma unroll
        for (int c = 0; c < 3; ++c) b[c] = bld16<SYS>(pr, slot0 + 1024 + lane * 16 + (5 + c) * 1024);
#pragma unroll
        for (int c = 0; c < 5; ++c) bst16<SC1>(wsr, dst + c * 1024, r.a[c]);
#pragma unroll
        for (int c = 0; c < 3; ++c) bst16<SC1>(wsr, dst + (5 + c) * 1024, b[c]);
        return;
    }
    u32x4 sd = r.sd;
    const unsigned len = 8u + side_bytes(h0 >> 16, h1);
    if (len > 384u) { if (lane >= 24 && (unsigned)lane * 16u < len) sd = bld16<SYS>(pr, slot0 + lane * 16); }
    *(LAS u32x4*)(smem + scr + lane * 16) = sd;
    const unsigned l3 = scr + 2048, l2 = l3 + CAP3 + 256;
    decode_levels(smem, h0, h1, scr + 8u, l3, l2, lane);
    const unsigned tq = (h0 & 0xffu) * 0x01010101u;
    unsigned e[16], m[16], ec = 0;
#pragma unroll
    for (int q = 0; q < 16; ++q) {
        const unsigned d = (r.a[4][q >> 2] >> (2 * (q & 3))) & 0x03030303u;
        m[q] = (d + 0x01010101u) & 0x04040404u; e[q] = tq - d;
    }
#pragma unroll
    for (int k = 0; k < 4; ++k) ec += (unsigned)__builtin_popcount(r.a[4][k] & (r.a[4][k] >> 1) & 0x55555555u);
    apply_escapes<16>(e, m, smem, l2 + wave_scan(ec) - ec);
#pragma unroll
    for (int c = 0; c < 8; ++c) {
        u32x4 o;
#pragma unroll
        for (int h = 0; h < 2; ++h) {
            const int q = 2 * c + h; const unsigned sm = r.a[q >> 2][q & 3];
            const unsigned lo = bfi(0x7f7f7f7fu, sm, e[q] << 7), hi = bfi(0x7f7f7f7fu, e[q] >> 1, sm);
            o[2 * h] = __builtin_amdgcn_perm(hi, lo, 0x05010400u); o[2 * h + 1] = __builtin_amdgcn_perm(hi, lo, 0x07030602u);
        }
        bst16<SC1>(wsr, dst + c * 1024, o);
    }
}

__device__ __forceinline__ unsigned strip_out_off(int r, int s, int lane) {
    const int tile = s >> 3, tm = tile >> 3, tn = tile & 7;
    const int row = (r >> 2) * RROWS + tm * 256 + (s & 7) * 32 + (lane >> 5);
    const int col = (r & 3) * RCOLS + tn * 256 + (lane & 31) * 8;
    return ((unsigned)row * ND + col) * 2u;
}

struct StripRegs { u32x4 mk; u32x4 lo[8]; u32x4 sd[2]; };
__device__ __forceinline__ void strip_issue(StripRegs& r, u32x4 pd, unsigned s, unsigned info, int lane) {
    if (info >> 30) return;
    const unsigned T = info & 0x3fffu, sl = (info >> 14) & 0xffu, slot = OFF_EO + s * SSLOT;
    const unsigned om = slot + lane * 16;
    ALD16(r.mk, pd, om, 0);
    { const unsigned o = (unsigned)lane < sl ? om : OOB; ALD16(r.sd[0], pd, o, 1024); }
    if (sl > 64u) { const unsigned o = (unsigned)lane + 64u < sl ? om : OOB; ALD16(r.sd[1], pd, o, 2048); }
#define STRIP_ROUND(k) if ((unsigned)(k) * 1024u < T) { const bool act = (unsigned)((k) * 1024 + lane * 16) < T; \
        const unsigned ol = act ? om + 3072 + ((k) >> 2) * 4096 : OOB; ALD16(r.lo[k], pd, ol, ((k) & 3) * 1024); }
    STRIP_ROUND(0) STRIP_ROUND(1) STRIP_ROUND(2) STRIP_ROUND(3) STRIP_ROUND(4) STRIP_ROUND(5) STRIP_ROUND(6) STRIP_ROUND(7)
}
#define STRIP_LANDED(r, fn, n, cnt) do { \
    asm volatile("s_waitcnt vmcnt(" #cnt ")" : "+v"(r.lo[0]), "+v"(r.lo[1]), "+v"(r.lo[2]), "+v"(r.lo[3]), "+v"(r.lo[4]), "+v"(r.lo[5]), "+v"(r.lo[6]), "+v"(r.lo[7]), \
        "+v"(r.mk), "+v"(r.sd[0]), "+v"(r.sd[1]), "+v"(fn), "+v"(n) :: "memory"); } while (0)
__device__ __forceinline__ void strip_stage1(const StripRegs& r, LAS unsigned char* smem, unsigned lbase, unsigned scr, unsigned info, int lane) {
    if (info >> 30) return;
    const unsigned T = info & 0x3fffu;
    const unsigned h0 = rfl(r.sd[0].x), h1 = rfl(r.sd[0].y);
    *(LAS u32x4*)(smem + scr + lane * 16) = r.sd[0];
    *(LAS u32x4*)(smem + scr + 1024 + lane * 16) = r.sd[1];
    const unsigned l3 = scr + 2048, l2 = l3 + CAP3 + 256;
    decode_levels(smem, h0, h1, scr + 8u + ((T + 31u) >> 5) * 4u, l3, l2, lane);
    const unsigned tq = (h0 & 0xffu) * 0x01010101u;
    unsigned x1[8], ec = 0;
#pragma unroll
    for (int k = 0; k < 8; ++k) {
        if ((unsigned)k * 1024u < T) {
            const unsigned z = __builtin_amdgcn_perm(0u, (unsigned)*(const LAS u16*)(smem + scr + 8 + k * 128 + lane * 2), 0x01010000u);
            x1[k] = (z & 0x000f000fu) | ((z >> 4) & 0x0f000f00u);
#pragma unroll
            for (int i = 0; i < 4; ++i) ec += (unsigned)__builtin_popcount((x1[k] >> i) & (r.lo[k][i] >> 7) & 0x01010101u);
        }
    }
    unsigned eo = l2 + wave_scan(ec) - ec;
    unsigned e[8][4];
#pragma unroll
    for (int k = 0; k < 8; ++k) {
        if ((unsigned)k * 1024u < T) {
            unsigned m[4];
#pragma unroll
            for (int i = 0; i < 4; ++i) {
                const unsigned d = ((x1[k] >> i) & 0x01010101u) | ((r.lo[k][i] >> 6) & 0x02020202u);
                m[i] = (d + 0x01010101u) & 0x04040404u; e[k][i] = tq - d;
            }
            eo = apply_escapes<4>(e[k], m, smem, eo);
        }
    }
#pragma unroll
    for (int k = 0; k < 8; ++k) {
        if ((unsigned)k * 1024u < T) {
            unsigned lo[4], hi[4];
#pragma unroll
            for (int i = 0; i < 4; ++i) { lo[i] = bfi(0x7f7f7f7fu, r.lo[k][i], e[k][i] << 7); hi[i] = (e[k][i] >> 1) & 0x7f7f7f7fu; }
            *(LAS u32x4*)(smem + lbase + k * 2048 + lane * 32) = (u32x4){lo[0], hi[0], lo[1], hi[1]};
            *(LAS u32x4*)(smem + lbase + k * 2048 + lane * 32 + 16) = (u32x4){lo[2], hi[2], lo[3], hi[3]};
        }
    }
}
__device__ __forceinline__ void strip_stage2(u32x4 mk, rsrc_t outr, rsrc_t por, LAS unsigned char* smem, unsigned lbase, int peer, unsigned s, unsigned info, int lane) {
    const unsigned oo = strip_out_off(peer, (int)s, lane);
    if (info >> 31) {
#pragma unroll
        for (int q = 0; q < 16; ++q) bst16<OAUX>(outr, oo + q * 2 * ND * 2, (u32x4){0x7fc07fc0u, 0x7fc07fc0u, 0x7fc07fc0u, 0x7fc07fc0u});
        return;
    }
    if (info >> 30) {
#pragma unroll
        for (int h = 0; h < 4; ++h) {
            u32x4 a[4];
#pragma unroll
            for (int q = 0; q < 4; ++q) a[q] = bld16<SYS>(por, oo + (h * 4 + q) * 2 * ND * 2);
#pragma unroll
            for (int q = 0; q < 4; ++q) bst16<OAUX>(outr, oo + (h * 4 + q) * 2 * ND * 2, a[q]);
        }
        return;
    }
    unsigned c2[8], x[8];
#pragma unroll
    for (int i = 0; i < 8; ++i) {
        const unsigned m16 = mk[i >> 1] >> ((i & 1) * 16);
        c2[i] = (unsigned)__builtin_popcount(m16 & 0xffu) | ((unsigned)__builtin_popcount((m16 >> 8) & 0xffu) << 16);
        x[i] = c2[i];
    }
#define SCAN_STEP(ctrl, rmask) _Pragma("unroll") for (int i = 0; i < 8; ++i) x[i] += (unsigned)__builtin_amdgcn_update_dpp(0, (int)x[i], ctrl, rmask, 0xf, false);
    SCAN_STEP(0x111, 0xf) SCAN_STEP(0x112, 0xf) SCAN_STEP(0x114, 0xf) SCAN_STEP(0x118, 0xf) SCAN_STEP(0x142, 0xa) SCAN_STEP(0x143, 0xc)
    unsigned pos = 0;
#pragma unroll
    for (int bt = 0; bt < 4; ++bt) {
        u32x4 sel[4], d[8]; unsigned off[8];
#pragma unroll
        for (int k = 0; k < 4; ++k) {
            const int i = bt * 2 + (k >> 1), hq = k & 1;
            const unsigned m8 = (mk[bt] >> (k * 8)) & 0xffu;
            sel[k] = *(const LAS u32x4*)(smem + SM_LUT + m8 * 16u);
            const unsigned tot = __builtin_amdgcn_readlane(x[i], 63);
            off[2 * k] = pos + (((x[i] - c2[i]) >> (hq * 16)) & 0xffffu);
            off[2 * k + 1] = off[2 * k] + (unsigned)__builtin_popcount(m8 & 15u);
            pos += (tot >> (hq * 16)) & 0xffffu;
        }
#pragma unroll
        for (int g = 0; g < 8; ++g) {
            const LAS u32x2* pl = (const LAS u32x2*)(smem + lbase + (off[g] & ~3u) * 2u);
            const u32x2 d0 = pl[0], d1 = pl[1];
            d[g] = (u32x4){d0.x, d0.y, d1.x, d1.y};
        }
#pragma unroll
        for (int k = 0; k < 4; ++k) {
            u32x4 o;
#pragma unroll
            for (int g = 0; g < 2; ++g) {
                const u32x4 dd = d[2 * k + g];
                const unsigned l4 = __builtin_amdgcn_alignbyte(dd.z, dd.x, off[2 * k + g]), h4 = __builtin_amdgcn_alignbyte(dd.w, dd.y, off[2 * k + g]);
                o[2 * g] = __builtin_amdgcn_perm(h4, l4, sel[k][2 * g]); o[2 * g + 1] = __builtin_amdgcn_perm(h4, l4, sel[k][2 * g + 1]);
            }
            bst16<OAUX>(outr, oo + (bt * 4 + k) * 2 * ND * 2, o);
        }
    }
}

__device__ __forceinline__ void pack_strip(const Params& p, rsrc_t wsr, rsrc_t outr, LAS unsigned char* smem, int s, int wid, int lane, unsigned seq) {
    const unsigned oo = strip_out_off(p.rank, s, lane);
    const unsigned lbase = (unsigned)wid * 16384u;
    unsigned pos = lbase, mx = 0, sg = 0, esum = 0, ts = 0;
    const unsigned dummy = SM_DUMMY + lane * 2;
    u32x4 mk = {0u, 0u, 0u, 0u};
#pragma unroll
    for (int q2 = 0; q2 < 8; ++q2) {
        u32x4 v[2]; unsigned m8[2];
#pragma unroll
        for (int hq = 0; hq < 2; ++hq) {
            const int q = q2 * 2 + hq, row = wid * 32 + 2 * q + (lane >> 5);
            v[hq] = *(const LAS u32x4*)(smem + row * 512 + (((lane & 31) ^ (row & 7)) * 16));
            bst16<SYS>(outr, oo + q * 2 * ND * 2, v[hq]);
            unsigned m = 0;
#pragma unroll
            for (int k = 0; k < 4; ++k) {
                sg |= v[hq][k];
                if (q2 < 2) {
                    mx = pkmax(mx, v[hq][k]);
                    esum = __builtin_amdgcn_udot4((v[hq][k] >> 7) & 0x00ff00ffu, 0x00010001u, esum, false);
                }
                const unsigned t = pkmin1(v[hq][k]);
                m |= ((t | (t >> 15)) & 3u) << (2 * k);
            }
            m8[hq] = m;
        }
        mk[q2 >> 1] |= (m8[0] | (m8[1] << 8)) << ((q2 & 1) * 16);
        const unsigned c2 = (unsigned)__builtin_popcount(m8[0]) | ((unsigned)__builtin_popcount(m8[1]) << 16);
        const unsigned inc = wave_scan(c2), exl = inc - c2, tot = __builtin_amdgcn_readlane(inc, 63);
#pragma unroll
        for (int hq = 0; hq < 2; ++hq) {
            unsigned a = pos + ((exl >> (hq * 16)) & 0xffffu) * 2u;
#pragma unroll
            for (int e = 0; e < 8; ++e) {
                const unsigned bit = (m8[hq] >> e) & 1u;
                *(LAS u16*)(smem + (bit ? a : dummy)) = (u16)(v[hq][e >> 1] >> ((e & 1) * 16));
                a += bit * 2u;
            }
            pos += ((tot >> (hq * 16)) & 0xffffu) * 2u;
        }
        if (q2 == 1) ts = (pos - lbase) >> 1;
    }
    const unsigned T = (pos - lbase) >> 1;
    STAMP(13);
    mx = (mx & 0xffffu) > (mx >> 16) ? (mx & 0xffffu) : (mx >> 16);
    const unsigned win = choose_window(wave_sum(esum), ts, (wave_max(mx) >> 7) & 0xffu);
    bool raw = FORCE_RAW_OUT || __builtin_amdgcn_ballot_w64((sg & 0x80008000u) != 0u) != 0ull;
    const unsigned slot = OFF_EO + (unsigned)s * SSLOT;
    unsigned sl = 0;
    if (!raw) {
        const unsigned top = win & 0xffu, top2 = top * 0x00010001u;
        if (lane < 16 && T + lane < 8192u) *(LAS u16*)(smem + lbase + (T + lane) * 2u) = (u16)(top << 7);
        bst16<SYS>(wsr, slot + lane * 16, mk);
        unsigned e4[8][4], m[8][4], ec = 0;
#pragma unroll
        for (int r = 0; r < 8; ++r) {
            if ((unsigned)r * 1024u < T) {
                const bool act = (unsigned)(r * 1024 + lane * 16) < T;
                const u32x4 a = *(const LAS u32x4*)(smem + lbase + r * 2048 + lane * 32), b = *(const LAS u32x4*)(smem + lbase + r * 2048 + lane * 32 + 16);
                const unsigned w8[8] = {a.x, a.y, a.z, a.w, b.x, b.y, b.z, b.w};
                u32x4 lo; unsigned x1 = 0;
#pragma unroll
                for (int k = 0; k < 4; ++k) {
                    const unsigned ea = (w8[2 * k] >> 7) & 0x00ff00ffu, eb = (w8[2 * k + 1] >> 7) & 0x00ff00ffu;
                    const unsigned d = __builtin_amdgcn_perm(pkmin(pksub(top2, eb), 0x00030003u), pkmin(pksub(top2, ea), 0x00030003u), 0x06040200u);
                    e4[r][k] = __builtin_amdgcn_perm(eb, ea, 0x06040200u);
                    lo[k] = bfi(0x7f7f7f7fu, __builtin_amdgcn_perm(w8[2 * k + 1], w8[2 * k], 0x06040200u), d << 6);
                    x1 |= (d & 0x01010101u) << k;
                    m[r][k] = act ? (d + 0x01010101u) & 0x04040404u : 0u; ec += (unsigned)__builtin_popcount(m[r][k]);
                }
                if (act) {
                    bst16<SYS>(wsr, slot + 3072 + r * 1024 + lane * 16, lo);
                    bst2<SYS>(wsr, slot + 1024 + 8 + r * 128 + lane * 2, __builtin_amdgcn_perm(0u, x1 | (x1 >> 4), 0x0c0c0200u));
                }
            }
        }
        STAMP(14);
        const unsigned einc = wave_scan(ec), n2 = __builtin_amdgcn_readlane(einc, 63);
        unsigned h1 = 0xffffffffu;
        const unsigned p1b = ((T + 31u) >> 5) * 4u;
        if (n2 <= CAP2 - 16u) {
            const unsigned l2 = SM_SCR + (unsigned)wid * 3072u, l3 = l2 + CAP2, l4 = l3 + CAP3 + 8;
            unsigned eo = l2 + einc - ec;
#pragma unroll
            for (int r = 0; r < 8; ++r) {
                if ((unsigned)r * 1024u < T) {
#pragma unroll
                    for (int k = 0; k < 4; ++k) eo = append_flagged(smem, e4[r][k], m[r][k], eo, dummy);
                }
            }
            h1 = encode_levels(smem, wsr, win, n2, l2, l3, l4, slot + 1024 + 8 + p1b, 2040u - p1b, dummy, lane);
        }
        if (h1 == 0xffffffffu) raw = true;
        else {
            if (lane == 0) bst8<SYS>(wsr, slot + 1024, (u32x2){win | (n2 << 16), h1});
            sl = (8u + p1b + side_bytes(n2, h1) + 15u) >> 4;
        }
    }
    STAMP(15);
    WAIT_V(0);
    if (lane < NR && lane != p.rank) {
        const unsigned info = raw ? (1u << 30) : (T | (sl << 14));
        char* pw = ((char*)p.ws_all[lane] + p.ws_off);
        __hip_atomic_store((gu64*)(pw + OFF_FOUT + ((unsigned)p.rank * 1024u + (unsigned)s) * 8u), ((u64)seq << 32) | info, __ATOMIC_RELAXED, __HIP_MEMORY_SCOPE_SYSTEM);
    }
}

}

__global__ void __launch_bounds__(512, 2) gemm_ar_fused(Params p) {
    __shared__ __attribute__((aligned(16))) unsigned char smem_[SMEM_BYTES];
    LAS unsigned char* smem = (LAS unsigned char*)smem_;
    const int tid = threadIdx.x, wid = rfl(tid >> 6), lane = tid & 63;
    char* ws = ((char*)p.ws_all[p.rank] + p.ws_off);
    const rsrc_t wsr = make_rsrc(ws, WS_USED);
    const rsrc_t outr = make_rsrc(p.out, OUT_BYTES);
    gu32* tmo = (gu32*)(ws + OFF_TMO);
    gu32* seqp = (gu32*)(ws + OFF_SEQ + (unsigned)(blockIdx.x * 8 + wid) * 4u);
    const unsigned seqv = __hip_atomic_load(seqp, __ATOMIC_RELAXED, __HIP_MEMORY_SCOPE_AGENT);
    STAMP(0);
    const int xcd = blockIdx.x & 7, wi = blockIdx.x >> 3;
    const int nsplit = xcd < 2 ? 16 : (xcd == 2 ? 8 : (xcd == 3 ? 4 : 0));
    const bool is_gemm = wi < 16 + nsplit;
    const int cw = xcd >= 4 ? (wi - 16) * 4 + xcd - 4 : (xcd == 3 ? 64 + wi - 20 : 76 + wi - 24);
    const bool is_link = !is_gemm && wid < NWV;
    if (!is_gemm) {
        if (tid == 0) { *(LAS unsigned*)(smem + SM_TOKEN) = 0u; *(LAS unsigned*)(smem + SM_TOKEN + 4) = 0xffffffffu; }
        WAIT_L(0); BAR;
    }
    if (!is_link) {
        Order od{0, 0};
        const int e = !is_gemm ? (wid - NWV) * 84 + cw
                    : xcd >= 4 ? NPD + (xcd >= 6 ? 0 : 256) + wid * 32 + wi * 2 + (xcd & 1)
                    : xcd < 2 ? NENC + 256 + wid * 64 + wi * 2 + xcd
                    : wi < 16 ? NENC + wid * 32 + wi * 2 + (xcd & 1) : NENC + 768 + wid * 12 + (xcd == 2 ? wi - 16 : wi - 8);
        if (e >= PDELAY) __builtin_amdgcn_s_sleep(PSLEEP);
        for (int t = e; t < 4096; t = (t == e ? NENC + NONCE + e : t + NENC)) {
            int j, g, u; od.locate<128, 32>(t, j, g, u);
            pack_unit(p, ws, wsr, smem, (unsigned)wid * 16384u, j, g, u, lane, seqv);
            if (e >= NENC) break;
        }
        STAMP(7);
        if (is_gemm) __syncthreads();
    }
    const unsigned seq = rfl(seqv) + 1u;
    if (lane == 0) __hip_atomic_store(seqp, seq, __ATOMIC_RELAXED, __HIP_MEMORY_SCOPE_AGENT);
    if (!is_gemm) {
        if (wid >= NLW) return;
        const int lq = cw % 7;
        const int peer = (p.rank + 1 + lq) & 7;
        const rsrc_t pr = make_rsrc((char*)p.ws_all[peer] + p.ws_off, WS_USED);
        const rsrc_t por = make_rsrc(p.out_all[peer], OUT_BYTES);
#pragma unroll
        for (int i = 0; i < 4; ++i) {
            const unsigned m8 = (unsigned)(i * 64 + lane); u32x4 sel;
#pragma unroll
            for (int g = 0; g < 2; ++g) {
                unsigned rk = 0;
#pragma unroll
                for (int h = 0; h < 2; ++h) {
                    unsigned w = 0;
#pragma unroll
                    for (int e = 0; e < 2; ++e) { const unsigned bit = (m8 >> (g * 4 + h * 2 + e)) & 1u; w |= (bit ? (rk | ((rk + 4u) << 8)) : 0x0c0cu) << (16 * e); rk += bit; }
                    sel[g * 2 + h] = w;
                }
            }
            *(LAS u32x4*)(smem + SM_LUT + m8 * 16u) = sel;
        }
        if (lane < 16) {
            unsigned rk = 4u, sel = 0;
#pragma unroll
            for (int k = 0; k < 4; ++k) { const unsigned bit = ((unsigned)lane >> k) & 1u; sel |= (bit ? rk : (unsigned)k) << (8 * k); rk += bit; }
            *(LAS unsigned*)(smem + SM_LUTE + lane * 4) = sel;
        }
        const unsigned esc = (unsigned)wid * 16384u + 8192u;
        bool dead = false;
        const u32x4 pd = make_desc((char*)p.ws_all[peer] + p.ws_off, WS_USED);
        const unsigned one = 1u;
        LAS unsigned* token = (LAS unsigned*)(smem + SM_TOKEN);
#define QNEXT(dst) do { if (lane == 0) asm volatile("global_atomic_add %0, %1, %2, off sc0" : "+v"(dst) : "v"(qp), "v"(one) : "memory"); } while (0)
#define TOKEN_TAKE() do { for (;;) { unsigned old = 1u; \
            if (lane == 0) old = __hip_atomic_exchange(token, 1u, __ATOMIC_RELAXED, __HIP_MEMORY_SCOPE_WORKGROUP); \
            if (rfl(old) == 0u) break; __builtin_amdgcn_s_sleep(1); } } while (0)
#define TOKEN_GIVE() do { if (lane == 0) __hip_atomic_store(token, 0u, __ATOMIC_RELAXED, __HIP_MEMORY_SCOPE_WORKGROUP); } while (0)
        {
            gu32* qp = (gu32*)(ws + OFF_Q + (unsigned)lq * 128u);
            const unsigned qbase = (seq - 1u) * (1536u + NT);
            LAS unsigned* utok = token + 1;
            gu64* finp = (gu64*)(ws + OFF_FIN + (unsigned)peer * 512u + (unsigned)lane * 8u);
            Order od{0, 0};
            u64 rdy0 = 0, rdy1 = 0, fv = 0;
            unsigned nq = 0;
            UnitRegs ur = {};
            if (wid == 0) {
                QNEXT(nq);
                UNIT_LANDED(ur, nq);
                if (lane == 0) __hip_atomic_store(utok, nq - qbase, __ATOMIC_RELAXED, __HIP_MEMORY_SCOPE_WORKGROUP);
            }
            unsigned pfo = 0;
#define PUBLISH(n) do { if (pfo) { WAIT_V(n); if (lane == 0) __hip_atomic_store((gu32*)(ws + pfo), seq, __ATOMIC_RELAXED, __HIP_MEMORY_SCOPE_AGENT); pfo = 0; } } while (0)
            for (;;) {
                unsigned i;
                for (;;) {
                    unsigned old = 0xffffffffu;
                    if (lane == 0) old = __hip_atomic_exchange(utok, 0xffffffffu, __ATOMIC_RELAXED, __HIP_MEMORY_SCOPE_WORKGROUP);
                    i = rfl(old);
                    if (i != 0xffffffffu) break;
                    if (pfo) PUBLISH(0); else __builtin_amdgcn_s_sleep(1);
                }
                if (i >= 1536u) { if (lane == 0) __hip_atomic_store(utok, i, __ATOMIC_RELAXED, __HIP_MEMORY_SCOPE_WORKGROUP); break; }
                QNEXT(nq);
                int j, g, u; od.locate<32, 16>((int)i, j, g, u);
                const unsigned word = i < 32u ? early_word(i) : (unsigned)(j * 8 + g), rb = word >> 1;
                if (!((((word & 1u) ? rdy1 : rdy0) >> rb) & 1ull) && !dead) {
                    PUBLISH(0);
                    const bool was0 = (rdy0 | rdy1) == 0ull;
                    const u64 t0 = __builtin_amdgcn_s_memrealtime(); unsigned n = 0;
                    for (;;) {
                        const u64 f2 = __hip_atomic_load(finp, __ATOMIC_RELAXED, __HIP_MEMORY_SCOPE_SYSTEM);
                        rdy0 = __builtin_amdgcn_ballot_w64((unsigned)f2 >= seq); rdy1 = __builtin_amdgcn_ballot_w64((unsigned)(f2 >> 32) >= seq);
                        if ((((word & 1u) ? rdy1 : rdy0) >> rb) & 1ull) break;
                        if ((++n & 63u) == 0u && give_up(tmo, seq, t0)) { dead = true; break; }
                        __builtin_amdgcn_s_sleep(2);
                    }
                    __builtin_amdgcn_fence(__ATOMIC_ACQUIRE, "wavefront");
                    if (was0) STAMP(1);
                }
                const int step = j * 8 + 1 + lq, li = g ? (g - 1) * 16 + u : u;
                const unsigned slot0 = unit_slot(p.rank, j, g, u);
                const unsigned dst0 = g ? OFF_XL + (unsigned)(step * 64 + li) * 8192u : OFF_WL + (unsigned)(step * 32 + li) * 8192u;
                const bool refresh = (~rdy0 & 0x7777777777777776ull) != 0ull || (~rdy1 & 0x3333333333333333ull) != 0ull;
                for (;;) {
                    unsigned old = 1u;
                    if (lane == 0) old = __hip_atomic_exchange(token, 1u, __ATOMIC_RELAXED, __HIP_MEMORY_SCOPE_WORKGROUP);
                    if (rfl(old) == 0u) break;
                    if (pfo) PUBLISH(0); else __builtin_amdgcn_s_sleep(1);
                }
                unit_issue(ur, pd, slot0, lane);
                asm volatile("s_waitcnt vmcnt(6)" : "+v"(nq) :: "memory");
                if (lane == 0) __hip_atomic_store(utok, nq - qbase, __ATOMIC_RELAXED, __HIP_MEMORY_SCOPE_WORKGROUP);
                WAIT_VN(EGU);
                TOKEN_GIVE();
                UNIT_LANDED(ur, nq);
                PUBLISH(0);
                if (refresh) asm volatile("global_load_dwordx2 %0, %1, off sc0 sc1" : "+v"(fv) : "v"(finp) : "memory");
                pfo = (g ? OFF_XF : OFF_WF) + (unsigned)(((li >> 2) * NSTEP + step) * 4 + (li & 3)) * 4u;
                unit_unpack(ur, wsr, pr, smem, esc, slot0, dst0, lane);
                if (refresh) {
                    asm volatile("s_waitcnt vmcnt(8)" : "+v"(fv) :: "memory");
                    rdy0 = __builtin_amdgcn_ballot_w64((unsigned)fv >= seq); rdy1 = __builtin_amdgcn_ballot_w64((unsigned)(fv >> 32) >= seq);
                }
            }
            PUBLISH(0);
            STAMP(3);
        }
        {
            gu32* qp = (gu32*)(ws + OFF_Q + (unsigned)(7 + lq) * 128u);
            const unsigned qbase = (seq - 1u) * (1024u + NT * NLW);
            gu64* fbase = (gu64*)(ws + OFF_FOUT + (unsigned)peer * 8192u);
            const unsigned lbase = (unsigned)wid * 16384u;
            unsigned nq = 0; u64 fn = 0;
            StripRegs sr = {};
            QNEXT(nq);
            asm volatile("s_waitcnt vmcnt(0)" : "+v"(nq) :: "memory");
            unsigned s = rfl(nq) - qbase;
            bool first = true;
            while (s < 1024u) {
                unsigned fh = rfl((unsigned)(fn >> 32)), info = rfl((unsigned)fn);
                if (fh < seq && !dead) {
                    const u64 t0 = __builtin_amdgcn_s_memrealtime(); unsigned n = 0;
                    for (;;) {
                        const u64 f2 = __hip_atomic_load(fbase + s, __ATOMIC_RELAXED, __HIP_MEMORY_SCOPE_SYSTEM);
                        fh = rfl((unsigned)(f2 >> 32)); info = rfl((unsigned)f2);
                        if (fh >= seq) break;
                        if ((++n & 63u) == 0u && give_up(tmo, seq, t0)) { dead = true; break; }
                        __builtin_amdgcn_s_sleep(4);
                    }
                }
                __builtin_amdgcn_fence(__ATOMIC_ACQUIRE, "wavefront");
                if (dead) info = 3u << 30;
                if (first) STAMP(4);
                first = false;
                TOKEN_TAKE();
                strip_issue(sr, pd, s, info, lane);
                QNEXT(nq);
                WAIT_VN(EGS);
                TOKEN_GIVE();
                STRIP_LANDED(sr, fn, nq, 0);
                const unsigned sn = rfl(nq) - qbase;
                { gu64* fa = fbase + (sn & 1023u); asm volatile("global_load_dwordx2 %0, %1, off sc0 sc1" : "+v"(fn) : "v"(fa) : "memory"); }
                strip_stage1(sr, smem, lbase, esc, info, lane);
                strip_stage2(sr.mk, outr, por, smem, lbase, peer, s, info, lane);
                asm volatile("s_waitcnt vmcnt(16)" : "+v"(fn) :: "memory");
                s = sn;
            }
            STAMP(6);
        }
        return;
    }

    const bool split = (wi & 15) < nsplit;
    const int wr = wid >> 2, wc = wid & 3;
    constexpr int HTB = 16384;
    constexpr size_t ASTEP = 64 * 8192, BSTEP = 32 * 8192;
    const unsigned ldsw = (unsigned)wid * 1024u;
    const unsigned smbase = (unsigned)(uintptr_t)smem;
#define SA(b, h) (((b) * 2 + (h)) * HTB)
#define SB(b, h) ((4 + (b) * 2 + (h)) * HTB)
#define STAGE(bufoff, gptr) do { _Pragma("unroll") for (int _i = 0; _i < 2; ++_i) \
        __builtin_amdgcn_global_load_lds((const unsigned*)((gptr) + voff[_i]), (LAS unsigned*)(smem + (bufoff) + ldsw + _i * 8192), 16, 0, SYS); } while (0)
#define LDA(dst, b, h) do { _Pragma("unroll") for (int m = 0; m < 4; ++m) _Pragma("unroll") for (int k = 0; k < 2; ++k) dst[m][k] = *(const LAS bf16x8*)(smem + SA(b, h) + aoff + m * 2048 + k * 1024); } while (0)
#define LDB(dst, b, h) do { _Pragma("unroll") for (int n = 0; n < 2; ++n) _Pragma("unroll") for (int k = 0; k < 2; ++k) dst[n][k] = *(const LAS bf16x8*)(smem + SB(b, h) + boff + n * 2048 + k * 1024); } while (0)
#define MMA(ai, bj, At, Bt) do { __builtin_amdgcn_s_setprio(1); _Pragma("unroll") for (int m = 0; m < 4; ++m) _Pragma("unroll") for (int n = 0; n < 2; ++n) _Pragma("unroll") for (int k = 0; k < 2; ++k) \
        acc[ai][bj][m][n] = __builtin_amdgcn_mfma_f32_16x16x32_bf16(Bt[n][k], At[m][k], acc[ai][bj][m][n], 0, 0, 0); __builtin_amdgcn_s_setprio(0); } while (0)
#define ZERO_ACC(a) _Pragma("unroll") for (int b = 0; b < 2; ++b) _Pragma("unroll") for (int m = 0; m < 4; ++m) _Pragma("unroll") for (int n = 0; n < 2; ++n) acc[a][b][m][n] = (f32x4){0.f, 0.f, 0.f, 0.f};
#define NEED(s) do { if (__builtin_expect((avail & 0xffffu) <= (unsigned)(s), 0)) { \
        if (wid == 0) { const unsigned s0 = avail & 0xffffu; unsigned av; const u64 t0 = __builtin_amdgcn_s_memrealtime(); unsigned n = 0; \
            const unsigned pn = (s0 + (lane & 31)) & 127u, po = pollo + SMAP(pn) * 16u; \
            for (;;) { const u32x4 f = bld16<SYS>(wsr, po); \
                const bool ok = (s0 + (lane & 31) >= (unsigned)te) || (f.x >= seq && f.y >= seq && f.z >= seq && f.w >= seq); \
                const u64 mm = __builtin_amdgcn_ballot_w64(ok); const unsigned both = (unsigned)mm & (unsigned)(mm >> 32); \
                av = s0 + (both == 0xffffffffu ? 32u : (unsigned)__builtin_ctz(~both)); \
                if (av > (unsigned)(s)) break; \
                if ((++n & 63u) == 0u && give_up(tmo, seq, t0)) { av = 0x10000u | NSTEP; break; } \
                __builtin_amdgcn_s_sleep(1); } \
            asm volatile("ds_write_b32 %0, %1\n\ts_waitcnt lgkmcnt(0)" :: "v"(smbase + SM_AVAIL), "v"(av) : "memory"); } \
        CFENCE; BAR; CFENCE; \
        unsigned av2; asm volatile("ds_read_b32 %0, %1\n\ts_waitcnt lgkmcnt(0)" : "=v"(av2) : "v"(smbase + SM_AVAIL) : "memory"); \
        avail = rfl(av2); } } while (0)
#define LOOP_STAMPS do { } while (0)
#define OPAQUE_U(dst, src) unsigned dst##_v = (unsigned)(src); asm volatile("" : "+v"(dst##_v)); const unsigned dst = rfl(dst##_v);
    for (int job = 0; job < 2; ++job) {
        int tid = threadIdx.x; asm volatile("" : "+v"(tid));
        const int lane = tid & 63, fr = lane & 15, fq = lane >> 4;
        const bool pair = split && job == 0;
        const int hsel = wi >> 4;
        const int atile = 44 + (xcd < 2 ? xcd * 32 + wi : (xcd == 2 ? 64 : 80) + (wi & 15) + nsplit * hsel);
        if (job && atile >= 128) return;
        const int tm = job ? atile >> 3 : (xcd >> 1) * 4 + ((wi & 15) >> 2), tn = job ? atile & 7 : (xcd & 1) * 4 + (wi & 3);
        const char* cA = ws + OFF_XL + (size_t)tm * 32768;
        const char* cB = ws + OFF_WL + (size_t)tn * 32768;
        const unsigned voff[2] = {(unsigned)tid * 16u, (unsigned)tid * 16u + 8192u};
        const int aoff = lds_byte(wr * 64 + fr, fq * 8), boff = lds_byte(wc * 32 + fr, fq * 8);
        const unsigned pollo = lane < 32 ? OFF_XF + (unsigned)(tm * NSTEP) * 16u : OFF_WF + (unsigned)(tn * NSTEP) * 16u;
        const int kh = tm < 8 ? KH1 : (tm < 12 ? KH2 : KH3);
        const int tb = job ? NSTEP - kh : 0, te = pair ? 64 : (job ? NSTEP : NSTEP - kh);
        const unsigned nl = tm < 8 ? (unsigned)NL1 : 16u;
        auto smap = [nl, pair, hsel](unsigned n) -> unsigned {
            if (pair) return (((n & ~3u) << 1) | (unsigned)(hsel << 2) | (n & 3u)) & 127u;
            if (n < nl) return n * 8u;
            const unsigned m = n - nl, full = (16u - nl) * 8u;
            if (m < full) return (m & 7u) == 7u ? (nl + (m >> 3)) * 8u : (m & ~7u) + 1u + (m & 7u);
            const unsigned r = m - full; return (16u - nl + r / 7u) * 8u + 1u + r % 7u;
        };
        const unsigned stab0 = smap((unsigned)lane), stab1 = smap((unsigned)lane + 64u);
#define SMAP(n) smap(n)
#define STAB(n) ((unsigned)__builtin_amdgcn_readlane((n) & 64 ? stab1 : stab0, (n) & 63))
        unsigned avail = (unsigned)tb;
        f32x4 acc[2][2][4][2];
        ZERO_ACC(0) ZERO_ACC(1)
        bf16x8 At[4][2], B0[2][2], B1[2][2];
        NEED(tb + 1);
        if (!job) STAMP(2);
        { const size_t s0 = STAB(tb); STAGE(SB(0, 0), cB + s0 * BSTEP); STAGE(SA(0, 0), cA + s0 * ASTEP); STAGE(SB(0, 1), cB + s0 * BSTEP + HTB); STAGE(SA(0, 1), cA + s0 * ASTEP + HTB); }
        if (wr == 1) BAR;
        WAIT_V(4); BAR;
        unsigned sp = STAB(tb + 1);
        { const size_t s1 = sp; STAGE(SB(1, 0), cB + s1 * BSTEP); STAGE(SA(1, 0), cA + s1 * ASTEP); STAGE(SB(1, 1), cB + s1 * BSTEP + HTB); }
        WAIT_V(6); BAR;
        for (int t = tb; t < te; t += 2) {
            const bool last = (t == te - 2);
            const int tn2 = last ? tb : t + 2;
            const unsigned s1 = sp, s2 = STAB(tn2), s3 = STAB(tn2 + 1);
            sp = s3;
            const char* a1 = cA + s1 * ASTEP;
            const char* a2 = cA + s2 * ASTEP; const char* b2 = cB + s2 * BSTEP;
            const char* a3 = cA + s3 * ASTEP; const char* b3 = cB + s3 * BSTEP;
            if (!last) NEED(t + 3);
            LOOP_STAMPS;
            LDB(B0, 0, 0); SCHED; LDA(At, 0, 0); STAGE(SA(1, 1), a1 + HTB);
            WAIT_L(8); BAR; WAIT_L(0); MMA(0, 0, At, B0); BAR; SCHED;
            LDB(B1, 0, 1); STAGE(SB(0, 0), b2);
            BAR; WAIT_L(0); MMA(0, 1, At, B1); BAR;
            LDA(At, 0, 1); STAGE(SA(0, 0), a2);
            BAR; WAIT_L(0); MMA(1, 0, At, B0); BAR; SCHED;
            STAGE(SB(0, 1), b2 + HTB);
            WAIT_V(6); BAR; MMA(1, 1, At, B1); BAR;
            LDB(B0, 1, 0); SCHED; LDA(At, 1, 0); STAGE(SA(0, 1), a2 + HTB);
            WAIT_L(8); BAR; WAIT_L(0); MMA(0, 0, At, B0); BAR; SCHED;
            LDB(B1, 1, 1); STAGE(SB(1, 0), b3);
            BAR; WAIT_L(0); MMA(0, 1, At, B1); BAR;
            LDA(At, 1, 1); STAGE(SA(1, 0), a3);
            BAR; WAIT_L(0); MMA(1, 0, At, B0); BAR; SCHED;
            STAGE(SB(1, 1), b3 + HTB);
            WAIT_V(6); BAR; MMA(1, 1, At, B1); BAR;
        }
        if (wr == 0) BAR;
        if (!job) STAMP(12);
        int tz = threadIdx.x; asm volatile("" : "+v"(tz));
        const int lane2 = tz & 63;
        const unsigned po = OFF_PART + (unsigned)(tm * 8 + tn) * 262144u + (unsigned)wid * 32768u + (unsigned)lane2 * 16u;
        gu32* pfl = (gu32*)(ws + OFF_PF + (unsigned)((tm * 8 + tn) * 16 + wid) * 4u);
        const unsigned take = (pair ? hsel != 0 : job != 0) ? 2u : 1u, give = 3u - take;
#pragma unroll
        for (int h = 0; h < 2; ++h) {
            OPAQUE_U(g1, (give >> h) & 1u)
            if (g1) {
#pragma unroll
                for (int q = 0; q < 16; ++q) bst16<SC1>(wsr, po + (h * 16 + q) * 1024, __builtin_bit_cast(u32x4, acc[h][(q >> 3) & 1][(q >> 1) & 3][q & 1]));
                WAIT_V(0);
                if (lane2 == 0) __hip_atomic_store(pfl + h * 8, seq, __ATOMIC_RELAXED, __HIP_MEMORY_SCOPE_AGENT);
            }
        }
#pragma unroll
        for (int h = 0; h < 2; ++h) {
            OPAQUE_U(t1, (take >> h) & 1u)
            if (t1) {
                if (rfl(__hip_atomic_load(pfl + h * 8, __ATOMIC_RELAXED, __HIP_MEMORY_SCOPE_AGENT)) < seq) {
                    const u64 t0 = __builtin_amdgcn_s_memrealtime(); unsigned n = 0;
                    while (rfl(__hip_atomic_load(pfl + h * 8, __ATOMIC_RELAXED, __HIP_MEMORY_SCOPE_AGENT)) < seq) {
                        if ((++n & 63u) == 0u && give_up(tmo, seq, t0)) break;
                        __builtin_amdgcn_s_sleep(1);
                    }
                }
                __builtin_amdgcn_fence(__ATOMIC_ACQUIRE, "wavefront");
                u32x4 pv[16];
#pragma unroll
                for (int q = 0; q < 16; ++q) pv[q] = bld16<SC1>(wsr, po + (h * 16 + q) * 1024);
#pragma unroll
                for (int q = 0; q < 16; ++q) { acc[h][(q >> 3) & 1][(q >> 1) & 3][q & 1] += __builtin_bit_cast(f32x4, pv[q]);
                    asm volatile("" : "+v"(acc[h][(q >> 3) & 1][(q >> 1) & 3][q & 1])); }
                CFENCE; SCHED;
            }
        }
        const unsigned tmov = __hip_atomic_load(tmo, __ATOMIC_RELAXED, __HIP_MEMORY_SCOPE_AGENT);
        WAIT_V(0); BAR;
        if (!job) STAMP(3);
        const int fr2 = lane2 & 15, fq2 = lane2 >> 4;
        const bool dead = (avail >> 16) != 0u || rfl(tmov) == seq;
#pragma unroll
        for (int ai = 0; ai < 2; ++ai) {
            OPAQUE_U(t1, (take >> ai) & 1u)
            if (t1) {
#pragma unroll
                for (int m = 0; m < 4; ++m) {
                    const int row = ai * 128 + wr * 64 + m * 16 + fr2;
#pragma unroll
                    for (int bj = 0; bj < 2; ++bj) {
                        f32x4 v0 = acc[ai][bj][m][0], v1 = acc[ai][bj][m][1];
#pragma unroll
                        for (int k = 0; k < 4; ++k) { v0[k] = v0[k] <= 0.f ? 0.f : v0[k]; v1[k] = v1[k] <= 0.f ? 0.f : v1[k]; }
                        u32x4 w; w.x = cvt_pk_bf16(v0[0], v0[1]); w.y = cvt_pk_bf16(v0[2], v0[3]); w.z = cvt_pk_bf16(v1[0], v1[1]); w.w = cvt_pk_bf16(v1[2], v1[3]);
                        if (dead) w = (u32x4){0x7fc07fc0u, 0x7fc07fc0u, 0x7fc07fc0u, 0x7fc07fc0u};
                        const int chunk = bj * 16 + wc * 4 + fq2;
                        *(LAS u32x4*)(smem + row * 512 + ((chunk ^ (row & 7)) * 16)) = w;
                    }
                }
            }
        }
        __syncthreads();
        if (!job) STAMP(4);
        if ((take >> wr) & 1u) pack_strip(p, wsr, outr, smem, (tm * 8 + tn) * 8 + wid, wid, lane2, seq);
        if (!job) STAMP(5); else STAMP(11);
        if (!split) return;
        __syncthreads();
    }
}

extern "C" void kernel_launch(int rank, int nranks, ncclComm_t comm,
                              hipStream_t stream,
                              void* const* d_in, const int* in_sizes, int n_in,
                              void* d_out, int out_size,
                              const int* in_dtypes, int out_dtype,
                              void* const* d_out_all,
                              void* const* d_ws_all, size_t ws_bytes) {
    (void)comm; (void)in_dtypes; (void)out_dtype;
    if (nranks != NR || n_in != 2 || (unsigned)rank >= (unsigned)NR || ws_bytes < WS_TOTAL || in_sizes[0] != ND * KS || in_sizes[1] != KS * ND || out_size != ND * ND) {
        fprintf(stderr, "kernel_launch: unexpected problem shape\n");
        return;
    }
    Params p;
    p.x = (const u16*)d_in[0]; p.w = (const u16*)d_in[1]; p.out = (u16*)d_out;
    p.out_all = d_out_all; p.ws_all = d_ws_all; p.rank = rank;
    p.ws_off = WS_AT_END ? (ws_bytes - WS_TOTAL) & ~(size_t)((2u << 20) - 1u) : 0;
    hipLaunchKernelGGL(gemm_ar_fused, dim3(NWG), dim3(512), 0, stream, p);
    HIP_CHECK(hipGetLastError());
}
```

```cpp
#include <hip/hip_runtime.h>
#include <rccl/rccl.h>
#include <cstdio>
#include <cstdint>

#define HIP_CHECK(expr) do { hipError_t _e = (expr); if (_e != hipSuccess) { \
    fprintf(stderr, "HIP error %s at %s:%d\n", hipGetErrorString(_e), __FILE__, __LINE__); \
    return; } } while (0)

#define LAS __attribute__((address_space(3)))
#define GAS __attribute__((address_space(1)))
typedef unsigned short u16;
typedef unsigned long long u64;
typedef short bf16x8 __attribute__((ext_vector_type(8)));
typedef short s16x4 __attribute__((ext_vector_type(4)));
typedef float f32x4 __attribute__((ext_vector_type(4)));
typedef unsigned u32x4 __attribute__((ext_vector_type(4)));
typedef unsigned u32x2 __attribute__((ext_vector_type(2)));
typedef u16 u16x2 __attribute__((ext_vector_type(2)));
typedef GAS unsigned gu32;
typedef GAS u64 gu64;

#define FORCE_RAW_IN 0
#define FORCE_RAW_OUT 0
#define NT 12
constexpr int NWV = 4;
#define NLW 6
#define EGU 1
#define EGS 2
#define WS_AT_END 1
#define OAUX 18
#define LAG1 0
#define LAG2 7
#define LAG3 14
#define KH1 12
#define KH2 22
#define KH3 42
#define NL1 4
#define PDELAY 336
#define PSLEEP 64

namespace {

constexpr int NR = 8;
constexpr int ND = 8192;
constexpr int KS = 1024;
constexpr int RROWS = 4096, RCOLS = 2048;
constexpr int NSTEP = 128;
constexpr int NWG = 256;
constexpr int NPD = 84 * 4;
constexpr int NENC = NPD + 512;
constexpr int NONCE = 256 + 512 + 96;
static_assert(KH1 % 2 == 0 && KH2 % 2 == 0 && KH3 % 2 == 0 && KH1 >= 2 && NLW >= NWV && NLW <= 8, "knobs");
__device__ __forceinline__ constexpr int lag(int k) { return k == 0 ? 0 : k == 1 ? LAG1 : k == 2 ? LAG2 : LAG3; }
constexpr unsigned long long DEADLINE = 40000000ull;

constexpr unsigned USLOT = 8192 + 1024;
constexpr unsigned SSLOT = 11264;
constexpr unsigned OFF_PART = 0;
constexpr unsigned OFF_EO = OFF_PART + (32u << 20);
constexpr unsigned OFF_XL = OFF_EO + (12u << 20);
constexpr unsigned OFF_WL = OFF_XL + (64u << 20);
constexpr unsigned OFF_EX = OFF_WL + (32u << 20);
constexpr unsigned OFF_EW = OFF_EX + (18u << 20);
constexpr unsigned OFF_C = OFF_EW + (18u << 20);
constexpr unsigned WS_USED = OFF_C + (1u << 20);
constexpr unsigned OFF_TMO = OFF_C + 0x80;
constexpr unsigned OFF_Q = OFF_C + 0x400;
constexpr unsigned OFF_ECNT = OFF_C + 0x1000;
constexpr unsigned OFF_FIN = OFF_C + 0x8000;
constexpr unsigned OFF_SEQ = OFF_C + 0xa000;
constexpr unsigned OFF_PF = OFF_C + 0xc000;
constexpr unsigned OFF_XF = OFF_C + 0x10000;
constexpr unsigned OFF_WF = OFF_C + 0x18000;
constexpr unsigned OFF_FOUT = OFF_C + 0x20000;
static_assert(2048u * USLOT <= (18u << 20), "unit area");
static_assert(1024u * SSLOT <= (12u << 20), "strip area");
constexpr unsigned OUT_BYTES = (unsigned)ND * ND * 2u;

constexpr int SM_TOKEN = 131072 + 32, SM_DUMMY = 131072 + 64, SM_LUT = 131072 + 256, SM_LUTE = SM_LUT + 4096, SM_SCR = SM_LUTE + 128;
constexpr int SM_AVAIL = SM_SCR + 24576 + 64;
constexpr int SMEM_BYTES = SM_AVAIL + 16;
constexpr unsigned CAP2 = 2304, CAP3 = 512, CAP4 = 240;

struct Params {
    const u16* x; const u16* w; u16* out;
    void* const* out_all; void* const* ws_all;
    int rank;
    size_t ws_off;
};

typedef __amdgpu_buffer_rsrc_t rsrc_t;
__device__ __forceinline__ rsrc_t make_rsrc(const void* base, unsigned bytes) {
    const u64 b = (u64)base;
    const unsigned lo = __builtin_amdgcn_readfirstlane((unsigned)b), hi = __builtin_amdgcn_readfirstlane((unsigned)(b >> 32));
    return __builtin_amdgcn_make_buffer_rsrc((void*)(((u64)hi << 32) | lo), 0, bytes, 0x00020000);
}
constexpr int SC1 = 16, SYS = 17;
template <int AUX> __device__ __forceinline__ u32x4 bld16(rsrc_t r, unsigned off) { return __builtin_amdgcn_raw_buffer_load_b128(r, off, 0, AUX); }
template <int AUX> __device__ __forceinline__ void bst16(rsrc_t r, unsigned off, u32x4 v) { __builtin_amdgcn_raw_buffer_store_b128(v, r, off, 0, AUX); }
template <int AUX> __device__ __forceinline__ void bst8(rsrc_t r, unsigned off, u32x2 v) { __builtin_amdgcn_raw_buffer_store_b64(v, r, off, 0, AUX); }
template <int AUX> __device__ __forceinline__ void bst4(rsrc_t r, unsigned off, unsigned v) { __builtin_amdgcn_raw_buffer_store_b32(v, r, off, 0, AUX); }
template <int AUX> __device__ __forceinline__ void bst2(rsrc_t r, unsigned off, unsigned v) { __builtin_amdgcn_raw_buffer_store_b16((u16)v, r, off, 0, AUX); }

__device__ __forceinline__ u32x4 make_desc(const void* base, unsigned bytes) {
    const u64 b = (u64)base;
    return (u32x4){(unsigned)__builtin_amdgcn_readfirstlane((unsigned)b), (unsigned)__builtin_amdgcn_readfirstlane((unsigned)(b >> 32)), bytes, 0x00020000u};
}
#define ALD16(dst, desc, off, imm) asm volatile("buffer_load_dwordx4 %0, %1, %2, 0 offen offset:%3 sc0 sc1" : "+v"(dst) : "v"(off), "s"(desc), "i"(imm) : "memory")
constexpr unsigned OOB = 0xfffff000u;

#define WAIT_V(n) asm volatile("s_waitcnt vmcnt(" #n ")" ::: "memory")
#define WAIT_VN(n) asm volatile("s_waitcnt vmcnt(%0)" :: "i"(n) : "memory")
#define WAIT_L(n) asm volatile("s_waitcnt lgkmcnt(" #n ")" ::: "memory")
#define BAR __builtin_amdgcn_s_barrier()
#define SCHED __builtin_amdgcn_sched_barrier(0)
#define CFENCE asm volatile("" ::: "memory")
constexpr size_t WS_TOTAL = WS_USED;
#define STAMP(k) do { } while (0)

__device__ __forceinline__ unsigned rfl(unsigned v) { return __builtin_amdgcn_readfirstlane(v); }
__device__ __forceinline__ unsigned pkmax(unsigned a, unsigned b) {
    u16x2 x = __builtin_bit_cast(u16x2, a), y = __builtin_bit_cast(u16x2, b);
    return __builtin_bit_cast(unsigned, __builtin_elementwise_max(x, y));
}
__device__ __forceinline__ unsigned pkmin(unsigned a, unsigned b) {
    u16x2 x = __builtin_bit_cast(u16x2, a), y = __builtin_bit_cast(u16x2, b);
    return __builtin_bit_cast(unsigned, __builtin_elementwise_min(x, y));
}
__device__ __forceinline__ unsigned wave_max(unsigned v) {
#define MAX_STEP(ctrl, rmask) { const unsigned t = (unsigned)__builtin_amdgcn_update_dpp(0, (int)v, ctrl, rmask, 0xf, false); v = v > t ? v : t; }
    MAX_STEP(0x111, 0xf) MAX_STEP(0x112, 0xf) MAX_STEP(0x114, 0xf) MAX_STEP(0x118, 0xf) MAX_STEP(0x142, 0xa) MAX_STEP(0x143, 0xc)
    return __builtin_amdgcn_readlane(v, 63);
}
__device__ __forceinline__ unsigned pkmin1(unsigned a) {
    unsigned r; asm("v_pk_min_u16 %0, %1, 1 op_sel_hi:[1,0]" : "=v"(r) : "v"(a)); return r;
}
__device__ __forceinline__ unsigned cvt_pk_bf16(float lo, float hi) { unsigned r; asm volatile("v_cvt_pk_bf16_f32 %0, %1, %2" : "=v"(r) : "v"(lo), "v"(hi)); return r; }
__device__ __forceinline__ int lds_byte(int r, int c) { const int st = (r >> 4) * 2 + (c >> 5), rr = r & 15, cc = c & 31, ob = rr * 64 + cc * 2; return st * 1024 + (ob ^ (((ob >> 9) & 1) << 5)); }

__device__ __forceinline__ unsigned wave_scan(unsigned x) {
    x += (unsigned)__builtin_amdgcn_update_dpp(0, (int)x, 0x111, 0xf, 0xf, false);
    x += (unsigned)__builtin_amdgcn_update_dpp(0, (int)x, 0x112, 0xf, 0xf, false);
    x += (unsigned)__builtin_amdgcn_update_dpp(0, (int)x, 0x114, 0xf, 0xf, false);
    x += (unsigned)__builtin_amdgcn_update_dpp(0, (int)x, 0x118, 0xf, 0xf, false);
    x += (unsigned)__builtin_amdgcn_update_dpp(0, (int)x, 0x142, 0xa, 0xf, false);
    x += (unsigned)__builtin_amdgcn_update_dpp(0, (int)x, 0x143, 0xc, 0xf, false);
    return x;
}
__device__ __forceinline__ unsigned wave_sum(unsigned v) { return __builtin_amdgcn_readlane(wave_scan(v), 63); }
__device__ __forceinline__ unsigned pksub(unsigned a, unsigned b) {
    u16x2 x = __builtin_bit_cast(u16x2, a), y = __builtin_bit_cast(u16x2, b);
    return __builtin_bit_cast(unsigned, (u16x2)(x - y));
}
__device__ __forceinline__ unsigned bfi(unsigned mask, unsigned a, unsigned b) { return (a & mask) | (b & ~mask); }
__device__ __forceinline__ unsigned choose_window(unsigned esum, unsigned n, unsigned emax) {
    if (!n) return 21u;
    const unsigned num = esum * 32u + n * 63u, den = n * 32u;
    unsigned t = num / den; const unsigned alt = (num - t * den) * 5u > den * 2u ? 1u : 0u;
    t = t > emax ? emax : t; t = t < 21u ? 21u : (t > 253u ? 253u : t);
    return t | (alt << 9);
}
template <int NQ> __device__ __forceinline__ unsigned apply_escapes(unsigned (&e)[NQ], const unsigned (&m)[NQ], LAS unsigned char* smem, unsigned off) {
    unsigned sel[NQ], o[NQ]; u32x2 w[NQ];
#pragma unroll
    for (int q = 0; q < NQ; ++q) {
        sel[q] = *(const LAS unsigned*)(smem + SM_LUTE + __builtin_amdgcn_udot4(m[q], 0x08040201u, 0u, false));
        o[q] = off; off += (unsigned)__builtin_popcount(m[q]);
        const LAS unsigned* pw = (const LAS unsigned*)(smem + (o[q] & ~3u));
        w[q] = (u32x2){pw[0], pw[1]};
    }
#pragma unroll
    for (int q = 0; q < NQ; ++q) e[q] = __builtin_amdgcn_perm(__builtin_amdgcn_alignbyte(w[q].y, w[q].x, o[q]), e[q], sel[q]);
    return off;
}
__device__ __forceinline__ unsigned append_flagged(LAS unsigned char* smem, unsigned e, unsigned m, unsigned off, unsigned dummy) {
#pragma unroll
    for (int k = 0; k < 4; ++k) {
        const unsigned bit = (m >> (8 * k + 2)) & 1u;
        smem[bit ? off : dummy] = (unsigned char)(e >> (8 * k));
        off += bit;
    }
    return off;
}
__device__ __forceinline__ void decode_levels(LAS unsigned char* smem, unsigned h0, unsigned h1, unsigned o2, unsigned l3, unsigned l2, int lane) {
    const unsigned t = h0 & 0xffu, alt = (h0 >> 9) & 1u;
    const unsigned n2 = (h0 >> 16) < CAP2 ? (h0 >> 16) : CAP2, n3 = (h1 & 0xffffu) < CAP3 ? (h1 & 0xffffu) : CAP3;
    const unsigned o3 = o2 + ((n2 + 15u) >> 4) * 4u, o4 = o3 + ((n3 + 7u) >> 3) * 4u;
    const unsigned b3q = (t - 6u + alt) * 0x01010101u, spq = (t + 1u + alt) * 0x01010101u;
    unsigned eo = o4;
    for (unsigned r = 0; r * 256u < n3; ++r) {
        const unsigned g = r * 64u + lane;
        unsigned c = *(const LAS u16*)(smem + o3 + g * 2u); c = g * 4u < n3 ? c : 0u;
        const unsigned z = __builtin_amdgcn_perm(0u, c, 0x01010000u);
        const unsigned d = (z & 0x000f000fu) | ((z >> 4) & 0x0f000f00u);
        const unsigned is15 = (d + 0x01010101u) & 0x10101010u, is14 = ((d + 0x02020202u) & 0x10101010u) ^ is15;
        unsigned e[1], m[1];
        e[0] = bfi((is14 << 4) - (is14 >> 4), spq, b3q - d); m[0] = is15 >> 2;
        const unsigned ec = (unsigned)__builtin_popcount(m[0]), inc = wave_scan(ec);
        apply_escapes<1>(e, m, smem, eo + inc - ec);
        eo += __builtin_amdgcn_readlane(inc, 63);
        *(LAS unsigned*)(smem + l3 + g * 4u) = e[0];
    }
    const unsigned tab = (t - 3u) | ((t - 4u) << 8) | ((alt ? t + 1u : t - 5u) << 16);
    eo = l3;
    for (unsigned r = 0; r * 1024u < n2; ++r) {
        const unsigned g = r * 64u + lane;
        unsigned c = *(const LAS unsigned*)(smem + o2 + g * 4u); c = g * 16u < n2 ? c : 0u;
        unsigned e[4], m[4];
#pragma unroll
        for (int k = 0; k < 4; ++k) { const unsigned d = (c >> (2 * k)) & 0x03030303u; e[k] = __builtin_amdgcn_perm(0u, tab, d); m[k] = (d + 0x01010101u) & 0x04040404u; }
        const unsigned ec = (unsigned)__builtin_popcount(c & (c >> 1) & 0x55555555u), inc = wave_scan(ec);
        apply_escapes<4>(e, m, smem, eo + inc - ec);
        eo += __builtin_amdgcn_readlane(inc, 63);
        if (g * 16u < n2) *(LAS u32x4*)(smem + l2 + g * 16u) = (u32x4){e[0], e[1], e[2], e[3]};
    }
}
__device__ __forceinline__ unsigned encode_levels(LAS unsigned char* smem, rsrc_t wsr, unsigned win, unsigned n2, unsigned l2, unsigned l3, unsigned l4, unsigned g2, unsigned room, unsigned dummy, int lane) {
    const unsigned t = win & 0xffu, alt = (win >> 9) & 1u;
    if (lane < 16) smem[l2 + n2 + lane] = (unsigned char)(t - 3u);
    const unsigned t3 = (t - 3u) * 0x00010001u, sc = (alt ? t + 1u : t - 5u) * 0x00010001u;
    unsigned n3 = 0;
    for (unsigned r = 0; r * 1024u < n2; ++r) {
        const unsigned g = r * 64u + lane; const bool act = g * 16u < n2;
        const u32x4 e = *(const LAS u32x4*)(smem + l2 + (act ? g : 0u) * 16u);
        unsigned c = 0, m[4], ec = 0;
#pragma unroll
        for (int k = 0; k < 4; ++k) {
            unsigned dd[2];
#pragma unroll
            for (int h = 0; h < 2; ++h) {
                const unsigned x = (e[k] >> (8 * h)) & 0x00ff00ffu;
                const unsigned a = pkmin(pksub(t3, x), 0x00020002u);
                dd[h] = a + ((a >> 1) & pkmin1(x ^ sc));
            }
            const unsigned d = dd[0] | (dd[1] << 8);
            c |= d << (2 * k); m[k] = act ? (d + 0x01010101u) & 0x04040404u : 0u; ec += (unsigned)__builtin_popcount(m[k]);
        }
        const unsigned inc = wave_scan(ec), tot = __builtin_amdgcn_readlane(inc, 63);
        if (n3 + tot > CAP3) return 0xffffffffu;
        if (act) bst4<SYS>(wsr, g2 + g * 4u, c);
        unsigned eo = l3 + n3 + inc - ec;
#pragma unroll
        for (int k = 0; k < 4; ++k) eo = append_flagged(smem, e[k], m[k], eo, dummy);
        n3 += tot;
    }
    const unsigned b3 = t - 6u + alt;
    if (lane < 8) smem[l3 + n3 + lane] = (unsigned char)b3;
    const unsigned g3 = g2 + ((n2 + 15u) >> 4) * 4u, g4 = g3 + ((n3 + 7u) >> 3) * 4u;
    if (g4 - g2 > room) return 0xffffffffu;
    const unsigned b32 = b3 * 0x00010001u, sp = (t + 1u + alt) * 0x00010001u;
    unsigned n4 = 0;
    for (unsigned r = 0; r * 256u < n3; ++r) {
        const unsigned g = r * 64u + lane; const bool act = g * 4u < ((n3 + 7u) & ~7u);
        const unsigned e = *(const LAS unsigned*)(smem + l3 + (act ? g : 0u) * 4u);
        unsigned dd[2];
#pragma unroll
        for (int h = 0; h < 2; ++h) {
            const unsigned x = (e >> (8 * h)) & 0x00ff00ffu;
            const unsigned a = pkmin(pksub(b32, x), 0x000e000eu);
            dd[h] = a + (((a + 0x00020002u) >> 4) & pkmin1(x ^ sp));
        }
        const unsigned d = dd[0] | (dd[1] << 8);
        const unsigned m = act ? ((d + 0x01010101u) & 0x10101010u) >> 2 : 0u;
        const unsigned ec = (unsigned)__builtin_popcount(m), inc = wave_scan(ec), tot = __builtin_amdgcn_readlane(inc, 63);
        if (n4 + tot > CAP4) return 0xffffffffu;
        const unsigned y = d | (d >> 4);
        if (act) bst2<SYS>(wsr, g3 + g * 2u, __builtin_amdgcn_perm(0u, y, 0x0c0c0200u));
        if (tot) append_flagged(smem, e, m, l4 + n4 + inc - ec, dummy);
        n4 += tot;
    }
    if (g4 - g2 + n4 > room) return 0xffffffffu;
    if (n4) { const unsigned lv = *(const LAS unsigned*)(smem + l4 + lane * 4); if ((unsigned)lane * 4u < n4) bst4<SYS>(wsr, g4 + lane * 4, lv); }
    return n3 | (n4 << 16);
}
__device__ __forceinline__ unsigned side_bytes(unsigned n2, unsigned h1) { return ((n2 + 15u) >> 4) * 4u + (((h1 & 0xffffu) + 7u) >> 3) * 4u + (h1 >> 16); }
struct Order {
    int ph, start;
    template <int NWU, int NXU> __device__ __forceinline__ void locate(int i, int& j, int& g, int& u) {
        for (;;) {
            int c = (ph < 16) ? NWU : 0;
#pragma unroll
            for (int k = 0; k < 4; ++k) c += ((unsigned)(ph - lag(k)) < 16u) ? NXU : 0;
            if (i < start + c) break;
            start += c; ++ph;
        }
        u = i - start; g = 0; j = ph;
        if (ph < 16) { if (u < NWU) return; u -= NWU; }
#pragma unroll
        for (int k = 0; k < 4; ++k) {
            if ((unsigned)(ph - lag(k)) < 16u) { if (u < NXU) { g = 1 + k; j = ph - lag(k); return; } u -= NXU; }
        }
    }
};

__device__ __forceinline__ bool give_up(gu32* tmo, unsigned seq, u64 t0) {
    bool dead = rfl(__hip_atomic_load(tmo, __ATOMIC_RELAXED, __HIP_MEMORY_SCOPE_AGENT)) == seq;
    if (!dead && __builtin_amdgcn_s_memrealtime() - t0 > DEADLINE) {
        __hip_atomic_store(tmo, seq, __ATOMIC_RELAXED, __HIP_MEMORY_SCOPE_AGENT);
        dead = true;
    }
    return dead;
}
__device__ __forceinline__ unsigned early_word(unsigned u) { return (u / 3u) * 8u + 5u + u % 3u; }

__device__ __forceinline__ void pack_unit(const Params& p, char* ws, rsrc_t wsr, LAS unsigned char* smem, unsigned lwo, int j, int g, int u, int lane, unsigned seqv) {
    const bool isx = g != 0; const int ub = isx ? (u >> 4) * 64 + (g - 1) * 16 + (u & 15) : u;
    const int ra = p.rank >> 2, rb = p.rank & 3;
    LAS unsigned char* lw = smem + lwo;
    u32x4 v[8];
    if (isx) {
        const u16* src = p.x + (size_t)(ub * 64 + (lane & 15)) * KS + j * 64 + (lane >> 4) * 8;
#pragma unroll
        for (int c = 0; c < 8; ++c) v[c] = *(const u32x4*)(src + (size_t)(c >> 1) * 16 * KS + (c & 1) * 32);
    } else {
        constexpr int P = 144;
        const u16* src = p.w + (size_t)(j * 64 + (lane >> 3)) * ND + ub * 64 + (lane & 7) * 8;
        u32x4 tmp[8];
#pragma unroll
        for (int i = 0; i < 8; ++i) tmp[i] = *(const u32x4*)(src + (size_t)i * 8 * ND);
#pragma unroll
        for (int i = 0; i < 8; ++i) *(LAS u32x4*)(lw + (i * 8 + (lane >> 3)) * P + (lane & 7) * 16) = tmp[i];
        const int q = (lane >> 2) & 3, pp = lane & 3;
#pragma unroll
        for (int c = 0; c < 8; ++c) {
            const int k0 = (c & 1) * 32 + (lane >> 4) * 8, rbk = c >> 1;
            const int col = (rbk >> 1) * 32 + 8 * pp + 4 * (rbk & 1);
            const s16x4 a = __builtin_amdgcn_ds_read_tr16_b64_v4i16((LAS s16x4*)(lw + (k0 + q) * P + col * 2));
            const s16x4 b = __builtin_amdgcn_ds_read_tr16_b64_v4i16((LAS s16x4*)(lw + (k0 + 4 + q) * P + col * 2));
            const u32x2 a2 = __builtin_bit_cast(u32x2, a), b2 = __builtin_bit_cast(u32x2, b);
            v[c] = (u32x4){a2.x, a2.y, b2.x, b2.y};
        }
    }
    const unsigned loff = (lane & 15) * 64 + (((lane >> 4) * 16) ^ (((lane >> 3) & 1) << 5));
    const int cls = isx ? (ub >> 6) : (ub >> 5);
    const bool own = isx ? (cls == ra) : (cls == rb);
    const int li = isx ? (ub & 63) : (ub & 31);
    if (own) {
        const unsigned dst = (isx ? OFF_XL + (unsigned)(j * 8 * 64 + li) * 8192u : OFF_WL + (unsigned)(j * 8 * 32 + li) * 8192u) + loff;
#pragma unroll
        for (int c = 0; c < 8; ++c) bst16<SC1>(wsr, dst + c * 1024, v[c]);
    }
    const unsigned slot0 = (isx ? OFF_EX : OFF_EW) + (unsigned)(j * 128 + ub) * USLOT;
    const unsigned pay = slot0 + 1024 + lane * 16;
    const bool early = j == 0 && g == 0;
    if (early) {
#pragma unroll
        for (int c = 0; c < 8; ++c) bst16<SYS>(wsr, pay + c * 1024, v[c]);
        if (lane == 0) bst4<SYS>(wsr, slot0, 1u << 8);
    } else {
    unsigned e4[16], sm[16], mx = 0, esum = 0;
#pragma unroll
    for (int q = 0; q < 16; ++q) {
        const unsigned a = v[q >> 1][(q & 1) * 2], b = v[q >> 1][(q & 1) * 2 + 1];
        const unsigned lo = __builtin_amdgcn_perm(b, a, 0x06040200u), hi = __builtin_amdgcn_perm(b, a, 0x07050301u);
        e4[q] = bfi(0x01010101u, lo >> 7, hi << 1);
        sm[q] = bfi(0x7f7f7f7fu, lo, hi);
        esum = __builtin_amdgcn_udot4(e4[q], 0x01010101u, esum, false);
        mx = pkmax(mx, a & 0x7fff7fffu); mx = pkmax(mx, b & 0x7fff7fffu);
    }
    mx = (mx & 0xffffu) > (mx >> 16) ? (mx & 0xffffu) : (mx >> 16);
    const unsigned win = choose_window(wave_sum(esum), 4096u, wave_max(mx) >> 7);
    const unsigned top2 = (win & 0xffu) * 0x00010001u;
    unsigned p2[4] = {0u, 0u, 0u, 0u}, m[16], ec = 0;
#pragma unroll
    for (int q = 0; q < 16; ++q) {
        const unsigned da = pkmin(pksub(top2, e4[q] & 0x00ff00ffu), 0x00030003u), db = pkmin(pksub(top2, (e4[q] >> 8) & 0x00ff00ffu), 0x00030003u);
        const unsigned d = da | (db << 8);
        p2[q >> 2] |= d << (2 * (q & 3));
        m[q] = (d + 0x01010101u) & 0x04040404u; ec += (unsigned)__builtin_popcount(m[q]);
    }
    const unsigned einc = wave_scan(ec), n2 = __builtin_amdgcn_readlane(einc, 63);
    unsigned h1 = 0xffffffffu;
    if (!FORCE_RAW_IN && n2 <= CAP2) {
        const unsigned l2 = lwo + 10240, l3 = l2 + CAP2 + 16, l4 = l3 + CAP3 + 16, dummy = SM_DUMMY + lane * 2;
        unsigned eo = l2 + einc - ec;
#pragma unroll
        for (int q = 0; q < 16; ++q) eo = append_flagged(smem, e4[q], m[q], eo, dummy);
        h1 = encode_levels(smem, wsr, win, n2, l2, l3, l4, slot0 + 8, 1016u, dummy, lane);
    }
    if (h1 == 0xffffffffu) {
#pragma unroll
        for (int c = 0; c < 8; ++c) bst16<SYS>(wsr, pay + c * 1024, v[c]);
        if (lane == 0) bst4<SYS>(wsr, slot0, 1u << 8);
    } else {
#pragma unroll
        for (int i = 0; i < 4; ++i) bst16<SYS>(wsr, pay + i * 1024, (u32x4){sm[4 * i], sm[4 * i + 1], sm[4 * i + 2], sm[4 * i + 3]});
        bst16<SYS>(wsr, pay + 4096, (u32x4){p2[0], p2[1], p2[2], p2[3]});
        if (lane == 0) bst8<SYS>(wsr, slot0, (u32x2){win | (n2 << 16), h1});
    }
    }
    WAIT_V(0);
    const unsigned seq = rfl(seqv) + 1u;
    if (own && lane == 0) {
        const unsigned f = (isx ? OFF_XF : OFF_WF) + (unsigned)(((li >> 2) * NSTEP + j * 8) * 4 + (li & 3)) * 4u;
        __hip_atomic_store((gu32*)(ws + f), seq, __ATOMIC_RELAXED, __HIP_MEMORY_SCOPE_AGENT);
    }
    const unsigned per = isx ? 16u : 32u;
    unsigned ticket = per - 1u;
    if (!early) {
        if (lane == 0) ticket = __hip_atomic_fetch_add((gu32*)(ws + OFF_ECNT + (unsigned)(j * 12 + (isx ? cls * 4 + g - 1 : 8 + cls)) * 128u), 1u, __ATOMIC_RELAXED, __HIP_MEMORY_SCOPE_AGENT);
        ticket = rfl(ticket);
    }
    if ((ticket & (per - 1u)) == per - 1u) {
        const bool want = lane < NR && lane != p.rank && (isx ? ((lane >> 2) == cls) : ((lane & 3) == cls));
        if (want) {
            char* pw = ((char*)p.ws_all[lane] + p.ws_off);
            const unsigned word = early ? early_word((unsigned)li) : (unsigned)(j * 8 + g);
            __hip_atomic_store((gu32*)(pw + OFF_FIN + (unsigned)p.rank * 512u + word * 4u), seq, __ATOMIC_RELAXED, __HIP_MEMORY_SCOPE_SYSTEM);
        }
    }
}

struct UnitRegs { u32x4 a[5]; u32x4 sd; };
__device__ __forceinline__ unsigned unit_slot(int rank, int j, int g, int u) {
    const int ub = g ? (rank >> 2) * 64 + (g - 1) * 16 + u : (rank & 3) * 32 + u;
    return (g ? OFF_EX : OFF_EW) + (unsigned)(j * 128 + ub) * USLOT;
}
__device__ __forceinline__ void unit_issue(UnitRegs& r, u32x4 pd, unsigned slot0, int lane) {
    const unsigned oh = lane < 24 ? slot0 + lane * 16 : OOB, o0 = slot0 + 1024 + lane * 16, o1 = o0 + 4096;
    ALD16(r.sd, pd, oh, 0);
    ALD16(r.a[0], pd, o0, 0); ALD16(r.a[1], pd, o0, 1024); ALD16(r.a[2], pd, o0, 2048); ALD16(r.a[3], pd, o0, 3072);
    ALD16(r.a[4], pd, o1, 0);
}
#define UNIT_LANDED(r, n) asm volatile("s_waitcnt vmcnt(0)" : "+v"(r.a[0]), "+v"(r.a[1]), "+v"(r.a[2]), "+v"(r.a[3]), "+v"(r.a[4]), "+v"(r.sd), "+v"(n) :: "memory")
__device__ __forceinline__ void unit_unpack(const UnitRegs& r, rsrc_t wsr, rsrc_t pr, LAS unsigned char* smem, unsigned scr, unsigned slot0, unsigned dst0, int lane) {
    const unsigned dst = dst0 + (lane & 15) * 64 + (((lane >> 4) * 16) ^ (((lane >> 3) & 1) << 5));
    const unsigned h0 = rfl(r.sd.x), h1 = rfl(r.sd.y);
    if ((h0 >> 8) & 1u) {
        u32x4 b[3];
#pragma unroll
        for (int c = 0; c < 3; ++c) b[c] = bld16<SYS>(pr, slot0 + 1024 + lane * 16 + (5 + c) * 1024);
#pragma unroll
        for (int c = 0; c < 5; ++c) bst16<SC1>(wsr, dst + c * 1024, r.a[c]);
#pragma unroll
        for (int c = 0; c < 3; ++c) bst16<SC1>(wsr, dst + (5 + c) * 1024, b[c]);
        return;
    }
    u32x4 sd = r.sd;
    const unsigned len = 8u + side_bytes(h0 >> 16, h1);
    if (len > 384u) { if (lane >= 24 && (unsigned)lane * 16u < len) sd = bld16<SYS>(pr, slot0 + lane * 16); }
    *(LAS u32x4*)(smem + scr + lane * 16) = sd;
    const unsigned l3 = scr + 2048, l2 = l3 + CAP3 + 256;
    decode_levels(smem, h0, h1, scr + 8u, l3, l2, lane);
    const unsigned tq = (h0 & 0xffu) * 0x01010101u;
    unsigned e[16], m[16], ec = 0;
#pragma unroll
    for (int q = 0; q < 16; ++q) {
        const unsigned d = (r.a[4][q >> 2] >> (2 * (q & 3))) & 0x03030303u;
        m[q] = (d + 0x01010101u) & 0x04040404u; e[q] = tq - d;
    }
#pragma unroll
    for (int k = 0; k < 4; ++k) ec += (unsigned)__builtin_popcount(r.a[4][k] & (r.a[4][k] >> 1) & 0x55555555u);
    apply_escapes<16>(e, m, smem, l2 + wave_scan(ec) - ec);
#pragma unroll
    for (int c = 0; c < 8; ++c) {
        u32x4 o;
#pragma unroll
        for (int h = 0; h < 2; ++h) {
            const int q = 2 * c + h; const unsigned sm = r.a[q >> 2][q & 3];
            const unsigned lo = bfi(0x7f7f7f7fu, sm, e[q] << 7), hi = bfi(0x7f7f7f7fu, e[q] >> 1, sm);
            o[2 * h] = __builtin_amdgcn_perm(hi, lo, 0x05010400u); o[2 * h + 1] = __builtin_amdgcn_perm(hi, lo, 0x07030602u);
        }
        bst16<SC1>(wsr, dst + c * 1024, o);
    }
}

__device__ __forceinline__ unsigned strip_out_off(int r, int s, int lane) {
    const int tile = s >> 3, tm = tile >> 3, tn = tile & 7;
    const int row = (r >> 2) * RROWS + tm * 256 + (s & 7) * 32 + (lane >> 5);
    const int col = (r & 3) * RCOLS + tn * 256 + (lane & 31) * 8;
    return ((unsigned)row * ND + col) * 2u;
}

struct StripRegs { u32x4 mk; u32x4 lo[8]; u32x4 sd[2]; };
__device__ __forceinline__ void strip_issue(StripRegs& r, u32x4 pd, unsigned s, unsigned info, int lane) {
    if (info >> 30) return;
    const unsigned T = info & 0x3fffu, sl = (info >> 14) & 0xffu, slot = OFF_EO + s * SSLOT;
    const unsigned om = slot + lane * 16;
    ALD16(r.mk, pd, om, 0);
    { const unsigned o = (unsigned)lane < sl ? om : OOB; ALD16(r.sd[0], pd, o, 1024); }
    if (sl > 64u) { const unsigned o = (unsigned)lane + 64u < sl ? om : OOB; ALD16(r.sd[1], pd, o, 2048); }
#define STRIP_ROUND(k) if ((unsigned)(k) * 1024u < T) { const bool act = (unsigned)((k) * 1024 + lane * 16) < T; \
        const unsigned ol = act ? om + 3072 + ((k) >> 2) * 4096 : OOB; ALD16(r.lo[k], pd, ol, ((k) & 3) * 1024); }
    STRIP_ROUND(0) STRIP_ROUND(1) STRIP_ROUND(2) STRIP_ROUND(3) STRIP_ROUND(4) STRIP_ROUND(5) STRIP_ROUND(6) STRIP_ROUND(7)
}
#define STRIP_LANDED(r, fn, n, cnt) do { \
    asm volatile("s_waitcnt vmcnt(" #cnt ")" : "+v"(r.lo[0]), "+v"(r.lo[1]), "+v"(r.lo[2]), "+v"(r.lo[3]), "+v"(r.lo[4]), "+v"(r.lo[5]), "+v"(r.lo[6]), "+v"(r.lo[7]), \
        "+v"(r.mk), "+v"(r.sd[0]), "+v"(r.sd[1]), "+v"(fn), "+v"(n) :: "memory"); } while (0)
__device__ __forceinline__ void strip_stage1(const StripRegs& r, LAS unsigned char* smem, unsigned lbase, unsigned scr, unsigned info, int lane) {
    if (info >> 30) return;
    const unsigned T = info & 0x3fffu;
    const unsigned h0 = rfl(r.sd[0].x), h1 = rfl(r.sd[0].y);
    *(LAS u32x4*)(smem + scr + lane * 16) = r.sd[0];
    *(LAS u32x4*)(smem + scr + 1024 + lane * 16) = r.sd[1];
    const unsigned l3 = scr + 2048, l2 = l3 + CAP3 + 256;
    decode_levels(smem, h0, h1, scr + 8u + ((T + 31u) >> 5) * 4u, l3, l2, lane);
    const unsigned tq = (h0 & 0xffu) * 0x01010101u;
    unsigned x1[8], ec = 0;
#pragma unroll
    for (int k = 0; k < 8; ++k) {
        if ((unsigned)k * 1024u < T) {
            const unsigned z = __builtin_amdgcn_perm(0u, (unsigned)*(const LAS u16*)(smem + scr + 8 + k * 128 + lane * 2), 0x01010000u);
            x1[k] = (z & 0x000f000fu) | ((z >> 4) & 0x0f000f00u);
#pragma unroll
            for (int i = 0; i < 4; ++i) ec += (unsigned)__builtin_popcount((x1[k] >> i) & (r.lo[k][i] >> 7) & 0x01010101u);
        }
    }
    unsigned eo = l2 + wave_scan(ec) - ec;
    unsigned e[8][4];
#pragma unroll
    for (int k = 0; k < 8; ++k) {
        if ((unsigned)k * 1024u < T) {
            unsigned m[4];
#pragma unroll
            for (int i = 0; i < 4; ++i) {
                const unsigned d = ((x1[k] >> i) & 0x01010101u) | ((r.lo[k][i] >> 6) & 0x02020202u);
                m[i] = (d + 0x01010101u) & 0x04040404u; e[k][i] = tq - d;
            }
            eo = apply_escapes<4>(e[k], m, smem, eo);
        }
    }
#pragma unroll
    for (int k = 0; k < 8; ++k) {
        if ((unsigned)k * 1024u < T) {
            unsigned lo[4], hi[4];
#pragma unroll
            for (int i = 0; i < 4; ++i) { lo[i] = bfi(0x7f7f7f7fu, r.lo[k][i], e[k][i] << 7); hi[i] = (e[k][i] >> 1) & 0x7f7f7f7fu; }
            *(LAS u32x4*)(smem + lbase + k * 2048 + lane * 32) = (u32x4){lo[0], hi[0], lo[1], hi[1]};
            *(LAS u32x4*)(smem + lbase + k * 2048 + lane * 32 + 16) = (u32x4){lo[2], hi[2], lo[3], hi[3]};
        }
    }
}
__device__ __forceinline__ void strip_stage2(u32x4 mk, rsrc_t outr, rsrc_t por, LAS unsigned char* smem, unsigned lbase, int peer, unsigned s, unsigned info, int lane) {
    const unsigned oo = strip_out_off(peer, (int)s, lane);
    if (info >> 31) {
#pragma unroll
        for (int q = 0; q < 16; ++q) bst16<OAUX>(outr, oo + q * 2 * ND * 2, (u32x4){0x7fc07fc0u, 0x7fc07fc0u, 0x7fc07fc0u, 0x7fc07fc0u});
        return;
    }
    if (info >> 30) {
#pragma unroll
        for (int h = 0; h < 4; ++h) {
            u32x4 a[4];
#pragma unroll
            for (int q = 0; q < 4; ++q) a[q] = bld16<SYS>(por, oo + (h * 4 + q) * 2 * ND * 2);
#pragma unroll
            for (int q = 0; q < 4; ++q) bst16<OAUX>(outr, oo + (h * 4 + q) * 2 * ND * 2, a[q]);
        }
        return;
    }
    unsigned c2[8], x[8];
#pragma unroll
    for (int i = 0; i < 8; ++i) {
        const unsigned m16 = mk[i >> 1] >> ((i & 1) * 16);
        c2[i] = (unsigned)__builtin_popcount(m16 & 0xffu) | ((unsigned)__builtin_popcount((m16 >> 8) & 0xffu) << 16);
        x[i] = c2[i];
    }
#define SCAN_STEP(ctrl, rmask) _Pragma("unroll") for (int i = 0; i < 8; ++i) x[i] += (unsigned)__builtin_amdgcn_update_dpp(0, (int)x[i], ctrl, rmask, 0xf, false);
    SCAN_STEP(0x111, 0xf) SCAN_STEP(0x112, 0xf) SCAN_STEP(0x114, 0xf) SCAN_STEP(0x118, 0xf) SCAN_STEP(0x142, 0xa) SCAN_STEP(0x143, 0xc)
    unsigned pos = 0;
#pragma unroll
    for (int bt = 0; bt < 4; ++bt) {
        u32x4 sel[4], d[8]; unsigned off[8];
#pragma unroll
        for (int k = 0; k < 4; ++k) {
            const int i = bt * 2 + (k >> 1), hq = k & 1;
            const unsigned m8 = (mk[bt] >> (k * 8)) & 0xffu;
            sel[k] = *(const LAS u32x4*)(smem + SM_LUT + m8 * 16u);
            const unsigned tot = __builtin_amdgcn_readlane(x[i], 63);
            off[2 * k] = pos + (((x[i] - c2[i]) >> (hq * 16)) & 0xffffu);
            off[2 * k + 1] = off[2 * k] + (unsigned)__builtin_popcount(m8 & 15u);
            pos += (tot >> (hq * 16)) & 0xffffu;
        }
#pragma unroll
        for (int g = 0; g < 8; ++g) {
            const LAS u32x2* pl = (const LAS u32x2*)(smem + lbase + (off[g] & ~3u) * 2u);
            const u32x2 d0 = pl[0], d1 = pl[1];
            d[g] = (u32x4){d0.x, d0.y, d1.x, d1.y};
        }
#pragma unroll
        for (int k = 0; k < 4; ++k) {
            u32x4 o;
#pragma unroll
            for (int g = 0; g < 2; ++g) {
                const u32x4 dd = d[2 * k + g];
                const unsigned l4 = __builtin_amdgcn_alignbyte(dd.z, dd.x, off[2 * k + g]), h4 = __builtin_amdgcn_alignbyte(dd.w, dd.y, off[2 * k + g]);
                o[2 * g] = __builtin_amdgcn_perm(h4, l4, sel[k][2 * g]); o[2 * g + 1] = __builtin_amdgcn_perm(h4, l4, sel[k][2 * g + 1]);
            }
            bst16<OAUX>(outr, oo + (bt * 4 + k) * 2 * ND * 2, o);
        }
    }
}

__device__ __forceinline__ void pack_strip(const Params& p, rsrc_t wsr, rsrc_t outr, LAS unsigned char* smem, int s, int wid, int lane, unsigned seq) {
    const unsigned oo = strip_out_off(p.rank, s, lane);
    const unsigned lbase = (unsigned)wid * 16384u;
    unsigned pos = lbase, mx = 0, sg = 0, esum = 0, ts = 0;
    const unsigned dummy = SM_DUMMY + lane * 2;
    u32x4 mk = {0u, 0u, 0u, 0u};
#pragma unroll
    for (int q2 = 0; q2 < 8; ++q2) {
        u32x4 v[2]; unsigned m8[2];
#pragma unroll
        for (int hq = 0; hq < 2; ++hq) {
            const int q = q2 * 2 + hq, row = wid * 32 + 2 * q + (lane >> 5);
            v[hq] = *(const LAS u32x4*)(smem + row * 512 + (((lane & 31) ^ (row & 7)) * 16));
            bst16<SYS>(outr, oo + q * 2 * ND * 2, v[hq]);
            unsigned m = 0;
#pragma unroll
            for (int k = 0; k < 4; ++k) {
                sg |= v[hq][k];
                if (q2 < 2) {
                    mx = pkmax(mx, v[hq][k]);
                    esum = __builtin_amdgcn_udot4((v[hq][k] >> 7) & 0x00ff00ffu, 0x00010001u, esum, false);
                }
                const unsigned t = pkmin1(v[hq][k]);
                m |= ((t | (t >> 15)) & 3u) << (2 * k);
            }
            m8[hq] = m;
        }
        mk[q2 >> 1] |= (m8[0] | (m8[1] << 8)) << ((q2 & 1) * 16);
        const unsigned c2 = (unsigned)__builtin_popcount(m8[0]) | ((unsigned)__builtin_popcount(m8[1]) << 16);
        const unsigned inc = wave_scan(c2), exl = inc - c2, tot = __builtin_amdgcn_readlane(inc, 63);
#pragma unroll
        for (int hq = 0; hq < 2; ++hq) {
            unsigned a = pos + ((exl >> (hq * 16)) & 0xffffu) * 2u;
#pragma unroll
            for (int e = 0; e < 8; ++e) {
                const unsigned bit = (m8[hq] >> e) & 1u;
                *(LAS u16*)(smem + (bit ? a : dummy)) = (u16)(v[hq][e >> 1] >> ((e & 1) * 16));
                a += bit * 2u;
            }
            pos += ((tot >> (hq * 16)) & 0xffffu) * 2u;
        }
        if (q2 == 1) ts = (pos - lbase) >> 1;
    }
    const unsigned T = (pos - lbase) >> 1;
    STAMP(13);
    mx = (mx & 0xffffu) > (mx >> 16) ? (mx & 0xffffu) : (mx >> 16);
    const unsigned win = choose_window(wave_sum(esum), ts, (wave_max(mx) >> 7) & 0xffu);
    bool raw = FORCE_RAW_OUT || __builtin_amdgcn_ballot_w64((sg & 0x80008000u) != 0u) != 0ull;
    const unsigned slot = OFF_EO + (unsigned)s * SSLOT;
    unsigned sl = 0;
    if (!raw) {
        const unsigned top = win & 0xffu, top2 = top * 0x00010001u;
        if (lane < 16 && T + lane < 8192u) *(LAS u16*)(smem + lbase + (T + lane) * 2u) = (u16)(top << 7);
        bst16<SYS>(wsr, slot + lane * 16, mk);
        unsigned e4[8][4], m[8][4], ec = 0;
#pragma unroll
        for (int r = 0; r < 8; ++r) {
            if ((unsigned)r * 1024u < T) {
                const bool act = (unsigned)(r * 1024 + lane * 16) < T;
                const u32x4 a = *(const LAS u32x4*)(smem + lbase + r * 2048 + lane * 32), b = *(const LAS u32x4*)(smem + lbase + r * 2048 + lane * 32 + 16);
                const unsigned w8[8] = {a.x, a.y, a.z, a.w, b.x, b.y, b.z, b.w};
                u32x4 lo; unsigned x1 = 0;
#pragma unroll
                for (int k = 0; k < 4; ++k) {
                    const unsigned ea = (w8[2 * k] >> 7) & 0x00ff00ffu, eb = (w8[2 * k + 1] >> 7) & 0x00ff00ffu;
                    const unsigned d = __builtin_amdgcn_perm(pkmin(pksub(top2, eb), 0x00030003u), pkmin(pksub(top2, ea), 0x00030003u), 0x06040200u);
                    e4[r][k] = __builtin_amdgcn_perm(eb, ea, 0x06040200u);
                    lo[k] = bfi(0x7f7f7f7fu, __builtin_amdgcn_perm(w8[2 * k + 1], w8[2 * k], 0x06040200u), d << 6);
                    x1 |= (d & 0x01010101u) << k;
                    m[r][k] = act ? (d + 0x01010101u) & 0x04040404u : 0u; ec += (unsigned)__builtin_popcount(m[r][k]);
                }
                if (act) {
                    bst16<SYS>(wsr, slot + 3072 + r * 1024 + lane * 16, lo);
                    bst2<SYS>(wsr, slot + 1024 + 8 + r * 128 + lane * 2, __builtin_amdgcn_perm(0u, x1 | (x1 >> 4), 0x0c0c0200u));
                }
            }
        }
        STAMP(14);
        const unsigned einc = wave_scan(ec), n2 = __builtin_amdgcn_readlane(einc, 63);
        unsigned h1 = 0xffffffffu;
        const unsigned p1b = ((T + 31u) >> 5) * 4u;
        if (n2 <= CAP2 - 16u) {
            const unsigned l2 = SM_SCR + (unsigned)wid * 3072u, l3 = l2 + CAP2, l4 = l3 + CAP3 + 8;
            unsigned eo = l2 + einc - ec;
#pragma unroll
            for (int r = 0; r < 8; ++r) {
                if ((unsigned)r * 1024u < T) {
#pragma unroll
                    for (int k = 0; k < 4; ++k) eo = append_flagged(smem, e4[r][k], m[r][k], eo, dummy);
                }
            }
            h1 = encode_levels(smem, wsr, win, n2, l2, l3, l4, slot + 1024 + 8 + p1b, 2040u - p1b, dummy, lane);
        }
        if (h1 == 0xffffffffu) raw = true;
        else {
            if (lane == 0) bst8<SYS>(wsr, slot + 1024, (u32x2){win | (n2 << 16), h1});
            sl = (8u + p1b + side_bytes(n2, h1) + 15u) >> 4;
        }
    }
    STAMP(15);
    WAIT_V(0);
    if (lane < NR && lane != p.rank) {
        const unsigned info = raw ? (1u << 30) : (T | (sl << 14));
        char* pw = ((char*)p.ws_all[lane] + p.ws_off);
        __hip_atomic_store((gu64*)(pw + OFF_FOUT + ((unsigned)p.rank * 1024u + (unsigned)s) * 8u), ((u64)seq << 32) | info, __ATOMIC_RELAXED, __HIP_MEMORY_SCOPE_SYSTEM);
    }
}

}

__global__ void __launch_bounds__(512, 2) gemm_ar_fused(Params p) {
    __shared__ __attribute__((aligned(16))) unsigned char smem_[SMEM_BYTES];
    LAS unsigned char* smem = (LAS unsigned char*)smem_;
    const int tid = threadIdx.x, wid = rfl(tid >> 6), lane = tid & 63;
    char* ws = ((char*)p.ws_all[p.rank] + p.ws_off);
    const rsrc_t wsr = make_rsrc(ws, WS_USED);
    const rsrc_t outr = make_rsrc(p.out, OUT_BYTES);
    gu32* tmo = (gu32*)(ws + OFF_TMO);
    gu32* seqp = (gu32*)(ws + OFF_SEQ + (unsigned)(blockIdx.x * 8 + wid) * 4u);
    const unsigned seqv = __hip_atomic_load(seqp, __ATOMIC_RELAXED, __HIP_MEMORY_SCOPE_AGENT);
    STAMP(0);
    const int xcd = blockIdx.x & 7, wi = blockIdx.x >> 3;
    const int nsplit = xcd < 2 ? 16 : (xcd == 2 ? 8 : (xcd == 3 ? 4 : 0));
    const bool is_gemm = wi < 16 + nsplit;
    const int cw = xcd >= 4 ? (wi - 16) * 4 + xcd - 4 : (xcd == 3 ? 64 + wi - 20 : 76 + wi - 24);
    const bool is_link = !is_gemm && wid < NWV;
    if (!is_gemm) {
        if (tid == 0) { *(LAS unsigned*)(smem + SM_TOKEN) = 0u; *(LAS unsigned*)(smem + SM_TOKEN + 4) = 0xffffffffu; }
        WAIT_L(0); BAR;
    }
    if (!is_link) {
        Order od{0, 0};
        const int e = !is_gemm ? (wid - NWV) * 84 + cw
                    : xcd >= 4 ? NPD + (xcd >= 6 ? 0 : 256) + wid * 32 + wi * 2 + (xcd & 1)
                    : xcd < 2 ? NENC + 256 + wid * 64 + wi * 2 + xcd
                    : wi < 16 ? NENC + wid * 32 + wi * 2 + (xcd & 1) : NENC + 768 + wid * 12 + (xcd == 2 ? wi - 16 : wi - 8);
        if (e >= PDELAY) __builtin_amdgcn_s_sleep(PSLEEP);
        for (int t = e; t < 4096; t = (t == e ? NENC + NONCE + e : t + NENC)) {
            int j, g, u; od.locate<128, 32>(t, j, g, u);
            pack_unit(p, ws, wsr, smem, (unsigned)wid * 16384u, j, g, u, lane, seqv);
            if (e >= NENC) break;
        }
        STAMP(7);
        if (is_gemm) __syncthreads();
    }
    const unsigned seq = rfl(seqv) + 1u;
    if (lane == 0) __hip_atomic_store(seqp, seq, __ATOMIC_RELAXED, __HIP_MEMORY_SCOPE_AGENT);
    if (!is_gemm) {
        if (wid >= NLW) return;
        const int lq = cw % 7;
        const int peer = (p.rank + 1 + lq) & 7;
        const rsrc_t pr = make_rsrc((char*)p.ws_all[peer] + p.ws_off, WS_USED);
        const rsrc_t por = make_rsrc(p.out_all[peer], OUT_BYTES);
#pragma unroll
        for (int i = 0; i < 4; ++i) {
            const unsigned m8 = (unsigned)(i * 64 + lane); u32x4 sel;
#pragma unroll
            for (int g = 0; g < 2; ++g) {
                unsigned rk = 0;
#pragma unroll
                for (int h = 0; h < 2; ++h) {
                    unsigned w = 0;
#pragma unroll
                    for (int e = 0; e < 2; ++e) { const unsigned bit = (m8 >> (g * 4 + h * 2 + e)) & 1u; w |= (bit ? (rk | ((rk + 4u) << 8)) : 0x0c0cu) << (16 * e); rk += bit; }
                    sel[g * 2 + h] = w;
                }
            }
            *(LAS u32x4*)(smem + SM_LUT + m8 * 16u) = sel;
        }
        if (lane < 16) {
            unsigned rk = 4u, sel = 0;
#pragma unroll
            for (int k = 0; k < 4; ++k) { const unsigned bit = ((unsigned)lane >> k) & 1u; sel |= (bit ? rk : (unsigned)k) << (8 * k); rk += bit; }
            *(LAS unsigned*)(smem + SM_LUTE + lane * 4) = sel;
        }
        const unsigned esc = (unsigned)wid * 16384u + 8192u;
        bool dead = false;
        const u32x4 pd = make_desc((char*)p.ws_all[peer] + p.ws_off, WS_USED);
        const unsigned one = 1u;
        LAS unsigned* token = (LAS unsigned*)(smem + SM_TOKEN);
        unsigned pfo = 0;
#define PUBLISH(n) do { if (pfo) { WAIT_V(n); if (lane == 0) __hip_atomic_store((gu32*)(ws + pfo), seq, __ATOMIC_RELAXED, __HIP_MEMORY_SCOPE_AGENT); pfo = 0; } } while (0)
#define QNEXT(dst) do { if (lane == 0) asm volatile("global_atomic_add %0, %1, %2, off sc0" : "+v"(dst) : "v"(qp), "v"(one) : "memory"); } while (0)
#define SPIN_GUARD(n, t0) (((++(n)) & 0xfffffu) == 0u && ((t0) ? __builtin_amdgcn_s_memrealtime() - (t0) > 2 * DEADLINE : ((t0) = __builtin_amdgcn_s_memrealtime(), false)))
#define TOKEN_TAKE() do { unsigned sn_ = 0; u64 st_ = 0; for (;;) { unsigned old = 1u; \
            if (lane == 0) old = __hip_atomic_exchange(token, 1u, __ATOMIC_RELAXED, __HIP_MEMORY_SCOPE_WORKGROUP); \
            if (rfl(old) == 0u) break; \
            if (SPIN_GUARD(sn_, st_)) break; \
            if (pfo) PUBLISH(0); else __builtin_amdgcn_s_sleep(1); } } while (0)
#define TOKEN_GIVE() do { if (lane == 0) __hip_atomic_store(token, 0u, __ATOMIC_RELAXED, __HIP_MEMORY_SCOPE_WORKGROUP); } while (0)
        {
            gu32* qp = (gu32*)(ws + OFF_Q + (unsigned)lq * 128u);
            const unsigned qbase = (seq - 1u) * (1536u + NT);
            LAS unsigned* utok = token + 1;
            gu64* finp = (gu64*)(ws + OFF_FIN + (unsigned)peer * 512u + (unsigned)lane * 8u);
            Order od{0, 0};
            u64 rdy0 = 0, rdy1 = 0, fv = 0;
            unsigned nq = 0;
            UnitRegs ur = {};
            if (wid == 0) {
                QNEXT(nq);
                UNIT_LANDED(ur, nq);
                if (lane == 0) __hip_atomic_store(utok, nq - qbase, __ATOMIC_RELAXED, __HIP_MEMORY_SCOPE_WORKGROUP);
            }
            for (;;) {
                unsigned i, sn_ = 0; u64 st_ = 0;
                for (;;) {
                    unsigned old = 0xffffffffu;
                    if (lane == 0) old = __hip_atomic_exchange(utok, 0xffffffffu, __ATOMIC_RELAXED, __HIP_MEMORY_SCOPE_WORKGROUP);
                    i = rfl(old);
                    if (i != 0xffffffffu) break;
                    if (SPIN_GUARD(sn_, st_)) { i = 0x7fffffffu; dead = true; __hip_atomic_store(tmo, seq, __ATOMIC_RELAXED, __HIP_MEMORY_SCOPE_AGENT); break; }
                    if (pfo) PUBLISH(0); else __builtin_amdgcn_s_sleep(1);
                }
                if (i >= 1536u) { if (lane == 0) __hip_atomic_store(utok, i, __ATOMIC_RELAXED, __HIP_MEMORY_SCOPE_WORKGROUP); break; }
                QNEXT(nq);
                int j, g, u; od.locate<32, 16>((int)i, j, g, u);
                const unsigned word = i < 32u ? early_word(i) : (unsigned)(j * 8 + g), rb = word >> 1;
                if (!((((word & 1u) ? rdy1 : rdy0) >> rb) & 1ull) && !dead) {
                    PUBLISH(0);
                    const bool was0 = (rdy0 | rdy1) == 0ull;
                    const u64 t0 = __builtin_amdgcn_s_memrealtime(); unsigned n = 0;
                    for (;;) {
                        const u64 f2 = __hip_atomic_load(finp, __ATOMIC_RELAXED, __HIP_MEMORY_SCOPE_SYSTEM);
                        rdy0 = __builtin_amdgcn_ballot_w64((unsigned)f2 >= seq); rdy1 = __builtin_amdgcn_ballot_w64((unsigned)(f2 >> 32) >= seq);
                        if ((((word & 1u) ? rdy1 : rdy0) >> rb) & 1ull) break;
                        if ((++n & 63u) == 0u && give_up(tmo, seq, t0)) { dead = true; break; }
                        __builtin_amdgcn_s_sleep(2);
                    }
                    __builtin_amdgcn_fence(__ATOMIC_ACQUIRE, "wavefront");
                    if (was0) STAMP(1);
                }
                const int step = j * 8 + 1 + lq, li = g ? (g - 1) * 16 + u : u;
                const unsigned slot0 = unit_slot(p.rank, j, g, u);
                const unsigned dst0 = g ? OFF_XL + (unsigned)(step * 64 + li) * 8192u : OFF_WL + (unsigned)(step * 32 + li) * 8192u;
                const bool refresh = (~rdy0 & 0x7777777777777776ull) != 0ull || (~rdy1 & 0x3333333333333333ull) != 0ull;
                TOKEN_TAKE();
                unit_issue(ur, pd, slot0, lane);
                asm volatile("s_waitcnt vmcnt(6)" : "+v"(nq) :: "memory");
                if (lane == 0) __hip_atomic_store(utok, nq - qbase, __ATOMIC_RELAXED, __HIP_MEMORY_SCOPE_WORKGROUP);
                WAIT_VN(EGU);
                TOKEN_GIVE();
                UNIT_LANDED(ur, nq);
                PUBLISH(0);
                if (refresh) asm volatile("global_load_dwordx2 %0, %1, off sc0 sc1" : "+v"(fv) : "v"(finp) : "memory");
                pfo = (g ? OFF_XF : OFF_WF) + (unsigned)(((li >> 2) * NSTEP + step) * 4 + (li & 3)) * 4u;
                unit_unpack(ur, wsr, pr, smem, esc, slot0, dst0, lane);
                if (refresh) {
                    asm volatile("s_waitcnt vmcnt(8)" : "+v"(fv) :: "memory");
                    rdy0 = __builtin_amdgcn_ballot_w64((unsigned)fv >= seq); rdy1 = __builtin_amdgcn_ballot_w64((unsigned)(fv >> 32) >= seq);
                }
            }
            PUBLISH(0);
            STAMP(3);
        }
        {
            gu32* qp = (gu32*)(ws + OFF_Q + (unsigned)(7 + lq) * 128u);
            const unsigned qbase = (seq - 1u) * (1024u + NT * NLW);
            gu64* fbase = (gu64*)(ws + OFF_FOUT + (unsigned)peer * 8192u);
            const unsigned lbase = (unsigned)wid * 16384u;
            unsigned nq = 0; u64 fn = 0;
            StripRegs sr = {};
            QNEXT(nq);
            asm volatile("s_waitcnt vmcnt(0)" : "+v"(nq) :: "memory");
            unsigned s = rfl(nq) - qbase;
            bool first = true;
            while (s < 1024u) {
                unsigned fh = rfl((unsigned)(fn >> 32)), info = rfl((unsigned)fn);
                if (fh < seq && !dead) {
                    const u64 t0 = __builtin_amdgcn_s_memrealtime(); unsigned n = 0;
                    for (;;) {
                        const u64 f2 = __hip_atomic_load(fbase + s, __ATOMIC_RELAXED, __HIP_MEMORY_SCOPE_SYSTEM);
                        fh = rfl((unsigned)(f2 >> 32)); info = rfl((unsigned)f2);
                        if (fh >= seq) break;
                        if ((++n & 63u) == 0u && give_up(tmo, seq, t0)) { dead = true; break; }
                        __builtin_amdgcn_s_sleep(4);
                    }
                }
                __builtin_amdgcn_fence(__ATOMIC_ACQUIRE, "wavefront");
                if (dead) info = 3u << 30;
                if (first) STAMP(4);
                first = false;
                TOKEN_TAKE();
                strip_issue(sr, pd, s, info, lane);
                QNEXT(nq);
                WAIT_VN(EGS);
                TOKEN_GIVE();
                STRIP_LANDED(sr, fn, nq, 0);
                const unsigned sn = rfl(nq) - qbase;
                { gu64* fa = fbase + (sn & 1023u); asm volatile("global_load_dwordx2 %0, %1, off sc0 sc1" : "+v"(fn) : "v"(fa) : "memory"); }
                strip_stage1(sr, smem, lbase, esc, info, lane);
                strip_stage2(sr.mk, outr, por, smem, lbase, peer, s, info, lane);
                asm volatile("s_waitcnt vmcnt(16)" : "+v"(fn) :: "memory");
                s = sn;
            }
            STAMP(6);
        }
        return;
    }

    const bool split = (wi & 15) < nsplit;
    const int wr = wid >> 2, wc = wid & 3;
    constexpr int HTB = 16384;
    constexpr size_t ASTEP = 64 * 8192, BSTEP = 32 * 8192;
    const unsigned ldsw = (unsigned)wid * 1024u;
    const unsigned smbase = (unsigned)(uintptr_t)smem;
#define SA(b, h) (((b) * 2 + (h)) * HTB)
#define SB(b, h) ((4 + (b) * 2 + (h)) * HTB)
#define STAGE(bufoff, gptr) do { _Pragma("unroll") for (int _i = 0; _i < 2; ++_i) \
        __builtin_amdgcn_global_load_lds((const unsigned*)((gptr) + voff[_i]), (LAS unsigned*)(smem + (bufoff) + ldsw + _i * 8192), 16, 0, SYS); } while (0)
#define LDA(dst, b, h) do { _Pragma("unroll") for (int m = 0; m < 4; ++m) _Pragma("unroll") for (int k = 0; k < 2; ++k) dst[m][k] = *(const LAS bf16x8*)(smem + SA(b, h) + aoff + m * 2048 + k * 1024); } while (0)
#define LDB(dst, b, h) do { _Pragma("unroll") for (int n = 0; n < 2; ++n) _Pragma("unroll") for (int k = 0; k < 2; ++k) dst[n][k] = *(const LAS bf16x8*)(smem + SB(b, h) + boff + n * 2048 + k * 1024); } while (0)
#define MMA(ai, bj, At, Bt) do { __builtin_amdgcn_s_setprio(1); _Pragma("unroll") for (int m = 0; m < 4; ++m) _Pragma("unroll") for (int n = 0; n < 2; ++n) _Pragma("unroll") for (int k = 0; k < 2; ++k) \
        acc[ai][bj][m][n] = __builtin_amdgcn_mfma_f32_16x16x32_bf16(Bt[n][k], At[m][k], acc[ai][bj][m][n], 0, 0, 0); __builtin_amdgcn_s_setprio(0); } while (0)
#define ZERO_ACC(a) _Pragma("unroll") for (int b = 0; b < 2; ++b) _Pragma("unroll") for (int m = 0; m < 4; ++m) _Pragma("unroll") for (int n = 0; n < 2; ++n) acc[a][b][m][n] = (f32x4){0.f, 0.f, 0.f, 0.f};
#define NEED(s) do { if (__builtin_expect((avail & 0xffffu) <= (unsigned)(s), 0)) { \
        if (wid == 0) { const unsigned s0 = avail & 0xffffu; unsigned av; const u64 t0 = __builtin_amdgcn_s_memrealtime(); unsigned n = 0; \
            const unsigned pn = (s0 + (lane & 31)) & 127u, po = pollo + SMAP(pn) * 16u; \
            for (;;) { const u32x4 f = bld16<SYS>(wsr, po); \
                const bool ok = (s0 + (lane & 31) >= (unsigned)te) || (f.x >= seq && f.y >= seq && f.z >= seq && f.w >= seq); \
                const u64 mm = __builtin_amdgcn_ballot_w64(ok); const unsigned both = (unsigned)mm & (unsigned)(mm >> 32); \
                av = s0 + (both == 0xffffffffu ? 32u : (unsigned)__builtin_ctz(~both)); \
                if (av > (unsigned)(s)) break; \
                if ((++n & 63u) == 0u && give_up(tmo, seq, t0)) { av = 0x10000u | NSTEP; break; } \
                __builtin_amdgcn_s_sleep(1); } \
            asm volatile("ds_write_b32 %0, %1\n\ts_waitcnt lgkmcnt(0)" :: "v"(smbase + SM_AVAIL), "v"(av) : "memory"); } \
        CFENCE; BAR; CFENCE; \
        unsigned av2; asm volatile("ds_read_b32 %0, %1\n\ts_waitcnt lgkmcnt(0)" : "=v"(av2) : "v"(smbase + SM_AVAIL) : "memory"); \
        avail = rfl(av2); } } while (0)
#define LOOP_STAMPS do { } while (0)
#define OPAQUE_U(dst, src) unsigned dst##_v = (unsigned)(src); asm volatile("" : "+v"(dst##_v)); const unsigned dst = rfl(dst##_v);
    for (int job = 0; job < 2; ++job) {
        int tid = threadIdx.x; asm volatile("" : "+v"(tid));
        const int lane = tid & 63, fr = lane & 15, fq = lane >> 4;
        const bool pair = split && job == 0;
        const int hsel = wi >> 4;
        const int atile = 44 + (xcd < 2 ? xcd * 32 + wi : (xcd == 2 ? 64 : 80) + (wi & 15) + nsplit * hsel);
        if (job && atile >= 128) return;
        const int tm = job ? atile >> 3 : (xcd >> 1) * 4 + ((wi & 15) >> 2), tn = job ? atile & 7 : (xcd & 1) * 4 + (wi & 3);
        const char* cA = ws + OFF_XL + (size_t)tm * 32768;
        const char* cB = ws + OFF_WL + (size_t)tn * 32768;
        const unsigned voff[2] = {(unsigned)tid * 16u, (unsigned)tid * 16u + 8192u};
        const int aoff = lds_byte(wr * 64 + fr, fq * 8), boff = lds_byte(wc * 32 + fr, fq * 8);
        const unsigned pollo = lane < 32 ? OFF_XF + (unsigned)(tm * NSTEP) * 16u : OFF_WF + (unsigned)(tn * NSTEP) * 16u;
        const int kh = tm < 8 ? KH1 : (tm < 12 ? KH2 : KH3);
        const int tb = job ? NSTEP - kh : 0, te = pair ? 64 : (job ? NSTEP : NSTEP - kh);
        const unsigned nl = tm < 8 ? (unsigned)NL1 : 16u;
        auto smap = [nl, pair, hsel](unsigned n) -> unsigned {
            if (pair) return (((n & ~3u) << 1) | (unsigned)(hsel << 2) | (n & 3u)) & 127u;
            if (n < nl) return n * 8u;
            const unsigned m = n - nl, full = (16u - nl) * 8u;
            if (m < full) return (m & 7u) == 7u ? (nl + (m >> 3)) * 8u : (m & ~7u) + 1u + (m & 7u);
            const unsigned r = m - full; return (16u - nl + r / 7u) * 8u + 1u + r % 7u;
        };
        const unsigned stab0 = smap((unsigned)lane), stab1 = smap((unsigned)lane + 64u);
#define SMAP(n) smap(n)
#define STAB(n) ((unsigned)__builtin_amdgcn_readlane((n) & 64 ? stab1 : stab0, (n) & 63))
        unsigned avail = (unsigned)tb;
        f32x4 acc[2][2][4][2];
        ZERO_ACC(0) ZERO_ACC(1)
        bf16x8 At[4][2], B0[2][2], B1[2][2];
        NEED(tb + 1);
        if (!job) STAMP(2);
        { const size_t s0 = STAB(tb); STAGE(SB(0, 0), cB + s0 * BSTEP); STAGE(SA(0, 0), cA + s0 * ASTEP); STAGE(SB(0, 1), cB + s0 * BSTEP + HTB); STAGE(SA(0, 1), cA + s0 * ASTEP + HTB); }
        if (wr == 1) BAR;
        WAIT_V(4); BAR;
        unsigned sp = STAB(tb + 1);
        { const size_t s1 = sp; STAGE(SB(1, 0), cB + s1 * BSTEP); STAGE(SA(1, 0), cA + s1 * ASTEP); STAGE(SB(1, 1), cB + s1 * BSTEP + HTB); }
        WAIT_V(6); BAR;
        for (int t = tb; t < te; t += 2) {
            const bool last = (t == te - 2);
            const int tn2 = last ? tb : t + 2;
            const unsigned s1 = sp, s2 = STAB(tn2), s3 = STAB(tn2 + 1);
            sp = s3;
            const char* a1 = cA + s1 * ASTEP;
            const char* a2 = cA + s2 * ASTEP; const char* b2 = cB + s2 * BSTEP;
            const char* a3 = cA + s3 * ASTEP; const char* b3 = cB + s3 * BSTEP;
            if (!last) NEED(t + 3);
            LOOP_STAMPS;
            LDB(B0, 0, 0); SCHED; LDA(At, 0, 0); STAGE(SA(1, 1), a1 + HTB);
            WAIT_L(8); BAR; WAIT_L(0); MMA(0, 0, At, B0); BAR; SCHED;
            LDB(B1, 0, 1); STAGE(SB(0, 0), b2);
            BAR; WAIT_L(0); MMA(0, 1, At, B1); BAR;
            LDA(At, 0, 1); STAGE(SA(0, 0), a2);
            BAR; WAIT_L(0); MMA(1, 0, At, B0); BAR; SCHED;
            STAGE(SB(0, 1), b2 + HTB);
            WAIT_V(6); BAR; MMA(1, 1, At, B1); BAR;
            LDB(B0, 1, 0); SCHED; LDA(At, 1, 0); STAGE(SA(0, 1), a2 + HTB);
            WAIT_L(8); BAR; WAIT_L(0); MMA(0, 0, At, B0); BAR; SCHED;
            LDB(B1, 1, 1); STAGE(SB(1, 0), b3);
            BAR; WAIT_L(0); MMA(0, 1, At, B1); BAR;
            LDA(At, 1, 1); STAGE(SA(1, 0), a3);
            BAR; WAIT_L(0); MMA(1, 0, At, B0); BAR; SCHED;
            STAGE(SB(1, 1), b3 + HTB);
            WAIT_V(6); BAR; MMA(1, 1, At, B1); BAR;
        }
        if (wr == 0) BAR;
        if (!job) STAMP(12);
        int tz = threadIdx.x; asm volatile("" : "+v"(tz));
        const int lane2 = tz & 63;
        const unsigned po = OFF_PART + (unsigned)(tm * 8 + tn) * 262144u + (unsigned)wid * 32768u + (unsigned)lane2 * 16u;
        gu32* pfl = (gu32*)(ws + OFF_PF + (unsigned)((tm * 8 + tn) * 16 + wid) * 4u);
        const unsigned take = (pair ? hsel != 0 : job != 0) ? 2u : 1u, give = 3u - take;
#pragma unroll
        for (int h = 0; h < 2; ++h) {
            OPAQUE_U(g1, (give >> h) & 1u)
            if (g1) {
#pragma unroll
                for (int q = 0; q < 16; ++q) bst16<SC1>(wsr, po + (h * 16 + q) * 1024, __builtin_bit_cast(u32x4, acc[h][(q >> 3) & 1][(q >> 1) & 3][q & 1]));
                WAIT_V(0);
                if (lane2 == 0) __hip_atomic_store(pfl + h * 8, seq, __ATOMIC_RELAXED, __HIP_MEMORY_SCOPE_AGENT);
            }
        }
#pragma unroll
        for (int h = 0; h < 2; ++h) {
            OPAQUE_U(t1, (take >> h) & 1u)
            if (t1) {
                if (rfl(__hip_atomic_load(pfl + h * 8, __ATOMIC_RELAXED, __HIP_MEMORY_SCOPE_AGENT)) < seq) {
                    const u64 t0 = __builtin_amdgcn_s_memrealtime(); unsigned n = 0;
                    while (rfl(__hip_atomic_load(pfl + h * 8, __ATOMIC_RELAXED, __HIP_MEMORY_SCOPE_AGENT)) < seq) {
                        if ((++n & 63u) == 0u && give_up(tmo, seq, t0)) break;
                        __builtin_amdgcn_s_sleep(1);
                    }
                }
                __builtin_amdgcn_fence(__ATOMIC_ACQUIRE, "wavefront");
                u32x4 pv[16];
#pragma unroll
                for (int q = 0; q < 16; ++q) pv[q] = bld16<SC1>(wsr, po + (h * 16 + q) * 1024);
#pragma unroll
                for (int q = 0; q < 16; ++q) { acc[h][(q >> 3) & 1][(q >> 1) & 3][q & 1] += __builtin_bit_cast(f32x4, pv[q]);
                    asm volatile("" : "+v"(acc[h][(q >> 3) & 1][(q >> 1) & 3][q & 1])); }
                CFENCE; SCHED;
            }
        }
        const unsigned tmov = __hip_atomic_load(tmo, __ATOMIC_RELAXED, __HIP_MEMORY_SCOPE_AGENT);
        WAIT_V(0); BAR;
        if (!job) STAMP(3);
        const int fr2 = lane2 & 15, fq2 = lane2 >> 4;
        const bool dead = (avail >> 16) != 0u || rfl(tmov) == seq;
#pragma unroll
        for (int ai = 0; ai < 2; ++ai) {
            OPAQUE_U(t1, (take >> ai) & 1u)
            if (t1) {
#pragma unroll
                for (int m = 0; m < 4; ++m) {
                    const int row = ai * 128 + wr * 64 + m * 16 + fr2;
#pragma unroll
                    for (int bj = 0; bj < 2; ++bj) {
                        f32x4 v0 = acc[ai][bj][m][0], v1 = acc[ai][bj][m][1];
#pragma unroll
                        for (int k = 0; k < 4; ++k) { v0[k] = v0[k] <= 0.f ? 0.f : v0[k]; v1[k] = v1[k] <= 0.f ? 0.f : v1[k]; }
                        u32x4 w; w.x = cvt_pk_bf16(v0[0], v0[1]); w.y = cvt_pk_bf16(v0[2], v0[3]); w.z = cvt_pk_bf16(v1[0], v1[1]); w.w = cvt_pk_bf16(v1[2], v1[3]);
                        if (dead) w = (u32x4){0x7fc07fc0u, 0x7fc07fc0u, 0x7fc07fc0u, 0x7fc07fc0u};
                        const int chunk = bj * 16 + wc * 4 + fq2;
                        *(LAS u32x4*)(smem + row * 512 + ((chunk ^ (row & 7)) * 16)) = w;
                    }
                }
            }
        }
        __syncthreads();
        if (!job) STAMP(4);
        if ((take >> wr) & 1u) pack_strip(p, wsr, outr, smem, (tm * 8 + tn) * 8 + wid, wid, lane2, seq);
        if (!job) STAMP(5); else STAMP(11);
        if (!split) return;
        __syncthreads();
    }
}

extern "C" void kernel_launch(int rank, int nranks, ncclComm_t comm,
                              hipStream_t stream,
                              void* const* d_in, const int* in_sizes, int n_in,
                              void* d_out, int out_size,
                              const int* in_dtypes, int out_dtype,
                              void* const* d_out_all,
                              void* const* d_ws_all, size_t ws_bytes) {
    (void)comm; (void)in_dtypes; (void)out_dtype;
    if (nranks != NR || n_in != 2 || (unsigned)rank >= (unsigned)NR || ws_bytes < WS_TOTAL || in_sizes[0] != ND * KS || in_sizes[1] != KS * ND || out_size != ND * ND) {
        fprintf(stderr, "kernel_launch: unexpected problem shape\n");
        return;
    }
    Params p;
    p.x = (const u16*)d_in[0]; p.w = (const u16*)d_in[1]; p.out = (u16*)d_out;
    p.out_all = d_out_all; p.ws_all = d_ws_all; p.rank = rank;
    p.ws_off = WS_AT_END ? (ws_bytes - WS_TOTAL) & ~(size_t)((2u << 20) - 1u) : 0;
    hipLaunchKernelGGL(gemm_ar_fused, dim3(NWG), dim3(512), 0, stream, p);
    HIP_CHECK(hipGetLastError());
}
```
